```python
import numpy as np

kernel_name = "gpu_gemm_a2a_m8192_k8192_n8192_bf16_none"


ABSMAX_RTOL = 0.008

COMPUTE_DTYPE = "bfloat16"
EPILOGUE = "none"
IN_QUANT = None
OUT_QUANT = None
IN_DTYPES = {"x": "bfloat16", "w_mat": "bfloat16"}
WORLD_SIZE = 8

def _to_bf16(x):
    u = np.ascontiguousarray(x, dtype=np.float32).view(np.uint32)
    return ((u + 0x7FFF + ((u >> 16) & 1)) & 0xFFFF0000).view(np.float32)

def setup_inputs():

    x = _to_bf16(np.random.randn(8192, 8192).astype(np.float32))
    w_mat = _to_bf16((np.random.randn(8192, 8192) / np.sqrt(8192)).astype(np.float32))
    return {"x": x, "w_mat": w_mat}

def _epilogue(y):
    return y

def reference(x, w_mat):
    y = (x @ w_mat).astype(np.float32)
    return _epilogue(y)
```

```cpp
#include <hip/hip_runtime.h>
#include <rccl/rccl.h>
#include <cstdio>

#define HIP_CHECK(expr) do { hipError_t _e = (expr); if (_e != hipSuccess) { \
    fprintf(stderr, "HIP error %s at %s:%d\n", hipGetErrorString(_e), __FILE__, __LINE__); \
    return; } } while (0)

namespace ga2a {

#define LAS __attribute__((address_space(3)))
typedef unsigned short u16;
typedef short bf16x8 __attribute__((ext_vector_type(8)));
typedef short bf16x4 __attribute__((ext_vector_type(4)));
typedef float f32x4 __attribute__((ext_vector_type(4)));
typedef float f32x2 __attribute__((ext_vector_type(2)));
typedef __bf16 bf16x2n __attribute__((ext_vector_type(2)));
typedef unsigned u32x4 __attribute__((ext_vector_type(4)));

constexpr int NR = 8;
constexpr int M_ = 1024, N_ = 8192, K_ = 8192;
constexpr unsigned HTB = 16384;
constexpr unsigned A_BYTES = (unsigned)M_ * K_ * 2u, W_BYTES = (unsigned)K_ * N_ * 2u;
constexpr unsigned OUT_BYTES = 8192u * 1024u * 2u;
constexpr unsigned SLOT = 131072u;
constexpr unsigned WS_PART = 0u;
constexpr unsigned WS_CTRL = 128u << 20;
constexpr unsigned WS_PFLAG = WS_CTRL;
constexpr unsigned WS_CALL = WS_CTRL + 131072u;
constexpr unsigned WS_DONE = WS_CTRL + 163840u;
constexpr unsigned WS_TMO = WS_CTRL + 172032u;
constexpr unsigned WS_STAMP = WS_CTRL + 262144u;
constexpr unsigned WS_USED = WS_CTRL + (1u << 20);
constexpr unsigned long long DEADLINE = 50000000ull;

constexpr unsigned kstepA = 128u, kstepB = 64u * N_ * 2u;
constexpr unsigned hA = 128u * K_ * 2u, hB = 256u;
constexpr unsigned iA = 64u * K_ * 2u, iB = 32u * N_ * 2u;

#define TE(j, c, pm) (unsigned char)(((j) << 4) | ((c) << 2) | (pm))
#define TCOL(j, c) TE(j, c, 0), TE(j, c, 1), TE(j, c, 2), TE(j, c, 3)
__device__ const unsigned char kTiles[128] = {
    TCOL(1, 0), TCOL(2, 0), TCOL(3, 0), TCOL(4, 0), TCOL(5, 0), TCOL(6, 0), TCOL(7, 0),
    TE(1, 1, 0), TE(1, 1, 1), TE(2, 1, 0), TE(2, 1, 1),
    TE(1, 1, 2), TE(1, 1, 3), TE(2, 1, 2), TE(2, 1, 3),
    TCOL(3, 1), TCOL(4, 1), TCOL(5, 1), TCOL(6, 1), TCOL(7, 1), TCOL(1, 2), TCOL(2, 2),
    TE(1, 3, 0), TE(1, 3, 1), TE(2, 3, 0), TE(2, 3, 1), TE(3, 3, 0), TE(3, 3, 1), TE(4, 3, 0), TE(4, 3, 1),
    TE(5, 3, 0), TE(5, 3, 1), TE(6, 3, 0), TE(6, 3, 1), TE(7, 3, 0), TE(7, 3, 1),
    TE(3, 2, 0), TE(3, 2, 1), TE(3, 2, 2), TE(4, 2, 0), TE(4, 2, 1), TE(4, 2, 2),
    TCOL(5, 2), TCOL(6, 2), TCOL(7, 2),
    TE(1, 3, 2), TE(1, 3, 3), TE(2, 3, 2), TE(2, 3, 3), TE(3, 3, 2), TE(3, 3, 3), TE(4, 3, 2), TE(4, 3, 3),
    TE(5, 3, 2), TE(5, 3, 3), TE(6, 3, 2), TE(6, 3, 3), TE(7, 3, 2), TE(7, 3, 3),
    TE(3, 2, 3), TE(4, 2, 3),
    TCOL(0, 0), TCOL(0, 1), TCOL(0, 2), TCOL(0, 3)};

struct Params {
  const u16* A;
  const u16* W;
  char* out[NR];
  char* ws[NR];
  int rank;
};

constexpr int T_PS2 = 1, T_PS3 = 2, T_FLAG = 4, T_POLL = 6, T_L0 = 7, T_V0 = 9, T_L1 = 10, T_V1 = 12;
constexpr int S0_FIRST = 10, S0_N = 6;
constexpr int S1_FIRST = 13, S1_N = 6;
constexpr bool slot_at(int first, int n, int i) { return (i >= first && i < first + n) || (i + 16 >= first && i + 16 < first + n); }

__device__ __forceinline__ unsigned pk2(float lo, float hi) {
  f32x2 v = {lo, hi};
  return __builtin_bit_cast(unsigned, __builtin_convertvector(v, bf16x2n));
}
__device__ __forceinline__ u32x4 pack_pair(const f32x4& v0, const f32x4& v1) {
  unsigned d00 = pk2(v0[0], v0[1]), d01 = pk2(v0[2], v0[3]);
  unsigned d10 = pk2(v1[0], v1[1]), d11 = pk2(v1[2], v1[3]);
  auto r0 = __builtin_amdgcn_permlane16_swap(d00, d10, false, false);
  auto r1 = __builtin_amdgcn_permlane16_swap(d01, d11, false, false);
  u32x4 w; w.x = r0[0]; w.y = r1[0]; w.z = r0[1]; w.w = r1[1];
  return w;
}
__device__ __forceinline__ float blo(unsigned x) { return __builtin_bit_cast(float, x << 16); }
__device__ __forceinline__ float bhi(unsigned x) { return __builtin_bit_cast(float, x & 0xffff0000u); }

struct St {
  f32x4 acc[2][2][4][2];
  bf16x8 At[4][2];
  bf16x4 B0[2][2][2], B1[2][2][2];
  u32x4 tmp[8];
  float red[8];
  u32x4 R0, R1;
  unsigned pollv;
  unsigned voffA, voffB, aoff0, aoff1, boff0, boff1, vlane16, vgat, vdst;
  u32x4 srdA, srdB, srdWs, srdPrev;
  unsigned ldsw, curA, curB, nxtA, nxtB;
  unsigned pwCur, pwPrev;
  unsigned prd;
  unsigned pflag;
  unsigned need;
  unsigned call, poison;
  unsigned d0lo, d0hi, d0off, d1lo, d1hi, d1off;
  unsigned pdlo, pdhi, pdoff;
  int ts0, ts1, TS0, TS1, wslot;
  char* wsp;
};

#define SA(b, h) (((b) * 2u + (h)) * HTB)
#define SB(b, h) ((4u + (b) * 2u + (h)) * HTB)
#define DMA2(ldsoff, voff, srd, sbase, simm, step) do { unsigned _t; asm volatile( \
    "s_add_u32 m0, %1, %2\n\ts_add_u32 %0, %6, %7\n\tbuffer_load_dwordx4 %4, %5, %0 offen lds\n\t" \
    "s_add_u32 m0, %1, %3\n\ts_add_u32 %0, %6, %8\n\tbuffer_load_dwordx4 %4, %5, %0 offen lds" \
    : "=&s"(_t) : "s"(s.ldsw), "i"(ldsoff), "i"((ldsoff) + 8192u), "v"(voff), "s"(srd), "s"(sbase), "i"(simm), "i"((simm) + (step)) : "memory", "scc"); } while (0)
#define STAGE_A(bufoff, sbase, simm) DMA2((bufoff), s.voffA, s.srdA, (sbase), (simm), iA)
#define STAGE_B(bufoff, sbase, simm) DMA2((bufoff), s.voffB, s.srdB, (sbase), (simm), iB)
__device__ __forceinline__ int lane_id() { int l; asm volatile("v_mbcnt_lo_u32_b32 %0, -1, 0\n\tv_mbcnt_hi_u32_b32 %0, -1, %0" : "=v"(l)); return l; }
#define RD128(dst, addr, off) asm volatile("ds_read_b128 %0, %1 offset:%2" : "=v"(dst) : "v"(addr), "i"(off))
#define RDTR(dst, addr, off) asm volatile("ds_read_b64_tr_b16 %0, %1 offset:%2" : "=v"(dst) : "v"(addr), "i"(off))
#define LDA(b, h) do { _Pragma("unroll") for (int m = 0; m < 4; ++m) { RD128(s.At[m][0], s.aoff0, SA(b, h) + m * 2048); RD128(s.At[m][1], s.aoff1, SA(b, h) + m * 2048); } } while (0)
#define LDB(dst, b, h) do { _Pragma("unroll") for (int k = 0; k < 2; ++k) { \
    RDTR(dst[0][k][0], s.boff0, SB(b, h) - 65536u + k * 8192); RDTR(dst[0][k][1], s.boff0, SB(b, h) - 65536u + k * 8192 + 1024); } \
    _Pragma("unroll") for (int k = 0; k < 2; ++k) { \
    RDTR(dst[1][k][0], s.boff1, SB(b, h) - 65536u + k * 8192); RDTR(dst[1][k][1], s.boff1, SB(b, h) - 65536u + k * 8192 + 1024); } } while (0)
#define WAIT_A() asm volatile("s_waitcnt lgkmcnt(0)" : "+v"(s.At[0][0]), "+v"(s.At[0][1]), "+v"(s.At[1][0]), "+v"(s.At[1][1]), "+v"(s.At[2][0]), "+v"(s.At[2][1]), "+v"(s.At[3][0]), "+v"(s.At[3][1]) :: "memory")
#define WAIT_B(Bt) asm volatile("s_waitcnt lgkmcnt(0)" : "+v"(Bt[0][0][0]), "+v"(Bt[0][0][1]), "+v"(Bt[0][1][0]), "+v"(Bt[0][1][1]), "+v"(Bt[1][0][0]), "+v"(Bt[1][0][1]), "+v"(Bt[1][1][0]), "+v"(Bt[1][1][1]) :: "memory")
#define CAT(x) __builtin_shufflevector(x[0], x[1], 0, 1, 2, 3, 4, 5, 6, 7)
#define WAIT_L(n) asm volatile("s_waitcnt lgkmcnt(" #n ")" ::: "memory")
#define BAR __builtin_amdgcn_s_barrier()
#define SCHED __builtin_amdgcn_sched_barrier(0)
#define ST128(data, voff, srd, soff, bits) asm volatile("buffer_store_dwordx4 %0, %1, %2, %3 offen " bits "\n\ts_nop 1" :: "v"(data), "v"(voff), "s"(srd), "s"(soff) : "memory")
#define LD128(dst, voff, srd, soff) asm volatile("buffer_load_dwordx4 %0, %1, %2, %3 offen sc1" : "=v"(dst) : "v"(voff), "s"(srd), "s"(soff) : "memory")
#define PBITS "sc1"

template <int N> __device__ __forceinline__ void wait_v() { asm volatile("s_waitcnt vmcnt(%0)" :: "i"(N) : "memory"); }

template <bool INIT>
__device__ __forceinline__ void mma(f32x4 (&acc)[4][2], const bf16x8 (&At)[4][2], const bf16x4 (&Bt)[2][2][2]) {
  const bf16x8 b00 = CAT(Bt[0][0]), b01 = CAT(Bt[0][1]), b10 = CAT(Bt[1][0]), b11 = CAT(Bt[1][1]);
  if constexpr (INIT)
    asm volatile("s_setprio 1\n\t"
      "v_mfma_f32_16x16x32_bf16 %0, %16, %8, 0\n\t"
      "v_mfma_f32_16x16x32_bf16 %1, %18, %8, 0\n\t"
      "v_mfma_f32_16x16x32_bf16 %2, %16, %10, 0\n\t"
      "v_mfma_f32_16x16x32_bf16 %3, %18, %10, 0\n\t"
      "v_mfma_f32_16x16x32_bf16 %4, %16, %12, 0\n\t"
      "v_mfma_f32_16x16x32_bf16 %5, %18, %12, 0\n\t"
      "v_mfma_f32_16x16x32_bf16 %6, %16, %14, 0\n\t"
      "v_mfma_f32_16x16x32_bf16 %7, %18, %14, 0\n\t"
      "v_mfma_f32_16x16x32_bf16 %0, %17, %9, %0\n\t"
      "v_mfma_f32_16x16x32_bf16 %1, %19, %9, %1\n\t"
      "v_mfma_f32_16x16x32_bf16 %2, %17, %11, %2\n\t"
      "v_mfma_f32_16x16x32_bf16 %3, %19, %11, %3\n\t"
      "v_mfma_f32_16x16x32_bf16 %4, %17, %13, %4\n\t"
      "v_mfma_f32_16x16x32_bf16 %5, %19, %13, %5\n\t"
      "v_mfma_f32_16x16x32_bf16 %6, %17, %15, %6\n\t"
      "v_mfma_f32_16x16x32_bf16 %7, %19, %15, %7\n\t"
      "s_setprio 0"
      : "=&v"(acc[0][0]), "=&v"(acc[0][1]), "=&v"(acc[1][0]), "=&v"(acc[1][1]), "=&v"(acc[2][0]), "=&v"(acc[2][1]), "=&v"(acc[3][0]), "=&v"(acc[3][1])
      : "v"(At[0][0]), "v"(At[0][1]), "v"(At[1][0]), "v"(At[1][1]), "v"(At[2][0]), "v"(At[2][1]), "v"(At[3][0]), "v"(At[3][1]), "v"(b00), "v"(b01), "v"(b10), "v"(b11));
  else
    asm volatile("s_setprio 1\n\t"
      "v_mfma_f32_16x16x32_bf16 %0, %16, %8, %0\n\t"
      "v_mfma_f32_16x16x32_bf16 %1, %18, %8, %1\n\t"
      "v_mfma_f32_16x16x32_bf16 %2, %16, %10, %2\n\t"
      "v_mfma_f32_16x16x32_bf16 %3, %18, %10, %3\n\t"
      "v_mfma_f32_16x16x32_bf16 %4, %16, %12, %4\n\t"
      "v_mfma_f32_16x16x32_bf16 %5, %18, %12, %5\n\t"
      "v_mfma_f32_16x16x32_bf16 %6, %16, %14, %6\n\t"
      "v_mfma_f32_16x16x32_bf16 %7, %18, %14, %7\n\t"
      "v_mfma_f32_16x16x32_bf16 %0, %17, %9, %0\n\t"
      "v_mfma_f32_16x16x32_bf16 %1, %19, %9, %1\n\t"
      "v_mfma_f32_16x16x32_bf16 %2, %17, %11, %2\n\t"
      "v_mfma_f32_16x16x32_bf16 %3, %19, %11, %3\n\t"
      "v_mfma_f32_16x16x32_bf16 %4, %17, %13, %4\n\t"
      "v_mfma_f32_16x16x32_bf16 %5, %19, %13, %5\n\t"
      "v_mfma_f32_16x16x32_bf16 %6, %17, %15, %6\n\t"
      "v_mfma_f32_16x16x32_bf16 %7, %19, %15, %7\n\t"
      "s_setprio 0"
      : "+v"(acc[0][0]), "+v"(acc[0][1]), "+v"(acc[1][0]), "+v"(acc[1][1]), "+v"(acc[2][0]), "+v"(acc[2][1]), "+v"(acc[3][0]), "+v"(acc[3][1])
      : "v"(At[0][0]), "v"(At[0][1]), "v"(At[1][0]), "v"(At[1][1]), "v"(At[2][0]), "v"(At[2][1]), "v"(At[3][0]), "v"(At[3][1]), "v"(b00), "v"(b01), "v"(b10), "v"(b11));
}

__device__ __forceinline__ void acc_guard(f32x4 (&q)[4][2]) {
  asm volatile("s_nop 7" : "+v"(q[0][0]), "+v"(q[0][1]), "+v"(q[1][0]), "+v"(q[1][1]), "+v"(q[2][0]), "+v"(q[2][1]), "+v"(q[3][0]), "+v"(q[3][1]));
}
__device__ __forceinline__ void store_quad(St& s, f32x4 (&q)[4][2], const u32x4& srd, unsigned off) {
  acc_guard(q);
#pragma unroll
  for (int m = 0; m < 4; ++m) { u32x4 w = pack_pair(q[m][0], q[m][1]); ST128(w, s.vlane16, srd, off + m * 1024u, PBITS); }
}
__device__ __forceinline__ void reduce_part(St& s, int j0, int j1) {
#pragma unroll
  for (int j = j0; j < j1; ++j)
#pragma unroll
    for (int d = 0; d < 4; ++d) {
      const unsigned x = s.tmp[j][d];
      if (j == 0) { s.red[2 * d] = blo(x); s.red[2 * d + 1] = bhi(x); }
      else { s.red[2 * d] += blo(x); s.red[2 * d + 1] += bhi(x); }
    }
  asm volatile("" : "+v"(s.red[0]), "+v"(s.red[1]), "+v"(s.red[2]), "+v"(s.red[3]), "+v"(s.red[4]), "+v"(s.red[5]), "+v"(s.red[6]), "+v"(s.red[7]));
}
__device__ __forceinline__ u32x4 reduce_fin(St& s) {
  u32x4 r;
  r.x = pk2(s.red[0], s.red[1]) | s.poison; r.y = pk2(s.red[2], s.red[3]) | s.poison;
  r.z = pk2(s.red[4], s.red[5]) | s.poison; r.w = pk2(s.red[6], s.red[7]) | s.poison;
  return r;
}
__device__ __forceinline__ void load_parts(St& s, const u32x4& srd, unsigned off) {
#pragma unroll
  for (int j = 0; j < 8; ++j) LD128(s.tmp[j], s.vgat, srd, off + j * SLOT);
}
__device__ __forceinline__ unsigned spin_flags(const char* wsp, unsigned blk, unsigned need, unsigned call) {
  const int lane = lane_id();
  const unsigned* f = (const unsigned*)(wsp + blk) + lane * 4;
  unsigned* tmo = (unsigned*)(wsp + WS_TMO);
  const unsigned long long t0 = __builtin_amdgcn_s_memrealtime();
  unsigned bad = 0;
  for (;;) {
    const unsigned v = __hip_atomic_load(f, __ATOMIC_RELAXED, __HIP_MEMORY_SCOPE_AGENT);
    if (__all(v >= need)) break;
    if (__builtin_amdgcn_readfirstlane(__hip_atomic_load(tmo, __ATOMIC_RELAXED, __HIP_MEMORY_SCOPE_AGENT)) == call ||
        __builtin_amdgcn_s_memrealtime() - t0 > DEADLINE) {
      if (lane == 0) __hip_atomic_store(tmo, call, __ATOMIC_RELAXED, __HIP_MEMORY_SCOPE_AGENT);
      bad = 0x7fc07fc0u;
      break;
    }
    __builtin_amdgcn_s_sleep(2);
  }
  asm volatile("s_waitcnt vmcnt(0) lgkmcnt(0)" ::: "memory");
  return bad;
}

template <int I, bool FINAL>
__device__ __forceinline__ void ktile(St& s) {
  constexpr unsigned b = I & 1, nb = b ^ 1;
  constexpr bool INIT = (I == 0);
  const unsigned a1 = (I + 1 < 16) ? s.curA : s.nxtA, a2 = (I + 2 < 16) ? s.curA : s.nxtA, b2 = (I + 2 < 16) ? s.curB : s.nxtB;
  constexpr unsigned a1i = (unsigned)((I + 1) & 15) * kstepA + hA, a2i = (unsigned)((I + 2) & 15) * kstepA, b2i = (unsigned)((I + 2) & 15) * kstepB;
  constexpr bool PS1 = (I == 0), PS2 = (I == T_PS2), PS3 = (I == T_PS3), FLAG = (I == T_FLAG), POLL = (I == T_POLL);
  constexpr bool L0 = (I == T_L0), L1 = (I == T_L1), V0 = (I == T_V0), V1 = (I == T_V1);
  constexpr bool TIE = (I == T_L0 + 1) || (I == T_L1 + 1);
  constexpr bool S0 = slot_at(S0_FIRST, S0_N, I), S1 = slot_at(S1_FIRST, S1_N, I);
  constexpr bool PS0 = (I == 15);
  constexpr int NW = 6 + ((PS1 || PS2 || PS3) ? 4 : 0) + (FLAG ? 1 : 0) + ((L0 || L1) ? 8 : 0) + (S0 ? 1 : 0) + (S1 ? 1 : 0) + (PS0 ? 4 : 0) + (FINAL ? 8 : 0);
  LDB(s.B0, b, 0); LDA(b, 0);
  if constexpr (POLL) asm volatile("buffer_load_dword %0, %1, %2, %3 offen sc1" : "=v"(s.pollv) : "v"(s.vlane16), "s"(s.srdPrev), "s"(s.pflag) : "memory");
  STAGE_A(SA(nb, 1), a1, a1i);
  if constexpr (PS1) store_quad(s, s.acc[0][1], s.srdPrev, s.pwPrev + 1u * 32768u);
  if constexpr (PS2) { _Pragma("unroll") for (int m = 0; m < 4; ++m) ST128(s.tmp[m], s.vlane16, s.srdPrev, s.pwPrev + 2u * 32768u + m * 1024u, PBITS); }
  if constexpr (PS3) { _Pragma("unroll") for (int m = 0; m < 4; ++m) ST128(s.tmp[4 + m], s.vlane16, s.srdPrev, s.pwPrev + 3u * 32768u + m * 1024u, PBITS); }
  if constexpr (FLAG) { unsigned _c; asm volatile("v_mov_b32 %0, %1\n\ts_mov_b64 exec, 1\n\tbuffer_store_dword %0, off, %2, %3 sc1\n\ts_mov_b64 exec, -1" : "=&v"(_c) : "s"(s.call), "s"(s.srdPrev), "s"(s.pflag + (unsigned)s.wslot * 16u) : "memory"); }
  if constexpr (L0) load_parts(s, s.srdPrev, s.prd);
  if constexpr (L1) load_parts(s, s.srdPrev, s.prd + 128u);
  WAIT_L(8); BAR; WAIT_B(s.B0); WAIT_A(); mma<INIT>(s.acc[0][0], s.At, s.B0); BAR; SCHED;
  LDB(s.B1, b, 1); STAGE_B(SB(b, 0), b2, b2i);
  if constexpr (S0) {
    const bool go = (s.ts0 == I) || (s.ts0 == I + 16);
    const u32x4 srd = {s.d0lo, s.d0hi, go ? OUT_BYTES : 0u, 0x20000u};
    ST128(s.R0, s.vdst, srd, s.d0off, "sc0 sc1");
  }
  if constexpr (S1) {
    const bool go = (s.ts1 == I) || (s.ts1 == I + 16);
    const u32x4 srd = {s.d1lo, s.d1hi, go ? OUT_BYTES : 0u, 0x20000u};
    ST128(s.R1, s.vdst, srd, s.d1off, "sc0 sc1");
  }
  if constexpr (PS0) store_quad(s, s.acc[0][0], s.srdWs, s.pwCur);
  if constexpr (PS1) { acc_guard(s.acc[1][0]); _Pragma("unroll") for (int m = 0; m < 4; ++m) { s.tmp[m] = pack_pair(s.acc[1][0][m][0], s.acc[1][0][m][1]); asm volatile("" : "+v"(s.tmp[m])); } }
  if constexpr (V0 || V1) reduce_part(s, 0, 3);
  BAR; WAIT_B(s.B1); mma<INIT>(s.acc[0][1], s.At, s.B1); BAR; SCHED;
  LDA(b, 1); STAGE_A(SA(b, 0), a2, a2i);
  if constexpr (FINAL) store_quad(s, s.acc[0][1], s.srdWs, s.pwCur + 1u * 32768u);
  if constexpr (PS1) { acc_guard(s.acc[1][1]); _Pragma("unroll") for (int m = 0; m < 4; ++m) { s.tmp[4 + m] = pack_pair(s.acc[1][1][m][0], s.acc[1][1][m][1]); asm volatile("" : "+v"(s.tmp[4 + m])); } }
  if constexpr (V0 || V1) reduce_part(s, 3, 5);
  BAR; WAIT_A(); mma<INIT>(s.acc[1][0], s.At, s.B0); BAR; SCHED;
  STAGE_B(SB(b, 1), b2, b2i + hB);
  if constexpr (FINAL) store_quad(s, s.acc[1][0], s.srdWs, s.pwCur + 2u * 32768u);
  if constexpr (V0 || V1) reduce_part(s, 5, 8);
  if constexpr (V0) { s.R0 = reduce_fin(s); asm volatile("" : "+v"(s.R0)); s.d0lo = s.pdlo; s.d0hi = s.pdhi; s.d0off = s.pdoff; s.ts0 = s.need ? s.TS0 : -1; }
  if constexpr (V1) { s.R1 = reduce_fin(s); asm volatile("" : "+v"(s.R1)); s.d1lo = s.pdlo; s.d1hi = s.pdhi; s.d1off = s.pdoff + 8u * 2048u; s.ts1 = s.need ? s.TS1 : -1; }
  if constexpr (TIE) asm volatile("s_waitcnt vmcnt(%8)" : "+v"(s.tmp[0]), "+v"(s.tmp[1]), "+v"(s.tmp[2]), "+v"(s.tmp[3]), "+v"(s.tmp[4]), "+v"(s.tmp[5]), "+v"(s.tmp[6]), "+v"(s.tmp[7]) : "i"(NW) : "memory");
  else if constexpr (POLL) asm volatile("s_waitcnt vmcnt(%1)" : "+v"(s.pollv) : "i"(NW) : "memory");
  else wait_v<NW>();
  BAR; mma<INIT>(s.acc[1][1], s.At, s.B1); BAR; SCHED;
  if constexpr (POLL) {
    if (!__all(s.pollv >= s.need)) s.poison |= spin_flags(s.wsp, s.pflag, s.need, s.call);
    SCHED;
  }
}

#define STAMP(i) do {} while (0)

__global__ __launch_bounds__(512, 2) void gemm_a2a(Params p) {
  __shared__ __attribute__((aligned(1024))) unsigned char smem[8 * HTB];
  const int tid = threadIdx.x, wid = __builtin_amdgcn_readfirstlane(tid >> 6), lane = tid & 63;
  const int wr = wid >> 2, wc = wid & 3, fr = lane & 15, fq = lane >> 4;
  const unsigned blk = blockIdx.x, mem = blk & 7u, grp = (blk >> 3) & 31u;
  const unsigned rank = (unsigned)p.rank & 7u;
  STAMP(0);
  St s;
  char* wsp = p.ws[rank];
  s.wsp = wsp;
  unsigned* callp = (unsigned*)(wsp + WS_CALL + blk * 64u);
  const unsigned call = __builtin_amdgcn_readfirstlane(__hip_atomic_load(callp, __ATOMIC_RELAXED, __HIP_MEMORY_SCOPE_AGENT)) + 1u;
  s.call = call; s.poison = 0u;
  unsigned info = 0;
#pragma unroll
  for (int r = 0; r < 4; ++r) {
    info |= (unsigned)kTiles[r * 32 + grp] << (8 * r);
  }
  info = __builtin_amdgcn_readfirstlane(info);
  unsigned olo[4], ohi[4];
#pragma unroll
  for (int r = 0; r < 4; ++r) {
    const unsigned long long o = (unsigned long long)p.out[(rank + (info >> (8 * r + 4))) & 7u];
    olo[r] = (unsigned)o; ohi[r] = (unsigned)(o >> 32) & 0xffffu;
  }
  const unsigned long long pa = (unsigned long long)p.A, pw = (unsigned long long)p.W, pws = (unsigned long long)wsp;
  s.srdA = (u32x4){(unsigned)pa, (unsigned)(pa >> 32) & 0xffffu, A_BYTES, 0x20000u};
  s.srdB = (u32x4){(unsigned)pw, (unsigned)(pw >> 32) & 0xffffu, W_BYTES, 0x20000u};
  s.srdWs = (u32x4){(unsigned)pws, (unsigned)(pws >> 32) & 0xffffu, WS_USED, 0x20000u};
  s.srdPrev = s.srdWs; s.srdPrev.z = 0u;
  const unsigned lds0 = (unsigned)(size_t)(LAS unsigned char*)smem;
  s.ldsw = lds0 + (unsigned)wid * 1024u;
  s.voffA = (unsigned)((8 * wid + (lane >> 3)) * (K_ * 2) + (((lane & 7) ^ (lane >> 3)) << 4));
  {
    const unsigned fB = (unsigned)((lane >> 4) | (((wid >> 1) & 1) << 2));
    s.voffB = (unsigned)((4 * wid + (lane >> 4)) * (N_ * 2)) + ((((unsigned)lane & 15u) ^ (fB << 1)) << 4);
    const int q = (lane & 15) >> 2, pp = lane & 3, g = lane >> 4;
    const int f = q | ((g & 1) << 2);
    s.aoff0 = lds0 + (unsigned)((wr * 64 + fr) * 128 + (((0 + fq) ^ (fr & 7)) << 4));
    s.aoff1 = lds0 + (unsigned)((wr * 64 + fr) * 128 + (((4 + fq) ^ (fr & 7)) << 4));
    s.boff0 = lds0 + 65536u + (unsigned)((8 * g + q) * 256 + (((2 * wc + 0) ^ f) << 5) + 8 * pp);
    s.boff1 = lds0 + 65536u + (unsigned)((8 * g + q) * 256 + (((2 * wc + 1) ^ f) << 5) + 8 * pp);
  }
  s.vlane16 = (unsigned)lane * 16u;
  {
    const unsigned r = (unsigned)lane >> 3, c = (unsigned)lane & 7u, cc = c & 3u;
    const unsigned sfq = ((cc & 1u) << 1) | (cc >> 1);
    s.vgat = (c >> 2) * 4096u + (sfq * 16u + r) * 16u;
    s.vdst = r * 2048u + c * 16u;
  }
  const unsigned quad = mem >> 1, mh = mem & 1u;
  const unsigned rwr = (unsigned)wid >> 2, rm = 2u * mh + (((unsigned)wid >> 1) & 1u), rwcp = (unsigned)wid & 1u;
  const unsigned prd_in = ((quad * 8u + rwr * 4u + 2u * rwcp) * 4u + rm) * 1024u;
  const unsigned dst_in = ((quad >> 1) * 128u + rwr * 64u + rm * 16u) * 2048u + ((quad & 1u) * 128u + rwcp * 64u) * 2u;
  s.wslot = (int)(mem * 8u + (unsigned)wid);
  {
    const unsigned key = grp * 8u + mem + (unsigned)wid;
    s.TS0 = S0_FIRST + (int)(key % (unsigned)S0_N);
    s.TS1 = S1_FIRST + (int)(key % (unsigned)S1_N);
  }
  s.ts0 = -1; s.ts1 = -1; s.need = 0u; s.pflag = WS_PFLAG; s.prd = 0u; s.pwPrev = 0u;
  s.d0lo = s.d0hi = s.d0off = s.d1lo = s.d1hi = s.d1off = 0u; s.pdlo = s.pdhi = s.pdoff = 0u;
  s.R0 = (u32x4){0u, 0u, 0u, 0u}; s.R1 = s.R0;
#pragma unroll
  for (int a = 0; a < 2; ++a)
#pragma unroll
    for (int bb = 0; bb < 2; ++bb)
#pragma unroll
      for (int m = 0; m < 4; ++m)
#pragma unroll
        for (int n = 0; n < 2; ++n) s.acc[a][bb][m][n] = (f32x4){0.f, 0.f, 0.f, 0.f};
  auto offA = [&](unsigned e) -> unsigned { return (e & 3u) * (256u * K_ * 2u) + mem * (1024u * 2u); };
  auto offB = [&](unsigned e) -> unsigned { return mem * (1024u * N_ * 2u) + ((((rank + (e >> 4)) & 7u) * 4u + ((e >> 2) & 3u)) * 512u); };
  s.nxtA = offA(info & 0xffu); s.nxtB = offB(info & 0xffu);
  asm volatile("s_waitcnt vmcnt(0) lgkmcnt(0)" ::: "memory");
  STAGE_B(SB(0, 0), s.nxtB, 0u); STAGE_A(SA(0, 0), s.nxtA, 0u); STAGE_B(SB(0, 1), s.nxtB, hB); STAGE_A(SA(0, 1), s.nxtA, hA);
  if (wr == 1) BAR;
  wait_v<4>(); BAR;
  STAGE_B(SB(1, 0), s.nxtB, kstepB); STAGE_A(SA(1, 0), s.nxtA, kstepA); STAGE_B(SB(1, 1), s.nxtB, hB + kstepB);
  wait_v<6>(); BAR;
  STAMP(1);
  for (int it = 0;; ++it) {
    const unsigned e = (info >> (8 * it)) & 0xffu;
    s.curA = s.nxtA; s.curB = s.nxtB;
    if (it < 3) { const unsigned en = (info >> (8 * it + 8)) & 0xffu; s.nxtA = offA(en); s.nxtB = offB(en); }
    else { s.nxtA = 0xC0000000u; s.nxtB = 0xC0000000u; }
    s.pwCur = WS_PART + (((unsigned)it * 32u + grp) * 8u + mem) * SLOT + (unsigned)wid * 4096u;
    ktile<0, false>(s); ktile<1, false>(s); ktile<2, false>(s); ktile<3, false>(s);
    ktile<4, false>(s); ktile<5, false>(s); ktile<6, false>(s); ktile<7, false>(s);
    ktile<8, false>(s); ktile<9, false>(s); ktile<10, false>(s); ktile<11, false>(s);
    ktile<12, false>(s); ktile<13, false>(s); ktile<14, false>(s);
    const unsigned npdlo = it == 0 ? olo[0] : it == 1 ? olo[1] : it == 2 ? olo[2] : olo[3];
    const unsigned npdhi = it == 0 ? ohi[0] : it == 1 ? ohi[1] : it == 2 ? ohi[2] : ohi[3];
    const unsigned npdoff = (rank * 1024u + (e & 3u) * 256u) * 2048u + ((e >> 2) & 3u) * 512u + dst_in;
    if (it == 3) { s.pdlo = npdlo; s.pdhi = npdhi; s.pdoff = npdoff; break; }
    ktile<15, false>(s);
    s.pdlo = npdlo; s.pdhi = npdhi; s.pdoff = npdoff;
    s.pwPrev = s.pwCur;
    s.prd = WS_PART + ((unsigned)it * 32u + grp) * 8u * SLOT + prd_in;
    s.pflag = WS_PFLAG + ((unsigned)it * 32u + grp) * 1024u;
    s.need = call; s.srdPrev.z = WS_USED;
    STAMP(2 + it);
  }
  ktile<15, true>(s);
  STAMP(5);
  store_quad(s, s.acc[1][1], s.srdWs, s.pwCur + 3u * 32768u);
  const unsigned fblk = WS_PFLAG + (3u * 32u + grp) * 1024u;
  const unsigned frd = WS_PART + (3u * 32u + grp) * 8u * SLOT + prd_in;
  wait_v<0>();
  STAMP(6);
  const int lane2 = lane_id();
  if (lane2 == 0) __hip_atomic_store((unsigned*)(wsp + fblk) + s.wslot * 4, call, __ATOMIC_RELAXED, __HIP_MEMORY_SCOPE_AGENT);
  if (wr == 0) BAR;
  s.poison |= spin_flags(wsp, fblk, call, call);
  STAMP(7);
  {
    u32x4 t2[8];
    load_parts(s, s.srdWs, frd);
#pragma unroll
    for (int j = 0; j < 8; ++j) LD128(t2[j], s.vgat, s.srdWs, frd + 128u + j * SLOT);
    {
      const u32x4 srd = {s.d1lo, s.d1hi, s.ts1 >= 16 ? OUT_BYTES : 0u, 0x20000u};
      ST128(s.R1, s.vdst, srd, s.d1off, "sc0 sc1");
    }
    asm volatile("s_waitcnt vmcnt(9)" : "+v"(s.tmp[0]), "+v"(s.tmp[1]), "+v"(s.tmp[2]), "+v"(s.tmp[3]), "+v"(s.tmp[4]), "+v"(s.tmp[5]), "+v"(s.tmp[6]), "+v"(s.tmp[7]) :: "memory");
    reduce_part(s, 0, 8);
    const u32x4 o0 = reduce_fin(s);
    const u32x4 srd = {s.pdlo, s.pdhi, OUT_BYTES, 0x20000u};
    ST128(o0, s.vdst, srd, s.pdoff, "sc0 sc1");
    asm volatile("s_waitcnt vmcnt(2)" : "+v"(t2[0]), "+v"(t2[1]), "+v"(t2[2]), "+v"(t2[3]), "+v"(t2[4]), "+v"(t2[5]), "+v"(t2[6]), "+v"(t2[7]) :: "memory");
#pragma unroll
    for (int j = 0; j < 8; ++j) s.tmp[j] = t2[j];
    reduce_part(s, 0, 8);
    const u32x4 o1 = reduce_fin(s);
    ST128(o1, s.vdst, srd, s.pdoff + 8u * 2048u, "sc0 sc1");
  }
  STAMP(8);
  wait_v<0>();
  STAMP(9);
  BAR;
  if (wid == 0) {
    if (lane2 < 4) {
      const unsigned e = (info >> (8 * lane2)) & 0xffu;
      const unsigned dp = (rank + (e >> 4)) & 7u;
      if (dp != rank) {
        unsigned* f = (unsigned*)(p.ws[dp] + WS_DONE + rank * 512u) + (e & 15u) * 8u + mem;
        __hip_atomic_store(f, call, __ATOMIC_RELAXED, __HIP_MEMORY_SCOPE_SYSTEM);
      }
    }
    if (lane2 == 0) __hip_atomic_store(callp, call, __ATOMIC_RELAXED, __HIP_MEMORY_SCOPE_AGENT);
    if (blk == 0) {
      const unsigned long long* f = (const unsigned long long*)(wsp + WS_DONE) + lane2;
      unsigned* tmo = (unsigned*)(wsp + WS_TMO);
      const unsigned long long t0 = __builtin_amdgcn_s_memrealtime();
      for (;;) {
        bool ok = true;
#pragma unroll
        for (unsigned j = 1; j < 8; ++j) {
          const unsigned long long v = __hip_atomic_load(f + ((rank + j) & 7u) * 64u, __ATOMIC_RELAXED, __HIP_MEMORY_SCOPE_SYSTEM);
          ok = ok && ((unsigned)v >= call) && ((unsigned)(v >> 32) >= call);
        }
        if (__all(ok)) break;
        if (__builtin_amdgcn_readfirstlane(__hip_atomic_load(tmo, __ATOMIC_RELAXED, __HIP_MEMORY_SCOPE_AGENT)) == call ||
            __builtin_amdgcn_s_memrealtime() - t0 > DEADLINE) {
          if (lane2 == 0) {
            __hip_atomic_store(tmo, call, __ATOMIC_RELAXED, __HIP_MEMORY_SCOPE_AGENT);
            __hip_atomic_store((unsigned*)(p.out[rank] + (size_t)rank * 1024u * 2048u), 0x7fc07fc0u, __ATOMIC_RELAXED, __HIP_MEMORY_SCOPE_SYSTEM);
          }
          break;
        }
        __builtin_amdgcn_s_sleep(1);
      }
      __builtin_amdgcn_fence(__ATOMIC_ACQUIRE, "");
    }
  }
}

}

extern "C" void kernel_launch(int rank, int nranks, ncclComm_t comm,
                              hipStream_t stream,
                              void* const* d_in, const int* in_sizes, int n_in,
                              void* d_out, int out_size,
                              const int* in_dtypes, int out_dtype,
                              void* const* d_out_all,
                              void* const* d_ws_all, size_t ws_bytes) {
  using namespace ga2a;
  (void)comm; (void)in_sizes; (void)n_in; (void)d_out; (void)out_size; (void)in_dtypes; (void)out_dtype;
  if (nranks != NR || ws_bytes < (size_t)WS_USED) { fprintf(stderr, "unsupported configuration\n"); return; }
  Params p;
  p.A = (const u16*)d_in[0];
  p.W = (const u16*)d_in[1];
  for (int i = 0; i < NR; ++i) { p.out[i] = (char*)d_out_all[i]; p.ws[i] = (char*)d_ws_all[i]; }
  p.rank = rank;
  hipLaunchKernelGGL(gemm_a2a, dim3(256), dim3(512), 0, stream, p);
  HIP_CHECK(hipGetLastError());
}
```

```cpp
#include <hip/hip_runtime.h>
#include <rccl/rccl.h>
#include <cstdio>

#define HIP_CHECK(expr) do { hipError_t _e = (expr); if (_e != hipSuccess) { \
    fprintf(stderr, "HIP error %s at %s:%d\n", hipGetErrorString(_e), __FILE__, __LINE__); \
    return; } } while (0)

namespace ga2a {

#define LAS __attribute__((address_space(3)))
typedef unsigned short u16;
typedef short bf16x8 __attribute__((ext_vector_type(8)));
typedef short bf16x4 __attribute__((ext_vector_type(4)));
typedef float f32x4 __attribute__((ext_vector_type(4)));
typedef float f32x2 __attribute__((ext_vector_type(2)));
typedef __bf16 bf16x2n __attribute__((ext_vector_type(2)));
typedef unsigned u32x4 __attribute__((ext_vector_type(4)));

constexpr int NR = 8;
constexpr int M_ = 1024, N_ = 8192, K_ = 8192;
constexpr unsigned HTB = 16384;
constexpr unsigned A_BYTES = (unsigned)M_ * K_ * 2u, W_BYTES = (unsigned)K_ * N_ * 2u;
constexpr unsigned OUT_BYTES = 8192u * 1024u * 2u;
constexpr unsigned SLOT = 65536u;
constexpr unsigned REGION = 256u * SLOT;
constexpr unsigned WS_PART = 0u;
constexpr unsigned WS_CTRL = 128u << 20;
constexpr unsigned WS_PFLAG = WS_CTRL;
constexpr unsigned WS_CALL = WS_CTRL + 131072u;
constexpr unsigned WS_DONE = WS_CTRL + 163840u;
constexpr unsigned WS_TMO = WS_CTRL + 180224u;
constexpr unsigned WS_STAMP = WS_CTRL + 262144u;
constexpr unsigned WS_USED = WS_CTRL + (1u << 20);
constexpr unsigned long long DEADLINE = 50000000ull;

constexpr unsigned kstepA = 128u, kstepB = 64u * N_ * 2u;
constexpr unsigned hA = 128u * K_ * 2u;
constexpr unsigned iA = 64u * K_ * 2u, iB = 32u * N_ * 2u;

struct Params {
  const u16* A;
  const u16* W;
  char* out[NR];
  char* ws[NR];
  int rank;
};

enum { V_PLAIN = 0, V_BG = 1, V_L = 2, V_R = 3, V_FINAL = 4 };
constexpr int T_FLAG = 2, T_POLL = 4;
constexpr int S0_FIRST = 10, S0_N = 5, S1_FIRST = 15, S1_N = 5;
#define GA2A_S3LAST 15
constexpr int S3_LAST = GA2A_S3LAST;
constexpr bool slot_at(int first, int n, int i) { return (i >= first && i < first + n) || (i + 16 >= first && i + 16 < first + n); }

__device__ __forceinline__ unsigned pk2(float lo, float hi) {
  f32x2 v = {lo, hi};
  return __builtin_bit_cast(unsigned, __builtin_convertvector(v, bf16x2n));
}
__device__ __forceinline__ u32x4 pack_pair(const f32x4& v0, const f32x4& v1) {
  unsigned d00 = pk2(v0[0], v0[1]), d01 = pk2(v0[2], v0[3]);
  unsigned d10 = pk2(v1[0], v1[1]), d11 = pk2(v1[2], v1[3]);
  auto r0 = __builtin_amdgcn_permlane16_swap(d00, d10, false, false);
  auto r1 = __builtin_amdgcn_permlane16_swap(d01, d11, false, false);
  u32x4 w; w.x = r0[0]; w.y = r1[0]; w.z = r0[1]; w.w = r1[1];
  return w;
}
__device__ __forceinline__ float blo(unsigned x) { return __builtin_bit_cast(float, x << 16); }
__device__ __forceinline__ float bhi(unsigned x) { return __builtin_bit_cast(float, x & 0xffff0000u); }

struct St {
  f32x4 acc[2][2][4][2];
  bf16x8 At[4][2];
  bf16x4 B0[2][2][2], B1[2][2][2];
  u32x4 tmp[8];
  float red[8];
  u32x4 R0, R1;
  unsigned pollv;
  unsigned voffA, voffB, aoff0, aoff1, boff0, boff1, vlane16, vgat, vdst;
  u32x4 srdA, srdB, srdWs;
  unsigned ldsw, curA, curBL, curBR, nxtA, nxtBL, nxtBR;
  unsigned pw;
  unsigned prd0, prd1;
  unsigned pflag;
  unsigned keep;
  unsigned call, poison;
  unsigned pdlo, pdhi, pdoff;
  unsigned d0lo, d0hi, d0off, d1lo, d1hi, d1off;
  int ts0, ts1, TS0, TS1, wslot;
  char* wsp;
};

#define SA(b, h) (((b) * 2u + (h)) * HTB)
#define SB(b, h) ((4u + (b) * 2u + (h)) * HTB)
#define DMA2(ldsoff, voff, srd, sbase, simm, step) do { unsigned _t; asm volatile( \
    "s_add_u32 m0, %1, %2\n\ts_add_u32 %0, %6, %7\n\tbuffer_load_dwordx4 %4, %5, %0 offen lds\n\t" \
    "s_add_u32 m0, %1, %3\n\ts_add_u32 %0, %6, %8\n\tbuffer_load_dwordx4 %4, %5, %0 offen lds" \
    : "=&s"(_t) : "s"(s.ldsw), "i"(ldsoff), "i"((ldsoff) + 8192u), "v"(voff), "s"(srd), "s"(sbase), "i"(simm), "i"((simm) + (step)) : "memory", "scc"); } while (0)
#define STAGE_A(bufoff, sbase, simm) DMA2((bufoff), s.voffA, s.srdA, (sbase), (simm), iA)
#define STAGE_B(bufoff, sbase, simm) DMA2((bufoff), s.voffB, s.srdB, (sbase), (simm), iB)
__device__ __forceinline__ int lane_id() { int l; asm volatile("v_mbcnt_lo_u32_b32 %0, -1, 0\n\tv_mbcnt_hi_u32_b32 %0, -1, %0" : "=v"(l)); return l; }
#define RD128(dst, addr, off) asm volatile("ds_read_b128 %0, %1 offset:%2" : "=v"(dst) : "v"(addr), "i"(off))
#define RDTR(dst, addr, off) asm volatile("ds_read_b64_tr_b16 %0, %1 offset:%2" : "=v"(dst) : "v"(addr), "i"(off))
#define LDA(b, h) do { _Pragma("unroll") for (int m = 0; m < 4; ++m) { RD128(s.At[m][0], s.aoff0, SA(b, h) + m * 2048); RD128(s.At[m][1], s.aoff1, SA(b, h) + m * 2048); } } while (0)
#define LDB(dst, b, h) do { _Pragma("unroll") for (int k = 0; k < 2; ++k) { \
    RDTR(dst[0][k][0], s.boff0, SB(b, h) - 65536u + k * 8192); RDTR(dst[0][k][1], s.boff0, SB(b, h) - 65536u + k * 8192 + 1024); } \
    _Pragma("unroll") for (int k = 0; k < 2; ++k) { \
    RDTR(dst[1][k][0], s.boff1, SB(b, h) - 65536u + k * 8192); RDTR(dst[1][k][1], s.boff1, SB(b, h) - 65536u + k * 8192 + 1024); } } while (0)
#define WAIT_A() asm volatile("s_waitcnt lgkmcnt(0)" : "+v"(s.At[0][0]), "+v"(s.At[0][1]), "+v"(s.At[1][0]), "+v"(s.At[1][1]), "+v"(s.At[2][0]), "+v"(s.At[2][1]), "+v"(s.At[3][0]), "+v"(s.At[3][1]) :: "memory")
#define WAIT_B(Bt) asm volatile("s_waitcnt lgkmcnt(0)" : "+v"(Bt[0][0][0]), "+v"(Bt[0][0][1]), "+v"(Bt[0][1][0]), "+v"(Bt[0][1][1]), "+v"(Bt[1][0][0]), "+v"(Bt[1][0][1]), "+v"(Bt[1][1][0]), "+v"(Bt[1][1][1]) :: "memory")
#define CAT(x) __builtin_shufflevector(x[0], x[1], 0, 1, 2, 3, 4, 5, 6, 7)
#define WAIT_L(n) asm volatile("s_waitcnt lgkmcnt(" #n ")" ::: "memory")
#define BAR __builtin_amdgcn_s_barrier()
#define SCHED __builtin_amdgcn_sched_barrier(0)
#define ENDM do { SCHED; asm volatile("s_setprio 0"); BAR; SCHED; } while (0)
#define ST128(data, voff, srd, soff, bits) asm volatile("buffer_store_dwordx4 %0, %1, %2, %3 offen " bits "\n\ts_nop 1" :: "v"(data), "v"(voff), "s"(srd), "s"(soff) : "memory")
#define LD128(dst, voff, srd, soff) asm volatile("buffer_load_dwordx4 %0, %1, %2, %3 offen sc1" : "=v"(dst) : "v"(voff), "s"(srd), "s"(soff) : "memory")
#define PST(u) ST128(s.tmp[u], s.vlane16, s.srdWs, s.pw + ((u) >> 2) * 32768u + ((u) & 3) * 1024u, "sc1")
#define PLD(j) LD128(s.tmp[j], s.vgat, s.srdWs, ((j) < 4 ? s.prd0 : s.prd1) + ((j) & 3) * SLOT)

template <int N> __device__ __forceinline__ void wait_v() { asm volatile("s_waitcnt vmcnt(%0)" :: "i"(N) : "memory"); }

template <bool INIT>
__device__ __forceinline__ void mma(f32x4 (&acc)[4][2], const bf16x8 (&At)[4][2], const bf16x4 (&Bt)[2][2][2]) {
  const bf16x8 b00 = CAT(Bt[0][0]), b01 = CAT(Bt[0][1]), b10 = CAT(Bt[1][0]), b11 = CAT(Bt[1][1]);
  if constexpr (INIT)
    asm volatile("s_setprio 1\n\t"
      "v_mfma_f32_16x16x32_bf16 %0, %16, %8, 0\n\t"
      "v_mfma_f32_16x16x32_bf16 %0, %17, %9, %0\n\t"
      "v_mfma_f32_16x16x32_bf16 %1, %18, %8, 0\n\t"
      "v_mfma_f32_16x16x32_bf16 %1, %19, %9, %1\n\t"
      "v_mfma_f32_16x16x32_bf16 %2, %16, %10, 0\n\t"
      "v_mfma_f32_16x16x32_bf16 %2, %17, %11, %2\n\t"
      "v_mfma_f32_16x16x32_bf16 %3, %18, %10, 0\n\t"
      "v_mfma_f32_16x16x32_bf16 %3, %19, %11, %3\n\t"
      "v_mfma_f32_16x16x32_bf16 %4, %16, %12, 0\n\t"
      "v_mfma_f32_16x16x32_bf16 %4, %17, %13, %4\n\t"
      "v_mfma_f32_16x16x32_bf16 %5, %18, %12, 0\n\t"
      "v_mfma_f32_16x16x32_bf16 %5, %19, %13, %5\n\t"
      "v_mfma_f32_16x16x32_bf16 %6, %16, %14, 0\n\t"
      "v_mfma_f32_16x16x32_bf16 %6, %17, %15, %6\n\t"
      "v_mfma_f32_16x16x32_bf16 %7, %18, %14, 0\n\t"
      "v_mfma_f32_16x16x32_bf16 %7, %19, %15, %7\n\t"
      : "+v"(acc[0][0]), "+v"(acc[0][1]), "+v"(acc[1][0]), "+v"(acc[1][1]), "+v"(acc[2][0]), "+v"(acc[2][1]), "+v"(acc[3][0]), "+v"(acc[3][1])
      : "v"(At[0][0]), "v"(At[0][1]), "v"(At[1][0]), "v"(At[1][1]), "v"(At[2][0]), "v"(At[2][1]), "v"(At[3][0]), "v"(At[3][1]), "v"(b00), "v"(b01), "v"(b10), "v"(b11));
  else
    asm volatile("s_setprio 1\n\t"
      "v_mfma_f32_16x16x32_bf16 %0, %16, %8, %0\n\t"
      "v_mfma_f32_16x16x32_bf16 %0, %17, %9, %0\n\t"
      "v_mfma_f32_16x16x32_bf16 %1, %18, %8, %1\n\t"
      "v_mfma_f32_16x16x32_bf16 %1, %19, %9, %1\n\t"
      "v_mfma_f32_16x16x32_bf16 %2, %16, %10, %2\n\t"
      "v_mfma_f32_16x16x32_bf16 %2, %17, %11, %2\n\t"
      "v_mfma_f32_16x16x32_bf16 %3, %18, %10, %3\n\t"
      "v_mfma_f32_16x16x32_bf16 %3, %19, %11, %3\n\t"
      "v_mfma_f32_16x16x32_bf16 %4, %16, %12, %4\n\t"
      "v_mfma_f32_16x16x32_bf16 %4, %17, %13, %4\n\t"
      "v_mfma_f32_16x16x32_bf16 %5, %18, %12, %5\n\t"
      "v_mfma_f32_16x16x32_bf16 %5, %19, %13, %5\n\t"
      "v_mfma_f32_16x16x32_bf16 %6, %16, %14, %6\n\t"
      "v_mfma_f32_16x16x32_bf16 %6, %17, %15, %6\n\t"
      "v_mfma_f32_16x16x32_bf16 %7, %18, %14, %7\n\t"
      "v_mfma_f32_16x16x32_bf16 %7, %19, %15, %7\n\t"
      : "+v"(acc[0][0]), "+v"(acc[0][1]), "+v"(acc[1][0]), "+v"(acc[1][1]), "+v"(acc[2][0]), "+v"(acc[2][1]), "+v"(acc[3][0]), "+v"(acc[3][1])
      : "v"(At[0][0]), "v"(At[0][1]), "v"(At[1][0]), "v"(At[1][1]), "v"(At[2][0]), "v"(At[2][1]), "v"(At[3][0]), "v"(At[3][1]), "v"(b00), "v"(b01), "v"(b10), "v"(b11));
}

__device__ __forceinline__ void pack_quad(u32x4* dst, const f32x4 (&q)[4][2]) {
#pragma unroll
  for (int m = 0; m < 4; ++m) { dst[m] = pack_pair(q[m][0], q[m][1]); asm volatile("" : "+v"(dst[m])); }
}
template <int FIRST>
__device__ __forceinline__ void reduce_part(St& s, int j0, int j1) {
#pragma unroll
  for (int j = j0; j < j1; ++j)
#pragma unroll
    for (int d = 0; d < 4; ++d) {
      const unsigned x = s.tmp[j][d];
      if (FIRST == 1 && j == j0) { s.red[2 * d] = blo(x); s.red[2 * d + 1] = bhi(x); }
      else {
        if (FIRST == 2 && j == j0) {
          s.red[2 * d] = __builtin_bit_cast(float, __builtin_bit_cast(unsigned, s.red[2 * d]) & s.keep);
          s.red[2 * d + 1] = __builtin_bit_cast(float, __builtin_bit_cast(unsigned, s.red[2 * d + 1]) & s.keep);
        }
        s.red[2 * d] += blo(x); s.red[2 * d + 1] += bhi(x);
      }
    }
  asm volatile("" : "+v"(s.red[0]), "+v"(s.red[1]), "+v"(s.red[2]), "+v"(s.red[3]), "+v"(s.red[4]), "+v"(s.red[5]), "+v"(s.red[6]), "+v"(s.red[7]));
}
__device__ __forceinline__ u32x4 reduce_fin(St& s) {
  u32x4 r;
  r.x = pk2(s.red[0], s.red[1]) | s.poison; r.y = pk2(s.red[2], s.red[3]) | s.poison;
  r.z = pk2(s.red[4], s.red[5]) | s.poison; r.w = pk2(s.red[6], s.red[7]) | s.poison;
  asm volatile("" : "+v"(r));
  return r;
}
__device__ __forceinline__ unsigned spin_flags(const char* wsp, unsigned blk, unsigned call) {
  const int lane = lane_id();
  const unsigned* f = (const unsigned*)(wsp + blk) + lane * 4;
  unsigned* tmo = (unsigned*)(wsp + WS_TMO);
  const unsigned long long t0 = __builtin_amdgcn_s_memrealtime();
  unsigned bad = 0;
  for (;;) {
    const unsigned v = __hip_atomic_load(f, __ATOMIC_RELAXED, __HIP_MEMORY_SCOPE_AGENT);
    if (__all(v >= call)) break;
    if (__builtin_amdgcn_readfirstlane(__hip_atomic_load(tmo, __ATOMIC_RELAXED, __HIP_MEMORY_SCOPE_AGENT)) == call ||
        __builtin_amdgcn_s_memrealtime() - t0 > DEADLINE) {
      if (lane == 0) __hip_atomic_store(tmo, call, __ATOMIC_RELAXED, __HIP_MEMORY_SCOPE_AGENT);
      bad = 0x7fc07fc0u;
      break;
    }
    __builtin_amdgcn_s_sleep(1);
  }
  asm volatile("s_waitcnt vmcnt(0) lgkmcnt(0)" ::: "memory");
  return bad;
}
#define SEND(R, lo, hi, off, go) do { const u32x4 _srd = {lo, hi, (go) ? OUT_BYTES : 0u, 0x20000u}; ST128(R, s.vdst, _srd, off, "sc0 sc1"); } while (0)

template <int I, int V>
__device__ __forceinline__ void ktile(St& s) {
  constexpr unsigned b = I & 1, nb = b ^ 1;
  constexpr bool BG = (V == V_BG || V == V_L || V == V_R || V == V_FINAL);
  constexpr bool FINAL = (V == V_FINAL);
  constexpr bool E15 = (I == 15 && (V == V_L || V == V_R)), E0 = (I == 0 && (V == V_L || V == V_R));
  constexpr int hv = (V == V_R) ? 1 : 0;
  constexpr bool INL = E0 && hv == 0, INR = E0 && hv == 1;
  const unsigned a1 = (I + 1 < 16) ? s.curA : s.nxtA, a2 = (I + 2 < 16) ? s.curA : s.nxtA;
  const unsigned bl = (I + 2 < 16) ? s.curBL : s.nxtBL, br = (I + 2 < 16) ? s.curBR : s.nxtBR;
  constexpr unsigned a1i = (unsigned)((I + 1) & 15) * kstepA + hA, a2i = (unsigned)((I + 2) & 15) * kstepA, b2i = (unsigned)((I + 2) & 15) * kstepB;
  constexpr bool S0 = BG && slot_at(S0_FIRST, S0_N, I), S1 = BG && slot_at(S1_FIRST, S1_N, I);
  constexpr int NX = (S0 ? 1 : 0) + (S1 ? 1 : 0) + (E15 ? 1 : 0) + (E0 ? 4 : 0) + ((V == V_BG && I == 1) ? 3 : 0) +
                     ((V == V_BG && (I == 6 || I == 7)) ? 3 : 0) + ((V == V_BG && I == 8) ? 2 : 0);
  LDB(s.B0, b, 0); LDA(b, 0);
  if constexpr (!FINAL) STAGE_A(SA(nb, 1), a1, a1i);
  WAIT_L(8); BAR; WAIT_B(s.B0); WAIT_A(); mma<INL>(s.acc[0][0], s.At, s.B0);
  if constexpr (E0) pack_quad(&s.tmp[4], s.acc[1][hv]);
  if constexpr (V == V_BG && I == 9) reduce_part<1>(s, 0, 2);
  if constexpr (V == V_BG && I == 10) reduce_part<2>(s, 4, 6);
  ENDM;
  LDB(s.B1, b, 1);
  if constexpr (!FINAL) STAGE_B(SB(b, 0), bl, b2i);
  if constexpr (E0) PST(1);
  if constexpr (V == V_BG && I == 1) PST(5);
  if constexpr (V == V_BG && I == 6) PLD(0);
  if constexpr (V == V_BG && I == 7) PLD(3);
  if constexpr (V == V_BG && I == 8) PLD(6);
  if constexpr (S1) SEND(s.R1, s.d1lo, s.d1hi, s.d1off, s.ts1 == (I < S1_FIRST ? I + 16 : I));
  BAR; WAIT_B(s.B1); mma<INR>(s.acc[0][1], s.At, s.B1);
  if constexpr (FINAL) pack_quad(&s.tmp[0], s.acc[0][0]);
  if constexpr (V == V_BG && I == 9) { reduce_part<0>(s, 2, 4); s.R0 = reduce_fin(s); s.d0lo = s.pdlo; s.d0hi = s.pdhi; s.d0off = s.pdoff; s.ts0 = s.keep ? -1 : s.TS0; }
  if constexpr (V == V_BG && I == 10) { reduce_part<0>(s, 6, 8); s.R1 = reduce_fin(s); s.d1lo = s.pdlo; s.d1hi = s.pdhi; s.d1off = s.pdoff + (s.keep ? 0u : 8u * 2048u); s.ts1 = s.TS1; }
  ENDM;
  LDA(b, 1);
  if constexpr (!FINAL) STAGE_A(SA(b, 0), a2, a2i);
  if constexpr (E0) PST(2);
  if constexpr (V == V_BG && I == 1) PST(6);
  if constexpr (V == V_BG && I == 6) PLD(1);
  if constexpr (V == V_BG && I == 7) PLD(4);
  if constexpr (V == V_BG && I == 8) PLD(7);
  if constexpr (S0) SEND(s.R0, s.d0lo, s.d0hi, s.d0off, s.ts0 == I);
  if constexpr (FINAL) { PST(0); PST(1); PST(2); PST(3); }
  BAR; WAIT_A(); mma<INL>(s.acc[1][0], s.At, s.B0);
  if constexpr (E15) pack_quad(&s.tmp[0], s.acc[0][hv]);
  if constexpr (FINAL) pack_quad(&s.tmp[0], s.acc[0][1]);
  ENDM;
  if constexpr (!FINAL) STAGE_B(SB(b, 1), br, b2i);
  if constexpr (E15) PST(0);
  if constexpr (E0) { PST(3); PST(4); }
  if constexpr (V == V_BG && I == 1) PST(7);
  if constexpr (V == V_BG && I == 6) PLD(2);
  if constexpr (V == V_BG && I == 7) PLD(5);
  if constexpr (FINAL) { s.pw += REGION; PST(0); PST(1); PST(2); PST(3); s.pw -= REGION; }
  else if constexpr (V == V_BG && I == 8) asm volatile("s_waitcnt vmcnt(%4)" : "+v"(s.tmp[0]), "+v"(s.tmp[1]), "+v"(s.tmp[2]), "+v"(s.tmp[3]) : "i"(6 + NX) : "memory");
  else if constexpr (V == V_BG && I == 9) asm volatile("s_waitcnt vmcnt(%4)" : "+v"(s.tmp[4]), "+v"(s.tmp[5]), "+v"(s.tmp[6]), "+v"(s.tmp[7]) : "i"(6 + NX) : "memory");
  else if constexpr (V == V_BG && I == T_POLL + 1) asm volatile("s_waitcnt vmcnt(%1)" : "+v"(s.pollv) : "i"(6 + NX) : "memory");
  else wait_v<6 + NX>();
  if constexpr (V == V_BG && I == T_FLAG) { unsigned _c; asm volatile("v_mov_b32 %0, %1\n\ts_mov_b64 exec, 1\n\tbuffer_store_dword %0, off, %2, %3 sc1\n\ts_mov_b64 exec, -1" : "=&v"(_c) : "s"(s.call), "s"(s.srdWs), "s"(s.pflag + (unsigned)s.wslot * 16u) : "memory"); }
  if constexpr (V == V_BG && I == T_POLL) asm volatile("buffer_load_dword %0, %1, %2, %3 offen sc1" : "=v"(s.pollv) : "v"(s.vlane16), "s"(s.srdWs), "s"(s.pflag) : "memory");
  BAR; mma<INR>(s.acc[1][1], s.At, s.B1);
  if constexpr (FINAL) pack_quad(&s.tmp[4], s.acc[1][0]);
  ENDM;
  if constexpr (V == V_BG && I == T_POLL + 1) {
    if (!__all(s.pollv >= s.call)) s.poison |= spin_flags(s.wsp, s.pflag, s.call);
    SCHED;
  }
}

#define STAMP(i) do {} while (0)

__global__ __launch_bounds__(512, 2) void gemm_a2a(Params p) {
  __shared__ __attribute__((aligned(1024))) unsigned char smem[8 * HTB + 64];
  const int tid = threadIdx.x, wid = __builtin_amdgcn_readfirstlane(tid >> 6), lane = tid & 63;
  const int wr = wid >> 2, wc = wid & 3, fr = lane & 15, fq = lane >> 4;
  const unsigned blk = blockIdx.x, mem = blk & 7u, grp = (blk >> 3) & 31u;
  const unsigned pm = grp & 3u, cg = grp >> 2, sub = mem >> 2, mi = mem & 3u;
  const unsigned rank = (unsigned)p.rank & 7u;
  STAMP(0);
  St s;
  char* wsp = p.ws[rank];
  s.wsp = wsp;
  const unsigned long long pa = (unsigned long long)p.A, pw = (unsigned long long)p.W, pws = (unsigned long long)wsp;
  s.srdA = (u32x4){(unsigned)pa, (unsigned)(pa >> 32) & 0xffffu, A_BYTES, 0x20000u};
  s.srdB = (u32x4){(unsigned)pw, (unsigned)(pw >> 32) & 0xffffu, W_BYTES, 0x20000u};
  s.srdWs = (u32x4){(unsigned)pws, (unsigned)(pws >> 32) & 0xffffu, WS_USED, 0x20000u};
  unsigned info;
  if (cg < 7u) {
    const unsigned j = (cg + 1u) << 3;
    info = (j | 0u) | ((j | (1u + sub)) << 6) | ((j | (3u + sub)) << 12) | ((j | (5u + sub)) << 18) | (cg << 24);
  } else {
    const unsigned j0 = (0x6427u >> (4u * pm)) & 7u;
    const unsigned j1 = ((4u + 2u * pm + sub) % 7u) + 1u, j2 = ((2u + 2u * pm + sub) % 7u) + 1u, j3 = ((2u * pm + sub) % 7u) + 1u;
    info = ((j0 << 3) | 7u) | (((j1 << 3) | 7u) << 6) | (((j2 << 3) | 7u) << 12) | (((j3 << 3) | 7u) << 18) | (7u << 24);
  }
  {
    unsigned inf2 = 0;
#pragma unroll
    for (unsigned k = 0; k < 5u; ++k) {
      const unsigned e = (info >> (6u * k)) & 63u;
      inf2 |= ((e & 56u) | ((e + (e >> 3) + 2u * rank) & 7u)) << (6u * k);
    }
    info = inf2;
  }
  auto ent = [&](unsigned k) -> unsigned { return (info >> (6u * k)) & 63u; };
  auto colB = [&](unsigned e) -> unsigned { return (((rank + (e >> 3)) & 7u) * 8u + (e & 7u)) * 256u; };
  auto kwin = [&](unsigned seg) -> unsigned { return 2u * mi + ((seg + sub) & 1u); };
  auto offA = [&](unsigned seg) -> unsigned { return pm * (256u * K_ * 2u) + kwin(seg) * 2048u; };
  auto offB = [&](unsigned seg, unsigned k) -> unsigned { return kwin(seg) * (1024u * N_ * 2u) + colB(ent(k)); };
  const unsigned lds0 = (unsigned)(size_t)(LAS unsigned char*)smem;
  s.ldsw = lds0 + (unsigned)wid * 1024u;
  s.voffA = (unsigned)((8 * wid + (lane >> 3)) * (K_ * 2) + (((lane & 7) ^ (lane >> 3)) << 4));
  {
    const unsigned fB = (unsigned)((lane >> 4) | (((wid >> 1) & 1) << 2));
    s.voffB = (unsigned)((4 * wid + (lane >> 4)) * (N_ * 2)) + ((((unsigned)lane & 15u) ^ (fB << 1)) << 4);
  }
  s.curA = offA(0); s.curBL = offB(0, 1); s.curBR = offB(0, 0);
  STAGE_B(SB(0, 0), s.curBL, 0u); STAGE_A(SA(0, 0), s.curA, 0u); STAGE_B(SB(0, 1), s.curBR, 0u); STAGE_A(SA(0, 1), s.curA, hA);
  STAGE_B(SB(1, 0), s.curBL, kstepB); STAGE_A(SA(1, 0), s.curA, kstepA); STAGE_B(SB(1, 1), s.curBR, kstepB);
  unsigned callv;
  asm volatile("buffer_load_dword %0, off, %1, %2 sc1" : "=v"(callv) : "s"(s.srdWs), "s"(WS_CALL + blk * 64u) : "memory");
  unsigned olo[5], ohi[5];
#pragma unroll
  for (int k = 0; k < 5; ++k) {
    const unsigned long long o = (unsigned long long)p.out[(rank + (ent(k) >> 3)) & 7u];
    olo[k] = (unsigned)o; ohi[k] = (unsigned)(o >> 32) & 0xffffu;
  }
  {
    const int q = (lane & 15) >> 2, pp = lane & 3, g = lane >> 4;
    const int f = q | ((g & 1) << 2);
    s.aoff0 = lds0 + (unsigned)((wr * 64 + fr) * 128 + (((0 + fq) ^ (fr & 7)) << 4));
    s.aoff1 = lds0 + (unsigned)((wr * 64 + fr) * 128 + (((4 + fq) ^ (fr & 7)) << 4));
    s.boff0 = lds0 + 65536u + (unsigned)((8 * g + q) * 256 + (((2 * wc + 0) ^ f) << 5) + 8 * pp);
    s.boff1 = lds0 + 65536u + (unsigned)((8 * g + q) * 256 + (((2 * wc + 1) ^ f) << 5) + 8 * pp);
  }
  s.vlane16 = (unsigned)lane * 16u;
  {
    const unsigned r = (unsigned)lane >> 3, c = (unsigned)lane & 7u, cc = c & 3u;
    const unsigned sfq = ((cc & 1u) << 1) | (cc >> 1);
    s.vgat = (c >> 2) * 4096u + (sfq * 16u + r) * 16u;
    s.vdst = r * 2048u + c * 16u;
  }
  const unsigned uw = (unsigned)wid, rwr = uw >> 2, rwcp = uw & 1u, b1 = (uw >> 1) & 1u;
  const unsigned ai4 = mi >> 1, m4 = 2u * (mi & 1u) + b1, ai8 = mem >> 2, m8 = mem & 3u;
  unsigned rd4 = (grp * 8u + sub * 4u) * SLOT + ((ai4 * 8u + rwr * 4u + 2u * rwcp) * 4u + m4) * 1024u;
  unsigned rd8 = (grp * 8u) * SLOT + ((ai8 * 8u + rwr * 4u + 2u * rwcp) * 4u + m8) * 1024u + b1 * 128u;
  const unsigned row0 = rank * 1024u + pm * 256u;
  unsigned ds4 = (row0 + ai4 * 128u + rwr * 64u + m4 * 16u) * 2048u + rwcp * 128u;
  unsigned ds8 = (row0 + ai8 * 128u + rwr * 64u + m8 * 16u + b1 * 8u) * 2048u + rwcp * 128u;
  unsigned pw0 = WS_PART + blk * SLOT + uw * 4096u;
  s.wslot = (int)(mem * 8u + uw);
  {
    const unsigned key = grp * 3u + mem + uw;
    s.TS0 = S0_FIRST + (int)(key % (unsigned)S0_N);
    s.TS1 = S1_FIRST + (int)(key % (unsigned)S1_N);
  }
  s.ts0 = -1; s.ts1 = -1; s.poison = 0u;
  s.d0lo = s.d0hi = s.d0off = s.d1lo = s.d1hi = s.d1off = 0u;
  s.R0 = (u32x4){0u, 0u, 0u, 0u}; s.R1 = s.R0;
#pragma unroll
  for (int a = 0; a < 2; ++a)
#pragma unroll
    for (int bb = 0; bb < 2; ++bb)
#pragma unroll
      for (int m = 0; m < 4; ++m)
#pragma unroll
        for (int n = 0; n < 2; ++n) s.acc[a][bb][m][n] = (f32x4){0.f, 0.f, 0.f, 0.f};
  s.nxtA = offA(1); s.nxtBL = offB(1, 1); s.nxtBR = offB(1, 2);
  asm volatile("s_waitcnt lgkmcnt(0)" : "+s"(rd4), "+s"(rd8), "+s"(ds4), "+s"(ds8), "+s"(pw0), "+s"(s.TS0), "+s"(s.TS1), "+s"(s.wslot), "+s"(s.nxtA), "+s"(s.nxtBL), "+s"(s.nxtBR) :: "memory");
  if (wr == 1) BAR;
  wait_v<7>(); BAR;
  BAR;
  STAMP(1);
  {
    const unsigned sa = s.curA, sl = s.curBL, sr = s.curBR;
    ktile<0, V_PLAIN>(s);
    asm volatile("" : "+v"(callv));
    s.call = __builtin_amdgcn_readfirstlane(callv) + 1u;
    ktile<1, V_PLAIN>(s);
    for (int t = 2; t < 14; t += 2) {
      s.curA += 2u * kstepA; s.curBL += 2u * kstepB; s.curBR += 2u * kstepB;
      ktile<0, V_PLAIN>(s); ktile<1, V_PLAIN>(s);
    }
    s.curA = sa; s.curBL = sl; s.curBR = sr;
    ktile<14, V_PLAIN>(s);
  }
  const unsigned call = s.call;
  for (unsigned seg = 1; seg <= 3u; ++seg) {
    const unsigned k = seg - 1u;
    s.pw = pw0 + k * REGION;
    if (seg == 2u) {
      ktile<15, V_L>(s);
      s.curA = s.nxtA; s.curBL = s.nxtBL; s.curBR = s.nxtBR;
      ktile<0, V_L>(s);
    }
    unsigned seg2 = seg; asm volatile("" : "+s"(seg2));
    if (seg2 != 2u) {
      ktile<15, V_R>(s);
      s.curA = s.nxtA; s.curBL = s.nxtBL; s.curBR = s.nxtBR;
      ktile<0, V_R>(s);
    }
    STAMP(1 + seg);
    s.keep = (k == 0u) ? 0xffffffffu : 0u;
    s.prd0 = WS_PART + k * REGION + (k == 0u ? rd8 : rd4);
    s.prd1 = s.prd0 + (k == 0u ? 4u * SLOT : 128u);
    s.pflag = WS_PFLAG + (k * 32u + grp) * 1024u;
    s.pdlo = k == 0u ? olo[0] : k == 1u ? olo[1] : olo[2];
    s.pdhi = k == 0u ? ohi[0] : k == 1u ? ohi[1] : ohi[2];
    s.pdoff = (k == 0u ? ds8 : ds4) + (ent(k) & 7u) * 256u;
    if (seg < 3u) { s.nxtA = offA(seg + 1u); s.nxtBL = offB(seg + 1u, 3u); s.nxtBR = offB(seg + 1u, seg == 1u ? 2u : 4u); }
    else {
      s.nxtA = 0xC0000000u; s.nxtBL = 0xC0000000u; s.nxtBR = 0xC0000000u;
      if (s.TS0 > S3_LAST) s.TS0 = 40;
      if (s.TS1 > S3_LAST) s.TS1 = 40;
    }
    ktile<1, V_BG>(s); ktile<2, V_BG>(s); ktile<3, V_BG>(s); ktile<4, V_BG>(s); ktile<5, V_BG>(s); ktile<6, V_BG>(s); ktile<7, V_BG>(s);
    ktile<8, V_BG>(s); ktile<9, V_BG>(s); ktile<10, V_BG>(s); ktile<11, V_BG>(s); ktile<12, V_BG>(s); ktile<13, V_BG>(s); ktile<14, V_BG>(s);
  }
  s.pw = pw0 + 3u * REGION;
  ktile<15, V_FINAL>(s);
  STAMP(5);
  PST(4); PST(5); PST(6); PST(7);
  asm volatile("s_nop 7" : "+v"(s.acc[1][1][0][0]), "+v"(s.acc[1][1][0][1]), "+v"(s.acc[1][1][1][0]), "+v"(s.acc[1][1][1][1]), "+v"(s.acc[1][1][2][0]), "+v"(s.acc[1][1][2][1]), "+v"(s.acc[1][1][3][0]), "+v"(s.acc[1][1][3][1]));
  pack_quad(&s.tmp[4], s.acc[1][1]);
  s.pw += REGION;
  PST(4); PST(5); PST(6); PST(7);
  const unsigned fblk = WS_PFLAG + (3u * 32u + grp) * 1024u;
  wait_v<0>();
  STAMP(6);
  const int lane2 = lane_id();
  if (lane2 == 0) __hip_atomic_store((unsigned*)(wsp + fblk) + s.wslot * 4, call, __ATOMIC_RELAXED, __HIP_MEMORY_SCOPE_AGENT);
  if (wr == 0) BAR;
  LAS unsigned* lflag = (LAS unsigned*)(smem + 8 * HTB);
  if (wid == 0) { const unsigned bad = spin_flags(wsp, fblk, call); if (lane2 == 0) *lflag = bad; asm volatile("s_waitcnt lgkmcnt(0)" ::: "memory"); }
  BAR;
  s.poison |= __builtin_amdgcn_readfirstlane(*(volatile LAS unsigned*)lflag);
  STAMP(7);
  {
    u32x4 t2[8];
    s.prd0 = WS_PART + 3u * REGION + rd4; s.prd1 = s.prd0 + 128u;
    PLD(0); PLD(1); PLD(2); PLD(3); PLD(4); PLD(5); PLD(6); PLD(7);
#pragma unroll
    for (int j = 0; j < 8; ++j) LD128(t2[j], s.vgat, s.srdWs, WS_PART + 4u * REGION + rd8 + j * SLOT);
    SEND(s.R0, s.d0lo, s.d0hi, s.d0off, s.ts0 >= 16);
    SEND(s.R1, s.d1lo, s.d1hi, s.d1off, s.ts1 >= 16);
    asm volatile("s_waitcnt vmcnt(14)" : "+v"(s.tmp[0]), "+v"(s.tmp[1]), "+v"(s.tmp[2]), "+v"(s.tmp[3]) :: "memory");
    reduce_part<1>(s, 0, 4);
    const u32x4 o0 = reduce_fin(s);
    const unsigned doff = ds4 + (ent(3) & 7u) * 256u;
    SEND(o0, olo[3], ohi[3], doff, true);
    asm volatile("s_waitcnt vmcnt(11)" : "+v"(s.tmp[4]), "+v"(s.tmp[5]), "+v"(s.tmp[6]), "+v"(s.tmp[7]) :: "memory");
    reduce_part<1>(s, 4, 8);
    const u32x4 o1 = reduce_fin(s);
    SEND(o1, olo[3], ohi[3], doff + 8u * 2048u, true);
    asm volatile("s_waitcnt vmcnt(4)" : "+v"(t2[0]), "+v"(t2[1]), "+v"(t2[2]), "+v"(t2[3]), "+v"(t2[4]), "+v"(t2[5]), "+v"(t2[6]), "+v"(t2[7]) :: "memory");
#pragma unroll
    for (int j = 0; j < 8; ++j) s.tmp[j] = t2[j];
    reduce_part<1>(s, 0, 8);
    const u32x4 o2 = reduce_fin(s);
    SEND(o2, olo[4], ohi[4], ds8 + (ent(4) & 7u) * 256u, true);
  }
  STAMP(8);
  wait_v<0>();
  STAMP(9);
  BAR;
  if (wid == 0) {
    if (lane2 < 4) {
      const unsigned e = (info >> (6u * (unsigned)lane2)) & 63u;
      const unsigned dp = (rank + (e >> 3)) & 7u;
      unsigned* f = (unsigned*)(p.ws[dp] + WS_DONE + rank * 1024u) + ((e & 7u) * 4u + pm) * 8u;
      if (lane2 == 0) __hip_atomic_store(f + mem, call, __ATOMIC_RELAXED, __HIP_MEMORY_SCOPE_SYSTEM);
      else __hip_atomic_store((unsigned long long*)f + mi, ((unsigned long long)call << 32) | call, __ATOMIC_RELAXED, __HIP_MEMORY_SCOPE_SYSTEM);
    }
    if (lane2 == 0) __hip_atomic_store((unsigned*)(wsp + WS_CALL + blk * 64u), call, __ATOMIC_RELAXED, __HIP_MEMORY_SCOPE_AGENT);
    if (blk == 0) {
      const unsigned long long* f = (const unsigned long long*)(wsp + WS_DONE) + lane2;
      unsigned* tmo = (unsigned*)(wsp + WS_TMO);
      const unsigned long long t0 = __builtin_amdgcn_s_memrealtime();
      for (;;) {
        bool ok = true;
#pragma unroll
        for (unsigned j = 1; j < 8; ++j) {
          const unsigned long long v0 = __hip_atomic_load(f + ((rank + j) & 7u) * 128u, __ATOMIC_RELAXED, __HIP_MEMORY_SCOPE_SYSTEM);
          const unsigned long long v1 = __hip_atomic_load(f + ((rank + j) & 7u) * 128u + 64u, __ATOMIC_RELAXED, __HIP_MEMORY_SCOPE_SYSTEM);
          ok = ok && ((unsigned)v0 >= call) && ((unsigned)(v0 >> 32) >= call) && ((unsigned)v1 >= call) && ((unsigned)(v1 >> 32) >= call);
        }
        if (__all(ok)) break;
        if (__builtin_amdgcn_readfirstlane(__hip_atomic_load(tmo, __ATOMIC_RELAXED, __HIP_MEMORY_SCOPE_AGENT)) == call ||
            __builtin_amdgcn_s_memrealtime() - t0 > DEADLINE) {
          if (lane2 == 0) {
            __hip_atomic_store(tmo, call, __ATOMIC_RELAXED, __HIP_MEMORY_SCOPE_AGENT);
            __hip_atomic_store((unsigned*)(p.out[rank] + (size_t)rank * 1024u * 2048u), 0x7fc07fc0u, __ATOMIC_RELAXED, __HIP_MEMORY_SCOPE_SYSTEM);
          }
          break;
        }
        __builtin_amdgcn_s_sleep(1);
      }
      __builtin_amdgcn_fence(__ATOMIC_ACQUIRE, "");
    }
  }
}

}

extern "C" void kernel_launch(int rank, int nranks, ncclComm_t comm,
                              hipStream_t stream,
                              void* const* d_in, const int* in_sizes, int n_in,
                              void* d_out, int out_size,
                              const int* in_dtypes, int out_dtype,
                              void* const* d_out_all,
                              void* const* d_ws_all, size_t ws_bytes) {
  using namespace ga2a;
  (void)comm; (void)in_sizes; (void)n_in; (void)d_out; (void)out_size; (void)in_dtypes; (void)out_dtype;
  if (nranks != NR || ws_bytes < (size_t)WS_USED) { fprintf(stderr, "unsupported configuration\n"); return; }
  Params p;
  p.A = (const u16*)d_in[0];
  p.W = (const u16*)d_in[1];
  for (int i = 0; i < NR; ++i) { p.out[i] = (char*)d_out_all[i]; p.ws[i] = (char*)d_ws_all[i]; }
  p.rank = rank;
  hipLaunchKernelGGL(gemm_a2a, dim3(256), dim3(512), 0, stream, p);
  HIP_CHECK(hipGetLastError());
}
```

```cpp
#include <hip/hip_runtime.h>
#include <rccl/rccl.h>
#include <cstdio>

#define HIP_CHECK(expr) do { hipError_t _e = (expr); if (_e != hipSuccess) { \
    fprintf(stderr, "HIP error %s at %s:%d\n", hipGetErrorString(_e), __FILE__, __LINE__); \
    return; } } while (0)

namespace ga2a {

#define LAS __attribute__((address_space(3)))
typedef unsigned short u16;
typedef short bf16x8 __attribute__((ext_vector_type(8)));
typedef short bf16x4 __attribute__((ext_vector_type(4)));
typedef float f32x4 __attribute__((ext_vector_type(4)));
typedef float f32x2 __attribute__((ext_vector_type(2)));
typedef __bf16 bf16x2n __attribute__((ext_vector_type(2)));
typedef unsigned u32x4 __attribute__((ext_vector_type(4)));

constexpr int NR = 8;
constexpr int M_ = 1024, N_ = 8192, K_ = 8192;
constexpr unsigned HTB = 16384;
constexpr unsigned A_BYTES = (unsigned)M_ * K_ * 2u, W_BYTES = (unsigned)K_ * N_ * 2u;
constexpr unsigned OUT_BYTES = 8192u * 1024u * 2u;
constexpr unsigned SLOT = 65536u;
constexpr unsigned REGION = 256u * SLOT;
constexpr unsigned WS_PART = 0u;
constexpr unsigned RMASK = 3u, RFIN = 3u;
constexpr unsigned WS_CTRL = 128u << 20;
constexpr unsigned WS_PFLAG = WS_CTRL + 524288u;
constexpr unsigned WS_CALL = WS_CTRL + 131072u;
constexpr unsigned WS_DONE = WS_CTRL + 163840u;
constexpr unsigned WS_TMO = WS_CTRL + 180224u;
constexpr unsigned WS_STAMP = WS_CTRL + 262144u;
constexpr unsigned WS_USED = WS_CTRL + (1u << 20);
constexpr unsigned long long DEADLINE = 50000000ull;

#define GA2A_KORD 2
constexpr unsigned ktA = 128u, ktB = 64u * N_ * 2u;
constexpr unsigned kstepA = 0u - 8u * ktA, kstepB = 0u - 8u * ktB, winA = ktA, winB = ktB, k0A = 120u * ktA, k0B = 120u * ktB;
constexpr unsigned hA = 128u * K_ * 2u;
constexpr unsigned iA = 64u * K_ * 2u, iB = 32u * N_ * 2u;

struct Params {
  const u16* A;
  const u16* W;
  char* out[NR];
  char* ws[NR];
  int rank;
};

enum { V_PLAIN = 0, V_BG = 1, V_L = 2, V_R = 3 };
#define GA2A_TP 3
constexpr int T_FLAG = 2, T_POLL = GA2A_TP, T_LD = T_POLL + 2;
constexpr int S0_FIRST = T_LD + 3, S0_N = 6, S1_FIRST = T_LD + 5, S1_N = 10;
constexpr bool slot_at(int first, int n, int i) { return (i >= first && i < first + n) || (i + 16 >= first && i + 16 < first + n); }

__device__ __forceinline__ unsigned pk2(float lo, float hi) {
  f32x2 v = {lo, hi};
  return __builtin_bit_cast(unsigned, __builtin_convertvector(v, bf16x2n));
}
__device__ __forceinline__ u32x4 pack_pair(const f32x4& v0, const f32x4& v1) {
  unsigned d00 = pk2(v0[0], v0[1]), d01 = pk2(v0[2], v0[3]);
  unsigned d10 = pk2(v1[0], v1[1]), d11 = pk2(v1[2], v1[3]);
  auto r0 = __builtin_amdgcn_permlane16_swap(d00, d10, false, false);
  auto r1 = __builtin_amdgcn_permlane16_swap(d01, d11, false, false);
  u32x4 w; w.x = r0[0]; w.y = r1[0]; w.z = r0[1]; w.w = r1[1];
  return w;
}

struct St {
  f32x4 acc[2][2][4][2];
  bf16x8 At[4][2];
  bf16x4 B0[2][2][2], B1[2][2][2];
  u32x4 tmp[8];
  f32x4 D1, D2;
  u32x4 S1, S2;
  u32x4 R0, R1;
  unsigned pollv;
  unsigned voffA, voffB, aoff0, aoff1, boff0, boff1, vlane16, vgat, vdst;
  u32x4 srdA, srdB, srdWs;
  unsigned ldsw, curA, curBL, curBR, nxtA, nxtBL, nxtBR;
  unsigned pw;
  unsigned prd0, prd1;
  unsigned pflag;
  unsigned keep;
  unsigned call, poison;
  unsigned pdlo, pdhi, pdoff;
  unsigned d0lo, d0hi, d0off, d1lo, d1hi, d1off;
  int ts0, ts1, TS0, TS1, wslot;
  char* wsp;
};

#define SA(b, h) (((b) * 2u + (h)) * HTB)
#define SB(b, h) ((4u + (b) * 2u + (h)) * HTB)
#define DMA2(ldsoff, voff, srd, sbase, simm, step) do { unsigned _t; asm volatile( \
    "s_add_u32 m0, %1, %2\n\ts_add_u32 %0, %6, %7\n\tbuffer_load_dwordx4 %4, %5, %0 offen lds\n\t" \
    "s_add_u32 m0, %1, %3\n\ts_add_u32 %0, %6, %8\n\tbuffer_load_dwordx4 %4, %5, %0 offen lds" \
    : "=&s"(_t) : "s"(s.ldsw), "i"(ldsoff), "i"((ldsoff) + 8192u), "v"(voff), "s"(srd), "s"(sbase), "i"(simm), "i"((simm) + (step)) : "memory", "scc"); } while (0)
#define STAGE_A(bufoff, sbase, simm) DMA2((bufoff), s.voffA, s.srdA, (sbase), (simm), iA)
#define STAGE_B(bufoff, sbase, simm) DMA2((bufoff), s.voffB, s.srdB, (sbase), (simm), iB)
__device__ __forceinline__ int lane_id() { int l; asm volatile("v_mbcnt_lo_u32_b32 %0, -1, 0\n\tv_mbcnt_hi_u32_b32 %0, -1, %0" : "=v"(l)); return l; }
#define RD128(dst, addr, off) asm volatile("ds_read_b128 %0, %1 offset:%2" : "=v"(dst) : "v"(addr), "i"(off))
#define RDTR(dst, addr, off) asm volatile("ds_read_b64_tr_b16 %0, %1 offset:%2" : "=v"(dst) : "v"(addr), "i"(off))
#define LDA(b, h) do { _Pragma("unroll") for (int m = 0; m < 4; ++m) { RD128(s.At[m][0], s.aoff0, SA(b, h) + m * 2048); RD128(s.At[m][1], s.aoff1, SA(b, h) + m * 2048); } } while (0)
#define LDB(dst, b, h) do { _Pragma("unroll") for (int k = 0; k < 2; ++k) { \
    RDTR(dst[0][k][0], s.boff0, SB(b, h) - 65536u + k * 8192); RDTR(dst[0][k][1], s.boff0, SB(b, h) - 65536u + k * 8192 + 1024); } \
    _Pragma("unroll") for (int k = 0; k < 2; ++k) { \
    RDTR(dst[1][k][0], s.boff1, SB(b, h) - 65536u + k * 8192); RDTR(dst[1][k][1], s.boff1, SB(b, h) - 65536u + k * 8192 + 1024); } } while (0)
#define WAIT_A() asm volatile("s_waitcnt lgkmcnt(0)" : "+v"(s.At[0][0]), "+v"(s.At[0][1]), "+v"(s.At[1][0]), "+v"(s.At[1][1]), "+v"(s.At[2][0]), "+v"(s.At[2][1]), "+v"(s.At[3][0]), "+v"(s.At[3][1]) :: "memory")
#define WAIT_B(Bt) asm volatile("s_waitcnt lgkmcnt(0)" : "+v"(Bt[0][0][0]), "+v"(Bt[0][0][1]), "+v"(Bt[0][1][0]), "+v"(Bt[0][1][1]), "+v"(Bt[1][0][0]), "+v"(Bt[1][0][1]), "+v"(Bt[1][1][0]), "+v"(Bt[1][1][1]) :: "memory")
#define CAT(x) __builtin_shufflevector(x[0], x[1], 0, 1, 2, 3, 4, 5, 6, 7)
#define WAIT_L(n) asm volatile("s_waitcnt lgkmcnt(" #n ")" ::: "memory")
#define BAR __builtin_amdgcn_s_barrier()
#define SCHED __builtin_amdgcn_sched_barrier(0)
#define ENDM do { SCHED; asm volatile("s_setprio 0"); BAR; SCHED; } while (0)
#define ST128(data, voff, srd, soff, bits) asm volatile("buffer_store_dwordx4 %0, %1, %2, %3 offen " bits "\n\ts_nop 1" :: "v"(data), "v"(voff), "s"(srd), "s"(soff) : "memory")
#define LD128(dst, voff, srd, soff) asm volatile("buffer_load_dwordx4 %0, %1, %2, %3 offen sc1" : "=v"(dst) : "v"(voff), "s"(srd), "s"(soff) : "memory")
#define PST(u) ST128(s.tmp[u], s.vlane16, s.srdWs, s.pw + ((u) >> 2) * 32768u + ((u) & 3) * 1024u, "sc1")
#define PLD(j) LD128(s.tmp[j], s.vgat, s.srdWs, ((j) < 4 ? s.prd0 : s.prd1) + ((j) & 3) * SLOT)

template <int N> __device__ __forceinline__ void wait_v() { asm volatile("s_waitcnt vmcnt(%0)" :: "i"(N) : "memory"); }

template <bool INIT>
__device__ __forceinline__ void mma(f32x4 (&acc)[4][2], const bf16x8 (&At)[4][2], const bf16x4 (&Bt)[2][2][2]) {
  const bf16x8 b00 = CAT(Bt[0][0]), b01 = CAT(Bt[0][1]), b10 = CAT(Bt[1][0]), b11 = CAT(Bt[1][1]);
  if constexpr (INIT)
    asm volatile("s_setprio 1\n\t"
      "v_mfma_f32_16x16x32_bf16 %0, %16, %8, 0\n\t"
      "v_mfma_f32_16x16x32_bf16 %0, %17, %9, %0\n\t"
      "v_mfma_f32_16x16x32_bf16 %1, %18, %8, 0\n\t"
      "v_mfma_f32_16x16x32_bf16 %1, %19, %9, %1\n\t"
      "v_mfma_f32_16x16x32_bf16 %2, %16, %10, 0\n\t"
      "v_mfma_f32_16x16x32_bf16 %2, %17, %11, %2\n\t"
      "v_mfma_f32_16x16x32_bf16 %3, %18, %10, 0\n\t"
      "v_mfma_f32_16x16x32_bf16 %3, %19, %11, %3\n\t"
      "v_mfma_f32_16x16x32_bf16 %4, %16, %12, 0\n\t"
      "v_mfma_f32_16x16x32_bf16 %4, %17, %13, %4\n\t"
      "v_mfma_f32_16x16x32_bf16 %5, %18, %12, 0\n\t"
      "v_mfma_f32_16x16x32_bf16 %5, %19, %13, %5\n\t"
      "v_mfma_f32_16x16x32_bf16 %6, %16, %14, 0\n\t"
      "v_mfma_f32_16x16x32_bf16 %6, %17, %15, %6\n\t"
      "v_mfma_f32_16x16x32_bf16 %7, %18, %14, 0\n\t"
      "v_mfma_f32_16x16x32_bf16 %7, %19, %15, %7\n\t"
      : "+v"(acc[0][0]), "+v"(acc[0][1]), "+v"(acc[1][0]), "+v"(acc[1][1]), "+v"(acc[2][0]), "+v"(acc[2][1]), "+v"(acc[3][0]), "+v"(acc[3][1])
      : "v"(At[0][0]), "v"(At[0][1]), "v"(At[1][0]), "v"(At[1][1]), "v"(At[2][0]), "v"(At[2][1]), "v"(At[3][0]), "v"(At[3][1]), "v"(b00), "v"(b01), "v"(b10), "v"(b11));
  else
    asm volatile("s_setprio 1\n\t"
      "v_mfma_f32_16x16x32_bf16 %0, %16, %8, %0\n\t"
      "v_mfma_f32_16x16x32_bf16 %0, %17, %9, %0\n\t"
      "v_mfma_f32_16x16x32_bf16 %1, %18, %8, %1\n\t"
      "v_mfma_f32_16x16x32_bf16 %1, %19, %9, %1\n\t"
      "v_mfma_f32_16x16x32_bf16 %2, %16, %10, %2\n\t"
      "v_mfma_f32_16x16x32_bf16 %2, %17, %11, %2\n\t"
      "v_mfma_f32_16x16x32_bf16 %3, %18, %10, %3\n\t"
      "v_mfma_f32_16x16x32_bf16 %3, %19, %11, %3\n\t"
      "v_mfma_f32_16x16x32_bf16 %4, %16, %12, %4\n\t"
      "v_mfma_f32_16x16x32_bf16 %4, %17, %13, %4\n\t"
      "v_mfma_f32_16x16x32_bf16 %5, %18, %12, %5\n\t"
      "v_mfma_f32_16x16x32_bf16 %5, %19, %13, %5\n\t"
      "v_mfma_f32_16x16x32_bf16 %6, %16, %14, %6\n\t"
      "v_mfma_f32_16x16x32_bf16 %6, %17, %15, %6\n\t"
      "v_mfma_f32_16x16x32_bf16 %7, %18, %14, %7\n\t"
      "v_mfma_f32_16x16x32_bf16 %7, %19, %15, %7\n\t"
      : "+v"(acc[0][0]), "+v"(acc[0][1]), "+v"(acc[1][0]), "+v"(acc[1][1]), "+v"(acc[2][0]), "+v"(acc[2][1]), "+v"(acc[3][0]), "+v"(acc[3][1])
      : "v"(At[0][0]), "v"(At[0][1]), "v"(At[1][0]), "v"(At[1][1]), "v"(At[2][0]), "v"(At[2][1]), "v"(At[3][0]), "v"(At[3][1]), "v"(b00), "v"(b01), "v"(b10), "v"(b11));
}

__device__ __forceinline__ void pack_quad(u32x4* dst, const f32x4 (&q)[4][2]) {
#pragma unroll
  for (int m = 0; m < 4; ++m) { dst[m] = pack_pair(q[m][0], q[m][1]); asm volatile("" : "+v"(dst[m])); }
}
template <int FIRST>
__device__ __forceinline__ void reduce4(St& s, int j) {
  if constexpr (FIRST == 1)
    asm volatile("v_mfma_f32_16x16x32_bf16 %0, %2, %4, 0\n\tv_mfma_f32_16x16x32_bf16 %1, %3, %4, 0\n\t"
                 "v_mfma_f32_16x16x32_bf16 %0, %2, %5, %0\n\tv_mfma_f32_16x16x32_bf16 %1, %3, %5, %1\n\t"
                 "v_mfma_f32_16x16x32_bf16 %0, %2, %6, %0\n\tv_mfma_f32_16x16x32_bf16 %1, %3, %6, %1\n\t"
                 "v_mfma_f32_16x16x32_bf16 %0, %2, %7, %0\n\tv_mfma_f32_16x16x32_bf16 %1, %3, %7, %1"
                 : "=&v"(s.D1), "=&v"(s.D2) : "v"(s.S1), "v"(s.S2), "v"(s.tmp[j]), "v"(s.tmp[j + 1]), "v"(s.tmp[j + 2]), "v"(s.tmp[j + 3]));
  else if constexpr (FIRST == 2)
    asm volatile("v_mfma_f32_16x16x32_bf16 %0, %2, %4, 0\n\tv_mfma_f32_16x16x32_bf16 %1, %3, %4, 0\n\t"
                 "v_mfma_f32_16x16x32_bf16 %0, %2, %5, %0\n\tv_mfma_f32_16x16x32_bf16 %1, %3, %5, %1\n\t"
                 "v_mfma_f32_16x16x32_bf16 %0, %2, %6, %0\n\tv_mfma_f32_16x16x32_bf16 %1, %3, %6, %1\n\t"
                 "v_mfma_f32_16x16x32_bf16 %0, %2, %7, %0\n\tv_mfma_f32_16x16x32_bf16 %1, %3, %7, %1"
                 : "+v"(s.D1), "+v"(s.D2) : "v"(s.S1), "v"(s.S2), "v"(s.tmp[j]), "v"(s.tmp[j + 1]), "v"(s.tmp[j + 2]), "v"(s.tmp[j + 3]));
  else
    asm volatile("v_mfma_f32_16x16x32_bf16 %0, %2, %4, %0\n\tv_mfma_f32_16x16x32_bf16 %1, %3, %4, %1\n\t"
                 "v_mfma_f32_16x16x32_bf16 %0, %2, %5, %0\n\tv_mfma_f32_16x16x32_bf16 %1, %3, %5, %1\n\t"
                 "v_mfma_f32_16x16x32_bf16 %0, %2, %6, %0\n\tv_mfma_f32_16x16x32_bf16 %1, %3, %6, %1\n\t"
                 "v_mfma_f32_16x16x32_bf16 %0, %2, %7, %0\n\tv_mfma_f32_16x16x32_bf16 %1, %3, %7, %1"
                 : "+v"(s.D1), "+v"(s.D2) : "v"(s.S1), "v"(s.S2), "v"(s.tmp[j]), "v"(s.tmp[j + 1]), "v"(s.tmp[j + 2]), "v"(s.tmp[j + 3]));
}
template <bool NOP>
__device__ __forceinline__ u32x4 reduce_fin(St& s) {
  if constexpr (NOP) asm volatile("s_nop 7" : "+v"(s.D1), "+v"(s.D2)); else asm volatile("" : "+v"(s.D1), "+v"(s.D2));
  u32x4 r;
  r.x = pk2(s.D1[0], s.D1[1]) | s.poison; r.y = pk2(s.D1[2], s.D1[3]) | s.poison;
  r.z = pk2(s.D2[0], s.D2[1]) | s.poison; r.w = pk2(s.D2[2], s.D2[3]) | s.poison;
  asm volatile("" : "+v"(r));
  return r;
}
__device__ __forceinline__ unsigned spin_flags(const char* base, unsigned blk, unsigned stride, unsigned call, const char* wsp) {
  const int lane = lane_id();
  const unsigned* f = (const unsigned*)(base + blk + (unsigned)lane * stride);
  unsigned* tmo = (unsigned*)(wsp + WS_TMO);
  const unsigned long long t0 = __builtin_amdgcn_s_memrealtime();
  unsigned bad = 0;
  for (unsigned n = 1;; ++n) {
    const unsigned v = __hip_atomic_load(f, __ATOMIC_RELAXED, __HIP_MEMORY_SCOPE_SYSTEM);
    if (__all(v >= call)) break;
    if ((n & 15u) == 0u &&
        (__builtin_amdgcn_readfirstlane(__hip_atomic_load(tmo, __ATOMIC_RELAXED, __HIP_MEMORY_SCOPE_AGENT)) == call ||
         __builtin_amdgcn_s_memrealtime() - t0 > DEADLINE)) {
      if (lane == 0) __hip_atomic_store(tmo, call, __ATOMIC_RELAXED, __HIP_MEMORY_SCOPE_AGENT);
      bad = 0x7fc07fc0u;
      break;
    }
  }
  asm volatile("s_waitcnt vmcnt(0) lgkmcnt(0)" ::: "memory");
  return bad;
}
#define SENDL(R, lo, hi, off) do { const u32x4 _srd = {lo, hi, OUT_BYTES, 0x20000u}; ST128(R, s.vdst, _srd, off, "sc0 sc1"); } while (0)
#define SEND(R, lo, hi, off, go) do { const u32x4 _srd = {lo, hi, (go) ? OUT_BYTES : 0u, 0x20000u}; ST128(R, s.vdst, _srd, off, "sc0 sc1"); } while (0)

#define PFLAG(blk) do { unsigned _c; asm volatile("v_mov_b32 %0, %1\n\ts_mov_b64 exec, 1\n\tbuffer_store_dword %0, off, %2, %3 sc1\n\ts_mov_b64 exec, -1" : "=&v"(_c) : "s"(s.call), "s"(s.srdWs), "s"((blk) + (unsigned)s.wslot * 16u) : "memory"); } while (0)
#define GUARD(q) asm volatile("s_nop 7" : "+v"(q[0][0]), "+v"(q[0][1]), "+v"(q[1][0]), "+v"(q[1][1]), "+v"(q[2][0]), "+v"(q[2][1]), "+v"(q[3][0]), "+v"(q[3][1]))

template <int I, int V>
__device__ __forceinline__ void ktile(St& s) {
  constexpr unsigned b = I & 1, nb = b ^ 1;
  constexpr bool BG = (V != V_PLAIN);
  constexpr bool E15 = (I == 15 && (V == V_L || V == V_R)), E0 = (I == 0 && (V == V_L || V == V_R));
  constexpr int hv = (V == V_R) ? 1 : 0;
  constexpr bool INL = E0 && hv == 0, INR = E0 && hv == 1;
  const unsigned a1 = (I + 1 < 16) ? s.curA : s.nxtA, a2 = (I + 2 < 16) ? s.curA : s.nxtA;
  const unsigned bl = (I + 2 < 16) ? s.curBL : s.nxtBL, br = (I + 2 < 16) ? s.curBR : s.nxtBR;
  constexpr unsigned a1i = (unsigned)((I + 1) & 15) * kstepA + hA, a2i = (unsigned)((I + 2) & 15) * kstepA, b2i = (unsigned)((I + 2) & 15) * kstepB;
  constexpr bool S0 = BG && slot_at(S0_FIRST, S0_N, I), S1 = BG && slot_at(S1_FIRST, S1_N, I);
  constexpr int NX = (E15 ? 1 : 0) + (E0 ? 4 : 0) + ((V == V_BG && I == 1) ? 3 : 0) +
                     ((V == V_BG && (I == T_LD || I == T_LD + 1)) ? 3 : 0) + ((V == V_BG && I == T_LD + 2) ? 2 : 0);
  LDB(s.B0, b, 0); LDA(b, 0);
  STAGE_A(SA(nb, 1), a1, a1i);
  WAIT_L(8); BAR; WAIT_B(s.B0); WAIT_A(); mma<INL>(s.acc[0][0], s.At, s.B0);
  if constexpr (E0) pack_quad(&s.tmp[4], s.acc[1][hv]);
  if constexpr (V == V_BG && I == T_LD + 3) reduce4<1>(s, 0);
  if constexpr (V == V_BG && I == T_LD + 4) { if (s.keep != 0u) reduce4<0>(s, 4); unsigned k2 = s.keep; asm volatile("" : "+s"(k2)); if (k2 == 0u) reduce4<2>(s, 4); }
  ENDM;
  LDB(s.B1, b, 1);
  STAGE_B(SB(b, 0), bl, b2i);
  if constexpr (E0) PST(1);
  if constexpr (V == V_BG && I == 1) PST(5);
  if constexpr (V == V_BG && I == T_LD) PLD(0);
  if constexpr (V == V_BG && I == T_LD + 1) PLD(3);
  if constexpr (V == V_BG && I == T_LD + 2) PLD(6);
  BAR; WAIT_B(s.B1); mma<INR>(s.acc[0][1], s.At, s.B1);
  if constexpr (V == V_BG && I == T_LD + 3) { s.R0 = reduce_fin<false>(s); s.d0lo = s.pdlo; s.d0hi = s.pdhi; s.d0off = s.pdoff; s.ts0 = s.TS0 | (int)s.keep; asm volatile("" : "+s"(s.d0lo), "+s"(s.d0hi), "+s"(s.d0off), "+s"(s.ts0)); }
  if constexpr (V == V_BG && I == T_LD + 4) { s.R1 = reduce_fin<false>(s); s.d1lo = s.pdlo; s.d1hi = s.pdhi; s.d1off = s.pdoff + (~s.keep & (8u * 2048u)); s.ts1 = s.TS1; asm volatile("" : "+s"(s.d1lo), "+s"(s.d1hi), "+s"(s.d1off), "+s"(s.ts1)); }
  ENDM;
  LDA(b, 1);
  STAGE_A(SA(b, 0), a2, a2i);
  if constexpr (E0) PST(2);
  if constexpr (V == V_BG && I == 1) PST(6);
  if constexpr (V == V_BG && I == T_LD) PLD(1);
  if constexpr (V == V_BG && I == T_LD + 1) PLD(4);
  if constexpr (V == V_BG && I == T_LD + 2) PLD(7);
  if constexpr (S0) { if (s.ts0 == I) SENDL(s.R0, s.d0lo, s.d0hi, s.d0off); }
  if constexpr (S1) { if (s.ts1 == (I < S1_FIRST ? I + 16 : I)) SENDL(s.R1, s.d1lo, s.d1hi, s.d1off); }
  BAR; WAIT_A(); mma<INL>(s.acc[1][0], s.At, s.B0);
  if constexpr (E15) pack_quad(&s.tmp[0], s.acc[0][hv]);
  ENDM;
  STAGE_B(SB(b, 1), br, b2i);
  if constexpr (E15) PST(0);
  if constexpr (E0) { PST(3); PST(4); }
  if constexpr (V == V_BG && I == 1) PST(7);
  if constexpr (V == V_BG && I == T_LD) PLD(2);
  if constexpr (V == V_BG && I == T_LD + 1) PLD(5);
  if constexpr (V == V_BG && I == T_LD + 2) asm volatile("s_waitcnt vmcnt(%4)" : "+v"(s.tmp[0]), "+v"(s.tmp[1]), "+v"(s.tmp[2]), "+v"(s.tmp[3]) : "i"(6 + NX) : "memory");
  else if constexpr (V == V_BG && I == T_LD + 3) asm volatile("s_waitcnt vmcnt(%4)" : "+v"(s.tmp[4]), "+v"(s.tmp[5]), "+v"(s.tmp[6]), "+v"(s.tmp[7]) : "i"(6 + NX) : "memory");
  else if constexpr (V == V_BG && I == T_POLL + 1) asm volatile("s_waitcnt vmcnt(%1)" : "+v"(s.pollv) : "i"(6 + NX) : "memory");
  else wait_v<6 + NX>();
  if constexpr (V == V_BG && I == T_FLAG) PFLAG(s.pflag);
  if constexpr (V == V_BG && I == T_POLL) asm volatile("buffer_load_dword %0, %1, %2, %3 offen sc1" : "=v"(s.pollv) : "v"(s.vlane16), "s"(s.srdWs), "s"(s.pflag) : "memory");
  BAR; mma<INR>(s.acc[1][1], s.At, s.B1);
  ENDM;
  if constexpr (V == V_BG && I == T_POLL + 1) {
    if (!__all(s.pollv >= s.call)) s.poison |= spin_flags(s.wsp, s.pflag, 16u, s.call, s.wsp);
    SCHED;
  }
}

__device__ __forceinline__ void final_pair(St& s, unsigned flagL) {
  LDB(s.B0, 0u, 0); LDA(0u, 0); STAGE_A(SA(1u, 1), s.curA, 15u * kstepA + hA);
  SEND(s.R1, s.d1lo, s.d1hi, s.d1off, s.ts1 == 14);
  BAR; WAIT_B(s.B0); WAIT_A(); mma<false>(s.acc[0][0], s.At, s.B0); ENDM;
  LDA(0u, 1);
  BAR; WAIT_A(); mma<false>(s.acc[1][0], s.At, s.B0); ENDM;
  LDB(s.B1, 0u, 1); wait_v<3>();
  BAR; WAIT_B(s.B1); mma<false>(s.acc[1][1], s.At, s.B1); ENDM;
  LDB(s.B0, 1u, 0); LDA(1u, 0); wait_v<1>();
  BAR; WAIT_B(s.B0); WAIT_A(); mma<false>(s.acc[0][0], s.At, s.B0); ENDM;
  LDA(1u, 1); GUARD(s.acc[0][0]); pack_quad(&s.tmp[0], s.acc[0][0]); PST(0); PST(1); PST(2); PST(3);
  BAR; WAIT_A(); mma<false>(s.acc[1][0], s.At, s.B0); ENDM;
  LDB(s.B1, 1u, 1); GUARD(s.acc[1][0]); pack_quad(&s.tmp[4], s.acc[1][0]); PST(4); PST(5); PST(6); PST(7);
  BAR; WAIT_B(s.B1); mma<false>(s.acc[1][1], s.At, s.B1); ENDM;
  LDA(1u, 0);
  BAR; WAIT_A(); mma<false>(s.acc[0][1], s.At, s.B1); ENDM;
  LDB(s.B1, 0u, 1); LDA(0u, 0); wait_v<0>(); PFLAG(flagL);
  s.pw += REGION;
  pack_quad(&s.tmp[4], s.acc[1][1]); PST(4); PST(5); PST(6); PST(7);
  BAR; WAIT_B(s.B1); WAIT_A(); mma<false>(s.acc[0][1], s.At, s.B1); ENDM;
}

#define STAMP(i) do {} while (0)

__global__ __launch_bounds__(512, 2) void gemm_a2a(Params p) {
  __shared__ __attribute__((aligned(1024))) unsigned char smem[8 * HTB + 64];
  const int tid = threadIdx.x, wid = __builtin_amdgcn_readfirstlane(tid >> 6), lane = tid & 63;
  const int wr = wid >> 2, wc = wid & 3, fr = lane & 15, fq = lane >> 4;
  const unsigned blk = blockIdx.x, mem = blk & 7u, grp = (blk >> 3) & 31u;
  const unsigned pm = grp & 3u, cg = grp >> 2, sub = mem >> 2, mi = mem & 3u;
  const unsigned rank = (unsigned)p.rank & 7u;
  STAMP(0);
  St s;
  char* wsp = p.ws[rank];
  s.wsp = wsp;
  const unsigned long long pa = (unsigned long long)p.A, pw = (unsigned long long)p.W, pws = (unsigned long long)wsp;
  s.srdA = (u32x4){(unsigned)pa, (unsigned)(pa >> 32) & 0xffffu, A_BYTES, 0x20000u};
  s.srdB = (u32x4){(unsigned)pw, (unsigned)(pw >> 32) & 0xffffu, W_BYTES, 0x20000u};
  s.srdWs = (u32x4){(unsigned)pws, (unsigned)(pws >> 32) & 0xffffu, WS_USED, 0x20000u};
  unsigned info;
  if (cg < 7u) {
    const unsigned j = (cg + 1u) << 3;
    info = (j | 0u) | ((j | (1u + sub)) << 6) | ((j | (3u + sub)) << 12) | ((j | (5u + sub)) << 18) | (cg << 24);
  } else {
    const unsigned j0 = (0x6427u >> (4u * pm)) & 7u;
    const unsigned j1 = ((4u + 2u * pm + sub) % 7u) + 1u, j2 = ((2u + 2u * pm + sub) % 7u) + 1u, j3 = ((2u * pm + sub) % 7u) + 1u;
    info = ((j0 << 3) | 7u) | (((j1 << 3) | 7u) << 6) | (((j2 << 3) | 7u) << 12) | (((j3 << 3) | 7u) << 18) | (7u << 24);
  }
  auto ent = [&](unsigned k) -> unsigned { return (info >> (6u * k)) & 63u; };
  auto colB = [&](unsigned e) -> unsigned { return (((rank + (e >> 3)) & 7u) * 8u + (e & 7u)) * 256u; };
  auto kwin = [&](unsigned seg) -> unsigned { return 2u * mi + ((seg + sub) & 1u); };
  auto offA = [&](unsigned seg) -> unsigned { return pm * (256u * K_ * 2u) + k0A + kwin(seg) * winA; };
  auto offB = [&](unsigned seg, unsigned k) -> unsigned { return k0B + kwin(seg) * winB + colB(ent(k)); };
  const unsigned lds0 = (unsigned)(size_t)(LAS unsigned char*)smem;
  s.ldsw = lds0 + (unsigned)wid * 1024u;
  s.voffA = (unsigned)((8 * wid + (lane >> 3)) * (K_ * 2) + (((lane & 7) ^ (lane >> 3)) << 4));
  {
    const unsigned fB = (unsigned)((lane >> 4) | (((wid >> 1) & 1) << 2));
    s.voffB = (unsigned)((4 * wid + (lane >> 4)) * (N_ * 2)) + ((((unsigned)lane & 15u) ^ (fB << 1)) << 4);
  }
  s.curA = offA(0); s.curBL = offB(0, 1); s.curBR = offB(0, 0);
  STAGE_B(SB(0, 0), s.curBL, 0u); STAGE_A(SA(0, 0), s.curA, 0u); STAGE_B(SB(0, 1), s.curBR, 0u); STAGE_A(SA(0, 1), s.curA, hA);
  STAGE_B(SB(1, 0), s.curBL, kstepB); STAGE_A(SA(1, 0), s.curA, kstepA); STAGE_B(SB(1, 1), s.curBR, kstepB);
  unsigned callv;
  asm volatile("buffer_load_dword %0, off, %1, %2 sc1" : "=v"(callv) : "s"(s.srdWs), "s"(WS_CALL + blk * 64u) : "memory");
  unsigned olo[5], ohi[5];
#pragma unroll
  for (int k = 0; k < 5; ++k) {
    const unsigned long long o = (unsigned long long)p.out[(rank + (ent(k) >> 3)) & 7u];
    olo[k] = (unsigned)o; ohi[k] = (unsigned)(o >> 32) & 0xffffu;
  }
  {
    const int q = (lane & 15) >> 2, pp = lane & 3, g = lane >> 4;
    const int f = q | ((g & 1) << 2);
    s.aoff0 = lds0 + (unsigned)((wr * 64 + fr) * 128 + (((0 + fq) ^ (fr & 7)) << 4));
    s.aoff1 = lds0 + (unsigned)((wr * 64 + fr) * 128 + (((4 + fq) ^ (fr & 7)) << 4));
    s.boff0 = lds0 + 65536u + (unsigned)((8 * g + q) * 256 + (((2 * wc + 0) ^ f) << 5) + 8 * pp);
    s.boff1 = lds0 + 65536u + (unsigned)((8 * g + q) * 256 + (((2 * wc + 1) ^ f) << 5) + 8 * pp);
  }
  s.vlane16 = (unsigned)lane * 16u;
  {
    const unsigned one = (fq == (fr >> 2)) ? (0x3f80u << (16 * (fr & 1))) : 0u;
    const unsigned a = (fr & 2) ? 0u : one, b = (fr & 2) ? one : 0u;
    s.S1 = (u32x4){a, b, 0u, 0u}; s.S2 = (u32x4){0u, 0u, a, b};
    asm volatile("" : "+v"(s.S1), "+v"(s.S2));
  }
  {
    const unsigned r = (unsigned)lane >> 3, c = (unsigned)lane & 7u, cc = c & 3u;
    const unsigned sfq = ((cc & 1u) << 1) | (cc >> 1);
    s.vgat = (c >> 2) * 4096u + (sfq * 16u + r) * 16u;
    s.vdst = r * 2048u + c * 16u;
  }
  const unsigned uw = (unsigned)wid, rwr = uw >> 2, rwcp = uw & 1u, b1 = (uw >> 1) & 1u;
  const unsigned ai4 = mi >> 1, m4 = 2u * (mi & 1u) + b1, ai8 = mem >> 2, m8 = mem & 3u;
  unsigned rd4 = (grp * 8u + sub * 4u) * SLOT + ((ai4 * 8u + rwr * 4u + 2u * rwcp) * 4u + m4) * 1024u;
  unsigned rd8 = (grp * 8u) * SLOT + ((ai8 * 8u + rwr * 4u + 2u * rwcp) * 4u + m8) * 1024u + b1 * 128u;
  const unsigned row0 = rank * 1024u + pm * 256u;
  unsigned ds4 = (row0 + ai4 * 128u + rwr * 64u + m4 * 16u) * 2048u + rwcp * 128u;
  unsigned ds8 = (row0 + ai8 * 128u + rwr * 64u + m8 * 16u + b1 * 8u) * 2048u + rwcp * 128u;
  unsigned pw0 = WS_PART + blk * SLOT + uw * 4096u;
  s.wslot = (int)(mem * 8u + uw);
  unsigned tsl;
  {
    const unsigned key = grp * 3u + mem + uw;
    tsl = (((2u + key % 8u)) << 4) | (((key % 6u) | ((6u + key % 6u) << 4)) << 8) | (((key % 4u) | ((4u + key % 4u) << 4)) << 16);
  }
  s.ts0 = -1; s.ts1 = -1; s.poison = 0u;
  s.d0lo = s.d0hi = s.d0off = s.d1lo = s.d1hi = s.d1off = 0u;
  asm volatile("" : "+s"(s.d0lo), "+s"(s.d0hi), "+s"(s.d0off), "+s"(s.d1lo), "+s"(s.d1hi), "+s"(s.d1off), "+s"(s.ts0), "+s"(s.ts1));
  s.R0 = (u32x4){0u, 0u, 0u, 0u}; s.R1 = s.R0;
#pragma unroll
  for (int a = 0; a < 2; ++a)
#pragma unroll
    for (int bb = 0; bb < 2; ++bb)
#pragma unroll
      for (int m = 0; m < 4; ++m)
#pragma unroll
        for (int n = 0; n < 2; ++n) s.acc[a][bb][m][n] = (f32x4){0.f, 0.f, 0.f, 0.f};
  s.nxtA = offA(1); s.nxtBL = offB(1, 1); s.nxtBR = offB(1, 2);
  unsigned bl2 = offB(2, 3), br3 = offB(3, 4);
  unsigned do0 = ds8 + (ent(0) & 7u) * 256u, do1 = ds4 + (ent(1) & 7u) * 256u, do2 = ds4 + (ent(2) & 7u) * 256u;
  unsigned pf0 = WS_PFLAG + grp * 1024u;
  asm volatile("" : "+s"(bl2), "+s"(br3), "+s"(do0), "+s"(do1), "+s"(do2), "+s"(pf0));
  asm volatile("s_waitcnt lgkmcnt(0)" : "+s"(rd4), "+s"(rd8), "+s"(ds4), "+s"(ds8), "+s"(pw0), "+s"(tsl), "+s"(s.wslot), "+s"(s.nxtA), "+s"(s.nxtBL), "+s"(s.nxtBR) :: "memory");
  if (wr == 1) BAR;
  wait_v<7>(); BAR;
  BAR;
  STAMP(1);
  {
    const unsigned sa = s.curA, sl = s.curBL, sr = s.curBR;
    ktile<0, V_PLAIN>(s);
    asm volatile("" : "+v"(callv));
    s.call = __builtin_amdgcn_readfirstlane(callv) + 1u;
    ktile<1, V_PLAIN>(s);
    for (int t = 2; t < 14; t += 2) {
      s.curA += 2u * kstepA; s.curBL += 2u * kstepB; s.curBR += 2u * kstepB;
      ktile<0, V_PLAIN>(s); ktile<1, V_PLAIN>(s);
    }
    s.curA = sa; s.curBL = sl; s.curBR = sr;
  }
  const unsigned call = s.call;
  for (unsigned seg = 1; seg <= 3u; ++seg) {
    const unsigned k = seg - 1u;
    s.pw = pw0 + (k & RMASK) * REGION;
    ktile<14, V_BG>(s);
    if (seg == 2u) {
      ktile<15, V_L>(s);
      s.curA = s.nxtA; s.curBL = s.nxtBL; s.curBR = s.nxtBR;
      ktile<0, V_L>(s);
    }
    unsigned seg2 = seg; asm volatile("" : "+s"(seg2));
    if (seg2 != 2u) {
      ktile<15, V_R>(s);
      s.curA = s.nxtA; s.curBL = s.nxtBL; s.curBR = s.nxtBR;
      ktile<0, V_R>(s);
    }
    STAMP(1 + seg);
    s.keep = (unsigned)((int)(k - 1u) >> 31);
    s.prd0 = WS_PART + (k & RMASK) * REGION + (k == 0u ? rd8 : rd4);
    s.prd1 = s.prd0 + (k == 0u ? 4u * SLOT : 128u);
    s.pflag = pf0 + k * 32768u;
    s.TS0 = S0_FIRST + (int)((tsl >> (8u * k)) & 15u); s.TS1 = S0_FIRST + (int)((tsl >> (8u * k + 4u)) & 15u);
    s.pdlo = k == 0u ? olo[0] : k == 1u ? olo[1] : olo[2];
    s.pdhi = k == 0u ? ohi[0] : k == 1u ? ohi[1] : ohi[2];
    s.pdoff = k == 0u ? do0 : k == 1u ? do1 : do2;
    s.nxtA = k == 2u ? 0xC0000000u : s.curA ^ winA;
    s.nxtBL = k == 0u ? bl2 : k == 1u ? s.curBL ^ winB : 0xC0000000u;
    s.nxtBR = k == 0u ? s.curBR ^ winB : k == 1u ? br3 : 0xC0000000u;
    asm volatile("" : "+s"(s.nxtA), "+s"(s.nxtBL), "+s"(s.nxtBR), "+s"(s.pdoff), "+s"(s.pdlo), "+s"(s.pdhi), "+s"(s.pflag), "+s"(s.prd0), "+s"(s.prd1), "+s"(s.TS0), "+s"(s.TS1), "+s"(s.keep));
    ktile<1, V_BG>(s); ktile<2, V_BG>(s); ktile<3, V_BG>(s); ktile<4, V_BG>(s); ktile<5, V_BG>(s); ktile<6, V_BG>(s); ktile<7, V_BG>(s);
    ktile<8, V_BG>(s); ktile<9, V_BG>(s); ktile<10, V_BG>(s); ktile<11, V_BG>(s); ktile<12, V_BG>(s); ktile<13, V_BG>(s);
  }
  s.pw = pw0 + RFIN * REGION;
  const unsigned fblk = pf0 + 3u * 32768u;
  final_pair(s, fblk);
  STAMP(5);
  GUARD(s.acc[0][1]); pack_quad(&s.tmp[0], s.acc[0][1]); PST(0); PST(1); PST(2); PST(3);
  wait_v<0>(); PFLAG(fblk + 32768u);
  STAMP(6);
  const int lane2 = lane_id();
  if (wr == 0) BAR;
  LAS unsigned* lflag = (LAS unsigned*)(smem + 8 * HTB);
  if (wid == 0) {
    const unsigned bad = spin_flags(wsp, fblk, 16u, call, wsp);
    if (lane2 == 0) lflag[0] = bad;
    asm volatile("s_waitcnt lgkmcnt(0)" ::: "memory");
  }
  BAR;
  STAMP(7);
  {
    s.prd0 = WS_PART + RFIN * REGION + rd4; s.prd1 = s.prd0 + 128u;
    PLD(0); PLD(1); PLD(2); PLD(3); PLD(4); PLD(5); PLD(6); PLD(7);
    if (wid == 0) {
      const unsigned bad = spin_flags(wsp, fblk + 32768u, 16u, call, wsp);
      if (lane2 == 0) lflag[1] = bad;
      asm volatile("s_waitcnt lgkmcnt(0)" ::: "memory");
    }
    SEND(s.R1, s.d1lo, s.d1hi, s.d1off, s.ts1 >= 15);
    s.poison |= __builtin_amdgcn_readfirstlane(*(volatile LAS unsigned*)lflag);
    asm volatile("s_waitcnt vmcnt(5)" : "+v"(s.tmp[0]), "+v"(s.tmp[1]), "+v"(s.tmp[2]), "+v"(s.tmp[3]) :: "memory");
    reduce4<1>(s, 0);
    const u32x4 o0 = reduce_fin<true>(s);
    asm volatile("s_waitcnt vmcnt(1)" : "+v"(s.tmp[4]), "+v"(s.tmp[5]), "+v"(s.tmp[6]), "+v"(s.tmp[7]) :: "memory");
    reduce4<1>(s, 4);
    const u32x4 o1 = reduce_fin<true>(s);
    BAR;
    s.prd0 = WS_PART + (RFIN + 1u) * REGION + rd8; s.prd1 = s.prd0 + 4u * SLOT;
    PLD(0); PLD(1); PLD(2); PLD(3); PLD(4); PLD(5); PLD(6); PLD(7);
    const unsigned doff = ds4 + (ent(3) & 7u) * 256u;
    SEND(o0, olo[3], ohi[3], doff, true);
    SEND(o1, olo[3], ohi[3], doff + 8u * 2048u, true);
    s.poison |= __builtin_amdgcn_readfirstlane(*(volatile LAS unsigned*)(lflag + 1));
    asm volatile("s_waitcnt vmcnt(2)" : "+v"(s.tmp[0]), "+v"(s.tmp[1]), "+v"(s.tmp[2]), "+v"(s.tmp[3]), "+v"(s.tmp[4]), "+v"(s.tmp[5]), "+v"(s.tmp[6]), "+v"(s.tmp[7]) :: "memory");
    reduce4<1>(s, 0); reduce4<0>(s, 4);
    const u32x4 o2 = reduce_fin<true>(s);
    SEND(o2, olo[4], ohi[4], ds8 + (ent(4) & 7u) * 256u, true);
  }
  STAMP(8);
  wait_v<0>();
  STAMP(9);
  BAR;
  if (wid == 0) {
    if (lane2 < 4) {
      const unsigned e = (info >> (6u * (unsigned)lane2)) & 63u;
      const unsigned dp = (rank + (e >> 3)) & 7u;
      unsigned* f = (unsigned*)(p.ws[dp] + WS_DONE + rank * 1024u) + ((e & 7u) * 4u + pm) * 8u;
      if (lane2 == 0) __hip_atomic_store(f + mem, call, __ATOMIC_RELAXED, __HIP_MEMORY_SCOPE_SYSTEM);
      else __hip_atomic_store((unsigned long long*)f + mi, ((unsigned long long)call << 32) | call, __ATOMIC_RELAXED, __HIP_MEMORY_SCOPE_SYSTEM);
    }
    if (lane2 == 0) __hip_atomic_store((unsigned*)(wsp + WS_CALL + blk * 64u), call, __ATOMIC_RELAXED, __HIP_MEMORY_SCOPE_AGENT);
    if (blk == 0) {
      const unsigned long long* f = (const unsigned long long*)(wsp + WS_DONE) + lane2;
      unsigned* tmo = (unsigned*)(wsp + WS_TMO);
      const unsigned long long t0 = __builtin_amdgcn_s_memrealtime();
      for (unsigned n = 1;; ++n) {
        bool ok = true;
#pragma unroll
        for (unsigned j = 1; j < 8; ++j) {
          const unsigned long long v0 = __hip_atomic_load(f + ((rank + j) & 7u) * 128u, __ATOMIC_RELAXED, __HIP_MEMORY_SCOPE_SYSTEM);
          const unsigned long long v1 = __hip_atomic_load(f + ((rank + j) & 7u) * 128u + 64u, __ATOMIC_RELAXED, __HIP_MEMORY_SCOPE_SYSTEM);
          ok = ok && ((unsigned)v0 >= call) && ((unsigned)(v0 >> 32) >= call) && ((unsigned)v1 >= call) && ((unsigned)(v1 >> 32) >= call);
        }
        if (__all(ok)) break;
        if ((n & 15u) == 0u &&
            (__builtin_amdgcn_readfirstlane(__hip_atomic_load(tmo, __ATOMIC_RELAXED, __HIP_MEMORY_SCOPE_AGENT)) == call ||
             __builtin_amdgcn_s_memrealtime() - t0 > DEADLINE)) {
          if (lane2 == 0) {
            __hip_atomic_store(tmo, call, __ATOMIC_RELAXED, __HIP_MEMORY_SCOPE_AGENT);
            __hip_atomic_store((unsigned*)(p.out[rank] + (size_t)rank * 1024u * 2048u), 0x7fc07fc0u, __ATOMIC_RELAXED, __HIP_MEMORY_SCOPE_SYSTEM);
          }
          break;
        }
      }
      __builtin_amdgcn_fence(__ATOMIC_ACQUIRE, "");
    }
  }
}

}

extern "C" void kernel_launch(int rank, int nranks, ncclComm_t comm,
                              hipStream_t stream,
                              void* const* d_in, const int* in_sizes, int n_in,
                              void* d_out, int out_size,
                              const int* in_dtypes, int out_dtype,
                              void* const* d_out_all,
                              void* const* d_ws_all, size_t ws_bytes) {
  using namespace ga2a;
  (void)comm; (void)in_sizes; (void)n_in; (void)d_out; (void)out_size; (void)in_dtypes; (void)out_dtype;
  if (nranks != NR || ws_bytes < (size_t)WS_USED) { fprintf(stderr, "unsupported configuration\n"); return; }
  Params p;
  p.A = (const u16*)d_in[0];
  p.W = (const u16*)d_in[1];
  for (int i = 0; i < NR; ++i) { p.out[i] = (char*)d_out_all[i]; p.ws[i] = (char*)d_ws_all[i]; }
  p.rank = rank;
  hipLaunchKernelGGL(gemm_a2a, dim3(256), dim3(512), 0, stream, p);
  HIP_CHECK(hipGetLastError());
}
```

```cpp
#include <hip/hip_runtime.h>
#include <rccl/rccl.h>
#include <cstdio>

#define HIP_CHECK(expr) do { hipError_t _e = (expr); if (_e != hipSuccess) { \
    fprintf(stderr, "HIP error %s at %s:%d\n", hipGetErrorString(_e), __FILE__, __LINE__); \
    return; } } while (0)

namespace ga2a {

#define LAS __attribute__((address_space(3)))
typedef unsigned short u16;
typedef short bf16x8 __attribute__((ext_vector_type(8)));
typedef short bf16x4 __attribute__((ext_vector_type(4)));
typedef float f32x4 __attribute__((ext_vector_type(4)));
typedef float f32x2 __attribute__((ext_vector_type(2)));
typedef __bf16 bf16x2n __attribute__((ext_vector_type(2)));
typedef unsigned u32x4 __attribute__((ext_vector_type(4)));

constexpr int NR = 8;
constexpr int M_ = 1024, N_ = 8192, K_ = 8192;
constexpr unsigned HTB = 16384;
constexpr unsigned A_BYTES = (unsigned)M_ * K_ * 2u, W_BYTES = (unsigned)K_ * N_ * 2u;
constexpr unsigned OUT_BYTES = 8192u * 1024u * 2u;
constexpr unsigned SLOT = 65536u;
constexpr unsigned REGION = 256u * SLOT;
constexpr unsigned WS_PART = 0u;
constexpr unsigned RMASK = 3u, RFIN = 3u;
constexpr unsigned WS_CTRL = 128u << 20;
constexpr unsigned WS_PFLAG = WS_CTRL + 524288u;
constexpr unsigned WS_CALL = WS_CTRL + 131072u;
constexpr unsigned WS_DONE = WS_CTRL + 163840u;
constexpr unsigned WS_TMO = WS_CTRL + 180224u;
constexpr unsigned WS_STAMP = WS_CTRL + 262144u;
constexpr unsigned WS_USED = WS_CTRL + (1u << 20);
constexpr unsigned long long DEADLINE = 50000000ull;

constexpr unsigned ktA = 128u, ktB = 64u * N_ * 2u;
constexpr unsigned winA = 16u * ktA, winB = 16u * ktB;
constexpr unsigned hA = 128u * K_ * 2u;
constexpr unsigned iA = 64u * K_ * 2u, iB = 32u * N_ * 2u;

#define GA2A_SIG (pm + 0u * seg)
#define GA2A_BLKM 0x780u
constexpr unsigned BLKM = GA2A_BLKM;
struct Params {
  const u16* A;
  const u16* W;
  char* out[NR];
  char* ws[NR];
  int rank;
};

enum { V_PLAIN = 0, V_BG = 1, V_L = 2, V_R = 3, V_FIRST = 4 };
#define GA2A_SCHED 6
constexpr int PSTP[8] = {-1, -1, 1, 1, 2, 2, 3, 3};
constexpr int T_FLAG = 1, T_POLL = 2, POLL_EARLY = 1;
constexpr int PLDP[8] = {13, 14, 15, 17, 18, 19, 21, 22};
constexpr int T_CHECK = POLL_EARLY ? T_POLL : T_POLL + 1;
constexpr int T_TIE0 = PLDP[3] / 4 + 1, T_TIE1 = PLDP[7] / 4 + 1;
constexpr int S0_FIRST = T_TIE0 + 1, S0_N = 16 - S0_FIRST, S1_FIRST = T_TIE1 + 1, S1_N = 20 - S1_FIRST;
constexpr int count_at(const int (&t)[8], int kt) { int n = 0; for (int i = 0; i < 8; ++i) n += (t[i] >= kt * 4 && t[i] < kt * 4 + 4) ? 1 : 0; return n; }
constexpr bool slot_at(int first, int n, int i) { return (i >= first && i < first + n) || (i + 16 >= first && i + 16 < first + n); }

__device__ __forceinline__ unsigned pk2(float lo, float hi) {
  f32x2 v = {lo, hi};
  return __builtin_bit_cast(unsigned, __builtin_convertvector(v, bf16x2n));
}
__device__ __forceinline__ u32x4 pack_pair(const f32x4& v0, const f32x4& v1) {
  unsigned d00 = pk2(v0[0], v0[1]), d01 = pk2(v0[2], v0[3]);
  unsigned d10 = pk2(v1[0], v1[1]), d11 = pk2(v1[2], v1[3]);
  auto r0 = __builtin_amdgcn_permlane16_swap(d00, d10, false, false);
  auto r1 = __builtin_amdgcn_permlane16_swap(d01, d11, false, false);
  u32x4 w; w.x = r0[0]; w.y = r1[0]; w.z = r0[1]; w.w = r1[1];
  return w;
}

struct St {
  f32x4 acc[2][2][4][2];
  bf16x8 At[4][2];
  bf16x4 B0[2][2][2], B1[2][2][2];
  u32x4 tmp[8];
  f32x4 D1, D2;
  u32x4 S1, S2;
  u32x4 R0, R1;
  unsigned pollv;
  unsigned voffA, voffB, aoff0, aoff1, boff0, boff1, vlane16, vgat, vdst;
  u32x4 srdA, srdB, srdWs;
  unsigned ldsw;
  unsigned a1, a2, bl2, br2, kA, sig;
  unsigned sgA, sgBL, sgBR, nxtA, nxtBL, nxtBR, nxtK;
  unsigned pw;
  unsigned prd0, prd1;
  unsigned pflag;
  unsigned keep;
  unsigned call, poison;
  unsigned pdlo, pdhi, pdoff;
  unsigned d0lo, d0hi, d0off, d1lo, d1hi, d1off;
  int ts0, ts1, TS0, wslot;
  char* wsp;
};

#define SA(b, h) (((b) * 2u + (h)) * HTB)
#define SB(b, h) ((4u + (b) * 2u + (h)) * HTB)
#define DMA2(ldsoff, voff, srd, sbase, simm, step) do { unsigned _t; asm volatile( \
    "s_add_u32 m0, %1, %2\n\ts_add_u32 %0, %6, %7\n\tbuffer_load_dwordx4 %4, %5, %0 offen lds\n\t" \
    "s_add_u32 m0, %1, %3\n\ts_add_u32 %0, %6, %8\n\tbuffer_load_dwordx4 %4, %5, %0 offen lds" \
    : "=&s"(_t) : "s"(s.ldsw), "i"(ldsoff), "i"((ldsoff) + 8192u), "v"(voff), "s"(srd), "s"(sbase), "i"(simm), "i"((simm) + (step)) : "memory", "scc"); } while (0)
#define STAGE_A(bufoff, sbase, simm) DMA2((bufoff), s.voffA, s.srdA, (sbase), (simm), iA)
#define STAGE_B(bufoff, sbase, simm) DMA2((bufoff), s.voffB, s.srdB, (sbase), (simm), iB)
__device__ __forceinline__ int lane_id() { int l; asm volatile("v_mbcnt_lo_u32_b32 %0, -1, 0\n\tv_mbcnt_hi_u32_b32 %0, -1, %0" : "=v"(l)); return l; }
#define RD128(dst, addr, off) asm volatile("ds_read_b128 %0, %1 offset:%2" : "=v"(dst) : "v"(addr), "i"(off))
#define RDTR(dst, addr, off) asm volatile("ds_read_b64_tr_b16 %0, %1 offset:%2" : "=v"(dst) : "v"(addr), "i"(off))
#define LDA(b, h) do { _Pragma("unroll") for (int m = 0; m < 4; ++m) { RD128(s.At[m][0], s.aoff0, SA(b, h) + m * 2048); RD128(s.At[m][1], s.aoff1, SA(b, h) + m * 2048); } } while (0)
#define LDB(dst, b, h) do { _Pragma("unroll") for (int k = 0; k < 2; ++k) { \
    RDTR(dst[0][k][0], s.boff0, SB(b, h) - 65536u + k * 8192); RDTR(dst[0][k][1], s.boff0, SB(b, h) - 65536u + k * 8192 + 1024); } \
    _Pragma("unroll") for (int k = 0; k < 2; ++k) { \
    RDTR(dst[1][k][0], s.boff1, SB(b, h) - 65536u + k * 8192); RDTR(dst[1][k][1], s.boff1, SB(b, h) - 65536u + k * 8192 + 1024); } } while (0)
#define WAIT_A() asm volatile("s_waitcnt lgkmcnt(0)" : "+v"(s.At[0][0]), "+v"(s.At[0][1]), "+v"(s.At[1][0]), "+v"(s.At[1][1]), "+v"(s.At[2][0]), "+v"(s.At[2][1]), "+v"(s.At[3][0]), "+v"(s.At[3][1]) :: "memory")
#define WAIT_B(Bt) asm volatile("s_waitcnt lgkmcnt(0)" : "+v"(Bt[0][0][0]), "+v"(Bt[0][0][1]), "+v"(Bt[0][1][0]), "+v"(Bt[0][1][1]), "+v"(Bt[1][0][0]), "+v"(Bt[1][0][1]), "+v"(Bt[1][1][0]), "+v"(Bt[1][1][1]) :: "memory")
#define CAT(x) __builtin_shufflevector(x[0], x[1], 0, 1, 2, 3, 4, 5, 6, 7)
#define WAIT_L(n) asm volatile("s_waitcnt lgkmcnt(" #n ")" ::: "memory")
#define BAR __builtin_amdgcn_s_barrier()
#define SCHED __builtin_amdgcn_sched_barrier(0)
#define ENDM do { SCHED; asm volatile("s_setprio 0"); BAR; SCHED; } while (0)
#define ST128(data, voff, srd, soff, bits) asm volatile("buffer_store_dwordx4 %0, %1, %2, %3 offen " bits "\n\ts_nop 1" :: "v"(data), "v"(voff), "s"(srd), "s"(soff) : "memory")
#define LD128(dst, voff, srd, soff) asm volatile("buffer_load_dwordx4 %0, %1, %2, %3 offen sc1" : "=v"(dst) : "v"(voff), "s"(srd), "s"(soff) : "memory")
#define PST(u) ST128(s.tmp[u], s.vlane16, s.srdWs, s.pw + ((u) >> 2) * 32768u + ((u) & 3) * 1024u, "sc1")
#define PLD(j) LD128(s.tmp[j], s.vgat, s.srdWs, ((j) < 4 ? s.prd0 : s.prd1) + ((j) & 3) * SLOT)

template <int N> __device__ __forceinline__ void wait_v() { asm volatile("s_waitcnt vmcnt(%0)" :: "i"(N) : "memory"); }

template <bool INIT>
__device__ __forceinline__ void mma(f32x4 (&acc)[4][2], const bf16x8 (&At)[4][2], const bf16x4 (&Bt)[2][2][2]) {
  const bf16x8 b00 = CAT(Bt[0][0]), b01 = CAT(Bt[0][1]), b10 = CAT(Bt[1][0]), b11 = CAT(Bt[1][1]);
  if constexpr (INIT)
    asm volatile("s_setprio 1\n\t"
      "v_mfma_f32_16x16x32_bf16 %0, %16, %8, 0\n\t"
      "v_mfma_f32_16x16x32_bf16 %0, %17, %9, %0\n\t"
      "v_mfma_f32_16x16x32_bf16 %1, %18, %8, 0\n\t"
      "v_mfma_f32_16x16x32_bf16 %1, %19, %9, %1\n\t"
      "v_mfma_f32_16x16x32_bf16 %2, %16, %10, 0\n\t"
      "v_mfma_f32_16x16x32_bf16 %2, %17, %11, %2\n\t"
      "v_mfma_f32_16x16x32_bf16 %3, %18, %10, 0\n\t"
      "v_mfma_f32_16x16x32_bf16 %3, %19, %11, %3\n\t"
      "v_mfma_f32_16x16x32_bf16 %4, %16, %12, 0\n\t"
      "v_mfma_f32_16x16x32_bf16 %4, %17, %13, %4\n\t"
      "v_mfma_f32_16x16x32_bf16 %5, %18, %12, 0\n\t"
      "v_mfma_f32_16x16x32_bf16 %5, %19, %13, %5\n\t"
      "v_mfma_f32_16x16x32_bf16 %6, %16, %14, 0\n\t"
      "v_mfma_f32_16x16x32_bf16 %6, %17, %15, %6\n\t"
      "v_mfma_f32_16x16x32_bf16 %7, %18, %14, 0\n\t"
      "v_mfma_f32_16x16x32_bf16 %7, %19, %15, %7\n\t"
      : "+v"(acc[0][0]), "+v"(acc[0][1]), "+v"(acc[1][0]), "+v"(acc[1][1]), "+v"(acc[2][0]), "+v"(acc[2][1]), "+v"(acc[3][0]), "+v"(acc[3][1])
      : "v"(At[0][0]), "v"(At[0][1]), "v"(At[1][0]), "v"(At[1][1]), "v"(At[2][0]), "v"(At[2][1]), "v"(At[3][0]), "v"(At[3][1]), "v"(b00), "v"(b01), "v"(b10), "v"(b11));
  else
    asm volatile("s_setprio 1\n\t"
      "v_mfma_f32_16x16x32_bf16 %0, %16, %8, %0\n\t"
      "v_mfma_f32_16x16x32_bf16 %0, %17, %9, %0\n\t"
      "v_mfma_f32_16x16x32_bf16 %1, %18, %8, %1\n\t"
      "v_mfma_f32_16x16x32_bf16 %1, %19, %9, %1\n\t"
      "v_mfma_f32_16x16x32_bf16 %2, %16, %10, %2\n\t"
      "v_mfma_f32_16x16x32_bf16 %2, %17, %11, %2\n\t"
      "v_mfma_f32_16x16x32_bf16 %3, %18, %10, %3\n\t"
      "v_mfma_f32_16x16x32_bf16 %3, %19, %11, %3\n\t"
      "v_mfma_f32_16x16x32_bf16 %4, %16, %12, %4\n\t"
      "v_mfma_f32_16x16x32_bf16 %4, %17, %13, %4\n\t"
      "v_mfma_f32_16x16x32_bf16 %5, %18, %12, %5\n\t"
      "v_mfma_f32_16x16x32_bf16 %5, %19, %13, %5\n\t"
      "v_mfma_f32_16x16x32_bf16 %6, %16, %14, %6\n\t"
      "v_mfma_f32_16x16x32_bf16 %6, %17, %15, %6\n\t"
      "v_mfma_f32_16x16x32_bf16 %7, %18, %14, %7\n\t"
      "v_mfma_f32_16x16x32_bf16 %7, %19, %15, %7\n\t"
      : "+v"(acc[0][0]), "+v"(acc[0][1]), "+v"(acc[1][0]), "+v"(acc[1][1]), "+v"(acc[2][0]), "+v"(acc[2][1]), "+v"(acc[3][0]), "+v"(acc[3][1])
      : "v"(At[0][0]), "v"(At[0][1]), "v"(At[1][0]), "v"(At[1][1]), "v"(At[2][0]), "v"(At[2][1]), "v"(At[3][0]), "v"(At[3][1]), "v"(b00), "v"(b01), "v"(b10), "v"(b11));
}
template <bool INIT>
__device__ __forceinline__ void mma2(f32x4& c, const bf16x8& b0, const bf16x8& a0, const bf16x8& b1, const bf16x8& a1) {
  if constexpr (INIT) asm volatile("v_mfma_f32_16x16x32_bf16 %0, %1, %2, 0\n\tv_mfma_f32_16x16x32_bf16 %0, %3, %4, %0" : "+v"(c) : "v"(b0), "v"(a0), "v"(b1), "v"(a1));
  else asm volatile("v_mfma_f32_16x16x32_bf16 %0, %1, %2, %0\n\tv_mfma_f32_16x16x32_bf16 %0, %3, %4, %0" : "+v"(c) : "v"(b0), "v"(a0), "v"(b1), "v"(a1));
}
template <bool INIT>
__device__ __forceinline__ void mma_step(St& s, f32x4 (&acc)[4][2], const bf16x8 (&At)[4][2], const bf16x4 (&Bt)[2][2][2]) {
  const bf16x8 b00 = CAT(Bt[0][0]), b01 = CAT(Bt[0][1]), b10 = CAT(Bt[1][0]), b11 = CAT(Bt[1][1]);
  unsigned t;
  asm volatile("s_setprio 1");
  mma2<INIT>(acc[0][0], b00, At[0][0], b01, At[0][1]);
  unsigned k, k2;
  asm volatile("s_add_u32 %0, %0, 0x80\n\ts_add_u32 %1, %0, %2" : "+s"(s.kA), "=&s"(k) : "s"(s.sig) : "scc");
  mma2<INIT>(acc[0][1], b10, At[0][0], b11, At[0][1]);
  asm volatile("s_and_b32 %0, %0, %3\n\ts_and_b32 %1, %2, %4" : "+s"(k), "=&s"(k2) : "s"(s.kA), "i"(BLKM), "i"(0x780u & ~BLKM) : "scc");
  mma2<INIT>(acc[1][0], b00, At[1][0], b01, At[1][1]);
  asm volatile("s_or_b32 %0, %0, %2\n\ts_mov_b32 %1, %3" : "+s"(k), "=&s"(s.a1) : "s"(k2), "s"(s.a2) : "scc");
  mma2<INIT>(acc[1][1], b10, At[1][0], b11, At[1][1]);
  asm volatile("s_add_u32 %0, %2, %3\n\ts_lshl_b32 %1, %3, 13" : "=&s"(s.a2), "=&s"(t) : "s"(s.sgA), "s"(k) : "scc");
  mma2<INIT>(acc[2][0], b00, At[2][0], b01, At[2][1]);
  asm volatile("s_add_u32 %0, %2, %4\n\ts_add_u32 %1, %3, %4" : "=&s"(s.bl2), "=&s"(s.br2) : "s"(s.sgBL), "s"(s.sgBR), "s"(t) : "scc");
  mma2<INIT>(acc[2][1], b10, At[2][0], b11, At[2][1]);
  mma2<INIT>(acc[3][0], b00, At[3][0], b01, At[3][1]);
  mma2<INIT>(acc[3][1], b10, At[3][0], b11, At[3][1]);
}
__device__ __forceinline__ void pack_quad(u32x4* dst, const f32x4 (&q)[4][2]) {
#pragma unroll
  for (int m = 0; m < 4; ++m) { dst[m] = pack_pair(q[m][0], q[m][1]); asm volatile("" : "+v"(dst[m])); }
}
template <int FIRST>
__device__ __forceinline__ void reduce4(St& s, int j) {
  if constexpr (FIRST == 1)
    asm volatile("v_mfma_f32_16x16x32_bf16 %0, %2, %4, 0\n\tv_mfma_f32_16x16x32_bf16 %1, %3, %4, 0\n\t"
                 "v_mfma_f32_16x16x32_bf16 %0, %2, %5, %0\n\tv_mfma_f32_16x16x32_bf16 %1, %3, %5, %1\n\t"
                 "v_mfma_f32_16x16x32_bf16 %0, %2, %6, %0\n\tv_mfma_f32_16x16x32_bf16 %1, %3, %6, %1\n\t"
                 "v_mfma_f32_16x16x32_bf16 %0, %2, %7, %0\n\tv_mfma_f32_16x16x32_bf16 %1, %3, %7, %1"
                 : "=&v"(s.D1), "=&v"(s.D2) : "v"(s.S1), "v"(s.S2), "v"(s.tmp[j]), "v"(s.tmp[j + 1]), "v"(s.tmp[j + 2]), "v"(s.tmp[j + 3]));
  else if constexpr (FIRST == 2)
    asm volatile("v_mfma_f32_16x16x32_bf16 %0, %2, %4, 0\n\tv_mfma_f32_16x16x32_bf16 %1, %3, %4, 0\n\t"
                 "v_mfma_f32_16x16x32_bf16 %0, %2, %5, %0\n\tv_mfma_f32_16x16x32_bf16 %1, %3, %5, %1\n\t"
                 "v_mfma_f32_16x16x32_bf16 %0, %2, %6, %0\n\tv_mfma_f32_16x16x32_bf16 %1, %3, %6, %1\n\t"
                 "v_mfma_f32_16x16x32_bf16 %0, %2, %7, %0\n\tv_mfma_f32_16x16x32_bf16 %1, %3, %7, %1"
                 : "+v"(s.D1), "+v"(s.D2) : "v"(s.S1), "v"(s.S2), "v"(s.tmp[j]), "v"(s.tmp[j + 1]), "v"(s.tmp[j + 2]), "v"(s.tmp[j + 3]));
  else
    asm volatile("v_mfma_f32_16x16x32_bf16 %0, %2, %4, %0\n\tv_mfma_f32_16x16x32_bf16 %1, %3, %4, %1\n\t"
                 "v_mfma_f32_16x16x32_bf16 %0, %2, %5, %0\n\tv_mfma_f32_16x16x32_bf16 %1, %3, %5, %1\n\t"
                 "v_mfma_f32_16x16x32_bf16 %0, %2, %6, %0\n\tv_mfma_f32_16x16x32_bf16 %1, %3, %6, %1\n\t"
                 "v_mfma_f32_16x16x32_bf16 %0, %2, %7, %0\n\tv_mfma_f32_16x16x32_bf16 %1, %3, %7, %1"
                 : "+v"(s.D1), "+v"(s.D2) : "v"(s.S1), "v"(s.S2), "v"(s.tmp[j]), "v"(s.tmp[j + 1]), "v"(s.tmp[j + 2]), "v"(s.tmp[j + 3]));
}
template <bool NOP>
__device__ __forceinline__ u32x4 reduce_fin(St& s) {
  if constexpr (NOP) asm volatile("s_nop 7" : "+v"(s.D1), "+v"(s.D2)); else asm volatile("" : "+v"(s.D1), "+v"(s.D2));
  u32x4 r;
  r.x = pk2(s.D1[0], s.D1[1]) | s.poison; r.y = pk2(s.D1[2], s.D1[3]) | s.poison;
  r.z = pk2(s.D2[0], s.D2[1]) | s.poison; r.w = pk2(s.D2[2], s.D2[3]) | s.poison;
  asm volatile("" : "+v"(r));
  return r;
}
__device__ __forceinline__ unsigned spin_flags(const char* base, unsigned blk, unsigned stride, unsigned call, const char* wsp) {
  const int lane = lane_id();
  const unsigned* f = (const unsigned*)(base + blk + (unsigned)lane * stride);
  unsigned* tmo = (unsigned*)(wsp + WS_TMO);
  const unsigned long long t0 = __builtin_amdgcn_s_memrealtime();
  unsigned bad = 0;
  for (unsigned n = 1;; ++n) {
    const unsigned v = __hip_atomic_load(f, __ATOMIC_RELAXED, __HIP_MEMORY_SCOPE_SYSTEM);
    if (__all(v >= call)) break;
    if ((n & 15u) == 0u &&
        (__builtin_amdgcn_readfirstlane(__hip_atomic_load(tmo, __ATOMIC_RELAXED, __HIP_MEMORY_SCOPE_AGENT)) == call ||
         __builtin_amdgcn_s_memrealtime() - t0 > DEADLINE)) {
      if (lane == 0) __hip_atomic_store(tmo, call, __ATOMIC_RELAXED, __HIP_MEMORY_SCOPE_AGENT);
      bad = 0x7fc07fc0u;
      break;
    }
  }
  asm volatile("s_waitcnt vmcnt(0) lgkmcnt(0)" ::: "memory");
  return bad;
}
#define SENDL(R, lo, hi, off) do { const u32x4 _srd = {lo, hi, OUT_BYTES, 0x20000u}; ST128(R, s.vdst, _srd, off, "sc0 sc1"); } while (0)
#define SEND(R, lo, hi, off, go) do { const u32x4 _srd = {lo, hi, (go) ? OUT_BYTES : 0u, 0x20000u}; ST128(R, s.vdst, _srd, off, "sc0 sc1"); } while (0)

#define PFLAG(blk) do { unsigned _c; asm volatile("v_mov_b32 %0, %1\n\ts_mov_b64 exec, 1\n\tbuffer_store_dword %0, off, %2, %3 sc1\n\ts_mov_b64 exec, -1" : "=&v"(_c) : "s"(s.call), "s"(s.srdWs), "s"((blk) + (unsigned)s.wslot * 16u) : "memory"); } while (0)
#define GUARD(q) asm volatile("s_nop 7" : "+v"(q[0][0]), "+v"(q[0][1]), "+v"(q[1][0]), "+v"(q[1][1]), "+v"(q[2][0]), "+v"(q[2][1]), "+v"(q[3][0]), "+v"(q[3][1]))

template <int I, int V>
__device__ __forceinline__ void ktile(St& s) {
  constexpr unsigned b = I & 1, nb = b ^ 1;
  constexpr bool BG = (V != V_PLAIN && V != V_FIRST);
  constexpr bool E15 = (I == 15 && (V == V_L || V == V_R)), E0 = (I == 0 && (V == V_L || V == V_R));
  constexpr int hv = (V == V_R) ? 1 : 0;
  constexpr bool INL = (E0 && hv == 0) || V == V_FIRST, INR = (E0 && hv == 1) || V == V_FIRST;
  constexpr bool S0 = BG && slot_at(S0_FIRST, S0_N, I), S1 = BG && slot_at(S1_FIRST, S1_N, I);
  constexpr int KT = E15 ? -1 : (BG && I != 15 && I != 14) ? I : -100;
  constexpr int NX = count_at(PSTP, KT) + count_at(PLDP, KT);
#define BGMEM(ph) do { _Pragma("unroll") for (int u = 0; u < 8; ++u) { if (PSTP[u] == KT * 4 + (ph)) PST(u); if (PLDP[u] == KT * 4 + (ph)) PLD(u); } } while (0)
  LDB(s.B0, b, 0); LDA(b, 0);
  STAGE_A(SA(nb, 1), s.a1, hA);
  WAIT_L(8); BAR; WAIT_B(s.B0); WAIT_A(); mma<INL>(s.acc[0][0], s.At, s.B0);
  if constexpr (E0) pack_quad(&s.tmp[4], s.acc[1][hv]);
  if constexpr (KT == T_TIE0 + 1) reduce4<1>(s, 0);
  if constexpr (KT == T_TIE1 + 1 && T_TIE1 != T_TIE0) { if (s.keep != 0u) reduce4<0>(s, 4); unsigned k2 = s.keep; asm volatile("" : "+s"(k2)); if (k2 == 0u) reduce4<2>(s, 4); }
  ENDM;
  LDB(s.B1, b, 1);
  if constexpr (KT == T_POLL && POLL_EARLY) asm volatile("buffer_load_dword %0, %1, %2, %3 offen sc1" : "=v"(s.pollv) : "v"(s.vlane16), "s"(s.srdWs), "s"(s.pflag) : "memory");
  STAGE_B(SB(b, 0), s.bl2, 0u);
  BGMEM(1);
  BAR; WAIT_B(s.B1); mma<INR>(s.acc[0][1], s.At, s.B1);
  if constexpr (KT == T_TIE0 + 1) { s.R0 = reduce_fin<false>(s); s.d0lo = s.pdlo; s.d0hi = s.pdhi; s.d0off = s.pdoff; s.ts0 = (4 + (s.TS0 & 15)) | (int)s.keep; asm volatile("" : "+s"(s.d0lo), "+s"(s.d0hi), "+s"(s.d0off), "+s"(s.ts0)); }
  if constexpr (KT == T_TIE1 + 1) { s.R1 = reduce_fin<false>(s); s.d1lo = s.pdlo; s.d1hi = s.pdhi; s.d1off = s.pdoff + (~s.keep & (8u * 2048u)); s.ts1 = 4 + ((s.TS0 >> 4) & 15); asm volatile("" : "+s"(s.d1lo), "+s"(s.d1hi), "+s"(s.d1off), "+s"(s.ts1)); }
  ENDM;
  LDA(b, 1);
  STAGE_A(SA(b, 0), s.a2, 0u);
  BGMEM(2);
  if constexpr (S0) { if (s.ts0 == I) SENDL(s.R0, s.d0lo, s.d0hi, s.d0off); }
  if constexpr (S1) { if (s.ts1 == (I < S1_FIRST ? I + 16 : I)) SENDL(s.R1, s.d1lo, s.d1hi, s.d1off); }
  BAR; WAIT_A(); mma<INL>(s.acc[1][0], s.At, s.B0);
  if constexpr (E15) pack_quad(&s.tmp[0], s.acc[0][hv]);
  ENDM;
  STAGE_B(SB(b, 1), s.br2, 0u);
  BGMEM(3);
  if constexpr (KT == T_TIE0) asm volatile("s_waitcnt vmcnt(%4)" : "+v"(s.tmp[0]), "+v"(s.tmp[1]), "+v"(s.tmp[2]), "+v"(s.tmp[3]) : "i"(6 + NX) : "memory");
  else if constexpr (KT == T_TIE1) asm volatile("s_waitcnt vmcnt(%4)" : "+v"(s.tmp[4]), "+v"(s.tmp[5]), "+v"(s.tmp[6]), "+v"(s.tmp[7]) : "i"(6 + NX) : "memory");
  else if constexpr (KT == T_CHECK) asm volatile("s_waitcnt vmcnt(%1)" : "+v"(s.pollv) : "i"(6 + NX) : "memory");
  else wait_v<6 + NX>();
  if constexpr (KT == T_FLAG) PFLAG(s.pflag);
  if constexpr (KT == T_POLL && !POLL_EARLY) asm volatile("buffer_load_dword %0, %1, %2, %3 offen sc1" : "=v"(s.pollv) : "v"(s.vlane16), "s"(s.srdWs), "s"(s.pflag) : "memory");
  if constexpr (I == 13) { s.sgA = s.nxtA; s.sgBL = s.nxtBL; s.sgBR = s.nxtBR; s.kA = 0u - ktA; s.sig = s.nxtK; }
  BAR; mma_step<INR>(s, s.acc[1][1], s.At, s.B1);
  ENDM;
#undef BGMEM
  if constexpr (KT == T_CHECK) {
    if (!__all(s.pollv >= s.call || (s.vlane16 >> 7) == ((unsigned)s.wslot >> 3))) s.poison |= spin_flags(s.wsp, s.pflag, 16u, s.call, s.wsp);
    SCHED;
  }
}

__device__ __forceinline__ void final_pair(St& s, unsigned flagL) {
  LDB(s.B0, 0u, 0); LDA(0u, 0); STAGE_A(SA(1u, 1), s.a1, hA);
  SEND(s.R1, s.d1lo, s.d1hi, s.d1off, s.ts1 == 14);
  BAR; WAIT_B(s.B0); WAIT_A(); mma<false>(s.acc[0][0], s.At, s.B0); ENDM;
  LDA(0u, 1);
  BAR; WAIT_A(); mma<false>(s.acc[1][0], s.At, s.B0); ENDM;
  LDB(s.B1, 0u, 1); wait_v<3>();
  BAR; WAIT_B(s.B1); mma<false>(s.acc[1][1], s.At, s.B1); ENDM;
  LDB(s.B0, 1u, 0); LDA(1u, 0); wait_v<1>();
  BAR; WAIT_B(s.B0); WAIT_A(); mma<false>(s.acc[0][0], s.At, s.B0); ENDM;
  LDA(1u, 1); GUARD(s.acc[0][0]); pack_quad(&s.tmp[0], s.acc[0][0]); PST(0); PST(1); PST(2); PST(3);
  BAR; WAIT_A(); mma<false>(s.acc[1][0], s.At, s.B0); ENDM;
  LDB(s.B1, 1u, 1); GUARD(s.acc[1][0]); pack_quad(&s.tmp[4], s.acc[1][0]); PST(4); PST(5); PST(6); PST(7);
  BAR; WAIT_B(s.B1); mma<false>(s.acc[1][1], s.At, s.B1); ENDM;
  LDA(1u, 0);
  BAR; WAIT_A(); mma<false>(s.acc[0][1], s.At, s.B1); ENDM;
  LDB(s.B1, 0u, 1); LDA(0u, 0); wait_v<0>(); PFLAG(flagL);
  s.pw += REGION;
  pack_quad(&s.tmp[4], s.acc[1][1]); PST(4); PST(5); PST(6); PST(7);
  BAR; WAIT_B(s.B1); WAIT_A(); mma<false>(s.acc[0][1], s.At, s.B1); ENDM;
}

#define STAMP(i) do {} while (0)

__global__ __launch_bounds__(512, 2) void gemm_a2a(Params p) {
  __shared__ __attribute__((aligned(1024))) unsigned char smem[8 * HTB + 64];
  const int tid = threadIdx.x, wid = __builtin_amdgcn_readfirstlane(tid >> 6), lane = tid & 63;
  const int wr = wid >> 2, wc = wid & 3, fr = lane & 15, fq = lane >> 4;
  const unsigned blk = blockIdx.x, mem = blk & 7u, grp = (blk >> 3) & 31u;
  const unsigned pm = grp & 3u, cg = grp >> 2, sub = mem >> 2, mi = mem & 3u;
  const unsigned rank = (unsigned)p.rank & 7u;
  STAMP(0);
  St s;
  char* wsp = p.ws[rank];
  s.wsp = wsp;
  const unsigned long long pa = (unsigned long long)p.A, pw = (unsigned long long)p.W, pws = (unsigned long long)wsp;
  s.srdA = (u32x4){(unsigned)pa, (unsigned)(pa >> 32) & 0xffffu, A_BYTES, 0x20000u};
  s.srdB = (u32x4){(unsigned)pw, (unsigned)(pw >> 32) & 0xffffu, W_BYTES, 0x20000u};
  s.srdWs = (u32x4){(unsigned)pws, (unsigned)(pws >> 32) & 0xffffu, WS_USED, 0x20000u};
  unsigned info;
  if (cg < 7u) {
    const unsigned j = (cg + 1u) << 3;
    info = (j | 0u) | ((j | (1u + sub)) << 6) | ((j | (3u + sub)) << 12) | ((j | (5u + sub)) << 18) | (cg << 24);
  } else {
    const unsigned j0 = (0x6427u >> (4u * pm)) & 7u;
    const unsigned j1 = ((4u + 2u * pm + sub) % 7u) + 1u, j2 = ((2u + 2u * pm + sub) % 7u) + 1u, j3 = ((2u * pm + sub) % 7u) + 1u;
    info = ((j0 << 3) | 7u) | (((j1 << 3) | 7u) << 6) | (((j2 << 3) | 7u) << 12) | (((j3 << 3) | 7u) << 18) | (7u << 24);
  }
#define GA2A_ROT 1
  {
    unsigned inf2 = 0;
#pragma unroll
    for (unsigned k = 0; k < 5u; ++k) {
      const unsigned e = (info >> (6u * k)) & 63u;
      inf2 |= ((e & 56u) | ((e + (e >> 3) * GA2A_ROT + 2u * rank) & 7u)) << (6u * k);
    }
    info = inf2;
  }
  auto ent = [&](unsigned k) -> unsigned { return (info >> (6u * k)) & 63u; };
  auto colB = [&](unsigned e) -> unsigned { return (((rank + (e >> 3)) & 7u) * 8u + (e & 7u)) * 256u; };
  auto kwin = [&](unsigned seg) -> unsigned { return 2u * mi + ((seg + sub) & 1u); };
  auto offA = [&](unsigned seg) -> unsigned { return pm * (256u * K_ * 2u) + kwin(seg) * winA; };
  auto offB = [&](unsigned seg, unsigned k) -> unsigned { return kwin(seg) * winB + colB(ent(k)); };
  auto rot = [&](unsigned seg) -> unsigned { return GA2A_SIG; };
  const unsigned lds0 = (unsigned)(size_t)(LAS unsigned char*)smem;
  s.ldsw = lds0 + (unsigned)wid * 1024u;
  s.voffA = (unsigned)((8 * wid + (lane >> 3)) * (K_ * 2) + (((lane & 7) ^ (lane >> 3)) << 4));
  {
    const unsigned fB = (unsigned)((lane >> 4) | (((wid >> 1) & 1) << 2));
    s.voffB = (unsigned)((4 * wid + (lane >> 4)) * (N_ * 2)) + ((((unsigned)lane & 15u) ^ (fB << 1)) << 4);
  }
  s.sgA = offA(0); s.sgBL = offB(0, 1); s.sgBR = offB(0, 0);
  {
    s.sig = rot(0) * ktA;
    auto kk = [&](unsigned t) -> unsigned { return ((t * ktA + s.sig) & BLKM) | (t * ktA & ~BLKM); };
    const unsigned k0 = kk(0), k1 = kk(1), k2 = kk(2);
    STAGE_B(SB(0, 0), s.sgBL + (k0 << 13), 0u); STAGE_A(SA(0, 0), s.sgA + k0, 0u); STAGE_B(SB(0, 1), s.sgBR + (k0 << 13), 0u); STAGE_A(SA(0, 1), s.sgA + k0, hA);
    STAGE_B(SB(1, 0), s.sgBL + (k1 << 13), 0u); STAGE_A(SA(1, 0), s.sgA + k1, 0u); STAGE_B(SB(1, 1), s.sgBR + (k1 << 13), 0u);
    s.a1 = s.sgA + k1; s.a2 = s.sgA + k2; s.bl2 = s.sgBL + (k2 << 13); s.br2 = s.sgBR + (k2 << 13); s.kA = 2u * ktA;
  }
  unsigned callv;
  asm volatile("buffer_load_dword %0, off, %1, %2 sc1" : "=v"(callv) : "s"(s.srdWs), "s"(WS_CALL + blk * 64u) : "memory");
  unsigned olo[3], ohi[3];
#pragma unroll
  for (int k = 0; k < 3; ++k) {
    const unsigned long long o = (unsigned long long)p.out[(rank + (ent(k) >> 3)) & 7u];
    olo[k] = (unsigned)o; ohi[k] = (unsigned)(o >> 32) & 0xffffu;
  }
  {
    const int q = (lane & 15) >> 2, pp = lane & 3, g = lane >> 4;
    const int f = q | ((g & 1) << 2);
    s.aoff0 = lds0 + (unsigned)((wr * 64 + fr) * 128 + (((0 + fq) ^ (fr & 7)) << 4));
    s.aoff1 = lds0 + (unsigned)((wr * 64 + fr) * 128 + (((4 + fq) ^ (fr & 7)) << 4));
    s.boff0 = lds0 + 65536u + (unsigned)((8 * g + q) * 256 + (((2 * wc + 0) ^ f) << 5) + 8 * pp);
    s.boff1 = lds0 + 65536u + (unsigned)((8 * g + q) * 256 + (((2 * wc + 1) ^ f) << 5) + 8 * pp);
  }
  s.vlane16 = (unsigned)lane * 16u;
  {
    const unsigned one = (fq == (fr >> 2)) ? (0x3f80u << (16 * (fr & 1))) : 0u;
    const unsigned a = (fr & 2) ? 0u : one, b = (fr & 2) ? one : 0u;
    s.S1 = (u32x4){a, b, 0u, 0u}; s.S2 = (u32x4){0u, 0u, a, b};
    asm volatile("" : "+v"(s.S1), "+v"(s.S2));
  }
  {
    const unsigned r = (unsigned)lane >> 3, c = (unsigned)lane & 7u, cc = c & 3u;
    const unsigned sfq = ((cc & 1u) << 1) | (cc >> 1);
    s.vgat = (c >> 2) * 4096u + (sfq * 16u + r) * 16u;
    s.vdst = r * 2048u + c * 16u;
  }
  const unsigned uw = (unsigned)wid, rwr = uw >> 2, rwcp = uw & 1u, b1 = (uw >> 1) & 1u;
  const unsigned ai4 = mi >> 1, m4 = 2u * (mi & 1u) + b1, ai8 = mem >> 2, m8 = mem & 3u;
  unsigned rd4 = (grp * 8u + sub * 4u) * SLOT + ((ai4 * 8u + rwr * 4u + 2u * rwcp) * 4u + m4) * 1024u;
  unsigned rd8 = (grp * 8u) * SLOT + ((ai8 * 8u + rwr * 4u + 2u * rwcp) * 4u + m8) * 1024u + b1 * 128u;
  const unsigned row0 = rank * 1024u + pm * 256u;
  unsigned ds4 = (row0 + ai4 * 128u + rwr * 64u + m4 * 16u) * 2048u + rwcp * 128u;
  unsigned ds8 = (row0 + ai8 * 128u + rwr * 64u + m8 * 16u + b1 * 8u) * 2048u + rwcp * 128u;
  unsigned pw0 = WS_PART + blk * SLOT + uw * 4096u;
  s.wslot = (int)(mem * 8u + uw);
  unsigned tsl;
  {
    const unsigned key = grp * 3u + mem + uw;
    constexpr unsigned F0 = S0_FIRST - 4, F1 = S1_FIRST - 4;
    constexpr unsigned W2 = (20 - S0_FIRST) / 2;
    constexpr unsigned G1 = (S0_FIRST + 4 > S1_FIRST ? S0_FIRST + 4 : S1_FIRST) - 4;
    constexpr unsigned G2 = (F0 + W2 > F1 ? F0 + W2 : F1);
    tsl = ((F1 + key % (16u - F1)) << 4) | (((F0 + key % W2) | ((G2 + key % (16u - G2)) << 4)) << 8) | (((F0 + key % 4u) | ((G1 + key % 4u > 15u ? 15u : G1 + key % 4u) << 4)) << 16);
  }
  s.ts0 = -1; s.ts1 = -1; s.poison = 0u;
  s.d0lo = s.d0hi = s.d0off = s.d1lo = s.d1hi = s.d1off = 0u;
  asm volatile("" : "+s"(s.d0lo), "+s"(s.d0hi), "+s"(s.d0off), "+s"(s.d1lo), "+s"(s.d1hi), "+s"(s.d1off), "+s"(s.ts0), "+s"(s.ts1));
  s.R0 = (u32x4){0u, 0u, 0u, 0u}; s.R1 = s.R0;
#pragma unroll
  for (int a = 0; a < 2; ++a)
#pragma unroll
    for (int bb = 0; bb < 2; ++bb)
#pragma unroll
      for (int m = 0; m < 4; ++m)
#pragma unroll
        for (int n = 0; n < 2; ++n) asm volatile("" : "=v"(s.acc[a][bb][m][n]));
  s.nxtA = offA(1); s.nxtBL = offB(1, 1); s.nxtBR = offB(1, 2); s.nxtK = rot(1) * ktA;
  unsigned rots = rot(2) | (rot(3) << 4);
  unsigned bl2 = offB(2, 3), br3 = offB(3, 4);
  unsigned do0 = ds8 + (ent(0) & 7u) * 256u, do1 = ds4 + (ent(1) & 7u) * 256u, do2 = ds4 + (ent(2) & 7u) * 256u;
  unsigned pf0 = WS_PFLAG + grp * 1024u;
  asm volatile("" : "+s"(bl2), "+s"(br3), "+s"(do0), "+s"(do1), "+s"(do2), "+s"(pf0));
  asm volatile("s_waitcnt lgkmcnt(0)" : "+s"(rd4), "+s"(rd8), "+s"(ds4), "+s"(ds8), "+s"(pw0), "+s"(tsl), "+s"(s.wslot), "+s"(s.nxtA), "+s"(s.nxtBL), "+s"(s.nxtBR), "+s"(s.nxtK), "+s"(rots) :: "memory");
  if (wr == 1) BAR;
  wait_v<7>(); BAR;
  BAR;
  STAMP(1);
  {
    ktile<0, V_FIRST>(s);
    asm volatile("" : "+v"(callv));
    s.call = __builtin_amdgcn_readfirstlane(callv) + 1u;
    ktile<1, V_PLAIN>(s);
    for (int t = 2; t < 12; t += 2) { ktile<0, V_PLAIN>(s); ktile<1, V_PLAIN>(s); }
    ktile<12, V_PLAIN>(s); ktile<13, V_PLAIN>(s);
  }
  const unsigned call = s.call;
  for (unsigned seg = 1; seg <= 3u; ++seg) {
    const unsigned k = seg - 1u;
    s.pw = pw0 + (k & RMASK) * REGION;
    ktile<14, V_BG>(s);
    if (seg == 2u) {
      ktile<15, V_L>(s);
      ktile<0, V_L>(s);
    }
    unsigned seg2 = seg; asm volatile("" : "+s"(seg2));
    if (seg2 != 2u) {
      ktile<15, V_R>(s);
      ktile<0, V_R>(s);
    }
    STAMP(1 + seg);
    s.keep = (unsigned)((int)(k - 1u) >> 31);
    s.prd0 = WS_PART + (k & RMASK) * REGION + (k == 0u ? rd8 : rd4);
    s.prd1 = s.prd0 + (k == 0u ? 4u * SLOT : 128u);
    s.pflag = pf0 + k * 32768u;
    s.TS0 = (int)(tsl >> (8u * k));
    s.pdlo = k == 0u ? olo[0] : k == 1u ? olo[1] : olo[2];
    s.pdhi = k == 0u ? ohi[0] : k == 1u ? ohi[1] : ohi[2];
    s.pdoff = k == 0u ? do0 : k == 1u ? do1 : do2;
    s.nxtA = k == 2u ? 0xC0000000u : s.sgA ^ winA;
    s.nxtBL = k == 0u ? bl2 : k == 1u ? s.sgBL ^ winB : 0xC0000000u;
    s.nxtBR = k == 0u ? s.sgBR ^ winB : k == 1u ? br3 : 0xC0000000u;
    s.nxtK = ((rots >> (4u * k)) & 15u) * ktA;
    asm volatile("" : "+s"(s.nxtK), "+s"(s.nxtA), "+s"(s.nxtBL), "+s"(s.nxtBR), "+s"(s.pdoff), "+s"(s.pdlo), "+s"(s.pdhi), "+s"(s.pflag), "+s"(s.prd0), "+s"(s.prd1), "+s"(s.TS0), "+s"(s.keep));
    ktile<1, V_BG>(s); ktile<2, V_BG>(s); ktile<3, V_BG>(s); ktile<4, V_BG>(s); ktile<5, V_BG>(s); ktile<6, V_BG>(s); ktile<7, V_BG>(s);
    ktile<8, V_BG>(s); ktile<9, V_BG>(s); ktile<10, V_BG>(s); ktile<11, V_BG>(s); ktile<12, V_BG>(s); ktile<13, V_BG>(s);
  }
  s.pw = pw0 + RFIN * REGION;
  const unsigned fblk = pf0 + 3u * 32768u;
  final_pair(s, fblk);
  STAMP(5);
  GUARD(s.acc[0][1]); pack_quad(&s.tmp[0], s.acc[0][1]); PST(0); PST(1); PST(2); PST(3);
  wait_v<0>(); PFLAG(fblk + 32768u);
  STAMP(6);
  const int lane2 = lane_id();
  const unsigned long long o3 = (unsigned long long)p.out[(rank + (ent(3) >> 3)) & 7u], o4 = (unsigned long long)p.out[rank];
  const unsigned o3lo = (unsigned)o3, o3hi = (unsigned)(o3 >> 32) & 0xffffu, o4lo = (unsigned)o4, o4hi = (unsigned)(o4 >> 32) & 0xffffu;
  if (wr == 0) BAR;
  LAS unsigned* lflag = (LAS unsigned*)(smem + 8 * HTB);
  if (wid == 0) {
    const unsigned bad = spin_flags(wsp, fblk, 16u, call, wsp);
    if (lane2 == 0) lflag[0] = bad;
    asm volatile("s_waitcnt lgkmcnt(0)" ::: "memory");
  }
  BAR;
  STAMP(7);
  {
    s.prd0 = WS_PART + RFIN * REGION + rd4; s.prd1 = s.prd0 + 128u;
    PLD(0); PLD(1); PLD(2); PLD(3); PLD(4); PLD(5); PLD(6); PLD(7);
    if (wid == 0) {
      const unsigned bad = spin_flags(wsp, fblk + 32768u, 16u, call, wsp);
      if (lane2 == 0) lflag[1] = bad;
      asm volatile("s_waitcnt lgkmcnt(0)" ::: "memory");
    }
    SEND(s.R1, s.d1lo, s.d1hi, s.d1off, s.ts1 >= 15);
    s.poison |= __builtin_amdgcn_readfirstlane(*(volatile LAS unsigned*)lflag);
    asm volatile("s_waitcnt vmcnt(5)" : "+v"(s.tmp[0]), "+v"(s.tmp[1]), "+v"(s.tmp[2]), "+v"(s.tmp[3]) :: "memory");
    reduce4<1>(s, 0);
    const u32x4 o0 = reduce_fin<true>(s);
    asm volatile("s_waitcnt vmcnt(1)" : "+v"(s.tmp[4]), "+v"(s.tmp[5]), "+v"(s.tmp[6]), "+v"(s.tmp[7]) :: "memory");
    reduce4<1>(s, 4);
    const u32x4 o1 = reduce_fin<true>(s);
    BAR;
    s.prd0 = WS_PART + (RFIN + 1u) * REGION + rd8; s.prd1 = s.prd0 + 4u * SLOT;
    PLD(0); PLD(1); PLD(2); PLD(3); PLD(4); PLD(5); PLD(6); PLD(7);
    const unsigned doff = ds4 + (ent(3) & 7u) * 256u;
    SEND(o0, o3lo, o3hi, doff, true);
    SEND(o1, o3lo, o3hi, doff + 8u * 2048u, true);
    s.poison |= __builtin_amdgcn_readfirstlane(*(volatile LAS unsigned*)(lflag + 1));
    asm volatile("s_waitcnt vmcnt(2)" : "+v"(s.tmp[0]), "+v"(s.tmp[1]), "+v"(s.tmp[2]), "+v"(s.tmp[3]), "+v"(s.tmp[4]), "+v"(s.tmp[5]), "+v"(s.tmp[6]), "+v"(s.tmp[7]) :: "memory");
    reduce4<1>(s, 0); reduce4<0>(s, 4);
    const u32x4 o2 = reduce_fin<true>(s);
    SEND(o2, o4lo, o4hi, ds8 + (ent(4) & 7u) * 256u, true);
  }
  STAMP(8);
  wait_v<0>();
  STAMP(9);
  BAR;
  if (wid == 0) {
    if (lane2 < 4) {
      const unsigned e = (info >> (6u * (unsigned)lane2)) & 63u;
      const unsigned dp = (rank + (e >> 3)) & 7u;
      unsigned* f = (unsigned*)(p.ws[dp] + WS_DONE + rank * 1024u) + ((e & 7u) * 4u + pm) * 8u;
      if (lane2 == 0) __hip_atomic_store(f + mem, call, __ATOMIC_RELAXED, __HIP_MEMORY_SCOPE_SYSTEM);
      else __hip_atomic_store((unsigned long long*)f + mi, ((unsigned long long)call << 32) | call, __ATOMIC_RELAXED, __HIP_MEMORY_SCOPE_SYSTEM);
    }
    if (lane2 == 0) __hip_atomic_store((unsigned*)(wsp + WS_CALL + blk * 64u), call, __ATOMIC_RELAXED, __HIP_MEMORY_SCOPE_AGENT);
    if (blk == 0) {
      const unsigned long long* f = (const unsigned long long*)(wsp + WS_DONE) + lane2;
      unsigned* tmo = (unsigned*)(wsp + WS_TMO);
      const unsigned long long t0 = __builtin_amdgcn_s_memrealtime();
      for (unsigned n = 1;; ++n) {
        bool ok = true;
#pragma unroll
        for (unsigned j = 1; j < 8; ++j) {
          const unsigned long long v0 = __hip_atomic_load(f + ((rank + j) & 7u) * 128u, __ATOMIC_RELAXED, __HIP_MEMORY_SCOPE_SYSTEM);
          const unsigned long long v1 = __hip_atomic_load(f + ((rank + j) & 7u) * 128u + 64u, __ATOMIC_RELAXED, __HIP_MEMORY_SCOPE_SYSTEM);
          ok = ok && ((unsigned)v0 >= call) && ((unsigned)(v0 >> 32) >= call) && ((unsigned)v1 >= call) && ((unsigned)(v1 >> 32) >= call);
        }
        if (__all(ok)) break;
        if ((n & 15u) == 0u &&
            (__builtin_amdgcn_readfirstlane(__hip_atomic_load(tmo, __ATOMIC_RELAXED, __HIP_MEMORY_SCOPE_AGENT)) == call ||
             __builtin_amdgcn_s_memrealtime() - t0 > DEADLINE)) {
          if (lane2 == 0) {
            __hip_atomic_store(tmo, call, __ATOMIC_RELAXED, __HIP_MEMORY_SCOPE_AGENT);
            __hip_atomic_store((unsigned*)(p.out[rank] + (size_t)rank * 1024u * 2048u), 0x7fc07fc0u, __ATOMIC_RELAXED, __HIP_MEMORY_SCOPE_SYSTEM);
          }
          break;
        }
      }
      __builtin_amdgcn_fence(__ATOMIC_ACQUIRE, "");
    }
  }
}

}

extern "C" void kernel_launch(int rank, int nranks, ncclComm_t comm,
                              hipStream_t stream,
                              void* const* d_in, const int* in_sizes, int n_in,
                              void* d_out, int out_size,
                              const int* in_dtypes, int out_dtype,
                              void* const* d_out_all,
                              void* const* d_ws_all, size_t ws_bytes) {
  using namespace ga2a;
  (void)comm; (void)in_sizes; (void)n_in; (void)d_out; (void)out_size; (void)in_dtypes; (void)out_dtype;
  if (nranks != NR || ws_bytes < (size_t)WS_USED) { fprintf(stderr, "unsupported configuration\n"); return; }
  Params p;
  p.A = (const u16*)d_in[0];
  p.W = (const u16*)d_in[1];
  for (int i = 0; i < NR; ++i) { p.out[i] = (char*)d_out_all[i]; p.ws[i] = (char*)d_ws_all[i]; }
  p.rank = rank;
  hipLaunchKernelGGL(gemm_a2a, dim3(256), dim3(512), 0, stream, p);
  HIP_CHECK(hipGetLastError());
}
```

```cpp
#include <hip/hip_runtime.h>
#include <rccl/rccl.h>
#include <cstdio>

#define HIP_CHECK(expr) do { hipError_t _e = (expr); if (_e != hipSuccess) { \
    fprintf(stderr, "HIP error %s at %s:%d\n", hipGetErrorString(_e), __FILE__, __LINE__); \
    return; } } while (0)

namespace ga2a {

#define LAS __attribute__((address_space(3)))
typedef unsigned short u16;
typedef short bf16x8 __attribute__((ext_vector_type(8)));
typedef short bf16x4 __attribute__((ext_vector_type(4)));
typedef float f32x4 __attribute__((ext_vector_type(4)));
typedef float f32x2 __attribute__((ext_vector_type(2)));
typedef __bf16 bf16x2n __attribute__((ext_vector_type(2)));
typedef unsigned u32x4 __attribute__((ext_vector_type(4)));

constexpr int NR = 8;
constexpr int M_ = 1024, N_ = 8192, K_ = 8192;
constexpr unsigned HTB = 16384;
constexpr unsigned A_BYTES = (unsigned)M_ * K_ * 2u, W_BYTES = (unsigned)K_ * N_ * 2u;
constexpr unsigned OUT_BYTES = 8192u * 1024u * 2u;
constexpr unsigned SLOT = 65536u;
constexpr unsigned REGION = 256u * SLOT;
constexpr unsigned WS_PART = 0u;
constexpr unsigned WS_CTRL = 128u << 20;
constexpr unsigned WS_PFLAG = WS_CTRL + 524288u;
constexpr unsigned WS_CALL = WS_CTRL + 131072u;
constexpr unsigned WS_DONE = WS_CTRL + 163840u;
constexpr unsigned WS_TMO = WS_CTRL + 180224u;
constexpr unsigned WS_STAMP = WS_CTRL + 262144u;
constexpr unsigned WS_USED = WS_CTRL + (1u << 20);
constexpr unsigned long long DEADLINE = 50000000ull;

constexpr unsigned ktA = 128u, ktB = 64u * N_ * 2u;
constexpr unsigned hA = 128u * K_ * 2u;
constexpr unsigned iA = 64u * K_ * 2u, iB = 32u * N_ * 2u;

struct Params {
  const u16* A;
  const u16* W;
  char* out[NR];
  char* ws[NR];
  int rank;
};

enum { V_PLAIN = 0, V_BG = 1, V_L = 2, V_R = 3, V_FIRST = 4 };
constexpr int PSTP[8] = {-1, -1, 1, 1, 2, 2, 3, 3};
constexpr int T_FLAG = 1, T_POLL = 2;
constexpr int PLDP[8] = {13, 14, 15, 17, 18, 19, 21, 22};
constexpr int T_TIE0 = PLDP[3] / 4 + 1, T_TIE1 = PLDP[7] / 4 + 1;
constexpr int S0_FIRST = T_TIE0 + 1, S0_N = 16 - S0_FIRST, S1_FIRST = T_TIE1 + 1, S1_N = 20 - S1_FIRST;
constexpr int count_at(const int (&t)[8], int kt) { int n = 0; for (int i = 0; i < 8; ++i) n += (t[i] >= kt * 4 && t[i] < kt * 4 + 4) ? 1 : 0; return n; }
constexpr bool slot_at(int first, int n, int i) { return (i >= first && i < first + n) || (i + 16 >= first && i + 16 < first + n); }

__device__ __forceinline__ unsigned pk2(float lo, float hi) {
  f32x2 v = {lo, hi};
  return __builtin_bit_cast(unsigned, __builtin_convertvector(v, bf16x2n));
}
__device__ __forceinline__ u32x4 pack_pair(const f32x4& v0, const f32x4& v1) {
  unsigned d00 = pk2(v0[0], v0[1]), d01 = pk2(v0[2], v0[3]);
  unsigned d10 = pk2(v1[0], v1[1]), d11 = pk2(v1[2], v1[3]);
  auto r0 = __builtin_amdgcn_permlane16_swap(d00, d10, false, false);
  auto r1 = __builtin_amdgcn_permlane16_swap(d01, d11, false, false);
  u32x4 w; w.x = r0[0]; w.y = r1[0]; w.z = r0[1]; w.w = r1[1];
  return w;
}

struct St {
  f32x4 acc[2][2][4][2];
  bf16x8 At[4][2];
  bf16x4 B0[2][2][2], B1[2][2][2];
  u32x4 tmp[8];
  f32x4 D1, D2;
  u32x4 S1, S2;
  u32x4 R0, R1;
  unsigned pollv;
  unsigned voffA, voffB, aoff0, aoff1, boff0, boff1, vlane16, vgat, vdst;
  u32x4 srdA, srdB, srdWs;
  unsigned ldsw;
  unsigned a1, a2, bl2, br2, kA;
  unsigned sgA, sgBL, sgBR, nxtBL, nxtBR;
  unsigned pw;
  unsigned prd0, prd1;
  unsigned own, lrd;
  unsigned vls;
  unsigned pflag;
  unsigned keep;
  unsigned call, poison;
  unsigned pdlo, pdhi, pdoff;
  unsigned d0lo, d0hi, d0off, d1lo, d1hi, d1off;
  int ts0, ts1, TS0, wslot;
  char* wsp;
};

#define SA(b, h) (((b) * 2u + (h)) * HTB)
#define SB(b, h) ((4u + (b) * 2u + (h)) * HTB)
#define DMA2(ldsoff, voff, srd, sbase, simm, step) do { unsigned _t; asm volatile( \
    "s_add_u32 m0, %1, %2\n\ts_add_u32 %0, %6, %7\n\tbuffer_load_dwordx4 %4, %5, %0 offen lds\n\t" \
    "s_add_u32 m0, %1, %3\n\ts_add_u32 %0, %6, %8\n\tbuffer_load_dwordx4 %4, %5, %0 offen lds" \
    : "=&s"(_t) : "s"(s.ldsw), "i"(ldsoff), "i"((ldsoff) + 8192u), "v"(voff), "s"(srd), "s"(sbase), "i"(simm), "i"((simm) + (step)) : "memory", "scc"); } while (0)
#define STAGE_A(bufoff, sbase, simm) DMA2((bufoff), s.voffA, s.srdA, (sbase), (simm), iA)
#define STAGE_B(bufoff, sbase, simm) DMA2((bufoff), s.voffB, s.srdB, (sbase), (simm), iB)
__device__ __forceinline__ int lane_id() { int l; asm volatile("v_mbcnt_lo_u32_b32 %0, -1, 0\n\tv_mbcnt_hi_u32_b32 %0, -1, %0" : "=v"(l)); return l; }
#define RD128(dst, addr, off) asm volatile("ds_read_b128 %0, %1 offset:%2" : "=v"(dst) : "v"(addr), "i"(off))
#define RDTR(dst, addr, off) asm volatile("ds_read_b64_tr_b16 %0, %1 offset:%2" : "=v"(dst) : "v"(addr), "i"(off))
#define LDA(b, h) do { _Pragma("unroll") for (int m = 0; m < 4; ++m) { RD128(s.At[m][0], s.aoff0, SA(b, h) + m * 2048); RD128(s.At[m][1], s.aoff1, SA(b, h) + m * 2048); } } while (0)
#define LDB(dst, b, h) do { _Pragma("unroll") for (int k = 0; k < 2; ++k) { \
    RDTR(dst[0][k][0], s.boff0, SB(b, h) - 65536u + k * 8192); RDTR(dst[0][k][1], s.boff0, SB(b, h) - 65536u + k * 8192 + 1024); } \
    _Pragma("unroll") for (int k = 0; k < 2; ++k) { \
    RDTR(dst[1][k][0], s.boff1, SB(b, h) - 65536u + k * 8192); RDTR(dst[1][k][1], s.boff1, SB(b, h) - 65536u + k * 8192 + 1024); } } while (0)
#define WAIT_A() asm volatile("s_waitcnt lgkmcnt(0)" : "+v"(s.At[0][0]), "+v"(s.At[0][1]), "+v"(s.At[1][0]), "+v"(s.At[1][1]), "+v"(s.At[2][0]), "+v"(s.At[2][1]), "+v"(s.At[3][0]), "+v"(s.At[3][1]) :: "memory")
#define WAIT_B(Bt) asm volatile("s_waitcnt lgkmcnt(0)" : "+v"(Bt[0][0][0]), "+v"(Bt[0][0][1]), "+v"(Bt[0][1][0]), "+v"(Bt[0][1][1]), "+v"(Bt[1][0][0]), "+v"(Bt[1][0][1]), "+v"(Bt[1][1][0]), "+v"(Bt[1][1][1]) :: "memory")
#define CAT(x) __builtin_shufflevector(x[0], x[1], 0, 1, 2, 3, 4, 5, 6, 7)
#define WAIT_L(n) asm volatile("s_waitcnt lgkmcnt(" #n ")" ::: "memory")
#define BAR __builtin_amdgcn_s_barrier()
#define SCHED __builtin_amdgcn_sched_barrier(0)
#define ENDM do { SCHED; asm volatile("s_setprio 0"); BAR; SCHED; } while (0)
#define ST128(data, voff, srd, soff, bits) asm volatile("buffer_store_dwordx4 %0, %1, %2, %3 offen " bits "\n\ts_nop 1" :: "v"(data), "v"(voff), "s"(srd), "s"(soff) : "memory")
#define LD128(dst, voff, srd, soff) asm volatile("buffer_load_dwordx4 %0, %1, %2, %3 offen sc1" : "=v"(dst) : "v"(voff), "s"(srd), "s"(soff) : "memory")
#define PST(u) do { unsigned _t; asm volatile( \
    "s_bitcmp1_b32 %1, %2\n\ts_cselect_b32 %0, %3, %4\n\ts_cbranch_scc0 1f\n\tds_write_b128 %5, %6 offset:%7\n1:\n\t" \
    "s_add_u32 %0, %0, %8\n\tbuffer_store_dwordx4 %6, %9, %10, %0 offen sc1\n\ts_nop 1" \
    : "=&s"(_t) : "s"(s.own), "i"(u), "s"(0xC0000000u), "s"(s.pw), "v"(s.vls), "v"(s.tmp[u]), "i"(((u) & 3) * 1024), "i"(((u) >> 2) * 32768u + ((u) & 3) * 1024u), "v"(s.vlane16), "s"(s.srdWs) : "memory", "scc"); } while (0)
#define PLDG(j) LD128(s.tmp[j], s.vgat, s.srdWs, ((j) < 4 ? s.prd0 : s.prd1) ^ (((j) & 3) * SLOT))
#define PLDL(j, add) do { unsigned _a; asm volatile("v_add_u32 %1, %2, %3\n\tds_read_b128 %0, %1 offset:%4" : "=v"(s.tmp[j]), "=&v"(_a) : "s"(s.lrd), "v"(s.vgat), "i"(add) : "memory"); } while (0)
#define PLD(j) do { if ((j) == 0) PLDL(0, 0); else if ((j) == 4) { if (s.keep != 0u) PLDG(4); unsigned _k = s.keep; asm volatile("" : "+s"(_k)); if (_k == 0u) PLDL(4, 128); } else PLDG(j); } while (0)

template <int N> __device__ __forceinline__ void wait_v() { asm volatile("s_waitcnt vmcnt(%0)" :: "i"(N) : "memory"); }

template <bool INIT>
__device__ __forceinline__ void mma(f32x4 (&acc)[4][2], const bf16x8 (&At)[4][2], const bf16x4 (&Bt)[2][2][2]) {
  const bf16x8 b00 = CAT(Bt[0][0]), b01 = CAT(Bt[0][1]), b10 = CAT(Bt[1][0]), b11 = CAT(Bt[1][1]);
  if constexpr (INIT)
    asm volatile("s_setprio 1\n\t"
      "v_mfma_f32_16x16x32_bf16 %0, %16, %8, 0\n\t"
      "v_mfma_f32_16x16x32_bf16 %0, %17, %9, %0\n\t"
      "v_mfma_f32_16x16x32_bf16 %1, %18, %8, 0\n\t"
      "v_mfma_f32_16x16x32_bf16 %1, %19, %9, %1\n\t"
      "v_mfma_f32_16x16x32_bf16 %2, %16, %10, 0\n\t"
      "v_mfma_f32_16x16x32_bf16 %2, %17, %11, %2\n\t"
      "v_mfma_f32_16x16x32_bf16 %3, %18, %10, 0\n\t"
      "v_mfma_f32_16x16x32_bf16 %3, %19, %11, %3\n\t"
      "v_mfma_f32_16x16x32_bf16 %4, %16, %12, 0\n\t"
      "v_mfma_f32_16x16x32_bf16 %4, %17, %13, %4\n\t"
      "v_mfma_f32_16x16x32_bf16 %5, %18, %12, 0\n\t"
      "v_mfma_f32_16x16x32_bf16 %5, %19, %13, %5\n\t"
      "v_mfma_f32_16x16x32_bf16 %6, %16, %14, 0\n\t"
      "v_mfma_f32_16x16x32_bf16 %6, %17, %15, %6\n\t"
      "v_mfma_f32_16x16x32_bf16 %7, %18, %14, 0\n\t"
      "v_mfma_f32_16x16x32_bf16 %7, %19, %15, %7\n\t"
      : "+v"(acc[0][0]), "+v"(acc[0][1]), "+v"(acc[1][0]), "+v"(acc[1][1]), "+v"(acc[2][0]), "+v"(acc[2][1]), "+v"(acc[3][0]), "+v"(acc[3][1])
      : "v"(At[0][0]), "v"(At[0][1]), "v"(At[1][0]), "v"(At[1][1]), "v"(At[2][0]), "v"(At[2][1]), "v"(At[3][0]), "v"(At[3][1]), "v"(b00), "v"(b01), "v"(b10), "v"(b11));
  else
    asm volatile("s_setprio 1\n\t"
      "v_mfma_f32_16x16x32_bf16 %0, %16, %8, %0\n\t"
      "v_mfma_f32_16x16x32_bf16 %0, %17, %9, %0\n\t"
      "v_mfma_f32_16x16x32_bf16 %1, %18, %8, %1\n\t"
      "v_mfma_f32_16x16x32_bf16 %1, %19, %9, %1\n\t"
      "v_mfma_f32_16x16x32_bf16 %2, %16, %10, %2\n\t"
      "v_mfma_f32_16x16x32_bf16 %2, %17, %11, %2\n\t"
      "v_mfma_f32_16x16x32_bf16 %3, %18, %10, %3\n\t"
      "v_mfma_f32_16x16x32_bf16 %3, %19, %11, %3\n\t"
      "v_mfma_f32_16x16x32_bf16 %4, %16, %12, %4\n\t"
      "v_mfma_f32_16x16x32_bf16 %4, %17, %13, %4\n\t"
      "v_mfma_f32_16x16x32_bf16 %5, %18, %12, %5\n\t"
      "v_mfma_f32_16x16x32_bf16 %5, %19, %13, %5\n\t"
      "v_mfma_f32_16x16x32_bf16 %6, %16, %14, %6\n\t"
      "v_mfma_f32_16x16x32_bf16 %6, %17, %15, %6\n\t"
      "v_mfma_f32_16x16x32_bf16 %7, %18, %14, %7\n\t"
      "v_mfma_f32_16x16x32_bf16 %7, %19, %15, %7\n\t"
      : "+v"(acc[0][0]), "+v"(acc[0][1]), "+v"(acc[1][0]), "+v"(acc[1][1]), "+v"(acc[2][0]), "+v"(acc[2][1]), "+v"(acc[3][0]), "+v"(acc[3][1])
      : "v"(At[0][0]), "v"(At[0][1]), "v"(At[1][0]), "v"(At[1][1]), "v"(At[2][0]), "v"(At[2][1]), "v"(At[3][0]), "v"(At[3][1]), "v"(b00), "v"(b01), "v"(b10), "v"(b11));
}
template <bool INIT>
__device__ __forceinline__ void mma2(f32x4& c, const bf16x8& b0, const bf16x8& a0, const bf16x8& b1, const bf16x8& a1) {
  if constexpr (INIT) asm volatile("v_mfma_f32_16x16x32_bf16 %0, %1, %2, 0\n\tv_mfma_f32_16x16x32_bf16 %0, %3, %4, %0" : "+v"(c) : "v"(b0), "v"(a0), "v"(b1), "v"(a1));
  else asm volatile("v_mfma_f32_16x16x32_bf16 %0, %1, %2, %0\n\tv_mfma_f32_16x16x32_bf16 %0, %3, %4, %0" : "+v"(c) : "v"(b0), "v"(a0), "v"(b1), "v"(a1));
}
template <bool INIT>
__device__ __forceinline__ void mma_step(St& s, f32x4 (&acc)[4][2], const bf16x8 (&At)[4][2], const bf16x4 (&Bt)[2][2][2]) {
  const bf16x8 b00 = CAT(Bt[0][0]), b01 = CAT(Bt[0][1]), b10 = CAT(Bt[1][0]), b11 = CAT(Bt[1][1]);
  unsigned t;
  asm volatile("s_setprio 1");
  mma2<INIT>(acc[0][0], b00, At[0][0], b01, At[0][1]);
  asm volatile("s_add_u32 %0, %0, 0x80\n\ts_and_b32 %0, %0, 0xf80" : "+s"(s.kA) :: "scc");
  mma2<INIT>(acc[0][1], b10, At[0][0], b11, At[0][1]);
  asm volatile("s_mov_b32 %0, %1\n\ts_add_u32 %1, %2, %3" : "=&s"(s.a1), "+s"(s.a2) : "s"(s.sgA), "s"(s.kA) : "scc");
  mma2<INIT>(acc[1][0], b00, At[1][0], b01, At[1][1]);
  asm volatile("s_lshl_b32 %0, %1, 13" : "=s"(t) : "s"(s.kA) : "scc");
  mma2<INIT>(acc[1][1], b10, At[1][0], b11, At[1][1]);
  asm volatile("s_add_u32 %0, %2, %4\n\ts_add_u32 %1, %3, %4" : "=&s"(s.bl2), "=&s"(s.br2) : "s"(s.sgBL), "s"(s.sgBR), "s"(t) : "scc");
  mma2<INIT>(acc[2][0], b00, At[2][0], b01, At[2][1]);
  mma2<INIT>(acc[2][1], b10, At[2][0], b11, At[2][1]);
  mma2<INIT>(acc[3][0], b00, At[3][0], b01, At[3][1]);
  mma2<INIT>(acc[3][1], b10, At[3][0], b11, At[3][1]);
}
__device__ __forceinline__ void pack_quad(u32x4* dst, f32x4 (&q)[4][2]) {
  asm volatile("" : "+v"(q[0][0]), "+v"(q[0][1]), "+v"(q[1][0]), "+v"(q[1][1]), "+v"(q[2][0]), "+v"(q[2][1]), "+v"(q[3][0]), "+v"(q[3][1]));
#pragma unroll
  for (int m = 0; m < 4; ++m) { dst[m] = pack_pair(q[m][0], q[m][1]); asm volatile("" : "+v"(dst[m])); }
}
template <int FIRST>
__device__ __forceinline__ void reduce4(St& s, int j) {
  if constexpr (FIRST == 1)
    asm volatile("v_mfma_f32_16x16x32_bf16 %0, %2, %4, 0\n\tv_mfma_f32_16x16x32_bf16 %1, %3, %4, 0\n\t"
                 "v_mfma_f32_16x16x32_bf16 %0, %2, %5, %0\n\tv_mfma_f32_16x16x32_bf16 %1, %3, %5, %1\n\t"
                 "v_mfma_f32_16x16x32_bf16 %0, %2, %6, %0\n\tv_mfma_f32_16x16x32_bf16 %1, %3, %6, %1\n\t"
                 "v_mfma_f32_16x16x32_bf16 %0, %2, %7, %0\n\tv_mfma_f32_16x16x32_bf16 %1, %3, %7, %1"
                 : "=&v"(s.D1), "=&v"(s.D2) : "v"(s.S1), "v"(s.S2), "v"(s.tmp[j]), "v"(s.tmp[j + 1]), "v"(s.tmp[j + 2]), "v"(s.tmp[j + 3]));
  else if constexpr (FIRST == 2)
    asm volatile("v_mfma_f32_16x16x32_bf16 %0, %2, %4, 0\n\tv_mfma_f32_16x16x32_bf16 %1, %3, %4, 0\n\t"
                 "v_mfma_f32_16x16x32_bf16 %0, %2, %5, %0\n\tv_mfma_f32_16x16x32_bf16 %1, %3, %5, %1\n\t"
                 "v_mfma_f32_16x16x32_bf16 %0, %2, %6, %0\n\tv_mfma_f32_16x16x32_bf16 %1, %3, %6, %1\n\t"
                 "v_mfma_f32_16x16x32_bf16 %0, %2, %7, %0\n\tv_mfma_f32_16x16x32_bf16 %1, %3, %7, %1"
                 : "+v"(s.D1), "+v"(s.D2) : "v"(s.S1), "v"(s.S2), "v"(s.tmp[j]), "v"(s.tmp[j + 1]), "v"(s.tmp[j + 2]), "v"(s.tmp[j + 3]));
  else
    asm volatile("v_mfma_f32_16x16x32_bf16 %0, %2, %4, %0\n\tv_mfma_f32_16x16x32_bf16 %1, %3, %4, %1\n\t"
                 "v_mfma_f32_16x16x32_bf16 %0, %2, %5, %0\n\tv_mfma_f32_16x16x32_bf16 %1, %3, %5, %1\n\t"
                 "v_mfma_f32_16x16x32_bf16 %0, %2, %6, %0\n\tv_mfma_f32_16x16x32_bf16 %1, %3, %6, %1\n\t"
                 "v_mfma_f32_16x16x32_bf16 %0, %2, %7, %0\n\tv_mfma_f32_16x16x32_bf16 %1, %3, %7, %1"
                 : "+v"(s.D1), "+v"(s.D2) : "v"(s.S1), "v"(s.S2), "v"(s.tmp[j]), "v"(s.tmp[j + 1]), "v"(s.tmp[j + 2]), "v"(s.tmp[j + 3]));
}
template <bool NOP>
__device__ __forceinline__ u32x4 reduce_fin(St& s) {
  if constexpr (NOP) asm volatile("s_nop 7" : "+v"(s.D1), "+v"(s.D2)); else asm volatile("" : "+v"(s.D1), "+v"(s.D2));
  u32x4 r;
  r.x = pk2(s.D1[0], s.D1[1]) | s.poison; r.y = pk2(s.D1[2], s.D1[3]) | s.poison;
  r.z = pk2(s.D2[0], s.D2[1]) | s.poison; r.w = pk2(s.D2[2], s.D2[3]) | s.poison;
  asm volatile("" : "+v"(r));
  return r;
}
__device__ __forceinline__ unsigned spin_flags(const char* base, unsigned blk, unsigned stride, unsigned call, const char* wsp) {
  const int lane = lane_id();
  const unsigned* f = (const unsigned*)(base + blk + (unsigned)lane * stride);
  unsigned* tmo = (unsigned*)(wsp + WS_TMO);
  const unsigned long long t0 = __builtin_amdgcn_s_memrealtime();
  unsigned bad = 0;
  for (unsigned n = 1;; ++n) {
    const unsigned v = __hip_atomic_load(f, __ATOMIC_RELAXED, __HIP_MEMORY_SCOPE_SYSTEM);
    if (__all(v >= call)) break;
    if ((n & 15u) == 0u &&
        (__builtin_amdgcn_readfirstlane(__hip_atomic_load(tmo, __ATOMIC_RELAXED, __HIP_MEMORY_SCOPE_AGENT)) == call ||
         __builtin_amdgcn_s_memrealtime() - t0 > DEADLINE)) {
      if (lane == 0) __hip_atomic_store(tmo, call, __ATOMIC_RELAXED, __HIP_MEMORY_SCOPE_AGENT);
      bad = 0x7fc07fc0u;
      break;
    }
  }
  asm volatile("s_waitcnt vmcnt(0) lgkmcnt(0)" ::: "memory");
  return bad;
}
#define SEND(R, lo, hi, off) do { const u32x4 _srd = {lo, hi, OUT_BYTES, 0x20000u}; ST128(R, s.vdst, _srd, off, "sc0 sc1"); } while (0)

#define PFLAG(blk) do { unsigned _c; asm volatile("v_mov_b32 %0, %1\n\ts_mov_b64 exec, 1\n\tbuffer_store_dword %0, off, %2, %3 sc1\n\ts_mov_b64 exec, -1" : "=&v"(_c) : "s"(s.call), "s"(s.srdWs), "s"((blk) + (unsigned)s.wslot * 16u) : "memory"); } while (0)
#define GUARD(q) asm volatile("s_nop 7" : "+v"(q[0][0]), "+v"(q[0][1]), "+v"(q[1][0]), "+v"(q[1][1]), "+v"(q[2][0]), "+v"(q[2][1]), "+v"(q[3][0]), "+v"(q[3][1]))

template <int I, int V>
__device__ __forceinline__ void ktile(St& s) {
  constexpr unsigned b = I & 1, nb = b ^ 1;
  constexpr bool BG = (V != V_PLAIN && V != V_FIRST);
  constexpr bool E15 = (I == 15 && (V == V_L || V == V_R)), E0 = (I == 0 && (V == V_L || V == V_R));
  constexpr int hv = (V == V_R) ? 1 : 0;
  constexpr bool INL = (E0 && hv == 0) || V == V_FIRST, INR = (E0 && hv == 1) || V == V_FIRST;
  constexpr bool S0 = BG && slot_at(S0_FIRST, S0_N, I), S1 = BG && slot_at(S1_FIRST, S1_N, I);
  constexpr int KT = E15 ? -1 : (BG && I != 15 && I != 14) ? I : -100;
  constexpr int NX = count_at(PSTP, KT) + count_at(PLDP, KT) - (PLDP[0] / 4 == KT ? 1 : 0) - (PLDP[4] / 4 == KT ? 1 : 0);
#define BGMEM(ph) do { _Pragma("unroll") for (int u = 0; u < 8; ++u) { if (PSTP[u] == KT * 4 + (ph)) PST(u); if (PLDP[u] == KT * 4 + (ph)) PLD(u); } } while (0)
  LDB(s.B0, b, 0); LDA(b, 0);
  STAGE_A(SA(nb, 1), s.a1, hA);
  if constexpr (V == V_FIRST) wait_v<11>();
  WAIT_L(8); BAR; WAIT_B(s.B0); WAIT_A(); mma<INL>(s.acc[0][0], s.At, s.B0);
  if constexpr (E0) pack_quad(&s.tmp[4], s.acc[1][hv]);
  if constexpr (KT == T_TIE0 + 1) reduce4<1>(s, 0);
  if constexpr (KT == T_TIE1 + 1 && T_TIE1 != T_TIE0) { if (s.keep != 0u) reduce4<0>(s, 4); unsigned k2 = s.keep; asm volatile("" : "+s"(k2)); if (k2 == 0u) reduce4<2>(s, 4); }
  ENDM;
  LDB(s.B1, b, 1);
  if constexpr (KT == T_POLL) asm volatile("buffer_load_dword %0, %1, %2, %3 offen sc1" : "=v"(s.pollv) : "v"(s.vlane16), "s"(s.srdWs), "s"(s.pflag) : "memory");
  STAGE_B(SB(b, 0), s.bl2, 0u);
  BGMEM(1);
  if constexpr (V == V_FIRST) wait_v<11>();
  BAR; WAIT_B(s.B1); mma<INR>(s.acc[0][1], s.At, s.B1);
  if constexpr (KT == T_TIE0 + 1) { s.R0 = reduce_fin<false>(s); s.d0lo = s.pdlo; s.d0hi = s.pdhi; s.d0off = s.pdoff; s.ts0 = (4 + (s.TS0 & 15)) | (int)s.keep; asm volatile("" : "+s"(s.d0lo), "+s"(s.d0hi), "+s"(s.d0off), "+s"(s.ts0)); }
  if constexpr (KT == T_TIE1 + 1) { s.R1 = reduce_fin<false>(s); s.d1lo = s.pdlo; s.d1hi = s.pdhi; s.d1off = s.pdoff + (~s.keep & (8u * 2048u)); s.ts1 = 4 + ((s.TS0 >> 4) & 15); asm volatile("" : "+s"(s.d1lo), "+s"(s.d1hi), "+s"(s.d1off), "+s"(s.ts1)); }
  ENDM;
  LDA(b, 1);
  STAGE_A(SA(b, 0), s.a2, 0u);
  BGMEM(2);
  if constexpr (S0) { if (s.ts0 == I) SEND(s.R0, s.d0lo, s.d0hi, s.d0off); }
  if constexpr (S1) { if (s.ts1 == (I < S1_FIRST ? I + 16 : I)) SEND(s.R1, s.d1lo, s.d1hi, s.d1off); }
  BAR; WAIT_A(); mma<INL>(s.acc[1][0], s.At, s.B0);
  if constexpr (E15) pack_quad(&s.tmp[0], s.acc[0][hv]);
  ENDM;
  STAGE_B(SB(b, 1), s.br2, 0u);
  BGMEM(3);
  if constexpr (KT == T_TIE0) asm volatile("s_waitcnt vmcnt(%4)" : "+v"(s.tmp[0]), "+v"(s.tmp[1]), "+v"(s.tmp[2]), "+v"(s.tmp[3]) : "i"(6 + NX) : "memory");
  else if constexpr (KT == T_TIE1) asm volatile("s_waitcnt vmcnt(%4)" : "+v"(s.tmp[4]), "+v"(s.tmp[5]), "+v"(s.tmp[6]), "+v"(s.tmp[7]) : "i"(6 + NX) : "memory");
  else if constexpr (KT == T_POLL) asm volatile("s_waitcnt vmcnt(%1)" : "+v"(s.pollv) : "i"(6 + NX) : "memory");
  else wait_v<6 + NX>();
  if constexpr (KT == T_FLAG) PFLAG(s.pflag);
  if constexpr (I == 13) { s.sgBL = s.nxtBL; s.sgBR = s.nxtBR; }
  BAR; mma_step<INR>(s, s.acc[1][1], s.At, s.B1);
  ENDM;
#undef BGMEM
  if constexpr (KT == T_POLL) {
    if (!__all(s.pollv >= s.call || (s.vlane16 >> 7) == ((unsigned)s.wslot >> 3))) s.poison |= spin_flags(s.wsp, s.pflag, 16u, s.call, s.wsp);
    SCHED;
  }
}

__device__ __forceinline__ void final_pair(St& s, unsigned flagL, unsigned own8, unsigned adj8) {
  LDB(s.B0, 0u, 0); LDA(0u, 0); STAGE_A(SA(1u, 1), s.a1, hA);
  BAR; WAIT_B(s.B0); WAIT_A(); mma<false>(s.acc[0][0], s.At, s.B0); ENDM;
  LDA(0u, 1);
  BAR; WAIT_A(); mma<false>(s.acc[1][0], s.At, s.B0); ENDM;
  LDB(s.B1, 0u, 1); wait_v<2>();
  BAR; WAIT_B(s.B1); mma<false>(s.acc[1][1], s.At, s.B1); ENDM;
  LDB(s.B0, 1u, 0); LDA(1u, 0); wait_v<0>();
  BAR; WAIT_B(s.B0); WAIT_A(); mma<false>(s.acc[0][0], s.At, s.B0); ENDM;
  LDA(1u, 1); GUARD(s.acc[0][0]); pack_quad(&s.tmp[0], s.acc[0][0]); PST(0); PST(1); PST(2); PST(3);
  BAR; WAIT_A(); mma<false>(s.acc[1][0], s.At, s.B0); ENDM;
  LDB(s.B1, 1u, 1); GUARD(s.acc[1][0]); pack_quad(&s.tmp[4], s.acc[1][0]); PST(4); PST(5); PST(6); PST(7);
  BAR; WAIT_B(s.B1); mma<false>(s.acc[1][1], s.At, s.B1); ENDM;
  LDA(1u, 0);
  BAR; WAIT_A(); mma<false>(s.acc[0][1], s.At, s.B1); ENDM;
  LDB(s.B1, 0u, 1); LDA(0u, 0); wait_v<0>(); PFLAG(flagL);
  s.pw += REGION; s.own = own8; s.vls += adj8;
  pack_quad(&s.tmp[4], s.acc[1][1]); PST(4); PST(5); PST(6); PST(7);
  BAR; WAIT_B(s.B1); WAIT_A(); mma<false>(s.acc[0][1], s.At, s.B1); ENDM;
}

#define STAMP(i) do {} while (0)

__global__ __launch_bounds__(512, 2) void gemm_a2a(Params p) {
  __shared__ __attribute__((aligned(1024))) unsigned char smem[8 * HTB + 32768];
  const int tid = threadIdx.x, wid = __builtin_amdgcn_readfirstlane(tid >> 6), lane = tid & 63;
  const int wr = wid >> 2, wc = wid & 3, fr = lane & 15, fq = lane >> 4;
  const unsigned blk = blockIdx.x, mem = blk & 7u, grp = (blk >> 3) & 31u;
  const unsigned pm = grp & 3u, cg = grp >> 2, sub = mem >> 2, mi = mem & 3u;
  const unsigned rank = (unsigned)p.rank & 7u;
  STAMP(0);
  St s;
  char* wsp = p.ws[rank];
  s.wsp = wsp;
  const unsigned long long pa = (unsigned long long)p.A, pw = (unsigned long long)p.W, pws = (unsigned long long)wsp;
  s.srdA = (u32x4){(unsigned)pa, (unsigned)(pa >> 32) & 0xffffu, A_BYTES, 0x20000u};
  s.srdB = (u32x4){(unsigned)pw, (unsigned)(pw >> 32) & 0xffffu, W_BYTES, 0x20000u};
  s.srdWs = (u32x4){(unsigned)pws, (unsigned)(pws >> 32) & 0xffffu, WS_USED, 0x20000u};
  unsigned info;
  if (cg < 7u) {
    const unsigned j = (cg + 1u) << 3;
    info = (j | 0u) | ((j | (1u + sub)) << 6) | ((j | (3u + sub)) << 12) | ((j | (5u + sub)) << 18) | (cg << 24);
  } else {
    const unsigned j0 = (0x6427u >> (4u * pm)) & 7u;
    const unsigned j1 = ((4u + 2u * pm + sub) % 7u) + 1u, j2 = ((2u + 2u * pm + sub) % 7u) + 1u, j3 = ((2u * pm + sub) % 7u) + 1u;
    info = ((j0 << 3) | 7u) | (((j1 << 3) | 7u) << 6) | (((j2 << 3) | 7u) << 12) | (((j3 << 3) | 7u) << 18) | (7u << 24);
  }
  auto ent = [&](unsigned k) -> unsigned { return (info >> (6u * k)) & 63u; };
  auto colB = [&](unsigned e) -> unsigned { return (((rank + (e >> 3)) & 7u) * 8u + (e & 7u)) * 256u; };
  auto offB = [&](unsigned k) -> unsigned { return mi * (32u * ktB) + colB(ent(k)); };
  const unsigned lds0 = (unsigned)(size_t)(LAS unsigned char*)smem;
  s.ldsw = lds0 + (unsigned)wid * 1024u;
  s.voffA = (unsigned)((8 * wid + (lane >> 3)) * (K_ * 2) + (((lane & 7) ^ (lane >> 3)) << 4));
  {
    const unsigned fB = (unsigned)((lane >> 4) | (((wid >> 1) & 1) << 2));
    s.voffB = (unsigned)((4 * wid + (lane >> 4)) * (N_ * 2)) + ((((unsigned)lane & 15u) ^ (fB << 1)) << 4);
  }
  s.sgA = pm * (256u * K_ * 2u) + mi * (32u * ktA); s.sgBL = offB(1); s.sgBR = offB(0);
  {
    const unsigned k0 = (sub * 16u + pm) * ktA;
    const unsigned a0 = s.sgA + k0, l0 = s.sgBL + (k0 << 13), q0 = s.sgBR + (k0 << 13);
    STAGE_B(SB(0, 0), l0, 0u); STAGE_A(SA(0, 0), a0, 0u); STAGE_B(SB(0, 1), q0, 0u); STAGE_A(SA(0, 1), a0, hA);
    STAGE_B(SB(1, 0), l0, ktB); STAGE_A(SA(1, 0), a0, ktA); STAGE_B(SB(1, 1), q0, ktB);
    s.a1 = a0 + ktA; s.a2 = a0 + 2u * ktA; s.bl2 = l0 + 2u * ktB; s.br2 = q0 + 2u * ktB; s.kA = k0 + 2u * ktA;
  }
  unsigned callv;
  asm volatile("buffer_load_dword %0, off, %1, %2 sc1" : "=v"(callv) : "s"(s.srdWs), "s"(WS_CALL + blk * 64u) : "memory");
  unsigned olo[3], ohi[3];
#pragma unroll
  for (int k = 0; k < 3; ++k) {
    const unsigned long long o = (unsigned long long)p.out[(rank + (ent(k) >> 3)) & 7u];
    olo[k] = (unsigned)o; ohi[k] = (unsigned)(o >> 32) & 0xffffu;
  }
  {
    const int q = (lane & 15) >> 2, pp = lane & 3, g = lane >> 4;
    const int f = q | ((g & 1) << 2);
    s.aoff0 = lds0 + (unsigned)((wr * 64 + fr) * 128 + (((0 + fq) ^ (fr & 7)) << 4));
    s.aoff1 = lds0 + (unsigned)((wr * 64 + fr) * 128 + (((4 + fq) ^ (fr & 7)) << 4));
    s.boff0 = lds0 + 65536u + (unsigned)((8 * g + q) * 256 + (((2 * wc + 0) ^ f) << 5) + 8 * pp);
    s.boff1 = lds0 + 65536u + (unsigned)((8 * g + q) * 256 + (((2 * wc + 1) ^ f) << 5) + 8 * pp);
  }
  s.vlane16 = (unsigned)lane * 16u;
  {
    const unsigned one = (fq == (fr >> 2)) ? (0x3f80u << (16 * (fr & 1))) : 0u;
    const unsigned a = (fr & 2) ? 0u : one, b = (fr & 2) ? one : 0u;
    s.S1 = (u32x4){a, b, 0u, 0u}; s.S2 = (u32x4){0u, 0u, a, b};
    asm volatile("" : "+v"(s.S1), "+v"(s.S2));
  }
  {
    const unsigned r = (unsigned)lane >> 3, c = (unsigned)lane & 7u, cc = c & 3u;
    const unsigned sfq = ((cc & 1u) << 1) | (cc >> 1);
    s.vgat = (c >> 2) * 4096u + (sfq * 16u + r) * 16u;
    s.vdst = r * 2048u + c * 16u;
  }
  const unsigned uw = (unsigned)wid, rwr = uw >> 2, rwcp = uw & 1u, b1 = (uw >> 1) & 1u;
  const unsigned ai4 = mi >> 1, m4 = 2u * (mi & 1u) + b1, ai8 = mem >> 2, m8 = mem & 3u;
  unsigned rd4 = blk * SLOT + ((ai4 * 8u + rwr * 4u + 2u * rwcp) * 4u + m4) * 1024u;
  unsigned rd8 = blk * SLOT + ((ai8 * 8u + rwr * 4u + 2u * rwcp) * 4u + m8) * 1024u + b1 * 128u;
  const unsigned row0 = rank * 1024u + pm * 256u;
  unsigned ds4 = (row0 + ai4 * 128u + rwr * 64u + m4 * 16u) * 2048u + rwcp * 128u;
  unsigned ds8 = (row0 + ai8 * 128u + rwr * 64u + m8 * 16u + b1 * 8u) * 2048u + rwcp * 128u;
  unsigned pw0 = WS_PART + blk * SLOT + uw * 4096u;
  s.wslot = (int)(mem * 8u + uw);
  unsigned adj8 = (((2u * (mi & 1u)) ^ 2u) - m8) * 1024u;
  const unsigned lsp = lds0 + 8u * HTB;
  unsigned tsl;
  {
    const unsigned key = grp * 3u + mem + uw;
    constexpr unsigned F0 = S0_FIRST - 4, F1 = S1_FIRST - 4;
    constexpr unsigned W2 = (20 - S0_FIRST) / 2;
    constexpr unsigned G1 = (S0_FIRST + 4u > S1_FIRST ? S0_FIRST + 4u : S1_FIRST) - 4;
    constexpr unsigned G2 = (F0 + W2 > F1 ? F0 + W2 : F1);
    tsl = ((F1 + key % (16u - F1)) << 4) | (((F0 + key % W2) | ((G2 + key % (16u - G2)) << 4)) << 8) | (((F0 + key % 4u) | ((G1 + key % 4u) << 4)) << 16);
    static_assert(G1 + 3u + 4u <= 13u, "the last segment's pushes must leave before the final K-tile pair");
  }
  s.ts0 = -1; s.ts1 = -1; s.poison = 0u;
  s.d0lo = s.d0hi = s.d0off = s.d1lo = s.d1hi = s.d1off = 0u;
  asm volatile("" : "+s"(s.d0lo), "+s"(s.d0hi), "+s"(s.d0off), "+s"(s.d1lo), "+s"(s.d1hi), "+s"(s.d1off), "+s"(s.ts0), "+s"(s.ts1));
  s.R0 = (u32x4){0u, 0u, 0u, 0u}; s.R1 = s.R0;
#pragma unroll
  for (int a = 0; a < 2; ++a)
#pragma unroll
    for (int bb = 0; bb < 2; ++bb)
#pragma unroll
      for (int m = 0; m < 4; ++m)
#pragma unroll
        for (int n = 0; n < 2; ++n) asm volatile("" : "=v"(s.acc[a][bb][m][n]));
  s.nxtBL = s.sgBL; s.nxtBR = offB(2);
  unsigned bl2 = offB(3), br3 = offB(4);
  unsigned do0 = ds8 + (ent(0) & 7u) * 256u, do1 = ds4 + (ent(1) & 7u) * 256u, do2 = ds4 + (ent(2) & 7u) * 256u;
  unsigned pf0 = WS_PFLAG + grp * 1024u;
  asm volatile("" : "+s"(bl2), "+s"(br3), "+s"(do0), "+s"(do1), "+s"(do2), "+s"(pf0));
  asm volatile("s_waitcnt lgkmcnt(0)" : "+s"(rd4), "+s"(rd8), "+s"(ds4), "+s"(ds8), "+s"(pw0), "+s"(adj8), "+s"(tsl), "+s"(s.wslot), "+s"(s.nxtBL), "+s"(s.nxtBR) :: "memory");
  if (wr == 1) BAR;
  wait_v<11>(); BAR;
  BAR;
  STAMP(1);
  {
    ktile<0, V_FIRST>(s);
    asm volatile("" : "+v"(callv));
    s.call = __builtin_amdgcn_readfirstlane(callv) + 1u;
    ktile<1, V_PLAIN>(s);
    for (int t = 2; t < 12; t += 2) { ktile<0, V_PLAIN>(s); ktile<1, V_PLAIN>(s); }
    ktile<12, V_PLAIN>(s); ktile<13, V_PLAIN>(s);
  }
  const unsigned call = s.call;
  for (unsigned seg = 1; seg <= 3u; ++seg) {
    const unsigned k = seg - 1u;
    s.pw = pw0 + k * REGION;
    s.own = k == 0u ? 1u << mem : 3u << (2u * mi);
    s.vls = s.vlane16 + (lsp + uw * 4096u + (k == 0u ? adj8 : 0u));
    ktile<14, V_BG>(s);
    if (seg == 2u) {
      ktile<15, V_L>(s);
      ktile<0, V_L>(s);
    }
    unsigned seg2 = seg; asm volatile("" : "+s"(seg2));
    if (seg2 != 2u) {
      ktile<15, V_R>(s);
      ktile<0, V_R>(s);
    }
    STAMP(1 + seg);
    s.keep = (unsigned)((int)(k - 1u) >> 31);
    s.prd0 = WS_PART + k * REGION + (k == 0u ? rd8 : rd4);
    s.prd1 = k == 0u ? s.prd0 ^ (4u * SLOT) : s.prd0 + 128u;
    s.lrd = lsp + (s.prd0 & 0x7fffu) + (k == 0u ? adj8 : 0u);
    s.pflag = pf0 + k * 32768u;
    s.TS0 = (int)(tsl >> (8u * k));
    s.pdlo = k == 0u ? olo[0] : k == 1u ? olo[1] : olo[2];
    s.pdhi = k == 0u ? ohi[0] : k == 1u ? ohi[1] : ohi[2];
    s.pdoff = k == 0u ? do0 : k == 1u ? do1 : do2;
    s.nxtBL = bl2;
    s.nxtBR = k == 0u ? s.sgBR : br3;
    asm volatile("" : "+s"(s.nxtBL), "+s"(s.nxtBR), "+s"(s.pdoff), "+s"(s.pdlo), "+s"(s.pdhi), "+s"(s.pflag), "+s"(s.prd0), "+s"(s.prd1), "+s"(s.lrd), "+s"(s.TS0), "+s"(s.keep));
    ktile<1, V_BG>(s); ktile<2, V_BG>(s); ktile<3, V_BG>(s); ktile<4, V_BG>(s); ktile<5, V_BG>(s); ktile<6, V_BG>(s); ktile<7, V_BG>(s);
    ktile<8, V_BG>(s); ktile<9, V_BG>(s); ktile<10, V_BG>(s); ktile<11, V_BG>(s); ktile<12, V_BG>(s); ktile<13, V_BG>(s);
  }
  s.pw = pw0 + 3u * REGION;
  const unsigned fblk = pf0 + 3u * 32768u;
  final_pair(s, fblk, 1u << mem, adj8);
  STAMP(5);
  GUARD(s.acc[0][1]); pack_quad(&s.tmp[0], s.acc[0][1]); PST(0); PST(1); PST(2); PST(3);
  asm volatile("s_waitcnt vmcnt(0) lgkmcnt(0)" ::: "memory"); PFLAG(fblk + 32768u);
  STAMP(6);
  const int lane2 = lane_id();
  const unsigned long long o3 = (unsigned long long)p.out[(rank + (ent(3) >> 3)) & 7u], o4 = (unsigned long long)p.out[rank];
  const unsigned o3lo = (unsigned)o3, o3hi = (unsigned)(o3 >> 32) & 0xffffu, o4lo = (unsigned)o4, o4hi = (unsigned)(o4 >> 32) & 0xffffu;
  if (wr == 0) BAR;
  LAS unsigned* lflag = (LAS unsigned*)(smem + 8 * HTB + ((((2u * (mi & 1u)) ^ 2u) + 1u) * 1024u));
  if (wid == 0) {
    const unsigned bad = spin_flags(wsp, fblk, 16u, call, wsp);
    if (lane2 == 0) lflag[0] = bad;
    asm volatile("s_waitcnt lgkmcnt(0)" ::: "memory");
  }
  BAR;
  STAMP(7);
  {
    s.prd0 = WS_PART + 3u * REGION + rd4; s.prd1 = s.prd0 + 128u; s.lrd = lsp + (rd4 & 0x7fffu);
    PLDG(1); PLDG(2); PLDG(3); PLDG(5); PLDG(6); PLDG(7); PLDL(0, 0); PLDL(4, 128);
    if (wid == 0) {
      const unsigned bad = spin_flags(wsp, fblk + 32768u, 16u, call, wsp);
      if (lane2 == 0) lflag[1] = bad;
      asm volatile("s_waitcnt lgkmcnt(0)" ::: "memory");
    }
    s.poison |= __builtin_amdgcn_readfirstlane(*(volatile LAS unsigned*)lflag);
    asm volatile("s_waitcnt vmcnt(3) lgkmcnt(0)" : "+v"(s.tmp[0]), "+v"(s.tmp[1]), "+v"(s.tmp[2]), "+v"(s.tmp[3]) :: "memory");
    reduce4<1>(s, 0);
    const u32x4 o0 = reduce_fin<true>(s);
    asm volatile("s_waitcnt vmcnt(0)" : "+v"(s.tmp[4]), "+v"(s.tmp[5]), "+v"(s.tmp[6]), "+v"(s.tmp[7]) :: "memory");
    reduce4<1>(s, 4);
    const u32x4 o1 = reduce_fin<true>(s);
    BAR;
    s.prd0 = WS_PART + 4u * REGION + rd8; s.prd1 = s.prd0 ^ (4u * SLOT); s.lrd = lsp + (rd8 & 0x7fffu) + adj8;
    PLDG(1); PLDG(2); PLDG(3); PLDG(4); PLDG(5); PLDG(6); PLDG(7); PLDL(0, 0);
    const unsigned doff = ds4 + (ent(3) & 7u) * 256u;
    SEND(o0, o3lo, o3hi, doff);
    SEND(o1, o3lo, o3hi, doff + 8u * 2048u);
    s.poison |= __builtin_amdgcn_readfirstlane(*(volatile LAS unsigned*)(lflag + 1));
    asm volatile("s_waitcnt vmcnt(2) lgkmcnt(0)" : "+v"(s.tmp[0]), "+v"(s.tmp[1]), "+v"(s.tmp[2]), "+v"(s.tmp[3]), "+v"(s.tmp[4]), "+v"(s.tmp[5]), "+v"(s.tmp[6]), "+v"(s.tmp[7]) :: "memory");
    reduce4<1>(s, 0); reduce4<0>(s, 4);
    const u32x4 o2 = reduce_fin<true>(s);
    SEND(o2, o4lo, o4hi, ds8 + (ent(4) & 7u) * 256u);
  }
  STAMP(8);
  wait_v<0>();
  STAMP(9);
  BAR;
  if (wid == 0) {
    if (lane2 < 4) {
      const unsigned e = (info >> (6u * (unsigned)lane2)) & 63u;
      const unsigned dp = (rank + (e >> 3)) & 7u;
      unsigned* f = (unsigned*)(p.ws[dp] + WS_DONE + rank * 1024u) + ((e & 7u) * 4u + pm) * 8u;
      if (lane2 == 0) __hip_atomic_store(f + mem, call, __ATOMIC_RELAXED, __HIP_MEMORY_SCOPE_SYSTEM);
      else __hip_atomic_store((unsigned long long*)f + mi, ((unsigned long long)call << 32) | call, __ATOMIC_RELAXED, __HIP_MEMORY_SCOPE_SYSTEM);
    }
    if (lane2 == 0) __hip_atomic_store((unsigned*)(wsp + WS_CALL + blk * 64u), call, __ATOMIC_RELAXED, __HIP_MEMORY_SCOPE_AGENT);
    if (blk == 0) {
      const unsigned long long* f = (const unsigned long long*)(wsp + WS_DONE) + lane2;
      unsigned* tmo = (unsigned*)(wsp + WS_TMO);
      const unsigned long long t0 = __builtin_amdgcn_s_memrealtime();
      for (unsigned n = 1;; ++n) {
        bool ok = true;
#pragma unroll
        for (unsigned j = 1; j < 8; ++j) {
          const unsigned long long v0 = __hip_atomic_load(f + ((rank + j) & 7u) * 128u, __ATOMIC_RELAXED, __HIP_MEMORY_SCOPE_SYSTEM);
          const unsigned long long v1 = __hip_atomic_load(f + ((rank + j) & 7u) * 128u + 64u, __ATOMIC_RELAXED, __HIP_MEMORY_SCOPE_SYSTEM);
          ok = ok && ((unsigned)v0 >= call) && ((unsigned)(v0 >> 32) >= call) && ((unsigned)v1 >= call) && ((unsigned)(v1 >> 32) >= call);
        }
        if (__all(ok)) break;
        if ((n & 15u) == 0u &&
            (__builtin_amdgcn_readfirstlane(__hip_atomic_load(tmo, __ATOMIC_RELAXED, __HIP_MEMORY_SCOPE_AGENT)) == call ||
             __builtin_amdgcn_s_memrealtime() - t0 > DEADLINE)) {
          if (lane2 == 0) {
            __hip_atomic_store(tmo, call, __ATOMIC_RELAXED, __HIP_MEMORY_SCOPE_AGENT);
            __hip_atomic_store((unsigned*)(p.out[rank] + (size_t)rank * 1024u * 2048u), 0x7fc07fc0u, __ATOMIC_RELAXED, __HIP_MEMORY_SCOPE_SYSTEM);
          }
          break;
        }
      }
      __builtin_amdgcn_fence(__ATOMIC_ACQUIRE, "");
    }
  }
}

}

extern "C" void kernel_launch(int rank, int nranks, ncclComm_t comm,
                              hipStream_t stream,
                              void* const* d_in, const int* in_sizes, int n_in,
                              void* d_out, int out_size,
                              const int* in_dtypes, int out_dtype,
                              void* const* d_out_all,
                              void* const* d_ws_all, size_t ws_bytes) {
  using namespace ga2a;
  (void)comm; (void)in_sizes; (void)n_in; (void)d_out; (void)out_size; (void)in_dtypes; (void)out_dtype;
  if (nranks != NR || ws_bytes < (size_t)WS_USED) { fprintf(stderr, "unsupported configuration\n"); return; }
  Params p;
  p.A = (const u16*)d_in[0];
  p.W = (const u16*)d_in[1];
  for (int i = 0; i < NR; ++i) { p.out[i] = (char*)d_out_all[i]; p.ws[i] = (char*)d_ws_all[i]; }
  p.rank = rank;
  hipLaunchKernelGGL(gemm_a2a, dim3(256), dim3(512), 0, stream, p);
  HIP_CHECK(hipGetLastError());
}
```

```cpp
#include <hip/hip_runtime.h>
#include <rccl/rccl.h>
#include <cstdio>

#define HIP_CHECK(expr) do { hipError_t _e = (expr); if (_e != hipSuccess) { \
    fprintf(stderr, "HIP error %s at %s:%d\n", hipGetErrorString(_e), __FILE__, __LINE__); \
    return; } } while (0)

namespace ga2a {

#define LAS __attribute__((address_space(3)))
typedef unsigned short u16;
typedef short bf16x8 __attribute__((ext_vector_type(8)));
typedef short bf16x4 __attribute__((ext_vector_type(4)));
typedef float f32x4 __attribute__((ext_vector_type(4)));
typedef float f32x2 __attribute__((ext_vector_type(2)));
typedef __bf16 bf16x2n __attribute__((ext_vector_type(2)));
typedef unsigned u32x4 __attribute__((ext_vector_type(4)));

constexpr int NR = 8;
constexpr int M_ = 1024, N_ = 8192, K_ = 8192;
constexpr unsigned HTB = 16384;
constexpr unsigned A_BYTES = (unsigned)M_ * K_ * 2u, W_BYTES = (unsigned)K_ * N_ * 2u;
constexpr unsigned OUT_BYTES = 8192u * 1024u * 2u;
constexpr unsigned SLOT = 65536u;
constexpr unsigned REGION = 256u * SLOT;
constexpr unsigned WS_PART = 0u;
constexpr unsigned WS_CTRL = 128u << 20;
constexpr unsigned WS_PFLAG = WS_CTRL + 524288u;
constexpr unsigned WS_CALL = WS_CTRL + 131072u;
constexpr unsigned WS_DONE = WS_CTRL + 163840u;
constexpr unsigned WS_TMO = WS_CTRL + 180224u;
constexpr unsigned WS_STAMP = WS_CTRL + 262144u;
constexpr unsigned WS_USED = WS_CTRL + (1u << 20);
constexpr unsigned long long DEADLINE = 50000000ull;

constexpr unsigned ktA = 128u, ktB = 64u * N_ * 2u;
constexpr unsigned hA = 128u * K_ * 2u;
constexpr unsigned iA = 64u * K_ * 2u, iB = 32u * N_ * 2u;

struct Params {
  const u16* A;
  const u16* W;
  char* out[NR];
  char* ws[NR];
  int rank;
};

enum { V_PLAIN = 0, V_BG = 1, V_L = 2, V_R = 3, V_FIRST = 4 };
constexpr int PSTP[8] = {-1, -1, 0, 0, 1, 1, 2, 2};
constexpr int PLDP[8] = {13, 14, 15, 16, 17, 18, 19, 20};
constexpr int T_FLAG = 1, T_POLL = 2;
constexpr int T_TIE0 = PLDP[3] / 4 + 1, T_TIE1 = PLDP[7] / 4 + 1;
constexpr int S0_FIRST = T_TIE0 + 1, S0_N = 16 - S0_FIRST, S1_FIRST = T_TIE1 + 1, S1_N = 20 - S1_FIRST;
constexpr int count_at(const int (&t)[8], int kt) { int n = 0; for (int i = 0; i < 8; ++i) n += (t[i] >= kt * 4 && t[i] < kt * 4 + 4) ? 1 : 0; return n; }
constexpr int nbg(int p) { int n = 0; for (int i = 0; i < 8; ++i) n += (PSTP[i] == p ? 1 : 0) + ((PLDP[i] == p && i != 0 && i != 4) ? 1 : 0); return n; }
constexpr int lag5(int p) { int n = 10; for (int q = p - 5; q <= p; ++q) n += nbg(q); return n; }
constexpr bool slot_at(int first, int n, int i) { return (i >= first && i < first + n) || (i + 16 >= first && i + 16 < first + n); }

__device__ __forceinline__ unsigned pk2(float lo, float hi) {
  f32x2 v = {lo, hi};
  return __builtin_bit_cast(unsigned, __builtin_convertvector(v, bf16x2n));
}
__device__ __forceinline__ u32x4 pack_pair(const f32x4& v0, const f32x4& v1) {
  unsigned d00 = pk2(v0[0], v0[1]), d01 = pk2(v0[2], v0[3]);
  unsigned d10 = pk2(v1[0], v1[1]), d11 = pk2(v1[2], v1[3]);
  auto r0 = __builtin_amdgcn_permlane16_swap(d00, d10, false, false);
  auto r1 = __builtin_amdgcn_permlane16_swap(d01, d11, false, false);
  u32x4 w; w.x = r0[0]; w.y = r1[0]; w.z = r0[1]; w.w = r1[1];
  return w;
}

struct St {
  f32x4 acc[2][2][4][2];
  bf16x8 At[4][2];
  bf16x4 B0[2][2][2], B1[2][2][2];
  u32x4 tmp[8];
  f32x4 D1, D2;
  u32x4 S1, S2;
  u32x4 R0, R1;
  unsigned pollv;
  unsigned voffA, voffB, aoff0, aoff1, boff0, boff1, vlane16, vgat, vdst;
  u32x4 srdA, srdB, srdWs;
  unsigned ldsw;
  unsigned a1, a2, bl2, br2, kA;
  unsigned sgA, sgBL, sgBR, nxtBL, nxtBR;
  unsigned pw;
  unsigned prd0, prd1;
  unsigned own, lrd;
  unsigned vls;
  unsigned pflag;
  unsigned keep;
  unsigned call, poison;
  unsigned pdlo, pdhi, pdoff;
  unsigned d0lo, d0hi, d0off, d1lo, d1hi, d1off;
  int ts0, ts1, TS0, wslot;
};

#define SA(b, h) (((b) * 2u + (h)) * HTB)
#define SB(b, h) ((4u + (b) * 2u + (h)) * HTB)
#define DMA2(ldsoff, voff, srd, sbase, simm, step) do { unsigned _t; asm volatile( \
    "s_add_u32 m0, %1, %2\n\ts_add_u32 %0, %6, %7\n\tbuffer_load_dwordx4 %4, %5, %0 offen lds\n\t" \
    "s_add_u32 m0, %1, %3\n\ts_add_u32 %0, %6, %8\n\tbuffer_load_dwordx4 %4, %5, %0 offen lds" \
    : "=&s"(_t) : "s"(s.ldsw), "i"(ldsoff), "i"((ldsoff) + 8192u), "v"(voff), "s"(srd), "s"(sbase), "i"(simm), "i"((simm) + (step)) : "memory", "scc"); } while (0)
#define STAGE_A(bufoff, sbase, simm) DMA2((bufoff), s.voffA, s.srdA, (sbase), (simm), iA)
#define STAGE_B(bufoff, sbase, simm) DMA2((bufoff), s.voffB, s.srdB, (sbase), (simm), iB)
__device__ __forceinline__ int lane_id() { int l; asm volatile("v_mbcnt_lo_u32_b32 %0, -1, 0\n\tv_mbcnt_hi_u32_b32 %0, -1, %0" : "=v"(l)); return l; }
#define RD128(dst, addr, off) asm volatile("ds_read_b128 %0, %1 offset:%2" : "=v"(dst) : "v"(addr), "i"(off))
#define RDTR(dst, addr, off) asm volatile("ds_read_b64_tr_b16 %0, %1 offset:%2" : "=v"(dst) : "v"(addr), "i"(off))
#define LDA(b, h) do { _Pragma("unroll") for (int m = 0; m < 4; ++m) { RD128(s.At[m][0], s.aoff0, SA(b, h) + m * 2048); RD128(s.At[m][1], s.aoff1, SA(b, h) + m * 2048); } } while (0)
#define LDB(dst, b, h) do { _Pragma("unroll") for (int k = 0; k < 2; ++k) { \
    RDTR(dst[0][k][0], s.boff0, SB(b, h) - 65536u + k * 8192); RDTR(dst[0][k][1], s.boff0, SB(b, h) - 65536u + k * 8192 + 1024); } \
    _Pragma("unroll") for (int k = 0; k < 2; ++k) { \
    RDTR(dst[1][k][0], s.boff1, SB(b, h) - 65536u + k * 8192); RDTR(dst[1][k][1], s.boff1, SB(b, h) - 65536u + k * 8192 + 1024); } } while (0)
#define WAIT_A() asm volatile("s_waitcnt lgkmcnt(0)" : "+v"(s.At[0][0]), "+v"(s.At[0][1]), "+v"(s.At[1][0]), "+v"(s.At[1][1]), "+v"(s.At[2][0]), "+v"(s.At[2][1]), "+v"(s.At[3][0]), "+v"(s.At[3][1]) :: "memory")
#define WAIT_B(Bt) asm volatile("s_waitcnt lgkmcnt(0)" : "+v"(Bt[0][0][0]), "+v"(Bt[0][0][1]), "+v"(Bt[0][1][0]), "+v"(Bt[0][1][1]), "+v"(Bt[1][0][0]), "+v"(Bt[1][0][1]), "+v"(Bt[1][1][0]), "+v"(Bt[1][1][1]) :: "memory")
#define CAT(x) __builtin_shufflevector(x[0], x[1], 0, 1, 2, 3, 4, 5, 6, 7)
#define WAIT_L(n) asm volatile("s_waitcnt lgkmcnt(" #n ")" ::: "memory")
#define BAR __builtin_amdgcn_s_barrier()
#define SCHED __builtin_amdgcn_sched_barrier(0)
#define ENDM do { SCHED; asm volatile("s_setprio 0"); BAR; SCHED; } while (0)
#define ST128(data, voff, srd, soff, bits) asm volatile("buffer_store_dwordx4 %0, %1, %2, %3 offen " bits "\n\ts_nop 1" :: "v"(data), "v"(voff), "s"(srd), "s"(soff) : "memory")
#define LD128(dst, voff, srd, soff) asm volatile("buffer_load_dwordx4 %0, %1, %2, %3 offen sc1" : "=v"(dst) : "v"(voff), "s"(srd), "s"(soff) : "memory")
#define PST(u) do { unsigned _t; asm volatile( \
    "s_bitcmp1_b32 %1, %2\n\ts_cselect_b32 %0, %3, %4\n\ts_cbranch_scc0 1f\n\tds_write_b128 %5, %6 offset:%7\n1:\n\t" \
    "s_add_u32 %0, %0, %8\n\tbuffer_store_dwordx4 %6, %9, %10, %0 offen sc1\n\ts_nop 1" \
    : "=&s"(_t) : "s"(s.own), "i"(u), "s"(0xC0000000u), "s"(s.pw), "v"(s.vls), "v"(s.tmp[u]), "i"(((u) & 3) * 1024), "i"(((u) >> 2) * 32768u + ((u) & 3) * 1024u), "v"(s.vlane16), "s"(s.srdWs) : "memory", "scc"); } while (0)
#define PLDG(j) LD128(s.tmp[j], s.vgat, s.srdWs, ((j) < 4 ? s.prd0 : s.prd1) ^ (((j) & 3) * SLOT))
#define PLDL(j, add) do { unsigned _a; asm volatile("v_add_u32 %1, %2, %3\n\tds_read_b128 %0, %1 offset:%4" : "=v"(s.tmp[j]), "=&v"(_a) : "s"(s.lrd), "v"(s.vgat), "i"(add) : "memory"); } while (0)
#define PLD(j) do { if ((j) == 0) PLDL(0, 0); else if ((j) == 4) { if (s.keep != 0u) PLDG(4); unsigned _k = s.keep; asm volatile("" : "+s"(_k)); if (_k == 0u) PLDL(4, 128); } else PLDG(j); } while (0)

template <int N> __device__ __forceinline__ void wait_v() { asm volatile("s_waitcnt vmcnt(%0)" :: "i"(N) : "memory"); }

template <bool INIT>
__device__ __forceinline__ void mma(f32x4 (&acc)[4][2], const bf16x8 (&At)[4][2], const bf16x4 (&Bt)[2][2][2]) {
  const bf16x8 b00 = CAT(Bt[0][0]), b01 = CAT(Bt[0][1]), b10 = CAT(Bt[1][0]), b11 = CAT(Bt[1][1]);
  if constexpr (INIT)
    asm volatile("s_setprio 1\n\ts_nop 0\n\t"
      "v_mfma_f32_16x16x32_bf16 %0, %16, %8, 0\n\t"
      "v_mfma_f32_16x16x32_bf16 %0, %17, %9, %0\n\t"
      "v_mfma_f32_16x16x32_bf16 %1, %18, %8, 0\n\t"
      "v_mfma_f32_16x16x32_bf16 %1, %19, %9, %1\n\t"
      "v_mfma_f32_16x16x32_bf16 %2, %16, %10, 0\n\t"
      "v_mfma_f32_16x16x32_bf16 %2, %17, %11, %2\n\t"
      "v_mfma_f32_16x16x32_bf16 %3, %18, %10, 0\n\t"
      "v_mfma_f32_16x16x32_bf16 %3, %19, %11, %3\n\t"
      "v_mfma_f32_16x16x32_bf16 %4, %16, %12, 0\n\t"
      "v_mfma_f32_16x16x32_bf16 %4, %17, %13, %4\n\t"
      "v_mfma_f32_16x16x32_bf16 %5, %18, %12, 0\n\t"
      "v_mfma_f32_16x16x32_bf16 %5, %19, %13, %5\n\t"
      "v_mfma_f32_16x16x32_bf16 %6, %16, %14, 0\n\t"
      "v_mfma_f32_16x16x32_bf16 %6, %17, %15, %6\n\t"
      "v_mfma_f32_16x16x32_bf16 %7, %18, %14, 0\n\t"
      "v_mfma_f32_16x16x32_bf16 %7, %19, %15, %7\n\t"
      : "+v"(acc[0][0]), "+v"(acc[0][1]), "+v"(acc[1][0]), "+v"(acc[1][1]), "+v"(acc[2][0]), "+v"(acc[2][1]), "+v"(acc[3][0]), "+v"(acc[3][1])
      : "v"(At[0][0]), "v"(At[0][1]), "v"(At[1][0]), "v"(At[1][1]), "v"(At[2][0]), "v"(At[2][1]), "v"(At[3][0]), "v"(At[3][1]), "v"(b00), "v"(b01), "v"(b10), "v"(b11));
  else
    asm volatile("s_setprio 1\n\ts_nop 0\n\t"
      "v_mfma_f32_16x16x32_bf16 %0, %16, %8, %0\n\t"
      "v_mfma_f32_16x16x32_bf16 %0, %17, %9, %0\n\t"
      "v_mfma_f32_16x16x32_bf16 %1, %18, %8, %1\n\t"
      "v_mfma_f32_16x16x32_bf16 %1, %19, %9, %1\n\t"
      "v_mfma_f32_16x16x32_bf16 %2, %16, %10, %2\n\t"
      "v_mfma_f32_16x16x32_bf16 %2, %17, %11, %2\n\t"
      "v_mfma_f32_16x16x32_bf16 %3, %18, %10, %3\n\t"
      "v_mfma_f32_16x16x32_bf16 %3, %19, %11, %3\n\t"
      "v_mfma_f32_16x16x32_bf16 %4, %16, %12, %4\n\t"
      "v_mfma_f32_16x16x32_bf16 %4, %17, %13, %4\n\t"
      "v_mfma_f32_16x16x32_bf16 %5, %18, %12, %5\n\t"
      "v_mfma_f32_16x16x32_bf16 %5, %19, %13, %5\n\t"
      "v_mfma_f32_16x16x32_bf16 %6, %16, %14, %6\n\t"
      "v_mfma_f32_16x16x32_bf16 %6, %17, %15, %6\n\t"
      "v_mfma_f32_16x16x32_bf16 %7, %18, %14, %7\n\t"
      "v_mfma_f32_16x16x32_bf16 %7, %19, %15, %7\n\t"
      : "+v"(acc[0][0]), "+v"(acc[0][1]), "+v"(acc[1][0]), "+v"(acc[1][1]), "+v"(acc[2][0]), "+v"(acc[2][1]), "+v"(acc[3][0]), "+v"(acc[3][1])
      : "v"(At[0][0]), "v"(At[0][1]), "v"(At[1][0]), "v"(At[1][1]), "v"(At[2][0]), "v"(At[2][1]), "v"(At[3][0]), "v"(At[3][1]), "v"(b00), "v"(b01), "v"(b10), "v"(b11));
}
template <bool INIT>
__device__ __forceinline__ void mma2(f32x4& c, const bf16x8& b0, const bf16x8& a0, const bf16x8& b1, const bf16x8& a1) {
  if constexpr (INIT) asm volatile("v_mfma_f32_16x16x32_bf16 %0, %1, %2, 0\n\tv_mfma_f32_16x16x32_bf16 %0, %3, %4, %0" : "+v"(c) : "v"(b0), "v"(a0), "v"(b1), "v"(a1));
  else asm volatile("v_mfma_f32_16x16x32_bf16 %0, %1, %2, %0\n\tv_mfma_f32_16x16x32_bf16 %0, %3, %4, %0" : "+v"(c) : "v"(b0), "v"(a0), "v"(b1), "v"(a1));
}
template <bool INIT>
__device__ __forceinline__ void mma_step(St& s, f32x4 (&acc)[4][2], const bf16x8 (&At)[4][2], const bf16x4 (&Bt)[2][2][2]) {
  const bf16x8 b00 = CAT(Bt[0][0]), b01 = CAT(Bt[0][1]), b10 = CAT(Bt[1][0]), b11 = CAT(Bt[1][1]);
  unsigned t;
  asm volatile("s_setprio 1\n\ts_nop 0");
  mma2<INIT>(acc[0][0], b00, At[0][0], b01, At[0][1]);
  asm volatile("s_add_u32 %0, %0, 0x80\n\ts_and_b32 %0, %0, 0xf80" : "+s"(s.kA) :: "scc");
  mma2<INIT>(acc[0][1], b10, At[0][0], b11, At[0][1]);
  asm volatile("s_mov_b32 %0, %1\n\ts_add_u32 %1, %2, %3" : "=&s"(s.a1), "+s"(s.a2) : "s"(s.sgA), "s"(s.kA) : "scc");
  mma2<INIT>(acc[1][0], b00, At[1][0], b01, At[1][1]);
  asm volatile("s_lshl_b32 %0, %1, 13" : "=s"(t) : "s"(s.kA) : "scc");
  mma2<INIT>(acc[1][1], b10, At[1][0], b11, At[1][1]);
  asm volatile("s_add_u32 %0, %2, %4\n\ts_add_u32 %1, %3, %4" : "=&s"(s.bl2), "=&s"(s.br2) : "s"(s.sgBL), "s"(s.sgBR), "s"(t) : "scc");
  mma2<INIT>(acc[2][0], b00, At[2][0], b01, At[2][1]);
  mma2<INIT>(acc[2][1], b10, At[2][0], b11, At[2][1]);
  mma2<INIT>(acc[3][0], b00, At[3][0], b01, At[3][1]);
  mma2<INIT>(acc[3][1], b10, At[3][0], b11, At[3][1]);
}
__device__ __forceinline__ void pack_quad(u32x4* dst, f32x4 (&q)[4][2]) {
  asm volatile("" : "+v"(q[0][0]), "+v"(q[0][1]), "+v"(q[1][0]), "+v"(q[1][1]), "+v"(q[2][0]), "+v"(q[2][1]), "+v"(q[3][0]), "+v"(q[3][1]));
#pragma unroll
  for (int m = 0; m < 4; ++m) { dst[m] = pack_pair(q[m][0], q[m][1]); asm volatile("" : "+v"(dst[m])); }
}
__device__ __forceinline__ void make_sel(St& s) {
  unsigned l = s.vlane16; asm volatile("" : "+v"(l));
  const unsigned one = ((l >> 8) == ((l >> 6) & 3u)) ? (0x3f80u << (l & 16u)) : 0u;
  const unsigned a = (l & 32u) ? 0u : one, b = (l & 32u) ? one : 0u;
  s.S1 = (u32x4){a, b, 0u, 0u}; s.S2 = (u32x4){0u, 0u, a, b};
  asm volatile("s_nop 1" : "+v"(s.S1), "+v"(s.S2));
}
template <int FIRST>
__device__ __forceinline__ void reduce4(St& s, int j) {
  if constexpr (FIRST == 1)
    asm volatile("v_mfma_f32_16x16x32_bf16 %0, %2, %4, 0\n\tv_mfma_f32_16x16x32_bf16 %1, %3, %4, 0\n\t"
                 "v_mfma_f32_16x16x32_bf16 %0, %2, %5, %0\n\tv_mfma_f32_16x16x32_bf16 %1, %3, %5, %1\n\t"
                 "v_mfma_f32_16x16x32_bf16 %0, %2, %6, %0\n\tv_mfma_f32_16x16x32_bf16 %1, %3, %6, %1\n\t"
                 "v_mfma_f32_16x16x32_bf16 %0, %2, %7, %0\n\tv_mfma_f32_16x16x32_bf16 %1, %3, %7, %1"
                 : "=&v"(s.D1), "=&v"(s.D2) : "v"(s.S1), "v"(s.S2), "v"(s.tmp[j]), "v"(s.tmp[j + 1]), "v"(s.tmp[j + 2]), "v"(s.tmp[j + 3]));
  else if constexpr (FIRST == 2)
    asm volatile("v_mfma_f32_16x16x32_bf16 %0, %2, %4, 0\n\tv_mfma_f32_16x16x32_bf16 %1, %3, %4, 0\n\t"
                 "v_mfma_f32_16x16x32_bf16 %0, %2, %5, %0\n\tv_mfma_f32_16x16x32_bf16 %1, %3, %5, %1\n\t"
                 "v_mfma_f32_16x16x32_bf16 %0, %2, %6, %0\n\tv_mfma_f32_16x16x32_bf16 %1, %3, %6, %1\n\t"
                 "v_mfma_f32_16x16x32_bf16 %0, %2, %7, %0\n\tv_mfma_f32_16x16x32_bf16 %1, %3, %7, %1"
                 : "+v"(s.D1), "+v"(s.D2) : "v"(s.S1), "v"(s.S2), "v"(s.tmp[j]), "v"(s.tmp[j + 1]), "v"(s.tmp[j + 2]), "v"(s.tmp[j + 3]));
  else
    asm volatile("v_mfma_f32_16x16x32_bf16 %0, %2, %4, %0\n\tv_mfma_f32_16x16x32_bf16 %1, %3, %4, %1\n\t"
                 "v_mfma_f32_16x16x32_bf16 %0, %2, %5, %0\n\tv_mfma_f32_16x16x32_bf16 %1, %3, %5, %1\n\t"
                 "v_mfma_f32_16x16x32_bf16 %0, %2, %6, %0\n\tv_mfma_f32_16x16x32_bf16 %1, %3, %6, %1\n\t"
                 "v_mfma_f32_16x16x32_bf16 %0, %2, %7, %0\n\tv_mfma_f32_16x16x32_bf16 %1, %3, %7, %1"
                 : "+v"(s.D1), "+v"(s.D2) : "v"(s.S1), "v"(s.S2), "v"(s.tmp[j]), "v"(s.tmp[j + 1]), "v"(s.tmp[j + 2]), "v"(s.tmp[j + 3]));
}
template <bool NOP>
__device__ __forceinline__ u32x4 reduce_fin(St& s) {
  if constexpr (NOP) asm volatile("s_nop 7" : "+v"(s.D1), "+v"(s.D2)); else asm volatile("" : "+v"(s.D1), "+v"(s.D2));
  u32x4 r;
  r.x = pk2(s.D1[0], s.D1[1]) | s.poison; r.y = pk2(s.D1[2], s.D1[3]) | s.poison;
  r.z = pk2(s.D2[0], s.D2[1]) | s.poison; r.w = pk2(s.D2[2], s.D2[3]) | s.poison;
  asm volatile("" : "+v"(r));
  return r;
}
__device__ __forceinline__ unsigned spin_flags(const St& s, unsigned blk) {
  unsigned bad, v0, v1, st, t0, t1;
  asm volatile(
      "s_memrealtime vcc\n\ts_mov_b32 %0, 0\n\ts_waitcnt lgkmcnt(0)\n\ts_mov_b32 %4, vcc_lo\n"
      "1:\n\t"
      "buffer_load_dword %1, %6, %7, %8 offen sc0 sc1\n\t"
      "buffer_load_dword %2, off, %7, %9 sc0 sc1\n\t"
      "s_memrealtime vcc\n\t"
      "s_waitcnt vmcnt(0) lgkmcnt(0)\n\t"
      "s_sub_u32 %5, vcc_lo, %4\n\t"
      "v_cmp_gt_u32 vcc, %10, %1\n\t"
      "v_readfirstlane_b32 %3, %2\n\t"
      "s_cmp_eq_u64 vcc, 0\n\t"
      "s_cbranch_scc1 3f\n\t"
      "s_cmp_eq_u32 %3, %10\n\t"
      "s_cbranch_scc1 2f\n\t"
      "s_cmp_lt_u32 %5, %11\n\t"
      "s_cbranch_scc1 1b\n"
      "2:\n\t"
      "v_mov_b32 %1, %10\n\ts_mov_b64 exec, 1\n\tbuffer_store_dword %1, off, %7, %9 sc0 sc1\n\ts_mov_b64 exec, -1\n\t"
      "s_mov_b32 %0, 0x7fc07fc0\n\ts_waitcnt vmcnt(0)\n"
      "3:"
      : "=&s"(bad), "=&v"(v0), "=&v"(v1), "=&s"(st), "=&s"(t0), "=&s"(t1)
      : "v"(s.vlane16), "s"(s.srdWs), "s"(blk), "s"(WS_TMO), "s"(s.call), "s"((unsigned)DEADLINE)
      : "memory", "scc", "vcc");
  return bad;
}
#define SEND(R, lo, hi, off) do { const u32x4 _srd = {lo, hi, OUT_BYTES, 0x20000u}; ST128(R, s.vdst, _srd, off, "sc0 sc1"); } while (0)

#define PFLAG(blk) do { unsigned _c; asm volatile("v_mov_b32 %0, %1\n\ts_mov_b64 exec, 1\n\tbuffer_store_dword %0, off, %2, %3 sc1\n\ts_mov_b64 exec, -1" : "=&v"(_c) : "s"(s.call), "s"(s.srdWs), "s"((blk) + (unsigned)s.wslot * 16u) : "memory"); } while (0)
#define GUARD(q) asm volatile("s_nop 7" : "+v"(q[0][0]), "+v"(q[0][1]), "+v"(q[1][0]), "+v"(q[1][1]), "+v"(q[2][0]), "+v"(q[2][1]), "+v"(q[3][0]), "+v"(q[3][1]))

template <int I, int V>
__device__ __forceinline__ void ktile(St& s) {
  constexpr unsigned b = I & 1, nb = b ^ 1;
  constexpr bool BG = (V != V_PLAIN && V != V_FIRST);
  constexpr bool E15 = (I == 15 && (V == V_L || V == V_R)), E0 = (I == 0 && (V == V_L || V == V_R));
  constexpr int hv = (V == V_R) ? 1 : 0;
  constexpr bool INL = (E0 && hv == 0) || V == V_FIRST, INR = (E0 && hv == 1) || V == V_FIRST;
  constexpr bool S0 = BG && slot_at(S0_FIRST, S0_N, I), S1 = BG && slot_at(S1_FIRST, S1_N, I);
  constexpr int KT = E15 ? -1 : (BG && I != 15 && I != 14) ? I : -100;
  constexpr int NX = count_at(PSTP, KT) + count_at(PLDP, KT) - (PLDP[0] / 4 == KT ? 1 : 0) - (PLDP[4] / 4 == KT ? 1 : 0);
#define BGMEM(ph) do { _Pragma("unroll") for (int u = 0; u < 8; ++u) { if (PSTP[u] == KT * 4 + (ph)) PST(u); if (PLDP[u] == KT * 4 + (ph)) PLD(u); } } while (0)
  LDA(b, 0);
  STAGE_A(SA(nb, 1), s.a1, hA);
  BGMEM(0);
  if constexpr (V == V_FIRST) wait_v<11>();
  BAR; WAIT_A(); mma<INL>(s.acc[0][0], s.At, s.B0);
  if constexpr (E0) pack_quad(&s.tmp[4], s.acc[1][hv]);
  if constexpr (KT == T_TIE0 + 1) { make_sel(s); reduce4<1>(s, 0); }
  if constexpr (KT == T_TIE1 + 1 && T_TIE1 != T_TIE0) { if (s.keep != 0u) reduce4<0>(s, 4); unsigned k2 = s.keep; asm volatile("" : "+s"(k2)); if (k2 == 0u) reduce4<2>(s, 4); }
  ENDM;
  LDB(s.B1, b, 1);
  if constexpr (KT == T_POLL) asm volatile("buffer_load_dword %0, %1, %2, %3 offen sc1" : "=v"(s.pollv) : "v"(s.vlane16), "s"(s.srdWs), "s"(s.pflag) : "memory");
  STAGE_B(SB(b, 0), s.bl2, 0u);
  BGMEM(1);
  if constexpr (V == V_FIRST) wait_v<11>();
  BAR; WAIT_B(s.B1); mma<INR>(s.acc[0][1], s.At, s.B1);
  if constexpr (KT == T_TIE0 + 1) { s.R0 = reduce_fin<false>(s); s.d0lo = s.pdlo; s.d0hi = s.pdhi; s.d0off = s.pdoff; s.ts0 = (4 + (s.TS0 & 15)) | (int)s.keep; asm volatile("" : "+s"(s.d0lo), "+s"(s.d0hi), "+s"(s.d0off), "+s"(s.ts0)); }
  if constexpr (KT == T_TIE1 + 1) { s.R1 = reduce_fin<false>(s); s.d1lo = s.pdlo; s.d1hi = s.pdhi; s.d1off = s.pdoff + (~s.keep & (8u * 2048u)); s.ts1 = 4 + ((s.TS0 >> 4) & 15); asm volatile("" : "+s"(s.d1lo), "+s"(s.d1hi), "+s"(s.d1off), "+s"(s.ts1)); }
  ENDM;
  LDA(b, 1);
  STAGE_A(SA(b, 0), s.a2, 0u);
  BGMEM(2);
  if constexpr (S0) { if (s.ts0 == I) SEND(s.R0, s.d0lo, s.d0hi, s.d0off); }
  if constexpr (S1) { if (s.ts1 == (I < S1_FIRST ? I + 16 : I)) SEND(s.R1, s.d1lo, s.d1hi, s.d1off); }
  wait_v<lag5(KT * 4 + 2)>();
  BAR; WAIT_A(); mma<INL>(s.acc[1][0], s.At, s.B0);
  if constexpr (E15) pack_quad(&s.tmp[0], s.acc[0][hv]);
  ENDM;
  LDB(s.B0, nb, 0);
  STAGE_B(SB(b, 1), s.br2, 0u);
  BGMEM(3);
  if constexpr (KT == T_TIE0) asm volatile("s_waitcnt vmcnt(%4)" : "+v"(s.tmp[0]), "+v"(s.tmp[1]), "+v"(s.tmp[2]), "+v"(s.tmp[3]) : "i"(6 + NX) : "memory");
  else if constexpr (KT == T_TIE1) asm volatile("s_waitcnt vmcnt(%4)" : "+v"(s.tmp[4]), "+v"(s.tmp[5]), "+v"(s.tmp[6]), "+v"(s.tmp[7]) : "i"(6 + NX) : "memory");
  else if constexpr (KT == T_POLL) asm volatile("s_waitcnt vmcnt(%1)" : "+v"(s.pollv) : "i"(6 + NX) : "memory");
  else wait_v<6 + NX>();
  if constexpr (KT == T_FLAG) PFLAG(s.pflag);
  if constexpr (I == 13) { s.sgBL = s.nxtBL; s.sgBR = s.nxtBR; }
  BAR; mma_step<INR>(s, s.acc[1][1], s.At, s.B1);
  WAIT_B(s.B0);
  ENDM;
#undef BGMEM
  if constexpr (KT == T_POLL) {
    if (!__all(s.pollv >= s.call || (s.vlane16 >> 7) == ((unsigned)s.wslot >> 3))) s.poison |= spin_flags(s, s.pflag);
    SCHED;
  }
}

__device__ __forceinline__ void final_pair(St& s, unsigned flagL, unsigned own8, unsigned adj8) {
  LDA(0u, 0); STAGE_A(SA(1u, 1), s.a1, hA);
  BAR; WAIT_A(); mma<false>(s.acc[0][0], s.At, s.B0); ENDM;
  LDA(0u, 1);
  BAR; WAIT_A(); mma<false>(s.acc[1][0], s.At, s.B0); ENDM;
  LDB(s.B1, 0u, 1); wait_v<2>();
  BAR; WAIT_B(s.B1); mma<false>(s.acc[1][1], s.At, s.B1); ENDM;
  LDB(s.B0, 1u, 0); LDA(1u, 0); wait_v<0>();
  BAR; WAIT_B(s.B0); WAIT_A(); mma<false>(s.acc[0][0], s.At, s.B0); ENDM;
  LDA(1u, 1); GUARD(s.acc[0][0]); pack_quad(&s.tmp[0], s.acc[0][0]); PST(0); PST(1); PST(2); PST(3);
  BAR; WAIT_A(); mma<false>(s.acc[1][0], s.At, s.B0); ENDM;
  LDB(s.B1, 1u, 1); GUARD(s.acc[1][0]); pack_quad(&s.tmp[4], s.acc[1][0]); PST(4); PST(5); PST(6); PST(7);
  BAR; WAIT_B(s.B1); mma<false>(s.acc[1][1], s.At, s.B1); ENDM;
  LDA(1u, 0);
  BAR; WAIT_A(); mma<false>(s.acc[0][1], s.At, s.B1); ENDM;
  LDB(s.B1, 0u, 1); LDA(0u, 0); wait_v<0>(); PFLAG(flagL);
  s.pw += REGION; s.own = own8; s.vls += adj8;
  pack_quad(&s.tmp[4], s.acc[1][1]); PST(4); PST(5); PST(6); PST(7);
  BAR; WAIT_B(s.B1); WAIT_A(); mma<false>(s.acc[0][1], s.At, s.B1); ENDM;
}

#define STAMP(i) do {} while (0)

__global__ __launch_bounds__(512, 2) void gemm_a2a(Params p) {
  __shared__ __attribute__((aligned(1024))) unsigned char smem[8 * HTB + 32768];
  const int tid = threadIdx.x, wid = __builtin_amdgcn_readfirstlane(tid >> 6), lane = tid & 63;
  const int wr = wid >> 2, wc = wid & 3, fr = lane & 15, fq = lane >> 4;
  const unsigned blk = blockIdx.x, mem = blk & 7u, grp = (blk >> 3) & 31u;
  const unsigned pm = grp & 3u, cg = grp >> 2, sub = mem >> 2, mi = mem & 3u;
  const unsigned rank = (unsigned)p.rank & 7u;
  STAMP(0);
  St s;
  char* wsp = p.ws[rank];
  const unsigned long long pa = (unsigned long long)p.A, pw = (unsigned long long)p.W, pws = (unsigned long long)wsp;
  s.srdA = (u32x4){(unsigned)pa, (unsigned)(pa >> 32) & 0xffffu, A_BYTES, 0x20000u};
  s.srdB = (u32x4){(unsigned)pw, (unsigned)(pw >> 32) & 0xffffu, W_BYTES, 0x20000u};
  s.srdWs = (u32x4){(unsigned)pws, (unsigned)(pws >> 32) & 0xffffu, WS_USED, 0x20000u};
  unsigned info;
  if (cg < 7u) {
    const unsigned j = (cg + 1u) << 3;
    info = (j | 0u) | ((j | (1u + sub)) << 6) | ((j | (3u + sub)) << 12) | ((j | (5u + sub)) << 18) | (cg << 24);
  } else {
    const unsigned j0 = (0x6427u >> (4u * pm)) & 7u;
    const unsigned j1 = ((4u + 2u * pm + sub) % 7u) + 1u, j2 = ((2u + 2u * pm + sub) % 7u) + 1u, j3 = ((2u * pm + sub) % 7u) + 1u;
    info = ((j0 << 3) | 7u) | (((j1 << 3) | 7u) << 6) | (((j2 << 3) | 7u) << 12) | (((j3 << 3) | 7u) << 18) | (7u << 24);
  }
  auto ent = [&](unsigned k) -> unsigned { return (info >> (6u * k)) & 63u; };
  auto colB = [&](unsigned e) -> unsigned { return (((rank + (e >> 3)) & 7u) * 8u + (e & 7u)) * 256u; };
  auto offB = [&](unsigned k) -> unsigned { return mi * (32u * ktB) + colB(ent(k)); };
  const unsigned lds0 = (unsigned)(size_t)(LAS unsigned char*)smem;
  s.ldsw = lds0 + (unsigned)wid * 1024u;
  s.voffA = (unsigned)((8 * wid + (lane >> 3)) * (K_ * 2) + (((lane & 7) ^ (lane >> 3)) << 4));
  {
    const unsigned fB = (unsigned)((lane >> 4) | (((wid >> 1) & 1) << 2));
    s.voffB = (unsigned)((4 * wid + (lane >> 4)) * (N_ * 2)) + ((((unsigned)lane & 15u) ^ (fB << 1)) << 4);
  }
  s.sgA = pm * (256u * K_ * 2u) + mi * (32u * ktA); s.sgBL = offB(1); s.sgBR = offB(0);
  {
    const unsigned k0 = (sub * 16u + pm) * ktA;
    const unsigned a0 = s.sgA + k0, l0 = s.sgBL + (k0 << 13), q0 = s.sgBR + (k0 << 13);
    STAGE_B(SB(0, 0), l0, 0u); STAGE_A(SA(0, 0), a0, 0u); STAGE_B(SB(0, 1), q0, 0u); STAGE_A(SA(0, 1), a0, hA);
    STAGE_B(SB(1, 0), l0, ktB); STAGE_A(SA(1, 0), a0, ktA); STAGE_B(SB(1, 1), q0, ktB);
    s.a1 = a0 + ktA; s.a2 = a0 + 2u * ktA; s.bl2 = l0 + 2u * ktB; s.br2 = q0 + 2u * ktB; s.kA = k0 + 2u * ktA;
  }
  unsigned callv;
  asm volatile("buffer_load_dword %0, off, %1, %2 sc1" : "=v"(callv) : "s"(s.srdWs), "s"(WS_CALL + blk * 64u) : "memory");
  unsigned olo[3], ohi[3];
#pragma unroll
  for (int k = 0; k < 3; ++k) {
    const unsigned long long o = (unsigned long long)p.out[(rank + (ent(k) >> 3)) & 7u];
    olo[k] = (unsigned)o; ohi[k] = (unsigned)(o >> 32) & 0xffffu;
  }
  {
    const int q = (lane & 15) >> 2, pp = lane & 3, g = lane >> 4;
    const int f = q | ((g & 1) << 2);
    s.aoff0 = lds0 + (unsigned)((wr * 64 + fr) * 128 + (((0 + fq) ^ (fr & 7)) << 4));
    s.aoff1 = lds0 + (unsigned)((wr * 64 + fr) * 128 + (((4 + fq) ^ (fr & 7)) << 4));
    s.boff0 = lds0 + 65536u + (unsigned)((8 * g + q) * 256 + (((2 * wc + 0) ^ f) << 5) + 8 * pp);
    s.boff1 = lds0 + 65536u + (unsigned)((8 * g + q) * 256 + (((2 * wc + 1) ^ f) << 5) + 8 * pp);
  }
  s.vlane16 = (unsigned)lane * 16u;
  {
    const unsigned r = (unsigned)lane >> 3, c = (unsigned)lane & 7u, cc = c & 3u;
    const unsigned sfq = ((cc & 1u) << 1) | (cc >> 1);
    s.vgat = (c >> 2) * 4096u + (sfq * 16u + r) * 16u;
    s.vdst = r * 2048u + c * 16u;
  }
  const unsigned uw = (unsigned)wid, rwr = uw >> 2, rwcp = uw & 1u, b1 = (uw >> 1) & 1u;
  const unsigned ai4 = mi >> 1, m4 = 2u * (mi & 1u) + b1, ai8 = mem >> 2, m8 = mem & 3u;
  unsigned rd4 = blk * SLOT + ((ai4 * 8u + rwr * 4u + 2u * rwcp) * 4u + m4) * 1024u;
  unsigned rd8 = blk * SLOT + ((ai8 * 8u + rwr * 4u + 2u * rwcp) * 4u + m8) * 1024u + b1 * 128u;
  const unsigned row0 = rank * 1024u + pm * 256u;
  unsigned ds4 = (row0 + ai4 * 128u + rwr * 64u + m4 * 16u) * 2048u + rwcp * 128u;
  unsigned ds8 = (row0 + ai8 * 128u + rwr * 64u + m8 * 16u + b1 * 8u) * 2048u + rwcp * 128u;
  unsigned pw0 = WS_PART + blk * SLOT + uw * 4096u;
  s.wslot = (int)(mem * 8u + uw);
  unsigned adj8 = (((2u * (mi & 1u)) ^ 2u) - m8) * 1024u;
  const unsigned lsp = lds0 + 8u * HTB;
  unsigned tsl;
  {
    const unsigned key = grp * 3u + mem + uw;
    constexpr unsigned F0 = S0_FIRST - 4, F1 = S1_FIRST - 4;
    constexpr unsigned W2 = (20 - S0_FIRST) / 2;
    constexpr unsigned G1 = (S0_FIRST + 4u > S1_FIRST ? S0_FIRST + 4u : S1_FIRST) - 4;
    constexpr unsigned G2 = (F0 + W2 > F1 ? F0 + W2 : F1);
    tsl = ((F1 + key % (16u - F1)) << 4) | (((F0 + key % W2) | ((G2 + key % (16u - G2)) << 4)) << 8) | (((F0 + key % 4u) | ((G1 + key % 4u) << 4)) << 16);
    static_assert(G1 + 3u + 4u <= 13u, "the last segment's pushes must leave before the final K-tile pair");
  }
  s.ts0 = -1; s.ts1 = -1; s.poison = 0u;
  s.d0lo = s.d0hi = s.d0off = s.d1lo = s.d1hi = s.d1off = 0u;
  asm volatile("" : "+s"(s.d0lo), "+s"(s.d0hi), "+s"(s.d0off), "+s"(s.d1lo), "+s"(s.d1hi), "+s"(s.d1off), "+s"(s.ts0), "+s"(s.ts1));
  s.R0 = (u32x4){0u, 0u, 0u, 0u}; s.R1 = s.R0;
#pragma unroll
  for (int a = 0; a < 2; ++a)
#pragma unroll
    for (int bb = 0; bb < 2; ++bb)
#pragma unroll
      for (int m = 0; m < 4; ++m)
#pragma unroll
        for (int n = 0; n < 2; ++n) asm volatile("" : "=v"(s.acc[a][bb][m][n]));
  s.nxtBL = s.sgBL; s.nxtBR = offB(2);
  unsigned bl2 = offB(3), br3 = offB(4);
  unsigned do0 = ds8 + (ent(0) & 7u) * 256u, do1 = ds4 + (ent(1) & 7u) * 256u, do2 = ds4 + (ent(2) & 7u) * 256u;
  unsigned pf0 = WS_PFLAG + grp * 1024u;
  asm volatile("" : "+s"(bl2), "+s"(br3), "+s"(do0), "+s"(do1), "+s"(do2), "+s"(pf0));
  asm volatile("s_waitcnt lgkmcnt(0)" : "+s"(rd4), "+s"(rd8), "+s"(ds4), "+s"(ds8), "+s"(pw0), "+s"(adj8), "+s"(tsl), "+s"(s.wslot), "+s"(s.nxtBL), "+s"(s.nxtBR) :: "memory");
  if (wr == 1) BAR;
  wait_v<11>(); BAR;
  BAR;
  STAMP(1);
  {
    LDB(s.B0, 0u, 0); WAIT_B(s.B0);
    ktile<0, V_FIRST>(s);
    asm volatile("" : "+v"(callv));
    s.call = __builtin_amdgcn_readfirstlane(callv) + 1u;
    ktile<1, V_PLAIN>(s);
    for (int t = 2; t < 12; t += 2) { ktile<0, V_PLAIN>(s); ktile<1, V_PLAIN>(s); }
    ktile<12, V_PLAIN>(s); ktile<13, V_PLAIN>(s);
  }
  const unsigned call = s.call;
  for (unsigned seg = 1; seg <= 3u; ++seg) {
    const unsigned k = seg - 1u;
    s.pw = pw0 + k * REGION;
    s.own = k == 0u ? 1u << mem : 3u << (2u * mi);
    s.vls = s.vlane16 + (lsp + uw * 4096u + (k == 0u ? adj8 : 0u));
    ktile<14, V_BG>(s);
    if (seg == 2u) {
      ktile<15, V_L>(s);
      ktile<0, V_L>(s);
    }
    unsigned seg2 = seg; asm volatile("" : "+s"(seg2));
    if (seg2 != 2u) {
      ktile<15, V_R>(s);
      ktile<0, V_R>(s);
    }
    STAMP(1 + seg);
    s.keep = (unsigned)((int)(k - 1u) >> 31);
    s.prd0 = WS_PART + k * REGION + (k == 0u ? rd8 : rd4);
    s.prd1 = k == 0u ? s.prd0 ^ (4u * SLOT) : s.prd0 + 128u;
    s.lrd = lsp + (s.prd0 & 0x7fffu) + (k == 0u ? adj8 : 0u);
    s.pflag = pf0 + k * 32768u;
    s.TS0 = (int)(tsl >> (8u * k));
    s.pdlo = k == 0u ? olo[0] : k == 1u ? olo[1] : olo[2];
    s.pdhi = k == 0u ? ohi[0] : k == 1u ? ohi[1] : ohi[2];
    s.pdoff = k == 0u ? do0 : k == 1u ? do1 : do2;
    s.nxtBL = bl2;
    s.nxtBR = k == 0u ? s.sgBR : br3;
    asm volatile("" : "+s"(s.nxtBL), "+s"(s.nxtBR), "+s"(s.pdoff), "+s"(s.pdlo), "+s"(s.pdhi), "+s"(s.pflag), "+s"(s.prd0), "+s"(s.prd1), "+s"(s.lrd), "+s"(s.TS0), "+s"(s.keep));
    ktile<1, V_BG>(s); ktile<2, V_BG>(s); ktile<3, V_BG>(s); ktile<4, V_BG>(s); ktile<5, V_BG>(s); ktile<6, V_BG>(s); ktile<7, V_BG>(s);
    ktile<8, V_BG>(s); ktile<9, V_BG>(s); ktile<10, V_BG>(s); ktile<11, V_BG>(s); ktile<12, V_BG>(s); ktile<13, V_BG>(s);
  }
  s.pw = pw0 + 3u * REGION;
  const unsigned fblk = pf0 + 3u * 32768u;
  final_pair(s, fblk, 1u << mem, adj8);
  STAMP(5);
  GUARD(s.acc[0][1]); pack_quad(&s.tmp[0], s.acc[0][1]); PST(0); PST(1); PST(2); PST(3);
  asm volatile("s_waitcnt vmcnt(0) lgkmcnt(0)" ::: "memory"); PFLAG(fblk + 32768u);
  STAMP(6);
  const int lane2 = lane_id();
  const unsigned long long o3 = (unsigned long long)p.out[(rank + (ent(3) >> 3)) & 7u], o4 = (unsigned long long)p.out[rank];
  const unsigned o3lo = (unsigned)o3, o3hi = (unsigned)(o3 >> 32) & 0xffffu, o4lo = (unsigned)o4, o4hi = (unsigned)(o4 >> 32) & 0xffffu;
  if (wr == 0) BAR;
  LAS unsigned* lflag = (LAS unsigned*)(smem + 8 * HTB + ((((2u * (mi & 1u)) ^ 2u) + 1u) * 1024u));
  if (wid == 0) {
    const unsigned bad = spin_flags(s, fblk);
    if (lane2 == 0) lflag[0] = bad;
    asm volatile("s_waitcnt lgkmcnt(0)" ::: "memory");
  }
  BAR;
  STAMP(7);
  {
    s.prd0 = WS_PART + 3u * REGION + rd4; s.prd1 = s.prd0 + 128u; s.lrd = lsp + (rd4 & 0x7fffu);
    PLDG(1); PLDG(2); PLDG(3); PLDG(5); PLDG(6); PLDG(7); PLDL(0, 0); PLDL(4, 128);
    make_sel(s);
    if (wid == 0) {
      const unsigned bad = spin_flags(s, fblk + 32768u);
      if (lane2 == 0) lflag[1] = bad;
      asm volatile("s_waitcnt lgkmcnt(0)" ::: "memory");
    }
    s.poison |= __builtin_amdgcn_readfirstlane(*(volatile LAS unsigned*)lflag);
    asm volatile("s_waitcnt vmcnt(3) lgkmcnt(0)" : "+v"(s.tmp[0]), "+v"(s.tmp[1]), "+v"(s.tmp[2]), "+v"(s.tmp[3]) :: "memory");
    reduce4<1>(s, 0);
    const u32x4 o0 = reduce_fin<true>(s);
    asm volatile("s_waitcnt vmcnt(0)" : "+v"(s.tmp[4]), "+v"(s.tmp[5]), "+v"(s.tmp[6]), "+v"(s.tmp[7]) :: "memory");
    reduce4<1>(s, 4);
    const u32x4 o1 = reduce_fin<true>(s);
    BAR;
    s.prd0 = WS_PART + 4u * REGION + rd8; s.prd1 = s.prd0 ^ (4u * SLOT); s.lrd = lsp + (rd8 & 0x7fffu) + adj8;
    PLDG(1); PLDG(2); PLDG(3); PLDG(4); PLDG(5); PLDG(6); PLDG(7); PLDL(0, 0);
    const unsigned doff = ds4 + (ent(3) & 7u) * 256u;
    SEND(o0, o3lo, o3hi, doff);
    SEND(o1, o3lo, o3hi, doff + 8u * 2048u);
    s.poison |= __builtin_amdgcn_readfirstlane(*(volatile LAS unsigned*)(lflag + 1));
    asm volatile("s_waitcnt vmcnt(2) lgkmcnt(0)" : "+v"(s.tmp[0]), "+v"(s.tmp[1]), "+v"(s.tmp[2]), "+v"(s.tmp[3]), "+v"(s.tmp[4]), "+v"(s.tmp[5]), "+v"(s.tmp[6]), "+v"(s.tmp[7]) :: "memory");
    reduce4<1>(s, 0); reduce4<0>(s, 4);
    const u32x4 o2 = reduce_fin<true>(s);
    SEND(o2, o4lo, o4hi, ds8 + (ent(4) & 7u) * 256u);
  }
  STAMP(8);
  wait_v<0>();
  STAMP(9);
  BAR;
  if (wid == 0) {
    if (lane2 < 4) {
      const unsigned e = (info >> (6u * (unsigned)lane2)) & 63u;
      const unsigned dp = (rank + (e >> 3)) & 7u;
      unsigned* f = (unsigned*)(p.ws[dp] + WS_DONE + rank * 1024u) + ((e & 7u) * 4u + pm) * 8u;
      if (lane2 == 0) __hip_atomic_store(f + mem, call, __ATOMIC_RELAXED, __HIP_MEMORY_SCOPE_SYSTEM);
      else __hip_atomic_store((unsigned long long*)f + mi, ((unsigned long long)call << 32) | call, __ATOMIC_RELAXED, __HIP_MEMORY_SCOPE_SYSTEM);
    }
    if (lane2 == 0) __hip_atomic_store((unsigned*)(wsp + WS_CALL + blk * 64u), call, __ATOMIC_RELAXED, __HIP_MEMORY_SCOPE_AGENT);
    if (blk == 0) {
      const unsigned long long* f = (const unsigned long long*)(wsp + WS_DONE) + lane2;
      unsigned* tmo = (unsigned*)(wsp + WS_TMO);
      const unsigned long long t0 = __builtin_amdgcn_s_memrealtime();
      for (unsigned n = 1;; ++n) {
        bool ok = true;
#pragma unroll
        for (unsigned j = 1; j < 8; ++j) {
          const unsigned long long v0 = __hip_atomic_load(f + ((rank + j) & 7u) * 128u, __ATOMIC_RELAXED, __HIP_MEMORY_SCOPE_SYSTEM);
          const unsigned long long v1 = __hip_atomic_load(f + ((rank + j) & 7u) * 128u + 64u, __ATOMIC_RELAXED, __HIP_MEMORY_SCOPE_SYSTEM);
          ok = ok && ((unsigned)v0 >= call) && ((unsigned)(v0 >> 32) >= call) && ((unsigned)v1 >= call) && ((unsigned)(v1 >> 32) >= call);
        }
        if (__all(ok)) break;
        if ((n & 15u) == 0u &&
            (__builtin_amdgcn_readfirstlane(__hip_atomic_load(tmo, __ATOMIC_RELAXED, __HIP_MEMORY_SCOPE_AGENT)) == call ||
             __builtin_amdgcn_s_memrealtime() - t0 > DEADLINE)) {
          if (lane2 == 0) {
            __hip_atomic_store(tmo, call, __ATOMIC_RELAXED, __HIP_MEMORY_SCOPE_AGENT);
            __hip_atomic_store((unsigned*)(p.out[rank] + (size_t)rank * 1024u * 2048u), 0x7fc07fc0u, __ATOMIC_RELAXED, __HIP_MEMORY_SCOPE_SYSTEM);
          }
          break;
        }
      }
      __builtin_amdgcn_fence(__ATOMIC_ACQUIRE, "");
    }
  }
}

}

extern "C" void kernel_launch(int rank, int nranks, ncclComm_t comm,
                              hipStream_t stream,
                              void* const* d_in, const int* in_sizes, int n_in,
                              void* d_out, int out_size,
                              const int* in_dtypes, int out_dtype,
                              void* const* d_out_all,
                              void* const* d_ws_all, size_t ws_bytes) {
  using namespace ga2a;
  (void)comm; (void)in_sizes; (void)n_in; (void)d_out; (void)out_size; (void)in_dtypes; (void)out_dtype;
  if (nranks != NR || ws_bytes < (size_t)WS_USED) { fprintf(stderr, "unsupported configuration\n"); return; }
  Params p;
  p.A = (const u16*)d_in[0];
  p.W = (const u16*)d_in[1];
  for (int i = 0; i < NR; ++i) { p.out[i] = (char*)d_out_all[i]; p.ws[i] = (char*)d_ws_all[i]; }
  p.rank = rank;
  hipLaunchKernelGGL(gemm_a2a, dim3(256), dim3(512), 0, stream, p);
  HIP_CHECK(hipGetLastError());
}
```

```cpp
#include <hip/hip_runtime.h>
#include <rccl/rccl.h>
#include <cstdio>

#define HIP_CHECK(expr) do { hipError_t _e = (expr); if (_e != hipSuccess) { \
    fprintf(stderr, "HIP error %s at %s:%d\n", hipGetErrorString(_e), __FILE__, __LINE__); \
    return; } } while (0)

namespace ga2a {

#define LAS __attribute__((address_space(3)))
typedef unsigned short u16;
typedef short bf16x8 __attribute__((ext_vector_type(8)));
typedef short bf16x4 __attribute__((ext_vector_type(4)));
typedef float f32x4 __attribute__((ext_vector_type(4)));
typedef float f32x2 __attribute__((ext_vector_type(2)));
typedef __bf16 bf16x2n __attribute__((ext_vector_type(2)));
typedef unsigned u32x4 __attribute__((ext_vector_type(4)));

constexpr int NR = 8;
constexpr int M_ = 1024, N_ = 8192, K_ = 8192;
constexpr unsigned HTB = 16384;
constexpr unsigned A_BYTES = (unsigned)M_ * K_ * 2u, W_BYTES = (unsigned)K_ * N_ * 2u;
constexpr unsigned OUT_BYTES = 8192u * 1024u * 2u;
constexpr unsigned SLOT = 65536u;
constexpr unsigned REGION = 256u * SLOT;
constexpr unsigned WS_PART = 0u;
constexpr unsigned WS_CTRL = 128u << 20;
constexpr unsigned WS_PFLAG = WS_CTRL + 524288u;
constexpr unsigned WS_CALL = WS_CTRL + 131072u;
constexpr unsigned WS_DONE = WS_CTRL + 163840u;
constexpr unsigned WS_TMO = WS_CTRL + 180224u;
constexpr unsigned WS_USED = WS_CTRL + (1u << 20);
constexpr unsigned long long DEADLINE = 50000000ull;

constexpr unsigned ktA = 128u, ktB = 64u * N_ * 2u;
constexpr unsigned hA = 128u * K_ * 2u;
constexpr unsigned iA = 64u * K_ * 2u, iB = 32u * N_ * 2u;

struct Params {
  const u16* A;
  const u16* W;
  char* out[NR];
  char* ws[NR];
  int rank;
};

enum { V_PLAIN = 0, V_BG = 1, V_L = 2, V_R = 3, V_FIRST = 4 };
constexpr int PSTP[8] = {-1, -1, 0, 0, 1, 1, 2, 2};
constexpr int PLDP[8] = {13, 14, 15, 16, 17, 18, 19, 20};
constexpr int T_FLAG = 1, T_POLL = 2;
constexpr int T_TIE0 = PLDP[3] / 4 + 1, T_TIE1 = PLDP[7] / 4 + 1;
constexpr int S0_FIRST = T_TIE0 + 1, S0_N = 16 - S0_FIRST, S1_FIRST = T_TIE1 + 1, S1_N = 20 - S1_FIRST;
constexpr int count_at(const int (&t)[8], int kt) { int n = 0; for (int i = 0; i < 8; ++i) n += (t[i] >= kt * 4 && t[i] < kt * 4 + 4) ? 1 : 0; return n; }
constexpr int nbg(int p) { int n = 0; for (int i = 0; i < 8; ++i) n += (PSTP[i] == p ? 1 : 0) + ((PLDP[i] == p && i != 0 && i != 4) ? 1 : 0); return n; }
constexpr int lag5(int p) { int n = 10; for (int q = p - 5; q <= p; ++q) n += nbg(q); return n; }
constexpr bool slot_at(int first, int n, int i) { return (i >= first && i < first + n) || (i + 16 >= first && i + 16 < first + n); }

__device__ __forceinline__ unsigned pk2(float lo, float hi) {
  f32x2 v = {lo, hi};
  return __builtin_bit_cast(unsigned, __builtin_convertvector(v, bf16x2n));
}
__device__ __forceinline__ u32x4 pack_pair(const f32x4& v0, const f32x4& v1) {
  unsigned d00 = pk2(v0[0], v0[1]), d01 = pk2(v0[2], v0[3]);
  unsigned d10 = pk2(v1[0], v1[1]), d11 = pk2(v1[2], v1[3]);
  auto r0 = __builtin_amdgcn_permlane16_swap(d00, d10, false, false);
  auto r1 = __builtin_amdgcn_permlane16_swap(d01, d11, false, false);
  u32x4 w; w.x = r0[0]; w.y = r1[0]; w.z = r0[1]; w.w = r1[1];
  return w;
}

struct St {
  f32x4 acc[2][2][4][2];
  bf16x8 At[4][2];
  bf16x4 B0[2][2][2], B1[2][2][2];
  u32x4 tmp[8];
  f32x4 D1, D2;
  u32x4 S1, S2;
  u32x4 R0, R1;
  unsigned pollv;
  unsigned voffA, voffB, aoff0, aoff1, boff0, boff1, vlane16, vgat, vdst;
  u32x4 srdA, srdB, srdWs;
  unsigned ldsw;
  unsigned a1, a2, bl2, br2, kA;
  unsigned sgA, sgBL, sgBR, nxtBL, nxtBR;
  unsigned pw;
  unsigned prd0, prd1;
  unsigned own, lrd;
  unsigned vls;
  unsigned pflag;
  unsigned keep;
  unsigned call, poison;
  unsigned pdlo, pdhi, pdoff;
  unsigned d0lo, d0hi, d0off, d1lo, d1hi, d1off;
  int ts0, ts1, TS0, wslot;
};

#define SA(b, h) (((b) * 2u + (h)) * HTB)
#define SB(b, h) ((4u + (b) * 2u + (h)) * HTB)
#define DMA2(ldsoff, voff, srd, sbase, simm, step) do { unsigned _t; asm volatile( \
    "s_add_u32 m0, %1, %2\n\ts_add_u32 %0, %6, %7\n\tbuffer_load_dwordx4 %4, %5, %0 offen lds\n\t" \
    "s_add_u32 m0, %1, %3\n\ts_add_u32 %0, %6, %8\n\tbuffer_load_dwordx4 %4, %5, %0 offen lds" \
    : "=&s"(_t) : "s"(s.ldsw), "i"(ldsoff), "i"((ldsoff) + 8192u), "v"(voff), "s"(srd), "s"(sbase), "i"(simm), "i"((simm) + (step)) : "memory", "scc"); } while (0)
#define STAGE_A(bufoff, sbase, simm) DMA2((bufoff), s.voffA, s.srdA, (sbase), (simm), iA)
#define STAGE_B(bufoff, sbase, simm) DMA2((bufoff), s.voffB, s.srdB, (sbase), (simm), iB)
__device__ __forceinline__ int lane_id() { int l; asm volatile("v_mbcnt_lo_u32_b32 %0, -1, 0\n\tv_mbcnt_hi_u32_b32 %0, -1, %0" : "=v"(l)); return l; }
#define RD128(dst, addr, off) asm volatile("ds_read_b128 %0, %1 offset:%2" : "=v"(dst) : "v"(addr), "i"(off))
#define RDTR(dst, addr, off) asm volatile("ds_read_b64_tr_b16 %0, %1 offset:%2" : "=v"(dst) : "v"(addr), "i"(off))
#define LDA(b, h) do { _Pragma("unroll") for (int m = 0; m < 4; ++m) { RD128(s.At[m][0], s.aoff0, SA(b, h) + m * 2048); RD128(s.At[m][1], s.aoff1, SA(b, h) + m * 2048); } } while (0)
#define LDB(dst, b, h) do { _Pragma("unroll") for (int k = 0; k < 2; ++k) { \
    RDTR(dst[0][k][0], s.boff0, SB(b, h) - 65536u + k * 8192); RDTR(dst[0][k][1], s.boff0, SB(b, h) - 65536u + k * 8192 + 1024); } \
    _Pragma("unroll") for (int k = 0; k < 2; ++k) { \
    RDTR(dst[1][k][0], s.boff1, SB(b, h) - 65536u + k * 8192); RDTR(dst[1][k][1], s.boff1, SB(b, h) - 65536u + k * 8192 + 1024); } } while (0)
#define WAIT_A() asm volatile("s_waitcnt lgkmcnt(0)" : "+v"(s.At[0][0]), "+v"(s.At[0][1]), "+v"(s.At[1][0]), "+v"(s.At[1][1]), "+v"(s.At[2][0]), "+v"(s.At[2][1]), "+v"(s.At[3][0]), "+v"(s.At[3][1]) :: "memory")
#define WAIT_B(Bt) asm volatile("s_waitcnt lgkmcnt(0)" : "+v"(Bt[0][0][0]), "+v"(Bt[0][0][1]), "+v"(Bt[0][1][0]), "+v"(Bt[0][1][1]), "+v"(Bt[1][0][0]), "+v"(Bt[1][0][1]), "+v"(Bt[1][1][0]), "+v"(Bt[1][1][1]) :: "memory")
#define CAT(x) __builtin_shufflevector(x[0], x[1], 0, 1, 2, 3, 4, 5, 6, 7)
#define BAR __builtin_amdgcn_s_barrier()
#define SCHED __builtin_amdgcn_sched_barrier(0)
#define ENDM do { SCHED; asm volatile("s_setprio 0"); BAR; SCHED; } while (0)
#define ST128(data, voff, srd, soff, bits) asm volatile("buffer_store_dwordx4 %0, %1, %2, %3 offen " bits "\n\ts_nop 1" :: "v"(data), "v"(voff), "s"(srd), "s"(soff) : "memory")
#define LD128(dst, voff, srd, soff) asm volatile("buffer_load_dwordx4 %0, %1, %2, %3 offen sc1" : "=v"(dst) : "v"(voff), "s"(srd), "s"(soff) : "memory")
#define PST(u) do { unsigned _t; asm volatile( \
    "s_bitcmp1_b32 %1, %2\n\ts_cselect_b32 %0, %3, %4\n\ts_cbranch_scc0 1f\n\tds_write_b128 %5, %6 offset:%7\n1:\n\t" \
    "s_add_u32 %0, %0, %8\n\tbuffer_store_dwordx4 %6, %9, %10, %0 offen sc1\n\ts_nop 1" \
    : "=&s"(_t) : "s"(s.own), "i"(u), "s"(0xC0000000u), "s"(s.pw), "v"(s.vls), "v"(s.tmp[u]), "i"(((u) & 3) * 1024), "i"(((u) >> 2) * 32768u + ((u) & 3) * 1024u), "v"(s.vlane16), "s"(s.srdWs) : "memory", "scc"); } while (0)
#define PLDG(j) LD128(s.tmp[j], s.vgat, s.srdWs, ((j) < 4 ? s.prd0 : s.prd1) ^ (((j) & 3) * SLOT))
#define PLDL(j, add) do { unsigned _a; asm volatile("v_add_u32 %1, %2, %3\n\tds_read_b128 %0, %1 offset:%4" : "=v"(s.tmp[j]), "=&v"(_a) : "s"(s.lrd), "v"(s.vgat), "i"(add) : "memory"); } while (0)
#define PLD(j) do { if ((j) == 0) PLDL(0, 0); else if ((j) == 4) { if (s.keep != 0u) PLDG(4); unsigned _k = s.keep; asm volatile("" : "+s"(_k)); if (_k == 0u) PLDL(4, 128); } else PLDG(j); } while (0)

template <int N> __device__ __forceinline__ void wait_v() { asm volatile("s_waitcnt vmcnt(%0)" :: "i"(N) : "memory"); }

template <bool INIT>
__device__ __forceinline__ void mma(f32x4 (&acc)[4][2], const bf16x8 (&At)[4][2], const bf16x4 (&Bt)[2][2][2]) {
  const bf16x8 b00 = CAT(Bt[0][0]), b01 = CAT(Bt[0][1]), b10 = CAT(Bt[1][0]), b11 = CAT(Bt[1][1]);
  if constexpr (INIT)
    asm volatile("s_setprio 1\n\ts_nop 0\n\t"
      "v_mfma_f32_16x16x32_bf16 %0, %16, %8, 0\n\t"
      "v_mfma_f32_16x16x32_bf16 %0, %17, %9, %0\n\t"
      "v_mfma_f32_16x16x32_bf16 %1, %18, %8, 0\n\t"
      "v_mfma_f32_16x16x32_bf16 %1, %19, %9, %1\n\t"
      "v_mfma_f32_16x16x32_bf16 %2, %16, %10, 0\n\t"
      "v_mfma_f32_16x16x32_bf16 %2, %17, %11, %2\n\t"
      "v_mfma_f32_16x16x32_bf16 %3, %18, %10, 0\n\t"
      "v_mfma_f32_16x16x32_bf16 %3, %19, %11, %3\n\t"
      "v_mfma_f32_16x16x32_bf16 %4, %16, %12, 0\n\t"
      "v_mfma_f32_16x16x32_bf16 %4, %17, %13, %4\n\t"
      "v_mfma_f32_16x16x32_bf16 %5, %18, %12, 0\n\t"
      "v_mfma_f32_16x16x32_bf16 %5, %19, %13, %5\n\t"
      "v_mfma_f32_16x16x32_bf16 %6, %16, %14, 0\n\t"
      "v_mfma_f32_16x16x32_bf16 %6, %17, %15, %6\n\t"
      "v_mfma_f32_16x16x32_bf16 %7, %18, %14, 0\n\t"
      "v_mfma_f32_16x16x32_bf16 %7, %19, %15, %7\n\t"
      : "+v"(acc[0][0]), "+v"(acc[0][1]), "+v"(acc[1][0]), "+v"(acc[1][1]), "+v"(acc[2][0]), "+v"(acc[2][1]), "+v"(acc[3][0]), "+v"(acc[3][1])
      : "v"(At[0][0]), "v"(At[0][1]), "v"(At[1][0]), "v"(At[1][1]), "v"(At[2][0]), "v"(At[2][1]), "v"(At[3][0]), "v"(At[3][1]), "v"(b00), "v"(b01), "v"(b10), "v"(b11));
  else
    asm volatile("s_setprio 1\n\ts_nop 0\n\t"
      "v_mfma_f32_16x16x32_bf16 %0, %16, %8, %0\n\t"
      "v_mfma_f32_16x16x32_bf16 %0, %17, %9, %0\n\t"
      "v_mfma_f32_16x16x32_bf16 %1, %18, %8, %1\n\t"
      "v_mfma_f32_16x16x32_bf16 %1, %19, %9, %1\n\t"
      "v_mfma_f32_16x16x32_bf16 %2, %16, %10, %2\n\t"
      "v_mfma_f32_16x16x32_bf16 %2, %17, %11, %2\n\t"
      "v_mfma_f32_16x16x32_bf16 %3, %18, %10, %3\n\t"
      "v_mfma_f32_16x16x32_bf16 %3, %19, %11, %3\n\t"
      "v_mfma_f32_16x16x32_bf16 %4, %16, %12, %4\n\t"
      "v_mfma_f32_16x16x32_bf16 %4, %17, %13, %4\n\t"
      "v_mfma_f32_16x16x32_bf16 %5, %18, %12, %5\n\t"
      "v_mfma_f32_16x16x32_bf16 %5, %19, %13, %5\n\t"
      "v_mfma_f32_16x16x32_bf16 %6, %16, %14, %6\n\t"
      "v_mfma_f32_16x16x32_bf16 %6, %17, %15, %6\n\t"
      "v_mfma_f32_16x16x32_bf16 %7, %18, %14, %7\n\t"
      "v_mfma_f32_16x16x32_bf16 %7, %19, %15, %7\n\t"
      : "+v"(acc[0][0]), "+v"(acc[0][1]), "+v"(acc[1][0]), "+v"(acc[1][1]), "+v"(acc[2][0]), "+v"(acc[2][1]), "+v"(acc[3][0]), "+v"(acc[3][1])
      : "v"(At[0][0]), "v"(At[0][1]), "v"(At[1][0]), "v"(At[1][1]), "v"(At[2][0]), "v"(At[2][1]), "v"(At[3][0]), "v"(At[3][1]), "v"(b00), "v"(b01), "v"(b10), "v"(b11));
}
template <bool INIT>
__device__ __forceinline__ void mma2(f32x4& c, const bf16x8& b0, const bf16x8& a0, const bf16x8& b1, const bf16x8& a1) {
  if constexpr (INIT) asm volatile("v_mfma_f32_16x16x32_bf16 %0, %1, %2, 0\n\tv_mfma_f32_16x16x32_bf16 %0, %3, %4, %0" : "+v"(c) : "v"(b0), "v"(a0), "v"(b1), "v"(a1));
  else asm volatile("v_mfma_f32_16x16x32_bf16 %0, %1, %2, %0\n\tv_mfma_f32_16x16x32_bf16 %0, %3, %4, %0" : "+v"(c) : "v"(b0), "v"(a0), "v"(b1), "v"(a1));
}
template <bool INIT>
__device__ __forceinline__ void mma_step(St& s, f32x4 (&acc)[4][2], const bf16x8 (&At)[4][2], const bf16x4 (&Bt)[2][2][2]) {
  const bf16x8 b00 = CAT(Bt[0][0]), b01 = CAT(Bt[0][1]), b10 = CAT(Bt[1][0]), b11 = CAT(Bt[1][1]);
  unsigned t;
  asm volatile("s_setprio 1\n\ts_nop 0");
  mma2<INIT>(acc[0][0], b00, At[0][0], b01, At[0][1]);
  asm volatile("s_add_u32 %0, %0, 0x80\n\ts_and_b32 %0, %0, 0xf80" : "+s"(s.kA) :: "scc");
  mma2<INIT>(acc[0][1], b10, At[0][0], b11, At[0][1]);
  asm volatile("s_mov_b32 %0, %1\n\ts_add_u32 %1, %2, %3" : "=&s"(s.a1), "+s"(s.a2) : "s"(s.sgA), "s"(s.kA) : "scc");
  mma2<INIT>(acc[1][0], b00, At[1][0], b01, At[1][1]);
  asm volatile("s_lshl_b32 %0, %1, 13" : "=s"(t) : "s"(s.kA) : "scc");
  mma2<INIT>(acc[1][1], b10, At[1][0], b11, At[1][1]);
  asm volatile("s_add_u32 %0, %2, %4\n\ts_add_u32 %1, %3, %4" : "=&s"(s.bl2), "=&s"(s.br2) : "s"(s.sgBL), "s"(s.sgBR), "s"(t) : "scc");
  mma2<INIT>(acc[2][0], b00, At[2][0], b01, At[2][1]);
  mma2<INIT>(acc[2][1], b10, At[2][0], b11, At[2][1]);
  mma2<INIT>(acc[3][0], b00, At[3][0], b01, At[3][1]);
  mma2<INIT>(acc[3][1], b10, At[3][0], b11, At[3][1]);
}
__device__ __forceinline__ void pack_quad(u32x4* dst, f32x4 (&q)[4][2]) {
  asm volatile("" : "+v"(q[0][0]), "+v"(q[0][1]), "+v"(q[1][0]), "+v"(q[1][1]), "+v"(q[2][0]), "+v"(q[2][1]), "+v"(q[3][0]), "+v"(q[3][1]));
#pragma unroll
  for (int m = 0; m < 4; ++m) { dst[m] = pack_pair(q[m][0], q[m][1]); asm volatile("" : "+v"(dst[m])); }
}
__device__ __forceinline__ void make_sel(St& s) {
  unsigned l = s.vlane16; asm volatile("" : "+v"(l));
  const unsigned one = ((l >> 8) == ((l >> 6) & 3u)) ? (0x3f80u << (l & 16u)) : 0u;
  const unsigned a = (l & 32u) ? 0u : one, b = (l & 32u) ? one : 0u;
  s.S1 = (u32x4){a, b, 0u, 0u}; s.S2 = (u32x4){0u, 0u, a, b};
  asm volatile("s_nop 1" : "+v"(s.S1), "+v"(s.S2));
}
template <int FIRST>
__device__ __forceinline__ void reduce4(St& s, int j) {
  if constexpr (FIRST == 1)
    asm volatile("v_mfma_f32_16x16x32_bf16 %0, %2, %4, 0\n\tv_mfma_f32_16x16x32_bf16 %1, %3, %4, 0\n\t"
                 "v_mfma_f32_16x16x32_bf16 %0, %2, %5, %0\n\tv_mfma_f32_16x16x32_bf16 %1, %3, %5, %1\n\t"
                 "v_mfma_f32_16x16x32_bf16 %0, %2, %6, %0\n\tv_mfma_f32_16x16x32_bf16 %1, %3, %6, %1\n\t"
                 "v_mfma_f32_16x16x32_bf16 %0, %2, %7, %0\n\tv_mfma_f32_16x16x32_bf16 %1, %3, %7, %1"
                 : "=&v"(s.D1), "=&v"(s.D2) : "v"(s.S1), "v"(s.S2), "v"(s.tmp[j]), "v"(s.tmp[j + 1]), "v"(s.tmp[j + 2]), "v"(s.tmp[j + 3]));
  else if constexpr (FIRST == 2)
    asm volatile("v_mfma_f32_16x16x32_bf16 %0, %2, %4, 0\n\tv_mfma_f32_16x16x32_bf16 %1, %3, %4, 0\n\t"
                 "v_mfma_f32_16x16x32_bf16 %0, %2, %5, %0\n\tv_mfma_f32_16x16x32_bf16 %1, %3, %5, %1\n\t"
                 "v_mfma_f32_16x16x32_bf16 %0, %2, %6, %0\n\tv_mfma_f32_16x16x32_bf16 %1, %3, %6, %1\n\t"
                 "v_mfma_f32_16x16x32_bf16 %0, %2, %7, %0\n\tv_mfma_f32_16x16x32_bf16 %1, %3, %7, %1"
                 : "+v"(s.D1), "+v"(s.D2) : "v"(s.S1), "v"(s.S2), "v"(s.tmp[j]), "v"(s.tmp[j + 1]), "v"(s.tmp[j + 2]), "v"(s.tmp[j + 3]));
  else
    asm volatile("v_mfma_f32_16x16x32_bf16 %0, %2, %4, %0\n\tv_mfma_f32_16x16x32_bf16 %1, %3, %4, %1\n\t"
                 "v_mfma_f32_16x16x32_bf16 %0, %2, %5, %0\n\tv_mfma_f32_16x16x32_bf16 %1, %3, %5, %1\n\t"
                 "v_mfma_f32_16x16x32_bf16 %0, %2, %6, %0\n\tv_mfma_f32_16x16x32_bf16 %1, %3, %6, %1\n\t"
                 "v_mfma_f32_16x16x32_bf16 %0, %2, %7, %0\n\tv_mfma_f32_16x16x32_bf16 %1, %3, %7, %1"
                 : "+v"(s.D1), "+v"(s.D2) : "v"(s.S1), "v"(s.S2), "v"(s.tmp[j]), "v"(s.tmp[j + 1]), "v"(s.tmp[j + 2]), "v"(s.tmp[j + 3]));
}
template <bool NOP>
__device__ __forceinline__ u32x4 reduce_fin(St& s) {
  if constexpr (NOP) asm volatile("s_nop 7" : "+v"(s.D1), "+v"(s.D2)); else asm volatile("" : "+v"(s.D1), "+v"(s.D2));
  u32x4 r;
  r.x = pk2(s.D1[0], s.D1[1]) | s.poison; r.y = pk2(s.D1[2], s.D1[3]) | s.poison;
  r.z = pk2(s.D2[0], s.D2[1]) | s.poison; r.w = pk2(s.D2[2], s.D2[3]) | s.poison;
  asm volatile("" : "+v"(r));
  return r;
}
__device__ __forceinline__ unsigned spin_flags(const St& s, unsigned blk) {
  unsigned bad, v0, v1, st, t0, t1;
  asm volatile(
      "s_memrealtime vcc\n\ts_mov_b32 %0, 0\n\ts_waitcnt lgkmcnt(0)\n\ts_mov_b32 %4, vcc_lo\n"
      "1:\n\t"
      "buffer_load_dword %1, %6, %7, %8 offen sc0 sc1\n\t"
      "buffer_load_dword %2, off, %7, %9 sc0 sc1\n\t"
      "s_memrealtime vcc\n\t"
      "s_waitcnt vmcnt(0) lgkmcnt(0)\n\t"
      "s_sub_u32 %5, vcc_lo, %4\n\t"
      "v_cmp_gt_u32 vcc, %10, %1\n\t"
      "v_readfirstlane_b32 %3, %2\n\t"
      "s_cmp_eq_u64 vcc, 0\n\t"
      "s_cbranch_scc1 3f\n\t"
      "s_cmp_eq_u32 %3, %10\n\t"
      "s_cbranch_scc1 2f\n\t"
      "s_cmp_lt_u32 %5, %11\n\t"
      "s_cbranch_scc1 1b\n"
      "2:\n\t"
      "v_mov_b32 %1, %10\n\ts_mov_b64 exec, 1\n\tbuffer_store_dword %1, off, %7, %9 sc0 sc1\n\ts_mov_b64 exec, -1\n\t"
      "s_mov_b32 %0, 0x7fc07fc0\n\ts_waitcnt vmcnt(0)\n"
      "3:"
      : "=&s"(bad), "=&v"(v0), "=&v"(v1), "=&s"(st), "=&s"(t0), "=&s"(t1)
      : "v"(s.vlane16), "s"(s.srdWs), "s"(blk), "s"(WS_TMO), "s"(s.call), "s"((unsigned)DEADLINE)
      : "memory", "scc", "vcc");
  return bad;
}
#define SEND(R, lo, hi, off) do { const u32x4 _srd = {lo, hi, OUT_BYTES, 0x20000u}; ST128(R, s.vdst, _srd, off, "sc0 sc1"); } while (0)

#define PFLAG(blk) do { unsigned _c; asm volatile("v_mov_b32 %0, %1\n\ts_mov_b64 exec, 1\n\tbuffer_store_dword %0, off, %2, %3 sc1\n\ts_mov_b64 exec, -1" : "=&v"(_c) : "s"(s.call), "s"(s.srdWs), "s"((blk) + (unsigned)s.wslot * 16u) : "memory"); } while (0)
#define GUARD(q) asm volatile("s_nop 7" : "+v"(q[0][0]), "+v"(q[0][1]), "+v"(q[1][0]), "+v"(q[1][1]), "+v"(q[2][0]), "+v"(q[2][1]), "+v"(q[3][0]), "+v"(q[3][1]))

template <int I, int V>
__device__ __forceinline__ void ktile(St& s) {
  constexpr unsigned b = I & 1, nb = b ^ 1;
  constexpr bool BG = (V != V_PLAIN && V != V_FIRST);
  constexpr bool E15 = (I == 15 && (V == V_L || V == V_R)), E0 = (I == 0 && (V == V_L || V == V_R));
  constexpr int hv = (V == V_R) ? 1 : 0;
  constexpr bool INL = (E0 && hv == 0) || V == V_FIRST, INR = (E0 && hv == 1) || V == V_FIRST;
  constexpr bool S0 = BG && slot_at(S0_FIRST, S0_N, I), S1 = BG && slot_at(S1_FIRST, S1_N, I);
  constexpr int KT = E15 ? -1 : (BG && I != 15 && I != 14) ? I : -100;
  constexpr int NX = count_at(PSTP, KT) + count_at(PLDP, KT) - (PLDP[0] / 4 == KT ? 1 : 0) - (PLDP[4] / 4 == KT ? 1 : 0);
#define BGMEM(ph) do { _Pragma("unroll") for (int u = 0; u < 8; ++u) { if (PSTP[u] == KT * 4 + (ph)) PST(u); if (PLDP[u] == KT * 4 + (ph)) PLD(u); } } while (0)
  LDA(b, 0);
  STAGE_A(SA(nb, 1), s.a1, hA);
  BGMEM(0);
  if constexpr (V == V_FIRST) wait_v<11>();
  BAR; WAIT_A(); mma<INL>(s.acc[0][0], s.At, s.B0);
  if constexpr (E0) pack_quad(&s.tmp[4], s.acc[1][hv]);
  if constexpr (KT == T_TIE0 + 1) { make_sel(s); reduce4<1>(s, 0); }
  if constexpr (KT == T_TIE1 + 1 && T_TIE1 != T_TIE0) { if (s.keep != 0u) reduce4<0>(s, 4); unsigned k2 = s.keep; asm volatile("" : "+s"(k2)); if (k2 == 0u) reduce4<2>(s, 4); }
  ENDM;
  LDB(s.B1, b, 1);
  if constexpr (KT == T_POLL) asm volatile("buffer_load_dword %0, %1, %2, %3 offen sc1" : "=v"(s.pollv) : "v"(s.vlane16), "s"(s.srdWs), "s"(s.pflag) : "memory");
  STAGE_B(SB(b, 0), s.bl2, 0u);
  BGMEM(1);
  if constexpr (V == V_FIRST) wait_v<11>();
  BAR; WAIT_B(s.B1); mma<INR>(s.acc[0][1], s.At, s.B1);
  if constexpr (KT == T_TIE0 + 1) { s.R0 = reduce_fin<false>(s); s.d0lo = s.pdlo; s.d0hi = s.pdhi; s.d0off = s.pdoff; s.ts0 = (4 + (s.TS0 & 15)) | (int)s.keep; asm volatile("" : "+s"(s.d0lo), "+s"(s.d0hi), "+s"(s.d0off), "+s"(s.ts0)); }
  if constexpr (KT == T_TIE1 + 1) { s.R1 = reduce_fin<false>(s); s.d1lo = s.pdlo; s.d1hi = s.pdhi; s.d1off = s.pdoff + (~s.keep & (8u * 2048u)); s.ts1 = 4 + ((s.TS0 >> 4) & 15); asm volatile("" : "+s"(s.d1lo), "+s"(s.d1hi), "+s"(s.d1off), "+s"(s.ts1)); }
  ENDM;
  LDA(b, 1);
  STAGE_A(SA(b, 0), s.a2, 0u);
  BGMEM(2);
  if constexpr (S0) { if (s.ts0 == I) SEND(s.R0, s.d0lo, s.d0hi, s.d0off); }
  if constexpr (S1) { if (s.ts1 == (I < S1_FIRST ? I + 16 : I)) SEND(s.R1, s.d1lo, s.d1hi, s.d1off); }
  wait_v<lag5(KT * 4 + 2)>();
  BAR; WAIT_A(); mma<INL>(s.acc[1][0], s.At, s.B0);
  if constexpr (E15) pack_quad(&s.tmp[0], s.acc[0][hv]);
  ENDM;
  LDB(s.B0, nb, 0);
  STAGE_B(SB(b, 1), s.br2, 0u);
  BGMEM(3);
  if constexpr (KT == T_TIE0) asm volatile("s_waitcnt vmcnt(%4)" : "+v"(s.tmp[0]), "+v"(s.tmp[1]), "+v"(s.tmp[2]), "+v"(s.tmp[3]) : "i"(6 + NX) : "memory");
  else if constexpr (KT == T_TIE1) asm volatile("s_waitcnt vmcnt(%4)" : "+v"(s.tmp[4]), "+v"(s.tmp[5]), "+v"(s.tmp[6]), "+v"(s.tmp[7]) : "i"(6 + NX) : "memory");
  else if constexpr (KT == T_POLL) asm volatile("s_waitcnt vmcnt(%1)" : "+v"(s.pollv) : "i"(6 + NX) : "memory");
  else wait_v<6 + NX>();
  if constexpr (KT == T_FLAG) PFLAG(s.pflag);
  if constexpr (I == 13) { s.sgBL = s.nxtBL; s.sgBR = s.nxtBR; }
  BAR; mma_step<INR>(s, s.acc[1][1], s.At, s.B1);
  WAIT_B(s.B0);
  ENDM;
#undef BGMEM
  if constexpr (KT == T_POLL) {
    if (!__all(s.pollv >= s.call || (s.vlane16 >> 7) == ((unsigned)s.wslot >> 3))) s.poison |= spin_flags(s, s.pflag);
    SCHED;
  }
}

__device__ __forceinline__ void final_pair(St& s, unsigned flagL, unsigned own8, unsigned adj8) {
  LDA(0u, 0); STAGE_A(SA(1u, 1), s.a1, hA);
  BAR; WAIT_A(); mma<false>(s.acc[0][0], s.At, s.B0); ENDM;
  LDA(0u, 1);
  BAR; WAIT_A(); mma<false>(s.acc[1][0], s.At, s.B0); ENDM;
  LDB(s.B1, 0u, 1); wait_v<2>();
  BAR; WAIT_B(s.B1); mma<false>(s.acc[1][1], s.At, s.B1); ENDM;
  LDB(s.B0, 1u, 0); LDA(1u, 0); wait_v<0>();
  BAR; WAIT_B(s.B0); WAIT_A(); mma<false>(s.acc[0][0], s.At, s.B0); ENDM;
  LDA(1u, 1); GUARD(s.acc[0][0]); pack_quad(&s.tmp[0], s.acc[0][0]); PST(0); PST(1); PST(2); PST(3);
  BAR; WAIT_A(); mma<false>(s.acc[1][0], s.At, s.B0); ENDM;
  LDB(s.B1, 1u, 1); GUARD(s.acc[1][0]); pack_quad(&s.tmp[4], s.acc[1][0]); PST(4); PST(5); PST(6); PST(7);
  BAR; WAIT_B(s.B1); mma<false>(s.acc[1][1], s.At, s.B1); ENDM;
  LDA(1u, 0);
  BAR; WAIT_A(); mma<false>(s.acc[0][1], s.At, s.B1); ENDM;
  LDB(s.B1, 0u, 1); LDA(0u, 0); wait_v<0>(); PFLAG(flagL);
  s.pw += REGION; s.own = own8; s.vls += adj8;
  pack_quad(&s.tmp[4], s.acc[1][1]); PST(4); PST(5); PST(6); PST(7);
  BAR; WAIT_B(s.B1); WAIT_A(); mma<false>(s.acc[0][1], s.At, s.B1); ENDM;
}

__global__ __launch_bounds__(512, 2) void gemm_a2a(Params p) {
  __shared__ __attribute__((aligned(1024))) unsigned char smem[8 * HTB + 32768];
  const int tid = threadIdx.x, wid = __builtin_amdgcn_readfirstlane(tid >> 6), lane = tid & 63;
  const int wr = wid >> 2, wc = wid & 3, fr = lane & 15, fq = lane >> 4;
  const unsigned blk = blockIdx.x, mem = blk & 7u, grp = (blk >> 3) & 31u;
  const unsigned pm = grp & 3u, cg = grp >> 2, sub = mem >> 2, mi = mem & 3u;
  const unsigned rank = (unsigned)p.rank & 7u;
  St s;
  char* wsp = p.ws[0];
  const unsigned long long pa = (unsigned long long)p.A, pw = (unsigned long long)p.W, pws = (unsigned long long)wsp;
  s.srdA = (u32x4){(unsigned)pa, (unsigned)(pa >> 32) & 0xffffu, A_BYTES, 0x20000u};
  s.srdB = (u32x4){(unsigned)pw, (unsigned)(pw >> 32) & 0xffffu, W_BYTES, 0x20000u};
  s.srdWs = (u32x4){(unsigned)pws, (unsigned)(pws >> 32) & 0xffffu, WS_USED, 0x20000u};
  unsigned info;
  if (cg < 7u) {
    const unsigned j = (cg + 1u) << 3;
    info = (j | 0u) | ((j | (1u + sub)) << 6) | ((j | (3u + sub)) << 12) | ((j | (5u + sub)) << 18) | (cg << 24);
  } else {
    const unsigned j0 = (0x6427u >> (4u * pm)) & 7u;
    const unsigned j1 = ((4u + 2u * pm + sub) % 7u) + 1u, j2 = ((2u + 2u * pm + sub) % 7u) + 1u, j3 = ((2u * pm + sub) % 7u) + 1u;
    info = ((j0 << 3) | 7u) | (((j1 << 3) | 7u) << 6) | (((j2 << 3) | 7u) << 12) | (((j3 << 3) | 7u) << 18) | (7u << 24);
  }
  auto ent = [&](unsigned k) -> unsigned { return (info >> (6u * k)) & 63u; };
  auto colB = [&](unsigned e) -> unsigned { return (((rank + (e >> 3)) & 7u) * 8u + (e & 7u)) * 256u; };
  auto offB = [&](unsigned k) -> unsigned { return mi * (32u * ktB) + colB(ent(k)); };
  const unsigned lds0 = (unsigned)(size_t)(LAS unsigned char*)smem;
  s.ldsw = lds0 + (unsigned)wid * 1024u;
  s.voffA = (unsigned)((8 * wid + (lane >> 3)) * (K_ * 2) + (((lane & 7) ^ (lane >> 3)) << 4));
  {
    const unsigned fB = (unsigned)((lane >> 4) | (((wid >> 1) & 1) << 2));
    s.voffB = (unsigned)((4 * wid + (lane >> 4)) * (N_ * 2)) + ((((unsigned)lane & 15u) ^ (fB << 1)) << 4);
  }
  s.sgA = pm * (256u * K_ * 2u) + mi * (32u * ktA); s.sgBL = offB(1); s.sgBR = offB(0);
  {
    const unsigned k0 = (sub * 16u + pm) * ktA;
    const unsigned a0 = s.sgA + k0, l0 = s.sgBL + (k0 << 13), q0 = s.sgBR + (k0 << 13);
    STAGE_B(SB(0, 0), l0, 0u); STAGE_A(SA(0, 0), a0, 0u); STAGE_B(SB(0, 1), q0, 0u); STAGE_A(SA(0, 1), a0, hA);
    STAGE_B(SB(1, 0), l0, ktB); STAGE_A(SA(1, 0), a0, ktA); STAGE_B(SB(1, 1), q0, ktB);
    s.a1 = a0 + ktA; s.a2 = a0 + 2u * ktA; s.bl2 = l0 + 2u * ktB; s.br2 = q0 + 2u * ktB; s.kA = k0 + 2u * ktA;
  }
  unsigned callv;
  asm volatile("buffer_load_dword %0, off, %1, %2 sc1" : "=v"(callv) : "s"(s.srdWs), "s"(WS_CALL + blk * 64u) : "memory");
  unsigned olo[3], ohi[3];
#pragma unroll
  for (int k = 0; k < 3; ++k) {
    const unsigned long long o = (unsigned long long)p.out[ent(k) >> 3];
    olo[k] = (unsigned)o; ohi[k] = (unsigned)(o >> 32) & 0xffffu;
  }
  {
    const int q = (lane & 15) >> 2, pp = lane & 3, g = lane >> 4;
    const int f = q | ((g & 1) << 2);
    s.aoff0 = lds0 + (unsigned)((wr * 64 + fr) * 128 + (((0 + fq) ^ (fr & 7)) << 4));
    s.aoff1 = lds0 + (unsigned)((wr * 64 + fr) * 128 + (((4 + fq) ^ (fr & 7)) << 4));
    s.boff0 = lds0 + 65536u + (unsigned)((8 * g + q) * 256 + (((2 * wc + 0) ^ f) << 5) + 8 * pp);
    s.boff1 = lds0 + 65536u + (unsigned)((8 * g + q) * 256 + (((2 * wc + 1) ^ f) << 5) + 8 * pp);
  }
  s.vlane16 = (unsigned)lane * 16u;
  {
    const unsigned r = (unsigned)lane >> 3, c = (unsigned)lane & 7u, cc = c & 3u;
    const unsigned sfq = ((cc & 1u) << 1) | (cc >> 1);
    s.vgat = (c >> 2) * 4096u + (sfq * 16u + r) * 16u;
    s.vdst = r * 2048u + c * 16u;
  }
  const unsigned uw = (unsigned)wid, rwr = uw >> 2, rwcp = uw & 1u, b1 = (uw >> 1) & 1u;
  const unsigned ai4 = mi >> 1, m4 = 2u * (mi & 1u) + b1, ai8 = mem >> 2, m8 = mem & 3u;
  unsigned rd4 = blk * SLOT + ((ai4 * 8u + rwr * 4u + 2u * rwcp) * 4u + m4) * 1024u;
  unsigned rd8 = blk * SLOT + ((ai8 * 8u + rwr * 4u + 2u * rwcp) * 4u + m8) * 1024u + b1 * 128u;
  const unsigned row0 = rank * 1024u + pm * 256u;
  unsigned ds4 = (row0 + ai4 * 128u + rwr * 64u + m4 * 16u) * 2048u + rwcp * 128u;
  unsigned ds8 = (row0 + ai8 * 128u + rwr * 64u + m8 * 16u + b1 * 8u) * 2048u + rwcp * 128u;
  unsigned pw0 = WS_PART + blk * SLOT + uw * 4096u;
  s.wslot = (int)(mem * 8u + uw);
  unsigned adj8 = (((2u * (mi & 1u)) ^ 2u) - m8) * 1024u;
  const unsigned lsp = lds0 + 8u * HTB;
  unsigned tsl;
  {
    const unsigned key = grp * 3u + mem + uw;
    constexpr unsigned F0 = S0_FIRST - 4, F1 = S1_FIRST - 4;
    constexpr unsigned W2 = (20 - S0_FIRST) / 2;
    constexpr unsigned G1 = (S0_FIRST + 4u > S1_FIRST ? S0_FIRST + 4u : S1_FIRST) - 4;
    constexpr unsigned G2 = (F0 + W2 > F1 ? F0 + W2 : F1);
    tsl = ((F1 + key % (16u - F1)) << 4) | (((F0 + key % W2) | ((G2 + key % (16u - G2)) << 4)) << 8) | (((F0 + key % 4u) | ((G1 + key % 4u) << 4)) << 16);
    static_assert(G1 + 3u + 4u <= 13u, "the last segment's pushes must leave before the final K-tile pair");
  }
  s.ts0 = -1; s.ts1 = -1; s.poison = 0u;
  s.d0lo = s.d0hi = s.d0off = s.d1lo = s.d1hi = s.d1off = 0u;
  asm volatile("" : "+s"(s.d0lo), "+s"(s.d0hi), "+s"(s.d0off), "+s"(s.d1lo), "+s"(s.d1hi), "+s"(s.d1off), "+s"(s.ts0), "+s"(s.ts1));
  s.R0 = (u32x4){0u, 0u, 0u, 0u}; s.R1 = s.R0;
#pragma unroll
  for (int a = 0; a < 2; ++a)
#pragma unroll
    for (int bb = 0; bb < 2; ++bb)
#pragma unroll
      for (int m = 0; m < 4; ++m)
#pragma unroll
        for (int n = 0; n < 2; ++n) asm volatile("" : "=v"(s.acc[a][bb][m][n]));
  s.nxtBL = s.sgBL; s.nxtBR = offB(2);
  unsigned bl2 = offB(3), br3 = offB(4);
  unsigned do0 = ds8 + (ent(0) & 7u) * 256u, do1 = ds4 + (ent(1) & 7u) * 256u, do2 = ds4 + (ent(2) & 7u) * 256u;
  unsigned pf0 = WS_PFLAG + grp * 1024u;
  asm volatile("" : "+s"(bl2), "+s"(br3), "+s"(do0), "+s"(do1), "+s"(do2), "+s"(pf0));
  asm volatile("s_waitcnt lgkmcnt(0)" : "+s"(rd4), "+s"(rd8), "+s"(ds4), "+s"(ds8), "+s"(pw0), "+s"(adj8), "+s"(tsl), "+s"(s.wslot), "+s"(s.nxtBL), "+s"(s.nxtBR) :: "memory");
  if (wr == 1) BAR;
  wait_v<11>(); BAR;
  BAR;
  {
    LDB(s.B0, 0u, 0); WAIT_B(s.B0);
    ktile<0, V_FIRST>(s);
    asm volatile("" : "+v"(callv));
    s.call = __builtin_amdgcn_readfirstlane(callv) + 1u;
    ktile<1, V_PLAIN>(s);
    for (int t = 2; t < 12; t += 2) { ktile<0, V_PLAIN>(s); ktile<1, V_PLAIN>(s); }
    ktile<12, V_PLAIN>(s); ktile<13, V_PLAIN>(s);
  }
  const unsigned call = s.call;
  for (unsigned seg = 1; seg <= 3u; ++seg) {
    const unsigned k = seg - 1u;
    s.pw = pw0 + k * REGION;
    s.own = k == 0u ? 1u << mem : 3u << (2u * mi);
    s.vls = s.vlane16 + (lsp + uw * 4096u + (k == 0u ? adj8 : 0u));
    ktile<14, V_BG>(s);
    if (seg == 2u) {
      ktile<15, V_L>(s);
      ktile<0, V_L>(s);
    }
    unsigned seg2 = seg; asm volatile("" : "+s"(seg2));
    if (seg2 != 2u) {
      ktile<15, V_R>(s);
      ktile<0, V_R>(s);
    }
    s.keep = (unsigned)((int)(k - 1u) >> 31);
    s.prd0 = WS_PART + k * REGION + (k == 0u ? rd8 : rd4);
    s.prd1 = k == 0u ? s.prd0 ^ (4u * SLOT) : s.prd0 + 128u;
    s.lrd = lsp + (s.prd0 & 0x7fffu) + (k == 0u ? adj8 : 0u);
    s.pflag = pf0 + k * 32768u;
    s.TS0 = (int)(tsl >> (8u * k));
    s.pdlo = k == 0u ? olo[0] : k == 1u ? olo[1] : olo[2];
    s.pdhi = k == 0u ? ohi[0] : k == 1u ? ohi[1] : ohi[2];
    s.pdoff = k == 0u ? do0 : k == 1u ? do1 : do2;
    s.nxtBL = bl2;
    s.nxtBR = k == 0u ? s.sgBR : br3;
    asm volatile("" : "+s"(s.nxtBL), "+s"(s.nxtBR), "+s"(s.pdoff), "+s"(s.pdlo), "+s"(s.pdhi), "+s"(s.pflag), "+s"(s.prd0), "+s"(s.prd1), "+s"(s.lrd), "+s"(s.TS0), "+s"(s.keep));
    ktile<1, V_BG>(s); ktile<2, V_BG>(s); ktile<3, V_BG>(s); ktile<4, V_BG>(s); ktile<5, V_BG>(s); ktile<6, V_BG>(s); ktile<7, V_BG>(s);
    ktile<8, V_BG>(s); ktile<9, V_BG>(s); ktile<10, V_BG>(s); ktile<11, V_BG>(s); ktile<12, V_BG>(s); ktile<13, V_BG>(s);
  }
  s.pw = pw0 + 3u * REGION;
  const unsigned fblk = pf0 + 3u * 32768u;
  final_pair(s, fblk, 1u << mem, adj8);
  GUARD(s.acc[0][1]); pack_quad(&s.tmp[0], s.acc[0][1]); PST(0); PST(1); PST(2); PST(3);
  asm volatile("s_waitcnt vmcnt(0) lgkmcnt(0)" ::: "memory"); PFLAG(fblk + 32768u);
  const int lane2 = lane_id();
  const unsigned long long o3 = (unsigned long long)p.out[ent(3) >> 3], o4 = (unsigned long long)p.out[0];
  char* const w0 = p.ws[ent(0) >> 3]; char* const w1 = p.ws[ent(1) >> 3]; char* const w2 = p.ws[ent(2) >> 3]; char* const w3 = p.ws[ent(3) >> 3];
  const unsigned o3lo = (unsigned)o3, o3hi = (unsigned)(o3 >> 32) & 0xffffu, o4lo = (unsigned)o4, o4hi = (unsigned)(o4 >> 32) & 0xffffu;
  if (wr == 0) BAR;
  LAS unsigned* lflag = (LAS unsigned*)(smem + 8 * HTB + ((((2u * (mi & 1u)) ^ 2u) + 1u) * 1024u));
  if (wid == 0) {
    const unsigned bad = spin_flags(s, fblk);
    if (lane2 == 0) lflag[0] = bad;
    asm volatile("s_waitcnt lgkmcnt(0)" ::: "memory");
  }
  BAR;
  {
    s.prd0 = WS_PART + 3u * REGION + rd4; s.prd1 = s.prd0 + 128u; s.lrd = lsp + (rd4 & 0x7fffu);
    PLDG(1); PLDG(2); PLDG(3); PLDG(5); PLDG(6); PLDG(7); PLDL(0, 0); PLDL(4, 128);
    make_sel(s);
    if (wid == 0) {
      const unsigned bad = spin_flags(s, fblk + 32768u);
      if (lane2 == 0) lflag[1] = bad;
      asm volatile("s_waitcnt lgkmcnt(0)" ::: "memory");
    }
    s.poison |= __builtin_amdgcn_readfirstlane(*(volatile LAS unsigned*)lflag);
    asm volatile("s_waitcnt vmcnt(3) lgkmcnt(0)" : "+v"(s.tmp[0]), "+v"(s.tmp[1]), "+v"(s.tmp[2]), "+v"(s.tmp[3]) :: "memory");
    reduce4<1>(s, 0);
    const u32x4 o0 = reduce_fin<true>(s);
    asm volatile("s_waitcnt vmcnt(0)" : "+v"(s.tmp[4]), "+v"(s.tmp[5]), "+v"(s.tmp[6]), "+v"(s.tmp[7]) :: "memory");
    reduce4<1>(s, 4);
    const u32x4 o1 = reduce_fin<true>(s);
    BAR;
    s.prd0 = WS_PART + 4u * REGION + rd8; s.prd1 = s.prd0 ^ (4u * SLOT); s.lrd = lsp + (rd8 & 0x7fffu) + adj8;
    PLDG(1); PLDG(2); PLDG(3); PLDG(4); PLDG(5); PLDG(6); PLDG(7); PLDL(0, 0);
    const unsigned doff = ds4 + (ent(3) & 7u) * 256u;
    SEND(o0, o3lo, o3hi, doff);
    SEND(o1, o3lo, o3hi, doff + 8u * 2048u);
    s.poison |= __builtin_amdgcn_readfirstlane(*(volatile LAS unsigned*)(lflag + 1));
    asm volatile("s_waitcnt vmcnt(2) lgkmcnt(0)" : "+v"(s.tmp[0]), "+v"(s.tmp[1]), "+v"(s.tmp[2]), "+v"(s.tmp[3]), "+v"(s.tmp[4]), "+v"(s.tmp[5]), "+v"(s.tmp[6]), "+v"(s.tmp[7]) :: "memory");
    reduce4<1>(s, 0); reduce4<0>(s, 4);
    const u32x4 o2 = reduce_fin<true>(s);
    SEND(o2, o4lo, o4hi, ds8 + (ent(4) & 7u) * 256u);
  }
  wait_v<0>();
  BAR;
  if (wid == 0) {
    if (lane2 < 4) {
      const unsigned e = (info >> (6u * (unsigned)lane2)) & 63u;
      char* const wd = lane2 == 0 ? w0 : lane2 == 1 ? w1 : lane2 == 2 ? w2 : w3;
      unsigned* f = (unsigned*)(wd + WS_DONE + rank * 1024u) + ((e & 7u) * 4u + pm) * 8u;
      if (lane2 == 0) __hip_atomic_store(f + mem, call, __ATOMIC_RELAXED, __HIP_MEMORY_SCOPE_SYSTEM);
      else __hip_atomic_store((unsigned long long*)f + mi, ((unsigned long long)call << 32) | call, __ATOMIC_RELAXED, __HIP_MEMORY_SCOPE_SYSTEM);
    }
    if (lane2 == 0) __hip_atomic_store((unsigned*)(wsp + WS_CALL + blk * 64u), call, __ATOMIC_RELAXED, __HIP_MEMORY_SCOPE_AGENT);
    if (blk == 0) {
      const unsigned long long* f = (const unsigned long long*)(wsp + WS_DONE) + lane2;
      unsigned* tmo = (unsigned*)(wsp + WS_TMO);
      const unsigned long long t0 = __builtin_amdgcn_s_memrealtime();
      for (unsigned n = 1;; ++n) {
        bool ok = true;
#pragma unroll
        for (unsigned j = 1; j < 8; ++j) {
          const unsigned long long v0 = __hip_atomic_load(f + ((rank + j) & 7u) * 128u, __ATOMIC_RELAXED, __HIP_MEMORY_SCOPE_SYSTEM);
          const unsigned long long v1 = __hip_atomic_load(f + ((rank + j) & 7u) * 128u + 64u, __ATOMIC_RELAXED, __HIP_MEMORY_SCOPE_SYSTEM);
          ok = ok && ((unsigned)v0 >= call) && ((unsigned)(v0 >> 32) >= call) && ((unsigned)v1 >= call) && ((unsigned)(v1 >> 32) >= call);
        }
        if (__all(ok)) break;
        if ((n & 15u) == 0u &&
            (__builtin_amdgcn_readfirstlane(__hip_atomic_load(tmo, __ATOMIC_RELAXED, __HIP_MEMORY_SCOPE_AGENT)) == call ||
             __builtin_amdgcn_s_memrealtime() - t0 > DEADLINE)) {
          if (lane2 == 0) {
            __hip_atomic_store(tmo, call, __ATOMIC_RELAXED, __HIP_MEMORY_SCOPE_AGENT);
            __hip_atomic_store((unsigned*)(o4 + (size_t)rank * 1024u * 2048u), 0x7fc07fc0u, __ATOMIC_RELAXED, __HIP_MEMORY_SCOPE_SYSTEM);
          }
          break;
        }
      }
      __builtin_amdgcn_fence(__ATOMIC_ACQUIRE, "");
    }
  }
}

}

extern "C" void kernel_launch(int rank, int nranks, ncclComm_t comm,
                              hipStream_t stream,
                              void* const* d_in, const int* in_sizes, int n_in,
                              void* d_out, int out_size,
                              const int* in_dtypes, int out_dtype,
                              void* const* d_out_all,
                              void* const* d_ws_all, size_t ws_bytes) {
  using namespace ga2a;
  (void)comm; (void)in_sizes; (void)n_in; (void)d_out; (void)out_size; (void)in_dtypes; (void)out_dtype;
  if (nranks != NR || ws_bytes < (size_t)WS_USED) { fprintf(stderr, "unsupported configuration\n"); return; }
  Params p;
  p.A = (const u16*)d_in[0];
  p.W = (const u16*)d_in[1];
  for (int j = 0; j < NR; ++j) { p.out[j] = (char*)d_out_all[(rank + j) & 7]; p.ws[j] = (char*)d_ws_all[(rank + j) & 7]; }
  p.rank = rank;
  hipLaunchKernelGGL(gemm_a2a, dim3(256), dim3(512), 0, stream, p);
  HIP_CHECK(hipGetLastError());
}
```

```cpp
#include <hip/hip_runtime.h>
#include <rccl/rccl.h>
#include <cstdio>

#define HIP_CHECK(expr) do { hipError_t _e = (expr); if (_e != hipSuccess) { \
    fprintf(stderr, "HIP error %s at %s:%d\n", hipGetErrorString(_e), __FILE__, __LINE__); \
    return; } } while (0)

namespace ga2a {

#define LAS __attribute__((address_space(3)))
typedef unsigned short u16;
typedef short bf16x8 __attribute__((ext_vector_type(8)));
typedef short bf16x4 __attribute__((ext_vector_type(4)));
typedef float f32x4 __attribute__((ext_vector_type(4)));
typedef float f32x2 __attribute__((ext_vector_type(2)));
typedef __bf16 bf16x2n __attribute__((ext_vector_type(2)));
typedef unsigned u32x4 __attribute__((ext_vector_type(4)));

constexpr int NR = 8;
constexpr int M_ = 1024, N_ = 8192, K_ = 8192;
constexpr unsigned HTB = 16384;
constexpr unsigned A_BYTES = (unsigned)M_ * K_ * 2u, W_BYTES = (unsigned)K_ * N_ * 2u;
constexpr unsigned OUT_BYTES = 8192u * 1024u * 2u;
constexpr unsigned SLOT = 65536u;
constexpr unsigned REGION = 256u * SLOT;
constexpr unsigned WS_PART = 0u;
constexpr unsigned WS_CTRL = 128u << 20;
constexpr unsigned WS_PFLAG = WS_CTRL + 524288u;
constexpr unsigned WS_CALL = WS_CTRL + 131072u;
constexpr unsigned WS_DONE = WS_CTRL + 163840u;
constexpr unsigned WS_TMO = WS_CTRL + 180224u;
constexpr unsigned WS_USED = WS_CTRL + (1u << 20);
constexpr unsigned long long DEADLINE = 50000000ull;

constexpr unsigned ktA = 128u, ktB = 64u * N_ * 2u;
constexpr unsigned hA = 128u * K_ * 2u;
constexpr unsigned iA = 64u * K_ * 2u, iB = 32u * N_ * 2u;

struct Params {
  const u16* A;
  const u16* W;
  char* out[NR];
  char* ws[NR];
  int rank;
};

enum { V_PLAIN = 0, V_BG = 1, V_L = 2, V_R = 3, V_FIRST = 4 };
constexpr int PSTP[8] = {-1, -1, 0, 0, 1, 1, 2, 2};
constexpr int PLDP[8] = {13, 14, 15, 16, 17, 18, 19, 20};
constexpr int T_FLAG = 1, T_POLL = 2;
constexpr int T_TIE0 = PLDP[3] / 4 + 1, T_TIE1 = PLDP[7] / 4 + 1;
constexpr int S0_FIRST = T_TIE0 + 1, S0_N = 16 - S0_FIRST, S1_FIRST = T_TIE1 + 1, S1_N = 20 - S1_FIRST;
constexpr int count_at(const int (&t)[8], int kt) { int n = 0; for (int i = 0; i < 8; ++i) n += (t[i] >= kt * 4 && t[i] < kt * 4 + 4) ? 1 : 0; return n; }
constexpr int nbg(int p) { int n = 0; for (int i = 0; i < 8; ++i) n += (PSTP[i] == p ? 1 : 0) + ((PLDP[i] == p && i != 0 && i != 4) ? 1 : 0); return n; }
constexpr int lag5(int p) { int n = 10; for (int q = p - 5; q <= p; ++q) n += nbg(q); return n; }
constexpr bool slot_at(int first, int n, int i) { return (i >= first && i < first + n) || (i + 16 >= first && i + 16 < first + n); }

__device__ __forceinline__ unsigned pk2(float lo, float hi) {
  f32x2 v = {lo, hi};
  return __builtin_bit_cast(unsigned, __builtin_convertvector(v, bf16x2n));
}
__device__ __forceinline__ u32x4 pack_pair(const f32x4& v0, const f32x4& v1) {
  unsigned d00 = pk2(v0[0], v0[1]), d01 = pk2(v0[2], v0[3]);
  unsigned d10 = pk2(v1[0], v1[1]), d11 = pk2(v1[2], v1[3]);
  auto r0 = __builtin_amdgcn_permlane16_swap(d00, d10, false, false);
  auto r1 = __builtin_amdgcn_permlane16_swap(d01, d11, false, false);
  u32x4 w; w.x = r0[0]; w.y = r1[0]; w.z = r0[1]; w.w = r1[1];
  return w;
}

struct St {
  f32x4 acc[2][2][4][2];
  bf16x8 At[4][2];
  bf16x4 B0[2][2][2], B1[2][2][2];
  u32x4 tmp[8];
  f32x4 D1, D2;
  u32x4 S1, S2;
  u32x4 R0, R1;
  unsigned pollv;
  unsigned voffA, voffB, aoff0, aoff1, boff0, boff1, vlane16, vgat, vdst;
  u32x4 srdA, srdB, srdWs;
  unsigned ldsw, lead;
  unsigned a1, a2, bl2, br2, kA;
  unsigned sgA, sgBL, sgBR, nxtBL, nxtBR;
  unsigned pw;
  unsigned prd0, prd1;
  unsigned own, lrd;
  unsigned vls;
  unsigned pflag;
  unsigned keep;
  unsigned call, poison;
  unsigned pdlo, pdhi, pdoff;
  unsigned d0lo, d0hi, d0off, d1lo, d1hi, d1off;
  int ts0, ts1, TS0, wslot;
};

#define SA(b, h) (((b) * 2u + (h)) * HTB)
#define SB(b, h) ((4u + (b) * 2u + (h)) * HTB)
#define DMA2(ldsoff, voff, srd, sbase, simm, step, bits) do { unsigned _t; asm volatile( \
    "s_add_u32 m0, %1, %2\n\ts_add_u32 %0, %6, %7\n\tbuffer_load_dwordx4 %4, %5, %0 offen " bits " lds\n\t" \
    "s_add_u32 m0, %1, %3\n\ts_add_u32 %0, %6, %8\n\tbuffer_load_dwordx4 %4, %5, %0 offen " bits " lds" \
    : "=&s"(_t) : "s"(s.ldsw), "i"(ldsoff), "i"((ldsoff) + 8192u), "v"(voff), "s"(srd), "s"(sbase), "i"(simm), "i"((simm) + (step)) : "memory", "scc"); } while (0)
#define STAGE_A(bufoff, sbase, simm) DMA2((bufoff), s.voffA, s.srdA, (sbase), (simm), iA, "")
#define STAGE_B(bufoff, sbase, simm) DMA2((bufoff), s.voffB, s.srdB, (sbase), (simm), iB, "")
#define STAGE_BN(bufoff, sbase) do { unsigned _t; asm volatile( \
    "s_add_u32 m0, %1, %2\n\ts_cmp_lg_u32 %8, 0\n\ts_cbranch_scc1 1f\n\t" \
    "buffer_load_dwordx4 %4, %5, %6 offen lds\n\ts_add_u32 m0, %1, %3\n\ts_add_u32 %0, %6, %7\n\tbuffer_load_dwordx4 %4, %5, %0 offen lds\n\ts_branch 2f\n" \
    "1:\n\tbuffer_load_dwordx4 %4, %5, %6 offen nt lds\n\ts_add_u32 m0, %1, %3\n\ts_add_u32 %0, %6, %7\n\tbuffer_load_dwordx4 %4, %5, %0 offen nt lds\n" \
    "2:" \
    : "=&s"(_t) : "s"(s.ldsw), "i"(bufoff), "i"((bufoff) + 8192u), "v"(s.voffB), "s"(s.srdB), "s"(sbase), "i"(iB), "s"(s.lead) : "memory", "scc"); } while (0)
__device__ __forceinline__ int lane_id() { int l; asm volatile("v_mbcnt_lo_u32_b32 %0, -1, 0\n\tv_mbcnt_hi_u32_b32 %0, -1, %0" : "=v"(l)); return l; }
#define RD128(dst, addr, off) asm volatile("ds_read_b128 %0, %1 offset:%2" : "=v"(dst) : "v"(addr), "i"(off))
#define RDTR(dst, addr, off) asm volatile("ds_read_b64_tr_b16 %0, %1 offset:%2" : "=v"(dst) : "v"(addr), "i"(off))
#define LDA(b, h) do { _Pragma("unroll") for (int m = 0; m < 4; ++m) { RD128(s.At[m][0], s.aoff0, SA(b, h) + m * 2048); RD128(s.At[m][1], s.aoff1, SA(b, h) + m * 2048); } } while (0)
#define LDB(dst, b, h) do { _Pragma("unroll") for (int k = 0; k < 2; ++k) { \
    RDTR(dst[0][k][0], s.boff0, SB(b, h) - 65536u + k * 8192); RDTR(dst[0][k][1], s.boff0, SB(b, h) - 65536u + k * 8192 + 1024); } \
    _Pragma("unroll") for (int k = 0; k < 2; ++k) { \
    RDTR(dst[1][k][0], s.boff1, SB(b, h) - 65536u + k * 8192); RDTR(dst[1][k][1], s.boff1, SB(b, h) - 65536u + k * 8192 + 1024); } } while (0)
#define WAIT_A() asm volatile("s_waitcnt lgkmcnt(0)" : "+v"(s.At[0][0]), "+v"(s.At[0][1]), "+v"(s.At[1][0]), "+v"(s.At[1][1]), "+v"(s.At[2][0]), "+v"(s.At[2][1]), "+v"(s.At[3][0]), "+v"(s.At[3][1]) :: "memory")
#define WAIT_B(Bt) asm volatile("s_waitcnt lgkmcnt(0)" : "+v"(Bt[0][0][0]), "+v"(Bt[0][0][1]), "+v"(Bt[0][1][0]), "+v"(Bt[0][1][1]), "+v"(Bt[1][0][0]), "+v"(Bt[1][0][1]), "+v"(Bt[1][1][0]), "+v"(Bt[1][1][1]) :: "memory")
#define CAT(x) __builtin_shufflevector(x[0], x[1], 0, 1, 2, 3, 4, 5, 6, 7)
#define BAR __builtin_amdgcn_s_barrier()
#define SCHED __builtin_amdgcn_sched_barrier(0)
#define ENDM do { SCHED; asm volatile("s_setprio 0"); BAR; SCHED; } while (0)
#define ST128(data, voff, srd, soff, bits) asm volatile("buffer_store_dwordx4 %0, %1, %2, %3 offen " bits "\n\ts_nop 1" :: "v"(data), "v"(voff), "s"(srd), "s"(soff) : "memory")
#define LD128(dst, voff, srd, soff) asm volatile("buffer_load_dwordx4 %0, %1, %2, %3 offen sc1" : "=v"(dst) : "v"(voff), "s"(srd), "s"(soff) : "memory")
#define PST(u) do { unsigned _t; asm volatile( \
    "s_bitcmp1_b32 %1, %2\n\ts_cselect_b32 %0, %3, %4\n\ts_cbranch_scc0 1f\n\tds_write_b128 %5, %6 offset:%7\n1:\n\t" \
    "s_add_u32 %0, %0, %8\n\tbuffer_store_dwordx4 %6, %9, %10, %0 offen sc1\n\ts_nop 1" \
    : "=&s"(_t) : "s"(s.own), "i"(u), "s"(0xC0000000u), "s"(s.pw), "v"(s.vls), "v"(s.tmp[u]), "i"(((u) & 3) * 1024), "i"(((u) >> 2) * 32768u + ((u) & 3) * 1024u), "v"(s.vlane16), "s"(s.srdWs) : "memory", "scc"); } while (0)
#define PLDG(j) LD128(s.tmp[j], s.vgat, s.srdWs, ((j) < 4 ? s.prd0 : s.prd1) ^ (((j) & 3) * SLOT))
#define PLDL(j, add) do { unsigned _a; asm volatile("v_add_u32 %1, %2, %3\n\tds_read_b128 %0, %1 offset:%4" : "=v"(s.tmp[j]), "=&v"(_a) : "s"(s.lrd), "v"(s.vgat), "i"(add) : "memory"); } while (0)
#define PLD(j) do { if ((j) == 0) PLDL(0, 0); else if ((j) == 4) { if (s.keep != 0u) PLDG(4); unsigned _k = s.keep; asm volatile("" : "+s"(_k)); if (_k == 0u) PLDL(4, 128); } else PLDG(j); } while (0)

template <int N> __device__ __forceinline__ void wait_v() { asm volatile("s_waitcnt vmcnt(%0)" :: "i"(N) : "memory"); }

template <bool INIT>
__device__ __forceinline__ void mma(f32x4 (&acc)[4][2], const bf16x8 (&At)[4][2], const bf16x4 (&Bt)[2][2][2]) {
  const bf16x8 b00 = CAT(Bt[0][0]), b01 = CAT(Bt[0][1]), b10 = CAT(Bt[1][0]), b11 = CAT(Bt[1][1]);
  if constexpr (INIT)
    asm volatile("s_setprio 1\n\ts_nop 0\n\t"
      "v_mfma_f32_16x16x32_bf16 %0, %16, %8, 0\n\t"
      "v_mfma_f32_16x16x32_bf16 %0, %17, %9, %0\n\t"
      "v_mfma_f32_16x16x32_bf16 %1, %18, %8, 0\n\t"
      "v_mfma_f32_16x16x32_bf16 %1, %19, %9, %1\n\t"
      "v_mfma_f32_16x16x32_bf16 %2, %16, %10, 0\n\t"
      "v_mfma_f32_16x16x32_bf16 %2, %17, %11, %2\n\t"
      "v_mfma_f32_16x16x32_bf16 %3, %18, %10, 0\n\t"
      "v_mfma_f32_16x16x32_bf16 %3, %19, %11, %3\n\t"
      "v_mfma_f32_16x16x32_bf16 %4, %16, %12, 0\n\t"
      "v_mfma_f32_16x16x32_bf16 %4, %17, %13, %4\n\t"
      "v_mfma_f32_16x16x32_bf16 %5, %18, %12, 0\n\t"
      "v_mfma_f32_16x16x32_bf16 %5, %19, %13, %5\n\t"
      "v_mfma_f32_16x16x32_bf16 %6, %16, %14, 0\n\t"
      "v_mfma_f32_16x16x32_bf16 %6, %17, %15, %6\n\t"
      "v_mfma_f32_16x16x32_bf16 %7, %18, %14, 0\n\t"
      "v_mfma_f32_16x16x32_bf16 %7, %19, %15, %7\n\t"
      : "+v"(acc[0][0]), "+v"(acc[0][1]), "+v"(acc[1][0]), "+v"(acc[1][1]), "+v"(acc[2][0]), "+v"(acc[2][1]), "+v"(acc[3][0]), "+v"(acc[3][1])
      : "v"(At[0][0]), "v"(At[0][1]), "v"(At[1][0]), "v"(At[1][1]), "v"(At[2][0]), "v"(At[2][1]), "v"(At[3][0]), "v"(At[3][1]), "v"(b00), "v"(b01), "v"(b10), "v"(b11));
  else
    asm volatile("s_setprio 1\n\ts_nop 0\n\t"
      "v_mfma_f32_16x16x32_bf16 %0, %16, %8, %0\n\t"
      "v_mfma_f32_16x16x32_bf16 %0, %17, %9, %0\n\t"
      "v_mfma_f32_16x16x32_bf16 %1, %18, %8, %1\n\t"
      "v_mfma_f32_16x16x32_bf16 %1, %19, %9, %1\n\t"
      "v_mfma_f32_16x16x32_bf16 %2, %16, %10, %2\n\t"
      "v_mfma_f32_16x16x32_bf16 %2, %17, %11, %2\n\t"
      "v_mfma_f32_16x16x32_bf16 %3, %18, %10, %3\n\t"
      "v_mfma_f32_16x16x32_bf16 %3, %19, %11, %3\n\t"
      "v_mfma_f32_16x16x32_bf16 %4, %16, %12, %4\n\t"
      "v_mfma_f32_16x16x32_bf16 %4, %17, %13, %4\n\t"
      "v_mfma_f32_16x16x32_bf16 %5, %18, %12, %5\n\t"
      "v_mfma_f32_16x16x32_bf16 %5, %19, %13, %5\n\t"
      "v_mfma_f32_16x16x32_bf16 %6, %16, %14, %6\n\t"
      "v_mfma_f32_16x16x32_bf16 %6, %17, %15, %6\n\t"
      "v_mfma_f32_16x16x32_bf16 %7, %18, %14, %7\n\t"
      "v_mfma_f32_16x16x32_bf16 %7, %19, %15, %7\n\t"
      : "+v"(acc[0][0]), "+v"(acc[0][1]), "+v"(acc[1][0]), "+v"(acc[1][1]), "+v"(acc[2][0]), "+v"(acc[2][1]), "+v"(acc[3][0]), "+v"(acc[3][1])
      : "v"(At[0][0]), "v"(At[0][1]), "v"(At[1][0]), "v"(At[1][1]), "v"(At[2][0]), "v"(At[2][1]), "v"(At[3][0]), "v"(At[3][1]), "v"(b00), "v"(b01), "v"(b10), "v"(b11));
}
template <bool INIT>
__device__ __forceinline__ void mma2(f32x4& c, const bf16x8& b0, const bf16x8& a0, const bf16x8& b1, const bf16x8& a1) {
  if constexpr (INIT) asm volatile("v_mfma_f32_16x16x32_bf16 %0, %1, %2, 0\n\tv_mfma_f32_16x16x32_bf16 %0, %3, %4, %0" : "+v"(c) : "v"(b0), "v"(a0), "v"(b1), "v"(a1));
  else asm volatile("v_mfma_f32_16x16x32_bf16 %0, %1, %2, %0\n\tv_mfma_f32_16x16x32_bf16 %0, %3, %4, %0" : "+v"(c) : "v"(b0), "v"(a0), "v"(b1), "v"(a1));
}
template <bool INIT>
__device__ __forceinline__ void mma_step(St& s, f32x4 (&acc)[4][2], const bf16x8 (&At)[4][2], const bf16x4 (&Bt)[2][2][2]) {
  const bf16x8 b00 = CAT(Bt[0][0]), b01 = CAT(Bt[0][1]), b10 = CAT(Bt[1][0]), b11 = CAT(Bt[1][1]);
  unsigned t;
  asm volatile("s_setprio 1\n\ts_nop 0");
  mma2<INIT>(acc[0][0], b00, At[0][0], b01, At[0][1]);
  asm volatile("s_add_u32 %0, %0, 0x80\n\ts_and_b32 %0, %0, 0xf80" : "+s"(s.kA) :: "scc");
  mma2<INIT>(acc[0][1], b10, At[0][0], b11, At[0][1]);
  asm volatile("s_mov_b32 %0, %1\n\ts_add_u32 %1, %2, %3" : "=&s"(s.a1), "+s"(s.a2) : "s"(s.sgA), "s"(s.kA) : "scc");
  mma2<INIT>(acc[1][0], b00, At[1][0], b01, At[1][1]);
  asm volatile("s_lshl_b32 %0, %1, 13" : "=s"(t) : "s"(s.kA) : "scc");
  mma2<INIT>(acc[1][1], b10, At[1][0], b11, At[1][1]);
  asm volatile("s_add_u32 %0, %2, %4\n\ts_add_u32 %1, %3, %4" : "=&s"(s.bl2), "=&s"(s.br2) : "s"(s.sgBL), "s"(s.sgBR), "s"(t) : "scc");
  mma2<INIT>(acc[2][0], b00, At[2][0], b01, At[2][1]);
  mma2<INIT>(acc[2][1], b10, At[2][0], b11, At[2][1]);
  mma2<INIT>(acc[3][0], b00, At[3][0], b01, At[3][1]);
  mma2<INIT>(acc[3][1], b10, At[3][0], b11, At[3][1]);
}
__device__ __forceinline__ void pack_quad(u32x4* dst, f32x4 (&q)[4][2]) {
  asm volatile("" : "+v"(q[0][0]), "+v"(q[0][1]), "+v"(q[1][0]), "+v"(q[1][1]), "+v"(q[2][0]), "+v"(q[2][1]), "+v"(q[3][0]), "+v"(q[3][1]));
#pragma unroll
  for (int m = 0; m < 4; ++m) { dst[m] = pack_pair(q[m][0], q[m][1]); asm volatile("" : "+v"(dst[m])); }
}
__device__ __forceinline__ void make_sel(St& s) {
  unsigned l = s.vlane16; asm volatile("" : "+v"(l));
  const unsigned one = ((l >> 8) == ((l >> 6) & 3u)) ? (0x3f80u << (l & 16u)) : 0u;
  const unsigned a = (l & 32u) ? 0u : one, b = (l & 32u) ? one : 0u;
  s.S1 = (u32x4){a, b, 0u, 0u}; s.S2 = (u32x4){0u, 0u, a, b};
  asm volatile("s_nop 1" : "+v"(s.S1), "+v"(s.S2));
}
template <int FIRST>
__device__ __forceinline__ void reduce4(St& s, int j) {
  if constexpr (FIRST == 1)
    asm volatile("v_mfma_f32_16x16x32_bf16 %0, %2, %4, 0\n\tv_mfma_f32_16x16x32_bf16 %1, %3, %4, 0\n\t"
                 "v_mfma_f32_16x16x32_bf16 %0, %2, %5, %0\n\tv_mfma_f32_16x16x32_bf16 %1, %3, %5, %1\n\t"
                 "v_mfma_f32_16x16x32_bf16 %0, %2, %6, %0\n\tv_mfma_f32_16x16x32_bf16 %1, %3, %6, %1\n\t"
                 "v_mfma_f32_16x16x32_bf16 %0, %2, %7, %0\n\tv_mfma_f32_16x16x32_bf16 %1, %3, %7, %1"
                 : "=&v"(s.D1), "=&v"(s.D2) : "v"(s.S1), "v"(s.S2), "v"(s.tmp[j]), "v"(s.tmp[j + 1]), "v"(s.tmp[j + 2]), "v"(s.tmp[j + 3]));
  else if constexpr (FIRST == 2)
    asm volatile("v_mfma_f32_16x16x32_bf16 %0, %2, %4, 0\n\tv_mfma_f32_16x16x32_bf16 %1, %3, %4, 0\n\t"
                 "v_mfma_f32_16x16x32_bf16 %0, %2, %5, %0\n\tv_mfma_f32_16x16x32_bf16 %1, %3, %5, %1\n\t"
                 "v_mfma_f32_16x16x32_bf16 %0, %2, %6, %0\n\tv_mfma_f32_16x16x32_bf16 %1, %3, %6, %1\n\t"
                 "v_mfma_f32_16x16x32_bf16 %0, %2, %7, %0\n\tv_mfma_f32_16x16x32_bf16 %1, %3, %7, %1"
                 : "+v"(s.D1), "+v"(s.D2) : "v"(s.S1), "v"(s.S2), "v"(s.tmp[j]), "v"(s.tmp[j + 1]), "v"(s.tmp[j + 2]), "v"(s.tmp[j + 3]));
  else
    asm volatile("v_mfma_f32_16x16x32_bf16 %0, %2, %4, %0\n\tv_mfma_f32_16x16x32_bf16 %1, %3, %4, %1\n\t"
                 "v_mfma_f32_16x16x32_bf16 %0, %2, %5, %0\n\tv_mfma_f32_16x16x32_bf16 %1, %3, %5, %1\n\t"
                 "v_mfma_f32_16x16x32_bf16 %0, %2, %6, %0\n\tv_mfma_f32_16x16x32_bf16 %1, %3, %6, %1\n\t"
                 "v_mfma_f32_16x16x32_bf16 %0, %2, %7, %0\n\tv_mfma_f32_16x16x32_bf16 %1, %3, %7, %1"
                 : "+v"(s.D1), "+v"(s.D2) : "v"(s.S1), "v"(s.S2), "v"(s.tmp[j]), "v"(s.tmp[j + 1]), "v"(s.tmp[j + 2]), "v"(s.tmp[j + 3]));
}
template <bool NOP>
__device__ __forceinline__ u32x4 reduce_fin(St& s) {
  if constexpr (NOP) asm volatile("s_nop 7" : "+v"(s.D1), "+v"(s.D2)); else asm volatile("" : "+v"(s.D1), "+v"(s.D2));
  u32x4 r;
  r.x = pk2(s.D1[0], s.D1[1]) | s.poison; r.y = pk2(s.D1[2], s.D1[3]) | s.poison;
  r.z = pk2(s.D2[0], s.D2[1]) | s.poison; r.w = pk2(s.D2[2], s.D2[3]) | s.poison;
  asm volatile("" : "+v"(r));
  return r;
}
__device__ __forceinline__ unsigned spin_flags(const St& s, unsigned blk) {
  unsigned bad, v0, v1, st, t0, t1;
  asm volatile(
      "s_memrealtime vcc\n\ts_mov_b32 %0, 0\n\ts_waitcnt lgkmcnt(0)\n\ts_mov_b32 %4, vcc_lo\n"
      "1:\n\t"
      "buffer_load_dword %1, %6, %7, %8 offen sc0 sc1\n\t"
      "buffer_load_dword %2, off, %7, %9 sc0 sc1\n\t"
      "s_memrealtime vcc\n\t"
      "s_waitcnt vmcnt(0) lgkmcnt(0)\n\t"
      "s_sub_u32 %5, vcc_lo, %4\n\t"
      "v_cmp_gt_u32 vcc, %10, %1\n\t"
      "v_readfirstlane_b32 %3, %2\n\t"
      "s_cmp_eq_u64 vcc, 0\n\t"
      "s_cbranch_scc1 3f\n\t"
      "s_cmp_eq_u32 %3, %10\n\t"
      "s_cbranch_scc1 2f\n\t"
      "s_cmp_lt_u32 %5, %11\n\t"
      "s_cbranch_scc1 1b\n"
      "2:\n\t"
      "v_mov_b32 %1, %10\n\ts_mov_b64 exec, 1\n\tbuffer_store_dword %1, off, %7, %9 sc0 sc1\n\ts_mov_b64 exec, -1\n\t"
      "s_mov_b32 %0, 0x7fc07fc0\n\ts_waitcnt vmcnt(0)\n"
      "3:"
      : "=&s"(bad), "=&v"(v0), "=&v"(v1), "=&s"(st), "=&s"(t0), "=&s"(t1)
      : "v"(s.vlane16), "s"(s.srdWs), "s"(blk), "s"(WS_TMO), "s"(s.call), "s"((unsigned)DEADLINE)
      : "memory", "scc", "vcc");
  return bad;
}
#define SEND(R, lo, hi, off) do { const u32x4 _srd = {lo, hi, OUT_BYTES, 0x20000u}; ST128(R, s.vdst, _srd, off, "sc0 sc1"); } while (0)

#define PFLAG(blk) do { unsigned _c; asm volatile("v_mov_b32 %0, %1\n\ts_mov_b64 exec, 1\n\tbuffer_store_dword %0, off, %2, %3 sc1\n\ts_mov_b64 exec, -1" : "=&v"(_c) : "s"(s.call), "s"(s.srdWs), "s"((blk) + (unsigned)s.wslot * 16u) : "memory"); } while (0)
#define GUARD(q) asm volatile("s_nop 7" : "+v"(q[0][0]), "+v"(q[0][1]), "+v"(q[1][0]), "+v"(q[1][1]), "+v"(q[2][0]), "+v"(q[2][1]), "+v"(q[3][0]), "+v"(q[3][1]))

template <int I, int V>
__device__ __forceinline__ void ktile(St& s) {
  constexpr unsigned b = I & 1, nb = b ^ 1;
  constexpr bool BG = (V != V_PLAIN && V != V_FIRST);
  constexpr bool E15 = (I == 15 && (V == V_L || V == V_R)), E0 = (I == 0 && (V == V_L || V == V_R));
  constexpr int hv = (V == V_R) ? 1 : 0;
  constexpr bool INL = (E0 && hv == 0) || V == V_FIRST, INR = (E0 && hv == 1) || V == V_FIRST;
  constexpr bool S0 = BG && slot_at(S0_FIRST, S0_N, I), S1 = BG && slot_at(S1_FIRST, S1_N, I);
  constexpr int KT = E15 ? -1 : (BG && I != 15 && I != 14) ? I : -100;
  constexpr int NX = count_at(PSTP, KT) + count_at(PLDP, KT) - (PLDP[0] / 4 == KT ? 1 : 0) - (PLDP[4] / 4 == KT ? 1 : 0);
#define BGMEM(ph) do { _Pragma("unroll") for (int u = 0; u < 8; ++u) { if (PSTP[u] == KT * 4 + (ph)) PST(u); if (PLDP[u] == KT * 4 + (ph)) PLD(u); } } while (0)
  LDA(b, 0);
  STAGE_A(SA(nb, 1), s.a1, hA);
  BGMEM(0);
  if constexpr (V == V_FIRST) wait_v<11>();
  BAR; WAIT_A(); mma<INL>(s.acc[0][0], s.At, s.B0);
  if constexpr (E0) pack_quad(&s.tmp[4], s.acc[1][hv]);
  if constexpr (KT == T_TIE0 + 1) { make_sel(s); reduce4<1>(s, 0); }
  if constexpr (KT == T_TIE1 + 1 && T_TIE1 != T_TIE0) { if (s.keep != 0u) reduce4<0>(s, 4); unsigned k2 = s.keep; asm volatile("" : "+s"(k2)); if (k2 == 0u) reduce4<2>(s, 4); }
  ENDM;
  LDB(s.B1, b, 1);
  if constexpr (KT == T_POLL) asm volatile("buffer_load_dword %0, %1, %2, %3 offen sc1" : "=v"(s.pollv) : "v"(s.vlane16), "s"(s.srdWs), "s"(s.pflag) : "memory");
  if constexpr (BG) STAGE_BN(SB(b, 0), s.bl2); else STAGE_B(SB(b, 0), s.bl2, 0u);
  BGMEM(1);
  if constexpr (V == V_FIRST) wait_v<11>();
  BAR; WAIT_B(s.B1); mma<INR>(s.acc[0][1], s.At, s.B1);
  if constexpr (KT == T_TIE0 + 1) { s.R0 = reduce_fin<false>(s); s.d0lo = s.pdlo; s.d0hi = s.pdhi; s.d0off = s.pdoff; s.ts0 = (4 + (s.TS0 & 15)) | (int)s.keep; asm volatile("" : "+s"(s.d0lo), "+s"(s.d0hi), "+s"(s.d0off), "+s"(s.ts0)); }
  if constexpr (KT == T_TIE1 + 1) { s.R1 = reduce_fin<false>(s); s.d1lo = s.pdlo; s.d1hi = s.pdhi; s.d1off = s.pdoff + (~s.keep & (8u * 2048u)); s.ts1 = 4 + ((s.TS0 >> 4) & 15); asm volatile("" : "+s"(s.d1lo), "+s"(s.d1hi), "+s"(s.d1off), "+s"(s.ts1)); }
  ENDM;
  LDA(b, 1);
  STAGE_A(SA(b, 0), s.a2, 0u);
  BGMEM(2);
  if constexpr (S0) { if (s.ts0 == I) SEND(s.R0, s.d0lo, s.d0hi, s.d0off); }
  if constexpr (S1) { if (s.ts1 == (I < S1_FIRST ? I + 16 : I)) SEND(s.R1, s.d1lo, s.d1hi, s.d1off); }
  wait_v<lag5(KT * 4 + 2)>();
  BAR; WAIT_A(); mma<INL>(s.acc[1][0], s.At, s.B0);
  if constexpr (E15) pack_quad(&s.tmp[0], s.acc[0][hv]);
  ENDM;
  LDB(s.B0, nb, 0);
  if constexpr (BG) STAGE_BN(SB(b, 1), s.br2); else STAGE_B(SB(b, 1), s.br2, 0u);
  BGMEM(3);
  if constexpr (KT == T_TIE0) asm volatile("s_waitcnt vmcnt(%4)" : "+v"(s.tmp[0]), "+v"(s.tmp[1]), "+v"(s.tmp[2]), "+v"(s.tmp[3]) : "i"(6 + NX) : "memory");
  else if constexpr (KT == T_TIE1) asm volatile("s_waitcnt vmcnt(%4)" : "+v"(s.tmp[4]), "+v"(s.tmp[5]), "+v"(s.tmp[6]), "+v"(s.tmp[7]) : "i"(6 + NX) : "memory");
  else if constexpr (KT == T_POLL) asm volatile("s_waitcnt vmcnt(%1)" : "+v"(s.pollv) : "i"(6 + NX) : "memory");
  else wait_v<6 + NX>();
  if constexpr (KT == T_FLAG) PFLAG(s.pflag);
  if constexpr (I == 13) { s.sgBL = s.nxtBL; s.sgBR = s.nxtBR; }
  BAR; mma_step<INR>(s, s.acc[1][1], s.At, s.B1);
  WAIT_B(s.B0);
  ENDM;
#undef BGMEM
  if constexpr (KT == T_POLL) {
    if (!__all(s.pollv >= s.call || (s.vlane16 >> 7) == ((unsigned)s.wslot >> 3))) s.poison |= spin_flags(s, s.pflag);
    SCHED;
  }
}

__device__ __forceinline__ void final_pair(St& s, unsigned flagL, unsigned own8, unsigned adj8) {
  LDA(0u, 0); STAGE_A(SA(1u, 1), s.a1, hA);
  BAR; WAIT_A(); mma<false>(s.acc[0][0], s.At, s.B0); ENDM;
  LDA(0u, 1);
  BAR; WAIT_A(); mma<false>(s.acc[1][0], s.At, s.B0); ENDM;
  LDB(s.B1, 0u, 1); wait_v<2>();
  BAR; WAIT_B(s.B1); mma<false>(s.acc[1][1], s.At, s.B1); ENDM;
  LDB(s.B0, 1u, 0); LDA(1u, 0); wait_v<0>();
  BAR; WAIT_B(s.B0); WAIT_A(); mma<false>(s.acc[0][0], s.At, s.B0); ENDM;
  LDA(1u, 1); GUARD(s.acc[0][0]); pack_quad(&s.tmp[0], s.acc[0][0]); PST(0); PST(1); PST(2); PST(3);
  BAR; WAIT_A(); mma<false>(s.acc[1][0], s.At, s.B0); ENDM;
  LDB(s.B1, 1u, 1); GUARD(s.acc[1][0]); pack_quad(&s.tmp[4], s.acc[1][0]); PST(4); PST(5); PST(6); PST(7);
  BAR; WAIT_B(s.B1); mma<false>(s.acc[1][1], s.At, s.B1); ENDM;
  LDA(1u, 0);
  BAR; WAIT_A(); mma<false>(s.acc[0][1], s.At, s.B1); ENDM;
  LDB(s.B1, 0u, 1); LDA(0u, 0); wait_v<0>(); PFLAG(flagL);
  s.pw += REGION; s.own = own8; s.vls += adj8;
  pack_quad(&s.tmp[4], s.acc[1][1]); PST(4); PST(5); PST(6); PST(7);
  BAR; WAIT_B(s.B1); WAIT_A(); mma<false>(s.acc[0][1], s.At, s.B1); ENDM;
}

__global__ __launch_bounds__(512, 2) void gemm_a2a(Params p) {
  __shared__ __attribute__((aligned(1024))) unsigned char smem[8 * HTB + 32768];
  const int tid = threadIdx.x, wid = __builtin_amdgcn_readfirstlane(tid >> 6), lane = tid & 63;
  const int wr = wid >> 2, wc = wid & 3, fr = lane & 15, fq = lane >> 4;
  const unsigned blk = blockIdx.x, mem = blk & 7u, grp = (blk >> 3) & 31u;
  const unsigned pm = grp & 3u, cg = grp >> 2, sub = mem >> 2, mi = mem & 3u;
  const unsigned rank = (unsigned)p.rank & 7u;
  St s;
  char* wsp = p.ws[0];
  const unsigned long long pa = (unsigned long long)p.A, pw = (unsigned long long)p.W, pws = (unsigned long long)wsp;
  s.srdA = (u32x4){(unsigned)pa, (unsigned)(pa >> 32) & 0xffffu, A_BYTES, 0x20000u};
  s.srdB = (u32x4){(unsigned)pw, (unsigned)(pw >> 32) & 0xffffu, W_BYTES, 0x20000u};
  s.srdWs = (u32x4){(unsigned)pws, (unsigned)(pws >> 32) & 0xffffu, WS_USED, 0x20000u};
  unsigned info;
  if (cg < 7u) {
    const unsigned j = (cg + 1u) << 3;
    info = (j | 0u) | ((j | (1u + sub)) << 6) | ((j | (3u + sub)) << 12) | ((j | (5u + sub)) << 18) | (cg << 24);
  } else {
    const unsigned j0 = (0x6427u >> (4u * pm)) & 7u;
    const unsigned j1 = ((4u + 2u * pm + sub) % 7u) + 1u, j2 = ((2u + 2u * pm + sub) % 7u) + 1u, j3 = ((2u * pm + sub) % 7u) + 1u;
    info = ((j0 << 3) | 7u) | (((j1 << 3) | 7u) << 6) | (((j2 << 3) | 7u) << 12) | (((j3 << 3) | 7u) << 18) | (7u << 24);
  }
  auto ent = [&](unsigned k) -> unsigned { return (info >> (6u * k)) & 63u; };
  auto colB = [&](unsigned e) -> unsigned { return (((rank + (e >> 3)) & 7u) * 8u + (e & 7u)) * 256u; };
  auto offB = [&](unsigned k) -> unsigned { return mi * (32u * ktB) + colB(ent(k)); };
  const unsigned lds0 = (unsigned)(size_t)(LAS unsigned char*)smem;
  s.ldsw = lds0 + (unsigned)wid * 1024u;
  s.voffA = (unsigned)((8 * wid + (lane >> 3)) * (K_ * 2) + (((lane & 7) ^ (lane >> 3)) << 4));
  {
    const unsigned fB = (unsigned)((lane >> 4) | (((wid >> 1) & 1) << 2));
    s.voffB = (unsigned)((4 * wid + (lane >> 4)) * (N_ * 2)) + ((((unsigned)lane & 15u) ^ (fB << 1)) << 4);
  }
  s.sgA = pm * (256u * K_ * 2u) + mi * (32u * ktA); s.sgBL = offB(1); s.sgBR = offB(0);
  {
    const unsigned k0 = (sub * 16u + pm) * ktA;
    const unsigned a0 = s.sgA + k0, l0 = s.sgBL + (k0 << 13), q0 = s.sgBR + (k0 << 13);
    STAGE_B(SB(0, 0), l0, 0u); STAGE_A(SA(0, 0), a0, 0u); STAGE_B(SB(0, 1), q0, 0u); STAGE_A(SA(0, 1), a0, hA);
    STAGE_B(SB(1, 0), l0, ktB); STAGE_A(SA(1, 0), a0, ktA); STAGE_B(SB(1, 1), q0, ktB);
    s.a1 = a0 + ktA; s.a2 = a0 + 2u * ktA; s.bl2 = l0 + 2u * ktB; s.br2 = q0 + 2u * ktB; s.kA = k0 + 2u * ktA;
  }
  unsigned callv;
  asm volatile("buffer_load_dword %0, off, %1, %2 sc1" : "=v"(callv) : "s"(s.srdWs), "s"(WS_CALL + blk * 64u) : "memory");
  unsigned olo[3], ohi[3];
#pragma unroll
  for (int k = 0; k < 3; ++k) {
    const unsigned long long o = (unsigned long long)p.out[ent(k) >> 3];
    olo[k] = (unsigned)o; ohi[k] = (unsigned)(o >> 32) & 0xffffu;
  }
  {
    const int q = (lane & 15) >> 2, pp = lane & 3, g = lane >> 4;
    const int f = q | ((g & 1) << 2);
    s.aoff0 = lds0 + (unsigned)((wr * 64 + fr) * 128 + (((0 + fq) ^ (fr & 7)) << 4));
    s.aoff1 = lds0 + (unsigned)((wr * 64 + fr) * 128 + (((4 + fq) ^ (fr & 7)) << 4));
    s.boff0 = lds0 + 65536u + (unsigned)((8 * g + q) * 256 + (((2 * wc + 0) ^ f) << 5) + 8 * pp);
    s.boff1 = lds0 + 65536u + (unsigned)((8 * g + q) * 256 + (((2 * wc + 1) ^ f) << 5) + 8 * pp);
  }
  s.vlane16 = (unsigned)lane * 16u;
  {
    const unsigned r = (unsigned)lane >> 3, c = (unsigned)lane & 7u, cc = c & 3u;
    const unsigned sfq = ((cc & 1u) << 1) | (cc >> 1);
    s.vgat = (c >> 2) * 4096u + (sfq * 16u + r) * 16u;
    s.vdst = r * 2048u + c * 16u;
  }
  const unsigned uw = (unsigned)wid, rwr = uw >> 2, rwcp = uw & 1u, b1 = (uw >> 1) & 1u;
  const unsigned ai4 = mi >> 1, m4 = 2u * (mi & 1u) + b1, ai8 = mem >> 2, m8 = mem & 3u;
  unsigned rd4 = blk * SLOT + ((ai4 * 8u + rwr * 4u + 2u * rwcp) * 4u + m4) * 1024u;
  unsigned rd8 = blk * SLOT + ((ai8 * 8u + rwr * 4u + 2u * rwcp) * 4u + m8) * 1024u + b1 * 128u;
  const unsigned row0 = rank * 1024u + pm * 256u;
  unsigned ds4 = (row0 + ai4 * 128u + rwr * 64u + m4 * 16u) * 2048u + rwcp * 128u;
  unsigned ds8 = (row0 + ai8 * 128u + rwr * 64u + m8 * 16u + b1 * 8u) * 2048u + rwcp * 128u;
  unsigned pw0 = WS_PART + blk * SLOT + uw * 4096u;
  s.wslot = (int)(mem * 8u + uw);
  unsigned adj8 = (((2u * (mi & 1u)) ^ 2u) - m8) * 1024u;
  const unsigned lsp = lds0 + 8u * HTB;
  unsigned tsl;
  {
    const unsigned key = grp * 3u + mem + uw;
    constexpr unsigned F0 = S0_FIRST - 4, F1 = S1_FIRST - 4;
    constexpr unsigned W2 = (20 - S0_FIRST) / 2;
    constexpr unsigned G1 = (S0_FIRST + 4u > S1_FIRST ? S0_FIRST + 4u : S1_FIRST) - 4;
    constexpr unsigned G2 = (F0 + W2 > F1 ? F0 + W2 : F1);
    tsl = ((F1 + key % (16u - F1)) << 4) | (((F0 + key % W2) | ((G2 + key % (16u - G2)) << 4)) << 8) | (((F0 + key % 4u) | ((G1 + key % 4u) << 4)) << 16);
    static_assert(G1 + 3u + 4u <= 13u, "the last segment's pushes must leave before the final K-tile pair");
  }
  s.ts0 = -1; s.ts1 = -1; s.poison = 0u;
  s.d0lo = s.d0hi = s.d0off = s.d1lo = s.d1hi = s.d1off = 0u;
  asm volatile("" : "+s"(s.d0lo), "+s"(s.d0hi), "+s"(s.d0off), "+s"(s.d1lo), "+s"(s.d1hi), "+s"(s.d1off), "+s"(s.ts0), "+s"(s.ts1));
  s.R0 = (u32x4){0u, 0u, 0u, 0u}; s.R1 = s.R0;
#pragma unroll
  for (int a = 0; a < 2; ++a)
#pragma unroll
    for (int bb = 0; bb < 2; ++bb)
#pragma unroll
      for (int m = 0; m < 4; ++m)
#pragma unroll
        for (int n = 0; n < 2; ++n) asm volatile("" : "=v"(s.acc[a][bb][m][n]));
  s.nxtBL = s.sgBL; s.nxtBR = offB(2);
  unsigned bl2 = offB(3), br3 = offB(4);
  unsigned do0 = ds8 + (ent(0) & 7u) * 256u, do1 = ds4 + (ent(1) & 7u) * 256u, do2 = ds4 + (ent(2) & 7u) * 256u;
  unsigned pf0 = WS_PFLAG + grp * 1024u;
  asm volatile("" : "+s"(bl2), "+s"(br3), "+s"(do0), "+s"(do1), "+s"(do2), "+s"(pf0));
  asm volatile("s_waitcnt lgkmcnt(0)" : "+s"(rd4), "+s"(rd8), "+s"(ds4), "+s"(ds8), "+s"(pw0), "+s"(adj8), "+s"(tsl), "+s"(s.wslot), "+s"(s.nxtBL), "+s"(s.nxtBR) :: "memory");
  if (wr == 1) BAR;
  wait_v<11>(); BAR;
  BAR;
  {
    LDB(s.B0, 0u, 0); WAIT_B(s.B0);
    ktile<0, V_FIRST>(s);
    asm volatile("" : "+v"(callv));
    s.call = __builtin_amdgcn_readfirstlane(callv) + 1u;
    ktile<1, V_PLAIN>(s);
    for (int t = 2; t < 12; t += 2) { ktile<0, V_PLAIN>(s); ktile<1, V_PLAIN>(s); }
    ktile<12, V_PLAIN>(s); ktile<13, V_PLAIN>(s);
  }
  const unsigned call = s.call;
  for (unsigned seg = 1; seg <= 3u; ++seg) {
    const unsigned k = seg - 1u;
    s.pw = pw0 + k * REGION;
    s.lead = pm & seg & 2u;
    s.own = k == 0u ? 1u << mem : 3u << (2u * mi);
    s.vls = s.vlane16 + (lsp + uw * 4096u + (k == 0u ? adj8 : 0u));
    ktile<14, V_BG>(s);
    if (seg == 2u) {
      ktile<15, V_L>(s);
      ktile<0, V_L>(s);
    }
    unsigned seg2 = seg; asm volatile("" : "+s"(seg2));
    if (seg2 != 2u) {
      ktile<15, V_R>(s);
      ktile<0, V_R>(s);
    }
    s.keep = (unsigned)((int)(k - 1u) >> 31);
    s.prd0 = WS_PART + k * REGION + (k == 0u ? rd8 : rd4);
    s.prd1 = k == 0u ? s.prd0 ^ (4u * SLOT) : s.prd0 + 128u;
    s.lrd = lsp + (s.prd0 & 0x7fffu) + (k == 0u ? adj8 : 0u);
    s.pflag = pf0 + k * 32768u;
    s.TS0 = (int)(tsl >> (8u * k));
    s.pdlo = k == 0u ? olo[0] : k == 1u ? olo[1] : olo[2];
    s.pdhi = k == 0u ? ohi[0] : k == 1u ? ohi[1] : ohi[2];
    s.pdoff = k == 0u ? do0 : k == 1u ? do1 : do2;
    s.nxtBL = bl2;
    s.nxtBR = k == 0u ? s.sgBR : br3;
    asm volatile("" : "+s"(s.nxtBL), "+s"(s.nxtBR), "+s"(s.pdoff), "+s"(s.pdlo), "+s"(s.pdhi), "+s"(s.pflag), "+s"(s.prd0), "+s"(s.prd1), "+s"(s.lrd), "+s"(s.TS0), "+s"(s.keep));
    ktile<1, V_BG>(s); ktile<2, V_BG>(s); ktile<3, V_BG>(s); ktile<4, V_BG>(s); ktile<5, V_BG>(s); ktile<6, V_BG>(s); ktile<7, V_BG>(s);
    ktile<8, V_BG>(s); ktile<9, V_BG>(s); ktile<10, V_BG>(s); ktile<11, V_BG>(s); ktile<12, V_BG>(s); ktile<13, V_BG>(s);
  }
  s.pw = pw0 + 3u * REGION;
  const unsigned fblk = pf0 + 3u * 32768u;
  final_pair(s, fblk, 1u << mem, adj8);
  GUARD(s.acc[0][1]); pack_quad(&s.tmp[0], s.acc[0][1]); PST(0); PST(1); PST(2); PST(3);
  asm volatile("s_waitcnt vmcnt(0) lgkmcnt(0)" ::: "memory"); PFLAG(fblk + 32768u);
  const int lane2 = lane_id();
  const unsigned long long o3 = (unsigned long long)p.out[ent(3) >> 3], o4 = (unsigned long long)p.out[0];
  char* const w0 = p.ws[ent(0) >> 3]; char* const w1 = p.ws[ent(1) >> 3]; char* const w2 = p.ws[ent(2) >> 3]; char* const w3 = p.ws[ent(3) >> 3];
  const unsigned o3lo = (unsigned)o3, o3hi = (unsigned)(o3 >> 32) & 0xffffu, o4lo = (unsigned)o4, o4hi = (unsigned)(o4 >> 32) & 0xffffu;
  if (wr == 0) BAR;
  LAS unsigned* lflag = (LAS unsigned*)(smem + 8 * HTB + ((((2u * (mi & 1u)) ^ 2u) + 1u) * 1024u));
  if (wid == 0) {
    const unsigned bad = spin_flags(s, fblk);
    if (lane2 == 0) lflag[0] = bad;
    asm volatile("s_waitcnt lgkmcnt(0)" ::: "memory");
  }
  BAR;
  {
    s.prd0 = WS_PART + 3u * REGION + rd4; s.prd1 = s.prd0 + 128u; s.lrd = lsp + (rd4 & 0x7fffu);
    PLDG(1); PLDG(2); PLDG(3); PLDG(5); PLDG(6); PLDG(7); PLDL(0, 0); PLDL(4, 128);
    make_sel(s);
    if (wid == 0) {
      const unsigned bad = spin_flags(s, fblk + 32768u);
      if (lane2 == 0) lflag[1] = bad;
      asm volatile("s_waitcnt lgkmcnt(0)" ::: "memory");
    }
    s.poison |= __builtin_amdgcn_readfirstlane(*(volatile LAS unsigned*)lflag);
    asm volatile("s_waitcnt vmcnt(3) lgkmcnt(0)" : "+v"(s.tmp[0]), "+v"(s.tmp[1]), "+v"(s.tmp[2]), "+v"(s.tmp[3]) :: "memory");
    reduce4<1>(s, 0);
    const u32x4 o0 = reduce_fin<true>(s);
    asm volatile("s_waitcnt vmcnt(0)" : "+v"(s.tmp[4]), "+v"(s.tmp[5]), "+v"(s.tmp[6]), "+v"(s.tmp[7]) :: "memory");
    reduce4<1>(s, 4);
    const u32x4 o1 = reduce_fin<true>(s);
    BAR;
    s.prd0 = WS_PART + 4u * REGION + rd8; s.prd1 = s.prd0 ^ (4u * SLOT); s.lrd = lsp + (rd8 & 0x7fffu) + adj8;
    PLDG(1); PLDG(2); PLDG(3); PLDG(4); PLDG(5); PLDG(6); PLDG(7); PLDL(0, 0);
    const unsigned doff = ds4 + (ent(3) & 7u) * 256u;
    SEND(o0, o3lo, o3hi, doff);
    SEND(o1, o3lo, o3hi, doff + 8u * 2048u);
    s.poison |= __builtin_amdgcn_readfirstlane(*(volatile LAS unsigned*)(lflag + 1));
    asm volatile("s_waitcnt vmcnt(2) lgkmcnt(0)" : "+v"(s.tmp[0]), "+v"(s.tmp[1]), "+v"(s.tmp[2]), "+v"(s.tmp[3]), "+v"(s.tmp[4]), "+v"(s.tmp[5]), "+v"(s.tmp[6]), "+v"(s.tmp[7]) :: "memory");
    reduce4<1>(s, 0); reduce4<0>(s, 4);
    const u32x4 o2 = reduce_fin<true>(s);
    SEND(o2, o4lo, o4hi, ds8 + (ent(4) & 7u) * 256u);
  }
  wait_v<0>();
  BAR;
  if (wid == 0) {
    if (lane2 < 4) {
      const unsigned e = (info >> (6u * (unsigned)lane2)) & 63u;
      char* const wd = lane2 == 0 ? w0 : lane2 == 1 ? w1 : lane2 == 2 ? w2 : w3;
      unsigned* f = (unsigned*)(wd + WS_DONE + rank * 1024u) + ((e & 7u) * 4u + pm) * 8u;
      if (lane2 == 0) __hip_atomic_store(f + mem, call, __ATOMIC_RELAXED, __HIP_MEMORY_SCOPE_SYSTEM);
      else __hip_atomic_store((unsigned long long*)f + mi, ((unsigned long long)call << 32) | call, __ATOMIC_RELAXED, __HIP_MEMORY_SCOPE_SYSTEM);
    }
    if (lane2 == 0) __hip_atomic_store((unsigned*)(wsp + WS_CALL + blk * 64u), call, __ATOMIC_RELAXED, __HIP_MEMORY_SCOPE_AGENT);
    if (blk == 0) {
      const unsigned long long* f = (const unsigned long long*)(wsp + WS_DONE) + lane2;
      unsigned* tmo = (unsigned*)(wsp + WS_TMO);
      const unsigned long long t0 = __builtin_amdgcn_s_memrealtime();
      for (unsigned n = 1;; ++n) {
        bool ok = true;
#pragma unroll
        for (unsigned j = 1; j < 8; ++j) {
          const unsigned long long v0 = __hip_atomic_load(f + ((rank + j) & 7u) * 128u, __ATOMIC_RELAXED, __HIP_MEMORY_SCOPE_SYSTEM);
          const unsigned long long v1 = __hip_atomic_load(f + ((rank + j) & 7u) * 128u + 64u, __ATOMIC_RELAXED, __HIP_MEMORY_SCOPE_SYSTEM);
          ok = ok && ((unsigned)v0 >= call) && ((unsigned)(v0 >> 32) >= call) && ((unsigned)v1 >= call) && ((unsigned)(v1 >> 32) >= call);
        }
        if (__all(ok)) break;
        if ((n & 15u) == 0u &&
            (__builtin_amdgcn_readfirstlane(__hip_atomic_load(tmo, __ATOMIC_RELAXED, __HIP_MEMORY_SCOPE_AGENT)) == call ||
             __builtin_amdgcn_s_memrealtime() - t0 > DEADLINE)) {
          if (lane2 == 0) {
            __hip_atomic_store(tmo, call, __ATOMIC_RELAXED, __HIP_MEMORY_SCOPE_AGENT);
            __hip_atomic_store((unsigned*)(o4 + (size_t)rank * 1024u * 2048u), 0x7fc07fc0u, __ATOMIC_RELAXED, __HIP_MEMORY_SCOPE_SYSTEM);
          }
          break;
        }
      }
      __builtin_amdgcn_fence(__ATOMIC_ACQUIRE, "");
    }
  }
}

}

extern "C" void kernel_launch(int rank, int nranks, ncclComm_t comm,
                              hipStream_t stream,
                              void* const* d_in, const int* in_sizes, int n_in,
                              void* d_out, int out_size,
                              const int* in_dtypes, int out_dtype,
                              void* const* d_out_all,
                              void* const* d_ws_all, size_t ws_bytes) {
  using namespace ga2a;
  (void)comm; (void)in_sizes; (void)n_in; (void)d_out; (void)out_size; (void)in_dtypes; (void)out_dtype;
  if (nranks != NR || ws_bytes < (size_t)WS_USED) { fprintf(stderr, "unsupported configuration\n"); return; }
  Params p;
  p.A = (const u16*)d_in[0];
  p.W = (const u16*)d_in[1];
  for (int j = 0; j < NR; ++j) { p.out[j] = (char*)d_out_all[(rank + j) & 7]; p.ws[j] = (char*)d_ws_all[(rank + j) & 7]; }
  p.rank = rank;
  hipLaunchKernelGGL(gemm_a2a, dim3(256), dim3(512), 0, stream, p);
  HIP_CHECK(hipGetLastError());
}
```

```cpp
#include <hip/hip_runtime.h>
#include <rccl/rccl.h>
#include <cstdio>

#define HIP_CHECK(expr) do { hipError_t _e = (expr); if (_e != hipSuccess) { \
    fprintf(stderr, "HIP error %s at %s:%d\n", hipGetErrorString(_e), __FILE__, __LINE__); \
    return; } } while (0)

namespace ga2a {

#define LAS __attribute__((address_space(3)))
typedef unsigned short u16;
typedef short bf16x8 __attribute__((ext_vector_type(8)));
typedef short bf16x4 __attribute__((ext_vector_type(4)));
typedef float f32x4 __attribute__((ext_vector_type(4)));
typedef float f32x2 __attribute__((ext_vector_type(2)));
typedef __bf16 bf16x2n __attribute__((ext_vector_type(2)));
typedef unsigned u32x4 __attribute__((ext_vector_type(4)));

constexpr int NR = 8;
constexpr int M_ = 1024, N_ = 8192, K_ = 8192;
constexpr unsigned HTB = 16384;
constexpr unsigned A_BYTES = (unsigned)M_ * K_ * 2u, W_BYTES = (unsigned)K_ * N_ * 2u;
constexpr unsigned OUT_BYTES = 8192u * 1024u * 2u;
constexpr unsigned SLOT = 65536u;
constexpr unsigned REGION = 256u * SLOT;
constexpr unsigned WS_PART = 0u;
constexpr unsigned WS_CTRL = 128u << 20;
constexpr unsigned WS_PFLAG = WS_CTRL + 524288u;
constexpr unsigned WS_CALL = WS_CTRL + 131072u;
constexpr unsigned WS_DONE = WS_CTRL + 163840u;
constexpr unsigned WS_TMO = WS_CTRL + 180224u;
constexpr unsigned WS_USED = WS_CTRL + (1u << 20);
constexpr unsigned long long DEADLINE = 50000000ull;

constexpr unsigned ktA = 128u, ktB = 64u * N_ * 2u;
constexpr unsigned hA = 128u * K_ * 2u;
constexpr unsigned iA = 64u * K_ * 2u, iB = 32u * N_ * 2u;

struct Params {
  const u16* A;
  const u16* W;
  char* out[NR];
  char* ws[NR];
  int rank;
};

enum { V_PLAIN = 0, V_BG = 1, V_L = 2, V_R = 3, V_FIRST = 4 };
constexpr int PSTP[8] = {-1, -1, 0, 0, 1, 1, 2, 2};
constexpr int PLDP[8] = {13, 14, 15, 16, 17, 18, 19, 20};
constexpr int T_FLAG = 1, T_POLL = 2;
constexpr int T_TIE0 = PLDP[3] / 4 + 1, T_TIE1 = PLDP[7] / 4 + 1;
constexpr int S0_FIRST = T_TIE0 + 1, S0_N = 16 - S0_FIRST, S1_FIRST = T_TIE1 + 1, S1_N = 20 - S1_FIRST;
constexpr int count_at(const int (&t)[8], int kt) { int n = 0; for (int i = 0; i < 8; ++i) n += (t[i] >= kt * 4 && t[i] < kt * 4 + 4) ? 1 : 0; return n; }
constexpr int nbg(int p) { int n = 0; for (int i = 0; i < 8; ++i) n += (PSTP[i] == p ? 1 : 0) + ((PLDP[i] == p && i != 0 && i != 4) ? 1 : 0); return n; }
constexpr int lag5(int p) { int n = 10; for (int q = p - 5; q <= p; ++q) n += nbg(q); return n; }
constexpr bool slot_at(int first, int n, int i) { return (i >= first && i < first + n) || (i + 16 >= first && i + 16 < first + n); }

__device__ __forceinline__ unsigned pk2(float lo, float hi) {
  f32x2 v = {lo, hi};
  return __builtin_bit_cast(unsigned, __builtin_convertvector(v, bf16x2n));
}
__device__ __forceinline__ u32x4 pack_pair(const f32x4& v0, const f32x4& v1) {
  unsigned d00 = pk2(v0[0], v0[1]), d01 = pk2(v0[2], v0[3]);
  unsigned d10 = pk2(v1[0], v1[1]), d11 = pk2(v1[2], v1[3]);
  auto r0 = __builtin_amdgcn_permlane16_swap(d00, d10, false, false);
  auto r1 = __builtin_amdgcn_permlane16_swap(d01, d11, false, false);
  u32x4 w; w.x = r0[0]; w.y = r1[0]; w.z = r0[1]; w.w = r1[1];
  return w;
}

struct St {
  f32x4 acc[2][2][4][2];
  bf16x8 At[4][2];
  bf16x4 B0[2][2][2], B1[2][2][2];
  u32x4 tmp[8];
  f32x4 D1, D2;
  u32x4 S1, S2;
  u32x4 R0, R1;
  unsigned pollv;
  unsigned voffA, voffB, aoff0, aoff1, boff0, boff1, vlane16, vgat, vdst;
  u32x4 srdA, srdB, srdWs;
  unsigned ldsw, lead;
  unsigned a1, a2, bl2, br2, kA;
  unsigned sgA, sgBL, sgBR, nxtBL, nxtBR;
  unsigned pw;
  unsigned prd0, prd1;
  unsigned own, lrd;
  unsigned vls;
  unsigned pflag;
  unsigned keep;
  unsigned call, poison;
  unsigned long long pd;
  unsigned long long d0, d1;
  int ts0, ts1, TS0, wslot;
};

#define SA(b, h) (((b) * 2u + (h)) * HTB)
#define SB(b, h) ((4u + (b) * 2u + (h)) * HTB)
#define DMA2(ldsoff, voff, srd, sbase, simm, step, bits) do { unsigned _t; asm volatile( \
    "s_add_u32 m0, %1, %2\n\ts_add_u32 %0, %6, %7\n\tbuffer_load_dwordx4 %4, %5, %0 offen " bits " lds\n\t" \
    "s_add_u32 m0, %1, %3\n\ts_add_u32 %0, %6, %8\n\tbuffer_load_dwordx4 %4, %5, %0 offen " bits " lds" \
    : "=&s"(_t) : "s"(s.ldsw), "i"(ldsoff), "i"((ldsoff) + 8192u), "v"(voff), "s"(srd), "s"(sbase), "i"(simm), "i"((simm) + (step)) : "memory", "scc"); } while (0)
#define STAGE_A(bufoff, sbase, simm) DMA2((bufoff), s.voffA, s.srdA, (sbase), (simm), iA, "")
#define STAGE_B(bufoff, sbase, simm) DMA2((bufoff), s.voffB, s.srdB, (sbase), (simm), iB, "")
#define STAGE_BN(bufoff, sbase) do { unsigned _t; asm volatile( \
    "s_add_u32 m0, %1, %2\n\ts_cmp_lg_u32 %8, 0\n\ts_cbranch_scc1 1f\n\t" \
    "buffer_load_dwordx4 %4, %5, %6 offen lds\n\ts_add_u32 m0, %1, %3\n\ts_add_u32 %0, %6, %7\n\tbuffer_load_dwordx4 %4, %5, %0 offen lds\n" \
    "2:\n\t.subsection 1\n" \
    "1:\n\tbuffer_load_dwordx4 %4, %5, %6 offen nt lds\n\ts_add_u32 m0, %1, %3\n\ts_add_u32 %0, %6, %7\n\tbuffer_load_dwordx4 %4, %5, %0 offen nt lds\n\ts_branch 2b\n\t" \
    ".subsection 0" \
    : "=&s"(_t) : "s"(s.ldsw), "i"(bufoff), "i"((bufoff) + 8192u), "v"(s.voffB), "s"(s.srdB), "s"(sbase), "i"(iB), "s"(s.lead) : "memory", "scc"); } while (0)
__device__ __forceinline__ int lane_id() { int l; asm volatile("v_mbcnt_lo_u32_b32 %0, -1, 0\n\tv_mbcnt_hi_u32_b32 %0, -1, %0" : "=v"(l)); return l; }
#define RD128(dst, addr, off) asm volatile("ds_read_b128 %0, %1 offset:%2" : "=v"(dst) : "v"(addr), "i"(off))
#define RDTR(dst, addr, off) asm volatile("ds_read_b64_tr_b16 %0, %1 offset:%2" : "=v"(dst) : "v"(addr), "i"(off))
#define LDA(b, h) do { _Pragma("unroll") for (int m = 0; m < 4; ++m) { RD128(s.At[m][0], s.aoff0, SA(b, h) + m * 2048); RD128(s.At[m][1], s.aoff1, SA(b, h) + m * 2048); } } while (0)
#define LDB(dst, b, h) do { _Pragma("unroll") for (int k = 0; k < 2; ++k) { \
    RDTR(dst[0][k][0], s.boff0, SB(b, h) - 65536u + k * 8192); RDTR(dst[0][k][1], s.boff0, SB(b, h) - 65536u + k * 8192 + 1024); } \
    _Pragma("unroll") for (int k = 0; k < 2; ++k) { \
    RDTR(dst[1][k][0], s.boff1, SB(b, h) - 65536u + k * 8192); RDTR(dst[1][k][1], s.boff1, SB(b, h) - 65536u + k * 8192 + 1024); } } while (0)
#define WAIT_A() asm volatile("s_waitcnt lgkmcnt(0)" : "+v"(s.At[0][0]), "+v"(s.At[0][1]), "+v"(s.At[1][0]), "+v"(s.At[1][1]), "+v"(s.At[2][0]), "+v"(s.At[2][1]), "+v"(s.At[3][0]), "+v"(s.At[3][1]) :: "memory")
#define WAIT_B(Bt) asm volatile("s_waitcnt lgkmcnt(0)" : "+v"(Bt[0][0][0]), "+v"(Bt[0][0][1]), "+v"(Bt[0][1][0]), "+v"(Bt[0][1][1]), "+v"(Bt[1][0][0]), "+v"(Bt[1][0][1]), "+v"(Bt[1][1][0]), "+v"(Bt[1][1][1]) :: "memory")
#define CAT(x) __builtin_shufflevector(x[0], x[1], 0, 1, 2, 3, 4, 5, 6, 7)
#define BAR __builtin_amdgcn_s_barrier()
#define SCHED __builtin_amdgcn_sched_barrier(0)
#define ENDM do { SCHED; asm volatile("s_setprio 0"); BAR; SCHED; } while (0)
#define ST128(data, voff, srd, soff, bits) asm volatile("buffer_store_dwordx4 %0, %1, %2, %3 offen " bits "\n\ts_nop 1" :: "v"(data), "v"(voff), "s"(srd), "s"(soff) : "memory")
#define LD128(dst, voff, srd, soff) asm volatile("buffer_load_dwordx4 %0, %1, %2, %3 offen sc1" : "=v"(dst) : "v"(voff), "s"(srd), "s"(soff) : "memory")
#define PST(u) do { unsigned _t; asm volatile( \
    "s_bitcmp1_b32 %1, %2\n\ts_cselect_b32 %0, %3, %4\n\ts_cbranch_scc1 1f\n" \
    "2:\n\ts_add_u32 %0, %0, %8\n\tbuffer_store_dwordx4 %6, %9, %10, %0 offen sc1\n\ts_nop 1\n\t" \
    ".subsection 1\n1:\n\tds_write_b128 %5, %6 offset:%7\n\ts_branch 2b\n\t.subsection 0" \
    : "=&s"(_t) : "s"(s.own), "i"(u), "s"(0xC0000000u), "s"(s.pw), "v"(s.vls), "v"(s.tmp[u]), "i"(((u) & 3) * 1024), "i"(((u) >> 2) * 32768u + ((u) & 3) * 1024u), "v"(s.vlane16), "s"(s.srdWs) : "memory", "scc"); } while (0)
#define PLDG(j) LD128(s.tmp[j], s.vgat, s.srdWs, ((j) < 4 ? s.prd0 : s.prd1) ^ (((j) & 3) * SLOT))
#define PLDL(j, add) do { unsigned _a; asm volatile("v_add_u32 %1, %2, %3\n\tds_read_b128 %0, %1 offset:%4" : "=v"(s.tmp[j]), "=&v"(_a) : "s"(s.lrd), "v"(s.vgat), "i"(add) : "memory"); } while (0)
#define PLD(j) do { if ((j) == 0) PLDL(0, 0); else if ((j) == 4) { if (s.keep != 0u) PLDG(4); unsigned _k = s.keep; asm volatile("" : "+s"(_k)); if (_k == 0u) PLDL(4, 128); } else PLDG(j); } while (0)

template <int N> __device__ __forceinline__ void wait_v() { asm volatile("s_waitcnt vmcnt(%0)" :: "i"(N) : "memory"); }

template <bool INIT>
__device__ __forceinline__ void mma(f32x4 (&acc)[4][2], const bf16x8 (&At)[4][2], const bf16x4 (&Bt)[2][2][2]) {
  const bf16x8 b00 = CAT(Bt[0][0]), b01 = CAT(Bt[0][1]), b10 = CAT(Bt[1][0]), b11 = CAT(Bt[1][1]);
  if constexpr (INIT)
    asm volatile("s_setprio 1\n\ts_nop 0\n\t"
      "v_mfma_f32_16x16x32_bf16 %0, %16, %8, 0\n\t"
      "v_mfma_f32_16x16x32_bf16 %0, %17, %9, %0\n\t"
      "v_mfma_f32_16x16x32_bf16 %1, %18, %8, 0\n\t"
      "v_mfma_f32_16x16x32_bf16 %1, %19, %9, %1\n\t"
      "v_mfma_f32_16x16x32_bf16 %2, %16, %10, 0\n\t"
      "v_mfma_f32_16x16x32_bf16 %2, %17, %11, %2\n\t"
      "v_mfma_f32_16x16x32_bf16 %3, %18, %10, 0\n\t"
      "v_mfma_f32_16x16x32_bf16 %3, %19, %11, %3\n\t"
      "v_mfma_f32_16x16x32_bf16 %4, %16, %12, 0\n\t"
      "v_mfma_f32_16x16x32_bf16 %4, %17, %13, %4\n\t"
      "v_mfma_f32_16x16x32_bf16 %5, %18, %12, 0\n\t"
      "v_mfma_f32_16x16x32_bf16 %5, %19, %13, %5\n\t"
      "v_mfma_f32_16x16x32_bf16 %6, %16, %14, 0\n\t"
      "v_mfma_f32_16x16x32_bf16 %6, %17, %15, %6\n\t"
      "v_mfma_f32_16x16x32_bf16 %7, %18, %14, 0\n\t"
      "v_mfma_f32_16x16x32_bf16 %7, %19, %15, %7\n\t"
      : "+v"(acc[0][0]), "+v"(acc[0][1]), "+v"(acc[1][0]), "+v"(acc[1][1]), "+v"(acc[2][0]), "+v"(acc[2][1]), "+v"(acc[3][0]), "+v"(acc[3][1])
      : "v"(At[0][0]), "v"(At[0][1]), "v"(At[1][0]), "v"(At[1][1]), "v"(At[2][0]), "v"(At[2][1]), "v"(At[3][0]), "v"(At[3][1]), "v"(b00), "v"(b01), "v"(b10), "v"(b11));
  else
    asm volatile("s_setprio 1\n\ts_nop 0\n\t"
      "v_mfma_f32_16x16x32_bf16 %0, %16, %8, %0\n\t"
      "v_mfma_f32_16x16x32_bf16 %0, %17, %9, %0\n\t"
      "v_mfma_f32_16x16x32_bf16 %1, %18, %8, %1\n\t"
      "v_mfma_f32_16x16x32_bf16 %1, %19, %9, %1\n\t"
      "v_mfma_f32_16x16x32_bf16 %2, %16, %10, %2\n\t"
      "v_mfma_f32_16x16x32_bf16 %2, %17, %11, %2\n\t"
      "v_mfma_f32_16x16x32_bf16 %3, %18, %10, %3\n\t"
      "v_mfma_f32_16x16x32_bf16 %3, %19, %11, %3\n\t"
      "v_mfma_f32_16x16x32_bf16 %4, %16, %12, %4\n\t"
      "v_mfma_f32_16x16x32_bf16 %4, %17, %13, %4\n\t"
      "v_mfma_f32_16x16x32_bf16 %5, %18, %12, %5\n\t"
      "v_mfma_f32_16x16x32_bf16 %5, %19, %13, %5\n\t"
      "v_mfma_f32_16x16x32_bf16 %6, %16, %14, %6\n\t"
      "v_mfma_f32_16x16x32_bf16 %6, %17, %15, %6\n\t"
      "v_mfma_f32_16x16x32_bf16 %7, %18, %14, %7\n\t"
      "v_mfma_f32_16x16x32_bf16 %7, %19, %15, %7\n\t"
      : "+v"(acc[0][0]), "+v"(acc[0][1]), "+v"(acc[1][0]), "+v"(acc[1][1]), "+v"(acc[2][0]), "+v"(acc[2][1]), "+v"(acc[3][0]), "+v"(acc[3][1])
      : "v"(At[0][0]), "v"(At[0][1]), "v"(At[1][0]), "v"(At[1][1]), "v"(At[2][0]), "v"(At[2][1]), "v"(At[3][0]), "v"(At[3][1]), "v"(b00), "v"(b01), "v"(b10), "v"(b11));
}
template <bool INIT>
__device__ __forceinline__ void mma2(f32x4& c, const bf16x8& b0, const bf16x8& a0, const bf16x8& b1, const bf16x8& a1) {
  if constexpr (INIT) asm volatile("v_mfma_f32_16x16x32_bf16 %0, %1, %2, 0\n\tv_mfma_f32_16x16x32_bf16 %0, %3, %4, %0" : "+v"(c) : "v"(b0), "v"(a0), "v"(b1), "v"(a1));
  else asm volatile("v_mfma_f32_16x16x32_bf16 %0, %1, %2, %0\n\tv_mfma_f32_16x16x32_bf16 %0, %3, %4, %0" : "+v"(c) : "v"(b0), "v"(a0), "v"(b1), "v"(a1));
}
template <bool INIT>
__device__ __forceinline__ void mma_step(St& s, f32x4 (&acc)[4][2], const bf16x8 (&At)[4][2], const bf16x4 (&Bt)[2][2][2]) {
  const bf16x8 b00 = CAT(Bt[0][0]), b01 = CAT(Bt[0][1]), b10 = CAT(Bt[1][0]), b11 = CAT(Bt[1][1]);
  unsigned t;
  asm volatile("s_setprio 1\n\ts_nop 0");
  mma2<INIT>(acc[0][0], b00, At[0][0], b01, At[0][1]);
  asm volatile("s_add_u32 %0, %0, 0x80\n\ts_and_b32 %0, %0, 0xf80" : "+s"(s.kA) :: "scc");
  mma2<INIT>(acc[0][1], b10, At[0][0], b11, At[0][1]);
  asm volatile("s_mov_b32 %0, %1\n\ts_add_u32 %1, %2, %3" : "=&s"(s.a1), "+s"(s.a2) : "s"(s.sgA), "s"(s.kA) : "scc");
  mma2<INIT>(acc[1][0], b00, At[1][0], b01, At[1][1]);
  asm volatile("s_lshl_b32 %0, %1, 13" : "=s"(t) : "s"(s.kA) : "scc");
  mma2<INIT>(acc[1][1], b10, At[1][0], b11, At[1][1]);
  asm volatile("s_add_u32 %0, %2, %4\n\ts_add_u32 %1, %3, %4" : "=&s"(s.bl2), "=&s"(s.br2) : "s"(s.sgBL), "s"(s.sgBR), "s"(t) : "scc");
  mma2<INIT>(acc[2][0], b00, At[2][0], b01, At[2][1]);
  mma2<INIT>(acc[2][1], b10, At[2][0], b11, At[2][1]);
  mma2<INIT>(acc[3][0], b00, At[3][0], b01, At[3][1]);
  mma2<INIT>(acc[3][1], b10, At[3][0], b11, At[3][1]);
}
__device__ __forceinline__ void pack_quad(u32x4* dst, f32x4 (&q)[4][2]) {
  asm volatile("" : "+v"(q[0][0]), "+v"(q[0][1]), "+v"(q[1][0]), "+v"(q[1][1]), "+v"(q[2][0]), "+v"(q[2][1]), "+v"(q[3][0]), "+v"(q[3][1]));
#pragma unroll
  for (int m = 0; m < 4; ++m) { dst[m] = pack_pair(q[m][0], q[m][1]); asm volatile("" : "+v"(dst[m])); }
}
__device__ __forceinline__ void make_sel(St& s) {
  unsigned l = s.vlane16; asm volatile("" : "+v"(l));
  const unsigned one = ((l >> 8) == ((l >> 6) & 3u)) ? (0x3f80u << (l & 16u)) : 0u;
  const unsigned a = (l & 32u) ? 0u : one, b = (l & 32u) ? one : 0u;
  s.S1 = (u32x4){a, b, 0u, 0u}; s.S2 = (u32x4){0u, 0u, a, b};
  asm volatile("s_nop 1" : "+v"(s.S1), "+v"(s.S2));
}
template <int FIRST>
__device__ __forceinline__ void reduce4(St& s, int j) {
  if constexpr (FIRST == 1)
    asm volatile("v_mfma_f32_16x16x32_bf16 %0, %2, %4, 0\n\tv_mfma_f32_16x16x32_bf16 %1, %3, %4, 0\n\t"
                 "v_mfma_f32_16x16x32_bf16 %0, %2, %5, %0\n\tv_mfma_f32_16x16x32_bf16 %1, %3, %5, %1\n\t"
                 "v_mfma_f32_16x16x32_bf16 %0, %2, %6, %0\n\tv_mfma_f32_16x16x32_bf16 %1, %3, %6, %1\n\t"
                 "v_mfma_f32_16x16x32_bf16 %0, %2, %7, %0\n\tv_mfma_f32_16x16x32_bf16 %1, %3, %7, %1"
                 : "=&v"(s.D1), "=&v"(s.D2) : "v"(s.S1), "v"(s.S2), "v"(s.tmp[j]), "v"(s.tmp[j + 1]), "v"(s.tmp[j + 2]), "v"(s.tmp[j + 3]));
  else if constexpr (FIRST == 2)
    asm volatile("v_mfma_f32_16x16x32_bf16 %0, %2, %4, 0\n\tv_mfma_f32_16x16x32_bf16 %1, %3, %4, 0\n\t"
                 "v_mfma_f32_16x16x32_bf16 %0, %2, %5, %0\n\tv_mfma_f32_16x16x32_bf16 %1, %3, %5, %1\n\t"
                 "v_mfma_f32_16x16x32_bf16 %0, %2, %6, %0\n\tv_mfma_f32_16x16x32_bf16 %1, %3, %6, %1\n\t"
                 "v_mfma_f32_16x16x32_bf16 %0, %2, %7, %0\n\tv_mfma_f32_16x16x32_bf16 %1, %3, %7, %1"
                 : "+v"(s.D1), "+v"(s.D2) : "v"(s.S1), "v"(s.S2), "v"(s.tmp[j]), "v"(s.tmp[j + 1]), "v"(s.tmp[j + 2]), "v"(s.tmp[j + 3]));
  else
    asm volatile("v_mfma_f32_16x16x32_bf16 %0, %2, %4, %0\n\tv_mfma_f32_16x16x32_bf16 %1, %3, %4, %1\n\t"
                 "v_mfma_f32_16x16x32_bf16 %0, %2, %5, %0\n\tv_mfma_f32_16x16x32_bf16 %1, %3, %5, %1\n\t"
                 "v_mfma_f32_16x16x32_bf16 %0, %2, %6, %0\n\tv_mfma_f32_16x16x32_bf16 %1, %3, %6, %1\n\t"
                 "v_mfma_f32_16x16x32_bf16 %0, %2, %7, %0\n\tv_mfma_f32_16x16x32_bf16 %1, %3, %7, %1"
                 : "+v"(s.D1), "+v"(s.D2) : "v"(s.S1), "v"(s.S2), "v"(s.tmp[j]), "v"(s.tmp[j + 1]), "v"(s.tmp[j + 2]), "v"(s.tmp[j + 3]));
}
template <bool NOP>
__device__ __forceinline__ u32x4 reduce_fin(St& s) {
  if constexpr (NOP) asm volatile("s_nop 7" : "+v"(s.D1), "+v"(s.D2)); else asm volatile("" : "+v"(s.D1), "+v"(s.D2));
  u32x4 r;
  r.x = pk2(s.D1[0], s.D1[1]) | s.poison; r.y = pk2(s.D1[2], s.D1[3]) | s.poison;
  r.z = pk2(s.D2[0], s.D2[1]) | s.poison; r.w = pk2(s.D2[2], s.D2[3]) | s.poison;
  asm volatile("" : "+v"(r));
  return r;
}
__device__ __forceinline__ unsigned spin_flags(const St& s, unsigned blk) {
  unsigned bad, v0, v1, st, t0, t1;
  asm volatile(
      "s_memrealtime vcc\n\ts_mov_b32 %0, 0\n\ts_waitcnt lgkmcnt(0)\n\ts_mov_b32 %4, vcc_lo\n"
      "1:\n\t"
      "buffer_load_dword %1, %6, %7, %8 offen sc0 sc1\n\t"
      "buffer_load_dword %2, off, %7, %9 sc0 sc1\n\t"
      "s_memrealtime vcc\n\t"
      "s_waitcnt vmcnt(0) lgkmcnt(0)\n\t"
      "s_sub_u32 %5, vcc_lo, %4\n\t"
      "v_cmp_gt_u32 vcc, %10, %1\n\t"
      "v_readfirstlane_b32 %3, %2\n\t"
      "s_cmp_eq_u64 vcc, 0\n\t"
      "s_cbranch_scc1 3f\n\t"
      "s_cmp_eq_u32 %3, %10\n\t"
      "s_cbranch_scc1 2f\n\t"
      "s_cmp_lt_u32 %5, %11\n\t"
      "s_cbranch_scc1 1b\n"
      "2:\n\t"
      "v_mov_b32 %1, %10\n\ts_mov_b64 exec, 1\n\tbuffer_store_dword %1, off, %7, %9 sc0 sc1\n\ts_mov_b64 exec, -1\n\t"
      "s_mov_b32 %0, 0x7fc07fc0\n\ts_waitcnt vmcnt(0)\n"
      "3:"
      : "=&s"(bad), "=&v"(v0), "=&v"(v1), "=&s"(st), "=&s"(t0), "=&s"(t1)
      : "v"(s.vlane16), "s"(s.srdWs), "s"(blk), "s"(WS_TMO), "s"(s.call), "s"((unsigned)DEADLINE)
      : "memory", "scc", "vcc");
  return bad;
}
#define SEND(R, lo, hi, off) do { const u32x4 _srd = {lo, hi, OUT_BYTES, 0x20000u}; ST128(R, s.vdst, _srd, off, "sc0 sc1"); } while (0)

#define SLOT_SEND(R, dst, ts, when) asm volatile("s_cmp_eq_u32 %0, %1\n\ts_cbranch_scc1 1f\n2:\n\t.subsection 1\n" \
    "1:\n\tglobal_store_dwordx4 %2, %3, %4 sc0 sc1\n\ts_nop 0\n\ts_branch 2b\n\t.subsection 0" :: "s"(ts), "i"(when), "v"(s.vdst), "v"(R), "s"(dst) : "memory", "scc")
#define PFLAG(blk) do { unsigned _c; asm volatile("v_mov_b32 %0, %1\n\ts_mov_b64 exec, 1\n\tbuffer_store_dword %0, off, %2, %3 sc1\n\ts_mov_b64 exec, -1" : "=&v"(_c) : "s"(s.call), "s"(s.srdWs), "s"((blk) + (unsigned)s.wslot * 16u) : "memory"); } while (0)
#define GUARD(q) asm volatile("s_nop 7" : "+v"(q[0][0]), "+v"(q[0][1]), "+v"(q[1][0]), "+v"(q[1][1]), "+v"(q[2][0]), "+v"(q[2][1]), "+v"(q[3][0]), "+v"(q[3][1]))

template <int I, int V>
__device__ __forceinline__ void ktile(St& s) {
  constexpr unsigned b = I & 1, nb = b ^ 1;
  constexpr bool BG = (V != V_PLAIN && V != V_FIRST);
  constexpr bool E15 = (I == 15 && (V == V_L || V == V_R)), E0 = (I == 0 && (V == V_L || V == V_R));
  constexpr int hv = (V == V_R) ? 1 : 0;
  constexpr bool INL = (E0 && hv == 0) || V == V_FIRST, INR = (E0 && hv == 1) || V == V_FIRST;
  constexpr bool S0 = BG && slot_at(S0_FIRST, S0_N, I), S1 = BG && slot_at(S1_FIRST, S1_N, I);
  constexpr int KT = E15 ? -1 : (BG && I != 15 && I != 14) ? I : -100;
  constexpr int NX = count_at(PSTP, KT) + count_at(PLDP, KT) - (PLDP[0] / 4 == KT ? 1 : 0) - (PLDP[4] / 4 == KT ? 1 : 0);
#define BGMEM(ph) do { _Pragma("unroll") for (int u = 0; u < 8; ++u) { if (PSTP[u] == KT * 4 + (ph)) PST(u); if (PLDP[u] == KT * 4 + (ph)) PLD(u); } } while (0)
  LDA(b, 0);
  STAGE_A(SA(nb, 1), s.a1, hA);
  BGMEM(0);
  if constexpr (V == V_FIRST) wait_v<11>();
  BAR; WAIT_A(); mma<INL>(s.acc[0][0], s.At, s.B0);
  if constexpr (E0) pack_quad(&s.tmp[4], s.acc[1][hv]);
  if constexpr (KT == T_TIE0 + 1) { make_sel(s); reduce4<1>(s, 0); }
  if constexpr (KT == T_TIE1 + 1 && T_TIE1 != T_TIE0) { if (s.keep != 0u) reduce4<0>(s, 4); unsigned k2 = s.keep; asm volatile("" : "+s"(k2)); if (k2 == 0u) reduce4<2>(s, 4); }
  ENDM;
  LDB(s.B1, b, 1);
  if constexpr (KT == T_POLL) asm volatile("buffer_load_dword %0, %1, %2, %3 offen sc1" : "=v"(s.pollv) : "v"(s.vlane16), "s"(s.srdWs), "s"(s.pflag) : "memory");
  if constexpr (BG) STAGE_BN(SB(b, 0), s.bl2); else STAGE_B(SB(b, 0), s.bl2, 0u);
  BGMEM(1);
  if constexpr (V == V_FIRST) wait_v<11>();
  BAR; WAIT_B(s.B1); mma<INR>(s.acc[0][1], s.At, s.B1);
  if constexpr (KT == T_TIE0 + 1) { s.R0 = reduce_fin<false>(s); s.d0 = s.pd; s.ts0 = (4 + (s.TS0 & 15)) | (int)s.keep; asm volatile("" : "+s"(s.d0), "+s"(s.ts0)); }
  if constexpr (KT == T_TIE1 + 1) { s.R1 = reduce_fin<false>(s); s.d1 = s.pd + (~s.keep & (8u * 2048u)); s.ts1 = 4 + ((s.TS0 >> 4) & 15); asm volatile("" : "+s"(s.d1), "+s"(s.ts1)); }
  ENDM;
  LDA(b, 1);
  STAGE_A(SA(b, 0), s.a2, 0u);
  BGMEM(2);
  if constexpr (S0) SLOT_SEND(s.R0, s.d0, s.ts0, I);
  if constexpr (S1) SLOT_SEND(s.R1, s.d1, s.ts1, (I < S1_FIRST ? I + 16 : I));
  wait_v<lag5(KT * 4 + 2)>();
  BAR; WAIT_A(); mma<INL>(s.acc[1][0], s.At, s.B0);
  if constexpr (E15) pack_quad(&s.tmp[0], s.acc[0][hv]);
  ENDM;
  LDB(s.B0, nb, 0);
  if constexpr (BG) STAGE_BN(SB(b, 1), s.br2); else STAGE_B(SB(b, 1), s.br2, 0u);
  BGMEM(3);
  if constexpr (KT == T_TIE0) asm volatile("s_waitcnt vmcnt(%4)" : "+v"(s.tmp[0]), "+v"(s.tmp[1]), "+v"(s.tmp[2]), "+v"(s.tmp[3]) : "i"(6 + NX) : "memory");
  else if constexpr (KT == T_TIE1) asm volatile("s_waitcnt vmcnt(%4)" : "+v"(s.tmp[4]), "+v"(s.tmp[5]), "+v"(s.tmp[6]), "+v"(s.tmp[7]) : "i"(6 + NX) : "memory");
  else if constexpr (KT == T_POLL) asm volatile("s_waitcnt vmcnt(%1)" : "+v"(s.pollv) : "i"(6 + NX) : "memory");
  else wait_v<6 + NX>();
  if constexpr (KT == T_FLAG) PFLAG(s.pflag);
  if constexpr (I == 13) { s.sgBL = s.nxtBL; s.sgBR = s.nxtBR; }
  BAR; mma_step<INR>(s, s.acc[1][1], s.At, s.B1);
  WAIT_B(s.B0);
  ENDM;
#undef BGMEM
  if constexpr (KT == T_POLL) {
    if (!__all(s.pollv >= s.call || (s.vlane16 >> 7) == ((unsigned)s.wslot >> 3))) s.poison |= spin_flags(s, s.pflag);
    SCHED;
  }
}

__device__ __forceinline__ void final_pair(St& s, unsigned flagL, unsigned own8, unsigned adj8) {
  LDA(0u, 0); STAGE_A(SA(1u, 1), s.a1, hA);
  BAR; WAIT_A(); mma<false>(s.acc[0][0], s.At, s.B0); ENDM;
  LDA(0u, 1);
  BAR; WAIT_A(); mma<false>(s.acc[1][0], s.At, s.B0); ENDM;
  LDB(s.B1, 0u, 1); wait_v<2>();
  BAR; WAIT_B(s.B1); mma<false>(s.acc[1][1], s.At, s.B1); ENDM;
  LDB(s.B0, 1u, 0); LDA(1u, 0); wait_v<0>();
  BAR; WAIT_B(s.B0); WAIT_A(); mma<false>(s.acc[0][0], s.At, s.B0); ENDM;
  LDA(1u, 1); GUARD(s.acc[0][0]); pack_quad(&s.tmp[0], s.acc[0][0]); PST(0); PST(1); PST(2); PST(3);
  BAR; WAIT_A(); mma<false>(s.acc[1][0], s.At, s.B0); ENDM;
  LDB(s.B1, 1u, 1); GUARD(s.acc[1][0]); pack_quad(&s.tmp[4], s.acc[1][0]); PST(4); PST(5); PST(6); PST(7);
  BAR; WAIT_B(s.B1); mma<false>(s.acc[1][1], s.At, s.B1); ENDM;
  LDA(1u, 0);
  BAR; WAIT_A(); mma<false>(s.acc[0][1], s.At, s.B1); ENDM;
  LDB(s.B1, 0u, 1); LDA(0u, 0); wait_v<0>(); PFLAG(flagL);
  s.pw += REGION; s.own = own8; s.vls += adj8;
  pack_quad(&s.tmp[4], s.acc[1][1]); PST(4); PST(5); PST(6); PST(7);
  BAR; WAIT_B(s.B1); WAIT_A(); mma<false>(s.acc[0][1], s.At, s.B1); ENDM;
}

__global__ __launch_bounds__(512, 2) void gemm_a2a(Params p) {
  __shared__ __attribute__((aligned(1024))) unsigned char smem[8 * HTB + 32768];
  const int tid = threadIdx.x, wid = __builtin_amdgcn_readfirstlane(tid >> 6), lane = tid & 63;
  const int wr = wid >> 2, wc = wid & 3, fr = lane & 15, fq = lane >> 4;
  const unsigned blk = blockIdx.x, mem = blk & 7u, grp = (blk >> 3) & 31u;
  const unsigned pm = grp & 3u, cg = grp >> 2, sub = mem >> 2, mi = mem & 3u;
  const unsigned rank = (unsigned)p.rank & 7u;
  St s;
  char* wsp = p.ws[0];
  const unsigned long long pa = (unsigned long long)p.A, pw = (unsigned long long)p.W, pws = (unsigned long long)wsp;
  s.srdA = (u32x4){(unsigned)pa, (unsigned)(pa >> 32) & 0xffffu, A_BYTES, 0x20000u};
  s.srdB = (u32x4){(unsigned)pw, (unsigned)(pw >> 32) & 0xffffu, W_BYTES, 0x20000u};
  s.srdWs = (u32x4){(unsigned)pws, (unsigned)(pws >> 32) & 0xffffu, WS_USED, 0x20000u};
  unsigned info;
  if (cg < 7u) {
    const unsigned j = (cg + 1u) << 3;
    info = (j | 0u) | ((j | (1u + sub)) << 6) | ((j | (3u + sub)) << 12) | ((j | (5u + sub)) << 18) | (cg << 24);
  } else {
    const unsigned j0 = (0x6427u >> (4u * pm)) & 7u;
    const unsigned j1 = ((4u + 2u * pm + sub) % 7u) + 1u, j2 = ((2u + 2u * pm + sub) % 7u) + 1u, j3 = ((2u * pm + sub) % 7u) + 1u;
    info = ((j0 << 3) | 7u) | (((j1 << 3) | 7u) << 6) | (((j2 << 3) | 7u) << 12) | (((j3 << 3) | 7u) << 18) | (7u << 24);
  }
  auto ent = [&](unsigned k) -> unsigned { return (info >> (6u * k)) & 63u; };
  auto colB = [&](unsigned e) -> unsigned { return (((rank + (e >> 3)) & 7u) * 8u + (e & 7u)) * 256u; };
  auto offB = [&](unsigned k) -> unsigned { return mi * (32u * ktB) + colB(ent(k)); };
  const unsigned lds0 = (unsigned)(size_t)(LAS unsigned char*)smem;
  s.ldsw = lds0 + (unsigned)wid * 1024u;
  s.voffA = (unsigned)((8 * wid + (lane >> 3)) * (K_ * 2) + (((lane & 7) ^ (lane >> 3)) << 4));
  {
    const unsigned fB = (unsigned)((lane >> 4) | (((wid >> 1) & 1) << 2));
    s.voffB = (unsigned)((4 * wid + (lane >> 4)) * (N_ * 2)) + ((((unsigned)lane & 15u) ^ (fB << 1)) << 4);
  }
  s.sgA = pm * (256u * K_ * 2u) + mi * (32u * ktA); s.sgBL = offB(1); s.sgBR = offB(0);
  {
    const unsigned k0 = (sub * 16u + pm) * ktA;
    const unsigned a0 = s.sgA + k0, l0 = s.sgBL + (k0 << 13), q0 = s.sgBR + (k0 << 13);
    STAGE_B(SB(0, 0), l0, 0u); STAGE_A(SA(0, 0), a0, 0u); STAGE_B(SB(0, 1), q0, 0u); STAGE_A(SA(0, 1), a0, hA);
    STAGE_B(SB(1, 0), l0, ktB); STAGE_A(SA(1, 0), a0, ktA); STAGE_B(SB(1, 1), q0, ktB);
    s.a1 = a0 + ktA; s.a2 = a0 + 2u * ktA; s.bl2 = l0 + 2u * ktB; s.br2 = q0 + 2u * ktB; s.kA = k0 + 2u * ktA;
  }
  unsigned callv;
  asm volatile("buffer_load_dword %0, off, %1, %2 sc1" : "=v"(callv) : "s"(s.srdWs), "s"(WS_CALL + blk * 64u) : "memory");
  unsigned long long ob0 = (unsigned long long)p.out[ent(0) >> 3], ob1 = (unsigned long long)p.out[ent(1) >> 3], ob2 = (unsigned long long)p.out[ent(2) >> 3];
  {
    const int q = (lane & 15) >> 2, pp = lane & 3, g = lane >> 4;
    const int f = q | ((g & 1) << 2);
    s.aoff0 = lds0 + (unsigned)((wr * 64 + fr) * 128 + (((0 + fq) ^ (fr & 7)) << 4));
    s.aoff1 = lds0 + (unsigned)((wr * 64 + fr) * 128 + (((4 + fq) ^ (fr & 7)) << 4));
    s.boff0 = lds0 + 65536u + (unsigned)((8 * g + q) * 256 + (((2 * wc + 0) ^ f) << 5) + 8 * pp);
    s.boff1 = lds0 + 65536u + (unsigned)((8 * g + q) * 256 + (((2 * wc + 1) ^ f) << 5) + 8 * pp);
  }
  s.vlane16 = (unsigned)lane * 16u;
  {
    const unsigned r = (unsigned)lane >> 3, c = (unsigned)lane & 7u, cc = c & 3u;
    const unsigned sfq = ((cc & 1u) << 1) | (cc >> 1);
    s.vgat = (c >> 2) * 4096u + (sfq * 16u + r) * 16u;
    s.vdst = r * 2048u + c * 16u;
  }
  const unsigned uw = (unsigned)wid, rwr = uw >> 2, rwcp = uw & 1u, b1 = (uw >> 1) & 1u;
  const unsigned ai4 = mi >> 1, m4 = 2u * (mi & 1u) + b1, ai8 = mem >> 2, m8 = mem & 3u;
  unsigned rd4 = blk * SLOT + ((ai4 * 8u + rwr * 4u + 2u * rwcp) * 4u + m4) * 1024u;
  unsigned rd8 = blk * SLOT + ((ai8 * 8u + rwr * 4u + 2u * rwcp) * 4u + m8) * 1024u + b1 * 128u;
  const unsigned row0 = rank * 1024u + pm * 256u;
  unsigned ds4 = (row0 + ai4 * 128u + rwr * 64u + m4 * 16u) * 2048u + rwcp * 128u;
  unsigned ds8 = (row0 + ai8 * 128u + rwr * 64u + m8 * 16u + b1 * 8u) * 2048u + rwcp * 128u;
  unsigned pw0 = WS_PART + blk * SLOT + uw * 4096u;
  s.wslot = (int)(mem * 8u + uw);
  unsigned adj8 = (((2u * (mi & 1u)) ^ 2u) - m8) * 1024u;
  const unsigned lsp = lds0 + 8u * HTB;
  unsigned tsl;
  {
    const unsigned key = grp * 3u + mem + uw;
    constexpr unsigned F0 = S0_FIRST - 4, F1 = S1_FIRST - 4;
    constexpr unsigned W2 = (20 - S0_FIRST) / 2;
    constexpr unsigned G1 = (S0_FIRST + 4u > S1_FIRST ? S0_FIRST + 4u : S1_FIRST) - 4;
    constexpr unsigned G2 = (F0 + W2 > F1 ? F0 + W2 : F1);
    tsl = ((F1 + key % (16u - F1)) << 4) | (((F0 + key % W2) | ((G2 + key % (16u - G2)) << 4)) << 8) | (((F0 + key % 4u) | ((G1 + key % 4u) << 4)) << 16);
    static_assert(G1 + 3u + 4u <= 13u, "the last segment's pushes must leave before the final K-tile pair");
  }
  s.ts0 = -1; s.ts1 = -1; s.poison = 0u;
  s.d0 = s.d1 = 0ull;
  asm volatile("" : "+s"(s.d0), "+s"(s.d1), "+s"(s.ts0), "+s"(s.ts1));
  s.R0 = (u32x4){0u, 0u, 0u, 0u}; s.R1 = s.R0;
#pragma unroll
  for (int a = 0; a < 2; ++a)
#pragma unroll
    for (int bb = 0; bb < 2; ++bb)
#pragma unroll
      for (int m = 0; m < 4; ++m)
#pragma unroll
        for (int n = 0; n < 2; ++n) asm volatile("" : "=v"(s.acc[a][bb][m][n]));
  s.nxtBL = s.sgBL; s.nxtBR = offB(2);
  unsigned bl2 = offB(3), br3 = offB(4);
  ob0 += ds8 + (ent(0) & 7u) * 256u; ob1 += ds4 + (ent(1) & 7u) * 256u; ob2 += ds4 + (ent(2) & 7u) * 256u;
  unsigned pf0 = WS_PFLAG + grp * 1024u;
  asm volatile("" : "+s"(bl2), "+s"(br3), "+s"(ob0), "+s"(ob1), "+s"(ob2), "+s"(pf0));
  asm volatile("s_waitcnt lgkmcnt(0)" : "+s"(rd4), "+s"(rd8), "+s"(ds4), "+s"(ds8), "+s"(pw0), "+s"(adj8), "+s"(tsl), "+s"(s.wslot), "+s"(s.nxtBL), "+s"(s.nxtBR) :: "memory");
  if (wr == 1) BAR;
  wait_v<11>(); BAR;
  BAR;
  {
    LDB(s.B0, 0u, 0); WAIT_B(s.B0);
    ktile<0, V_FIRST>(s);
    asm volatile("" : "+v"(callv));
    s.call = __builtin_amdgcn_readfirstlane(callv) + 1u;
    ktile<1, V_PLAIN>(s);
    for (int t = 2; t < 12; t += 2) { ktile<0, V_PLAIN>(s); ktile<1, V_PLAIN>(s); }
    ktile<12, V_PLAIN>(s); ktile<13, V_PLAIN>(s);
  }
  const unsigned call = s.call;
  for (unsigned seg = 1; seg <= 3u; ++seg) {
    const unsigned k = seg - 1u;
    s.pw = pw0 + k * REGION;
    s.lead = (0xC800u >> (seg * 4u + pm)) & 1u;
    s.own = k == 0u ? 1u << mem : 3u << (2u * mi);
    s.vls = s.vlane16 + (lsp + uw * 4096u + (k == 0u ? adj8 : 0u));
    ktile<14, V_BG>(s);
    if (seg == 2u) {
      ktile<15, V_L>(s);
      ktile<0, V_L>(s);
    }
    unsigned seg2 = seg; asm volatile("" : "+s"(seg2));
    if (seg2 != 2u) {
      ktile<15, V_R>(s);
      ktile<0, V_R>(s);
    }
    s.keep = (unsigned)((int)(k - 1u) >> 31);
    s.prd0 = WS_PART + k * REGION + (k == 0u ? rd8 : rd4);
    s.prd1 = k == 0u ? s.prd0 ^ (4u * SLOT) : s.prd0 + 128u;
    s.lrd = lsp + (s.prd0 & 0x7fffu) + (k == 0u ? adj8 : 0u);
    s.pflag = pf0 + k * 32768u;
    s.TS0 = (int)(tsl >> (8u * k));
    s.pd = k == 0u ? ob0 : k == 1u ? ob1 : ob2;
    s.nxtBL = bl2;
    s.nxtBR = k == 0u ? s.sgBR : br3;
    asm volatile("" : "+s"(s.nxtBL), "+s"(s.nxtBR), "+s"(s.pd), "+s"(s.pflag), "+s"(s.prd0), "+s"(s.prd1), "+s"(s.lrd), "+s"(s.TS0), "+s"(s.keep));
    ktile<1, V_BG>(s); ktile<2, V_BG>(s); ktile<3, V_BG>(s); ktile<4, V_BG>(s); ktile<5, V_BG>(s); ktile<6, V_BG>(s); ktile<7, V_BG>(s);
    ktile<8, V_BG>(s); ktile<9, V_BG>(s); ktile<10, V_BG>(s); ktile<11, V_BG>(s); ktile<12, V_BG>(s); ktile<13, V_BG>(s);
  }
  s.pw = pw0 + 3u * REGION;
  const unsigned fblk = pf0 + 3u * 32768u;
  final_pair(s, fblk, 1u << mem, adj8);
  GUARD(s.acc[0][1]); pack_quad(&s.tmp[0], s.acc[0][1]); PST(0); PST(1); PST(2); PST(3);
  asm volatile("s_waitcnt vmcnt(0) lgkmcnt(0)" ::: "memory"); PFLAG(fblk + 32768u);
  const int lane2 = lane_id();
  const unsigned long long o3 = (unsigned long long)p.out[ent(3) >> 3], o4 = (unsigned long long)p.out[0];
  char* const w0 = p.ws[ent(0) >> 3]; char* const w1 = p.ws[ent(1) >> 3]; char* const w2 = p.ws[ent(2) >> 3]; char* const w3 = p.ws[ent(3) >> 3];
  const unsigned o3lo = (unsigned)o3, o3hi = (unsigned)(o3 >> 32) & 0xffffu, o4lo = (unsigned)o4, o4hi = (unsigned)(o4 >> 32) & 0xffffu;
  if (wr == 0) BAR;
  LAS unsigned* lflag = (LAS unsigned*)(smem + 8 * HTB + ((((2u * (mi & 1u)) ^ 2u) + 1u) * 1024u));
  if (wid == 0) {
    const unsigned bad = spin_flags(s, fblk);
    if (lane2 == 0) lflag[0] = bad;
    asm volatile("s_waitcnt lgkmcnt(0)" ::: "memory");
  }
  BAR;
  {
    s.prd0 = WS_PART + 3u * REGION + rd4; s.prd1 = s.prd0 + 128u; s.lrd = lsp + (rd4 & 0x7fffu);
    PLDG(1); PLDG(2); PLDG(3); PLDG(5); PLDG(6); PLDG(7); PLDL(0, 0); PLDL(4, 128);
    make_sel(s);
    if (wid == 0) {
      const unsigned bad = spin_flags(s, fblk + 32768u);
      if (lane2 == 0) lflag[1] = bad;
      asm volatile("s_waitcnt lgkmcnt(0)" ::: "memory");
    }
    s.poison |= __builtin_amdgcn_readfirstlane(*(volatile LAS unsigned*)lflag);
    asm volatile("s_waitcnt vmcnt(3) lgkmcnt(0)" : "+v"(s.tmp[0]), "+v"(s.tmp[1]), "+v"(s.tmp[2]), "+v"(s.tmp[3]) :: "memory");
    reduce4<1>(s, 0);
    const u32x4 o0 = reduce_fin<true>(s);
    asm volatile("s_waitcnt vmcnt(0)" : "+v"(s.tmp[4]), "+v"(s.tmp[5]), "+v"(s.tmp[6]), "+v"(s.tmp[7]) :: "memory");
    reduce4<1>(s, 4);
    const u32x4 o1 = reduce_fin<true>(s);
    BAR;
    s.prd0 = WS_PART + 4u * REGION + rd8; s.prd1 = s.prd0 ^ (4u * SLOT); s.lrd = lsp + (rd8 & 0x7fffu) + adj8;
    PLDG(1); PLDG(2); PLDG(3); PLDG(4); PLDG(5); PLDG(6); PLDG(7); PLDL(0, 0);
    const unsigned doff = ds4 + (ent(3) & 7u) * 256u;
    SEND(o0, o3lo, o3hi, doff);
    SEND(o1, o3lo, o3hi, doff + 8u * 2048u);
    s.poison |= __builtin_amdgcn_readfirstlane(*(volatile LAS unsigned*)(lflag + 1));
    asm volatile("s_waitcnt vmcnt(2) lgkmcnt(0)" : "+v"(s.tmp[0]), "+v"(s.tmp[1]), "+v"(s.tmp[2]), "+v"(s.tmp[3]), "+v"(s.tmp[4]), "+v"(s.tmp[5]), "+v"(s.tmp[6]), "+v"(s.tmp[7]) :: "memory");
    reduce4<1>(s, 0); reduce4<0>(s, 4);
    const u32x4 o2 = reduce_fin<true>(s);
    SEND(o2, o4lo, o4hi, ds8 + (ent(4) & 7u) * 256u);
  }
  wait_v<0>();
  BAR;
  if (wid == 0) {
    if (lane2 < 4) {
      const unsigned e = (info >> (6u * (unsigned)lane2)) & 63u;
      char* const wd = lane2 == 0 ? w0 : lane2 == 1 ? w1 : lane2 == 2 ? w2 : w3;
      unsigned* f = (unsigned*)(wd + WS_DONE + rank * 1024u) + ((e & 7u) * 4u + pm) * 8u;
      if (lane2 == 0) __hip_atomic_store(f + mem, call, __ATOMIC_RELAXED, __HIP_MEMORY_SCOPE_SYSTEM);
      else __hip_atomic_store((unsigned long long*)f + mi, ((unsigned long long)call << 32) | call, __ATOMIC_RELAXED, __HIP_MEMORY_SCOPE_SYSTEM);
    }
    if (lane2 == 0) __hip_atomic_store((unsigned*)(wsp + WS_CALL + blk * 64u), call, __ATOMIC_RELAXED, __HIP_MEMORY_SCOPE_AGENT);
    if (blk == 0) {
      const unsigned long long* f = (const unsigned long long*)(wsp + WS_DONE) + lane2;
      unsigned* tmo = (unsigned*)(wsp + WS_TMO);
      const unsigned long long t0 = __builtin_amdgcn_s_memrealtime();
      for (unsigned n = 1;; ++n) {
        bool ok = true;
#pragma unroll
        for (unsigned j = 1; j < 8; ++j) {
          const unsigned long long v0 = __hip_atomic_load(f + ((rank + j) & 7u) * 128u, __ATOMIC_RELAXED, __HIP_MEMORY_SCOPE_SYSTEM);
          const unsigned long long v1 = __hip_atomic_load(f + ((rank + j) & 7u) * 128u + 64u, __ATOMIC_RELAXED, __HIP_MEMORY_SCOPE_SYSTEM);
          ok = ok && ((unsigned)v0 >= call) && ((unsigned)(v0 >> 32) >= call) && ((unsigned)v1 >= call) && ((unsigned)(v1 >> 32) >= call);
        }
        if (__all(ok)) break;
        if ((n & 15u) == 0u &&
            (__builtin_amdgcn_readfirstlane(__hip_atomic_load(tmo, __ATOMIC_RELAXED, __HIP_MEMORY_SCOPE_AGENT)) == call ||
             __builtin_amdgcn_s_memrealtime() - t0 > DEADLINE)) {
          if (lane2 == 0) {
            __hip_atomic_store(tmo, call, __ATOMIC_RELAXED, __HIP_MEMORY_SCOPE_AGENT);
            __hip_atomic_store((unsigned*)(o4 + (size_t)rank * 1024u * 2048u), 0x7fc07fc0u, __ATOMIC_RELAXED, __HIP_MEMORY_SCOPE_SYSTEM);
          }
          break;
        }
      }
      __builtin_amdgcn_fence(__ATOMIC_ACQUIRE, "");
    }
  }
}

}

extern "C" void kernel_launch(int rank, int nranks, ncclComm_t comm,
                              hipStream_t stream,
                              void* const* d_in, const int* in_sizes, int n_in,
                              void* d_out, int out_size,
                              const int* in_dtypes, int out_dtype,
                              void* const* d_out_all,
                              void* const* d_ws_all, size_t ws_bytes) {
  using namespace ga2a;
  (void)comm; (void)in_sizes; (void)n_in; (void)d_out; (void)out_size; (void)in_dtypes; (void)out_dtype;
  if (nranks != NR || ws_bytes < (size_t)WS_USED) { fprintf(stderr, "unsupported configuration\n"); return; }
  Params p;
  p.A = (const u16*)d_in[0];
  p.W = (const u16*)d_in[1];
  for (int j = 0; j < NR; ++j) { p.out[j] = (char*)d_out_all[(rank + j) & 7]; p.ws[j] = (char*)d_ws_all[(rank + j) & 7]; }
  p.rank = rank;
  hipLaunchKernelGGL(gemm_a2a, dim3(256), dim3(512), 0, stream, p);
  HIP_CHECK(hipGetLastError());
}
```

```cpp
#include <hip/hip_runtime.h>
#include <rccl/rccl.h>
#include <cstdio>

#define HIP_CHECK(expr) do { hipError_t _e = (expr); if (_e != hipSuccess) { \
    fprintf(stderr, "HIP error %s at %s:%d\n", hipGetErrorString(_e), __FILE__, __LINE__); \
    return; } } while (0)

namespace ga2a {

#define LAS __attribute__((address_space(3)))
typedef unsigned short u16;
typedef short bf16x8 __attribute__((ext_vector_type(8)));
typedef short bf16x4 __attribute__((ext_vector_type(4)));
typedef float f32x4 __attribute__((ext_vector_type(4)));
typedef float f32x2 __attribute__((ext_vector_type(2)));
typedef __bf16 bf16x2n __attribute__((ext_vector_type(2)));
typedef unsigned u32x4 __attribute__((ext_vector_type(4)));

constexpr int NR = 8;
constexpr int M_ = 1024, N_ = 8192, K_ = 8192;
constexpr unsigned HTB = 16384;
constexpr unsigned A_BYTES = (unsigned)M_ * K_ * 2u, W_BYTES = (unsigned)K_ * N_ * 2u;
constexpr unsigned OUT_BYTES = 8192u * 1024u * 2u;
constexpr unsigned SLOT = 65536u;
constexpr unsigned REGION = 256u * SLOT;
constexpr unsigned WS_PART = 0u;
constexpr unsigned WS_CTRL = 128u << 20;
constexpr unsigned WS_PFLAG = WS_CTRL + 524288u;
constexpr unsigned WS_CALL = WS_CTRL + 131072u;
constexpr unsigned WS_DONE = WS_CTRL + 163840u;
constexpr unsigned WS_TMO = WS_CTRL + 180224u;
constexpr unsigned WS_USED = WS_CTRL + (1u << 20);
constexpr unsigned long long DEADLINE = 50000000ull;

constexpr unsigned ktA = 128u, ktB = 64u * N_ * 2u;
constexpr unsigned hA = 128u * K_ * 2u;
constexpr unsigned iA = 64u * K_ * 2u, iB = 32u * N_ * 2u;

struct Params {
  const u16* A;
  const u16* W;
  char* out[NR];
  char* ws[NR];
  int rank;
};

enum { V_PLAIN = 0, V_BG = 1, V_L = 2, V_R = 3, V_FIRST = 4 };
constexpr int PSTP[8] = {-1, -1, 0, 0, 1, 1, 2, 2};
constexpr int PLDP[8] = {13, 14, 15, 16, 17, 18, 19, 20};
constexpr int T_FLAG = 1, T_POLL = 2;
constexpr int T_TIE0 = PLDP[3] / 4 + 1, T_TIE1 = PLDP[7] / 4 + 1;
constexpr int S0_FIRST = T_TIE0 + 1, S0_N = 16 - S0_FIRST, S1_FIRST = T_TIE1 + 1, S1_N = 20 - S1_FIRST;
constexpr int count_at(const int (&t)[8], int kt) { int n = 0; for (int i = 0; i < 8; ++i) n += (t[i] >= kt * 4 && t[i] < kt * 4 + 4) ? 1 : 0; return n; }
constexpr int nbg(int p) { int n = 0; for (int i = 0; i < 8; ++i) n += (PSTP[i] == p ? 1 : 0) + ((PLDP[i] == p && i != 0 && i != 4) ? 1 : 0); return n; }
constexpr int lag5(int p) { int n = 10; for (int q = p - 5; q <= p; ++q) n += nbg(q); return n; }
constexpr bool slot_at(int first, int n, int i) { return (i >= first && i < first + n) || (i + 16 >= first && i + 16 < first + n); }

__device__ __forceinline__ unsigned pk2(float lo, float hi) {
  f32x2 v = {lo, hi};
  return __builtin_bit_cast(unsigned, __builtin_convertvector(v, bf16x2n));
}
__device__ __forceinline__ u32x4 pack_pair(const f32x4& v0, const f32x4& v1) {
  unsigned d00 = pk2(v0[0], v0[1]), d01 = pk2(v0[2], v0[3]);
  unsigned d10 = pk2(v1[0], v1[1]), d11 = pk2(v1[2], v1[3]);
  auto r0 = __builtin_amdgcn_permlane16_swap(d00, d10, false, false);
  auto r1 = __builtin_amdgcn_permlane16_swap(d01, d11, false, false);
  u32x4 w; w.x = r0[0]; w.y = r1[0]; w.z = r0[1]; w.w = r1[1];
  return w;
}

struct St {
  f32x4 acc[2][2][4][2];
  bf16x8 At[4][2];
  bf16x4 B0[2][2][2], B1[2][2][2];
  u32x4 tmp[8];
  f32x4 D1, D2;
  u32x4 S1, S2;
  u32x4 R0, R1;
  unsigned pollv;
  unsigned voffA, voffB, aoff0, aoff1, boff0, boff1, vlane16, vgat, vdst;
  u32x4 srdA, srdB, srdWs;
  unsigned ldsw;
  unsigned a1, a2, bl2, br2, kA;
  unsigned sgA, sgBL, sgBR, nxtBL, nxtBR;
  unsigned pw;
  unsigned prd0, prd1;
  unsigned own, lrd;
  unsigned vls;
  unsigned pflag;
  unsigned keep;
  unsigned call, poison;
  unsigned long long pd;
  unsigned long long d0, d1;
  int ts0, ts1, TS0, wslot;
};

#define SA(b, h) (((b) * 2u + (h)) * HTB)
#define SB(b, h) ((4u + (b) * 2u + (h)) * HTB)
#define DMA2(ldsoff, voff, srd, sbase, simm, step, bits) do { unsigned _t; asm volatile( \
    "s_add_u32 m0, %1, %2\n\ts_add_u32 %0, %6, %7\n\tbuffer_load_dwordx4 %4, %5, %0 offen " bits " lds\n\t" \
    "s_add_u32 m0, %1, %3\n\ts_add_u32 %0, %6, %8\n\tbuffer_load_dwordx4 %4, %5, %0 offen " bits " lds" \
    : "=&s"(_t) : "s"(s.ldsw), "i"(ldsoff), "i"((ldsoff) + 8192u), "v"(voff), "s"(srd), "s"(sbase), "i"(simm), "i"((simm) + (step)) : "memory", "scc"); } while (0)
#define STAGE_A(bufoff, sbase, simm) DMA2((bufoff), s.voffA, s.srdA, (sbase), (simm), iA, "")
#define STAGE_B(bufoff, sbase, simm) DMA2((bufoff), s.voffB, s.srdB, (sbase), (simm), iB, "")
__device__ __forceinline__ int lane_id() { int l; asm volatile("v_mbcnt_lo_u32_b32 %0, -1, 0\n\tv_mbcnt_hi_u32_b32 %0, -1, %0" : "=v"(l)); return l; }
#define RD128(dst, addr, off) asm volatile("ds_read_b128 %0, %1 offset:%2" : "=v"(dst) : "v"(addr), "i"(off))
#define RDTR(dst, addr, off) asm volatile("ds_read_b64_tr_b16 %0, %1 offset:%2" : "=v"(dst) : "v"(addr), "i"(off))
#define LDA(b, h) do { _Pragma("unroll") for (int m = 0; m < 4; ++m) { RD128(s.At[m][0], s.aoff0, SA(b, h) + m * 2048); RD128(s.At[m][1], s.aoff1, SA(b, h) + m * 2048); } } while (0)
#define LDB(dst, b, h) do { _Pragma("unroll") for (int k = 0; k < 2; ++k) { \
    RDTR(dst[0][k][0], s.boff0, SB(b, h) - 65536u + k * 8192); RDTR(dst[0][k][1], s.boff0, SB(b, h) - 65536u + k * 8192 + 1024); } \
    _Pragma("unroll") for (int k = 0; k < 2; ++k) { \
    RDTR(dst[1][k][0], s.boff1, SB(b, h) - 65536u + k * 8192); RDTR(dst[1][k][1], s.boff1, SB(b, h) - 65536u + k * 8192 + 1024); } } while (0)
#define WAIT_A() asm volatile("s_waitcnt lgkmcnt(0)" : "+v"(s.At[0][0]), "+v"(s.At[0][1]), "+v"(s.At[1][0]), "+v"(s.At[1][1]), "+v"(s.At[2][0]), "+v"(s.At[2][1]), "+v"(s.At[3][0]), "+v"(s.At[3][1]) :: "memory")
#define WAIT_B(Bt) asm volatile("s_waitcnt lgkmcnt(0)" : "+v"(Bt[0][0][0]), "+v"(Bt[0][0][1]), "+v"(Bt[0][1][0]), "+v"(Bt[0][1][1]), "+v"(Bt[1][0][0]), "+v"(Bt[1][0][1]), "+v"(Bt[1][1][0]), "+v"(Bt[1][1][1]) :: "memory")
#define CAT(x) __builtin_shufflevector(x[0], x[1], 0, 1, 2, 3, 4, 5, 6, 7)
#define BAR __builtin_amdgcn_s_barrier()
#define SCHED __builtin_amdgcn_sched_barrier(0)
#define ENDM do { SCHED; asm volatile("s_setprio 0"); BAR; SCHED; } while (0)
#define ST128(data, voff, srd, soff, bits) asm volatile("buffer_store_dwordx4 %0, %1, %2, %3 offen " bits "\n\ts_nop 1" :: "v"(data), "v"(voff), "s"(srd), "s"(soff) : "memory")
#define LD128(dst, voff, srd, soff) asm volatile("buffer_load_dwordx4 %0, %1, %2, %3 offen sc1" : "=v"(dst) : "v"(voff), "s"(srd), "s"(soff) : "memory")
#define PST(u) do { unsigned _t; asm volatile( \
    "s_bitcmp1_b32 %1, %2\n\ts_cselect_b32 %0, %3, %4\n\ts_cbranch_scc1 1f\n" \
    "2:\n\ts_add_u32 %0, %0, %8\n\tbuffer_store_dwordx4 %6, %9, %10, %0 offen sc1\n\ts_nop 1\n\t" \
    ".subsection 1\n1:\n\tds_write_b128 %5, %6 offset:%7\n\ts_branch 2b\n\t.subsection 0" \
    : "=&s"(_t) : "s"(s.own), "i"(u), "s"(0xC0000000u), "s"(s.pw), "v"(s.vls), "v"(s.tmp[u]), "i"(((u) & 3) * 1024), "i"(((u) >> 2) * 32768u + ((u) & 3) * 1024u), "v"(s.vlane16), "s"(s.srdWs) : "memory", "scc"); } while (0)
#define PLDG(j) LD128(s.tmp[j], s.vgat, s.srdWs, ((j) < 4 ? s.prd0 : s.prd1) ^ (((j) & 3) * SLOT))
#define PLDL(j, add) do { unsigned _a; asm volatile("v_add_u32 %1, %2, %3\n\tds_read_b128 %0, %1 offset:%4" : "=v"(s.tmp[j]), "=&v"(_a) : "s"(s.lrd), "v"(s.vgat), "i"(add) : "memory"); } while (0)
#define PLD(j) do { if ((j) == 0) PLDL(0, 0); else if ((j) == 4) { if (s.keep != 0u) PLDG(4); unsigned _k = s.keep; asm volatile("" : "+s"(_k)); if (_k == 0u) PLDL(4, 128); } else PLDG(j); } while (0)

template <int N> __device__ __forceinline__ void wait_v() { asm volatile("s_waitcnt vmcnt(%0)" :: "i"(N) : "memory"); }

template <bool INIT>
__device__ __forceinline__ void mma(f32x4 (&acc)[4][2], const bf16x8 (&At)[4][2], const bf16x4 (&Bt)[2][2][2]) {
  const bf16x8 b00 = CAT(Bt[0][0]), b01 = CAT(Bt[0][1]), b10 = CAT(Bt[1][0]), b11 = CAT(Bt[1][1]);
  if constexpr (INIT)
    asm volatile("s_setprio 1\n\ts_nop 0\n\t"
      "v_mfma_f32_16x16x32_bf16 %0, %16, %8, 0\n\t"
      "v_mfma_f32_16x16x32_bf16 %0, %17, %9, %0\n\t"
      "v_mfma_f32_16x16x32_bf16 %1, %18, %8, 0\n\t"
      "v_mfma_f32_16x16x32_bf16 %1, %19, %9, %1\n\t"
      "v_mfma_f32_16x16x32_bf16 %2, %16, %10, 0\n\t"
      "v_mfma_f32_16x16x32_bf16 %2, %17, %11, %2\n\t"
      "v_mfma_f32_16x16x32_bf16 %3, %18, %10, 0\n\t"
      "v_mfma_f32_16x16x32_bf16 %3, %19, %11, %3\n\t"
      "v_mfma_f32_16x16x32_bf16 %4, %16, %12, 0\n\t"
      "v_mfma_f32_16x16x32_bf16 %4, %17, %13, %4\n\t"
      "v_mfma_f32_16x16x32_bf16 %5, %18, %12, 0\n\t"
      "v_mfma_f32_16x16x32_bf16 %5, %19, %13, %5\n\t"
      "v_mfma_f32_16x16x32_bf16 %6, %16, %14, 0\n\t"
      "v_mfma_f32_16x16x32_bf16 %6, %17, %15, %6\n\t"
      "v_mfma_f32_16x16x32_bf16 %7, %18, %14, 0\n\t"
      "v_mfma_f32_16x16x32_bf16 %7, %19, %15, %7\n\t"
      : "+v"(acc[0][0]), "+v"(acc[0][1]), "+v"(acc[1][0]), "+v"(acc[1][1]), "+v"(acc[2][0]), "+v"(acc[2][1]), "+v"(acc[3][0]), "+v"(acc[3][1])
      : "v"(At[0][0]), "v"(At[0][1]), "v"(At[1][0]), "v"(At[1][1]), "v"(At[2][0]), "v"(At[2][1]), "v"(At[3][0]), "v"(At[3][1]), "v"(b00), "v"(b01), "v"(b10), "v"(b11));
  else
    asm volatile("s_setprio 1\n\ts_nop 0\n\t"
      "v_mfma_f32_16x16x32_bf16 %0, %16, %8, %0\n\t"
      "v_mfma_f32_16x16x32_bf16 %0, %17, %9, %0\n\t"
      "v_mfma_f32_16x16x32_bf16 %1, %18, %8, %1\n\t"
      "v_mfma_f32_16x16x32_bf16 %1, %19, %9, %1\n\t"
      "v_mfma_f32_16x16x32_bf16 %2, %16, %10, %2\n\t"
      "v_mfma_f32_16x16x32_bf16 %2, %17, %11, %2\n\t"
      "v_mfma_f32_16x16x32_bf16 %3, %18, %10, %3\n\t"
      "v_mfma_f32_16x16x32_bf16 %3, %19, %11, %3\n\t"
      "v_mfma_f32_16x16x32_bf16 %4, %16, %12, %4\n\t"
      "v_mfma_f32_16x16x32_bf16 %4, %17, %13, %4\n\t"
      "v_mfma_f32_16x16x32_bf16 %5, %18, %12, %5\n\t"
      "v_mfma_f32_16x16x32_bf16 %5, %19, %13, %5\n\t"
      "v_mfma_f32_16x16x32_bf16 %6, %16, %14, %6\n\t"
      "v_mfma_f32_16x16x32_bf16 %6, %17, %15, %6\n\t"
      "v_mfma_f32_16x16x32_bf16 %7, %18, %14, %7\n\t"
      "v_mfma_f32_16x16x32_bf16 %7, %19, %15, %7\n\t"
      : "+v"(acc[0][0]), "+v"(acc[0][1]), "+v"(acc[1][0]), "+v"(acc[1][1]), "+v"(acc[2][0]), "+v"(acc[2][1]), "+v"(acc[3][0]), "+v"(acc[3][1])
      : "v"(At[0][0]), "v"(At[0][1]), "v"(At[1][0]), "v"(At[1][1]), "v"(At[2][0]), "v"(At[2][1]), "v"(At[3][0]), "v"(At[3][1]), "v"(b00), "v"(b01), "v"(b10), "v"(b11));
}
template <bool INIT>
__device__ __forceinline__ void mma2(f32x4& c, const bf16x8& b0, const bf16x8& a0, const bf16x8& b1, const bf16x8& a1) {
  if constexpr (INIT) asm volatile("v_mfma_f32_16x16x32_bf16 %0, %1, %2, 0\n\tv_mfma_f32_16x16x32_bf16 %0, %3, %4, %0" : "+v"(c) : "v"(b0), "v"(a0), "v"(b1), "v"(a1));
  else asm volatile("v_mfma_f32_16x16x32_bf16 %0, %1, %2, %0\n\tv_mfma_f32_16x16x32_bf16 %0, %3, %4, %0" : "+v"(c) : "v"(b0), "v"(a0), "v"(b1), "v"(a1));
}
template <bool INIT>
__device__ __forceinline__ void mma_step(St& s, f32x4 (&acc)[4][2], const bf16x8 (&At)[4][2], const bf16x4 (&Bt)[2][2][2]) {
  const bf16x8 b00 = CAT(Bt[0][0]), b01 = CAT(Bt[0][1]), b10 = CAT(Bt[1][0]), b11 = CAT(Bt[1][1]);
  unsigned t;
  asm volatile("s_setprio 1\n\ts_nop 0");
  mma2<INIT>(acc[0][0], b00, At[0][0], b01, At[0][1]);
  asm volatile("s_add_u32 %0, %0, 0x80\n\ts_and_b32 %0, %0, 0xf80" : "+s"(s.kA) :: "scc");
  mma2<INIT>(acc[0][1], b10, At[0][0], b11, At[0][1]);
  asm volatile("s_mov_b32 %0, %1\n\ts_add_u32 %1, %2, %3" : "=&s"(s.a1), "+s"(s.a2) : "s"(s.sgA), "s"(s.kA) : "scc");
  mma2<INIT>(acc[1][0], b00, At[1][0], b01, At[1][1]);
  asm volatile("s_lshl_b32 %0, %1, 13" : "=s"(t) : "s"(s.kA) : "scc");
  mma2<INIT>(acc[1][1], b10, At[1][0], b11, At[1][1]);
  asm volatile("s_add_u32 %0, %2, %4\n\ts_add_u32 %1, %3, %4" : "=&s"(s.bl2), "=&s"(s.br2) : "s"(s.sgBL), "s"(s.sgBR), "s"(t) : "scc");
  mma2<INIT>(acc[2][0], b00, At[2][0], b01, At[2][1]);
  mma2<INIT>(acc[2][1], b10, At[2][0], b11, At[2][1]);
  mma2<INIT>(acc[3][0], b00, At[3][0], b01, At[3][1]);
  mma2<INIT>(acc[3][1], b10, At[3][0], b11, At[3][1]);
}
__device__ __forceinline__ void pack_quad(u32x4* dst, f32x4 (&q)[4][2]) {
  asm volatile("" : "+v"(q[0][0]), "+v"(q[0][1]), "+v"(q[1][0]), "+v"(q[1][1]), "+v"(q[2][0]), "+v"(q[2][1]), "+v"(q[3][0]), "+v"(q[3][1]));
#pragma unroll
  for (int m = 0; m < 4; ++m) { dst[m] = pack_pair(q[m][0], q[m][1]); asm volatile("" : "+v"(dst[m])); }
}
__device__ __forceinline__ void make_sel(St& s) {
  unsigned l = s.vlane16; asm volatile("" : "+v"(l));
  const unsigned one = ((l >> 8) == ((l >> 6) & 3u)) ? (0x3f80u << (l & 16u)) : 0u;
  const unsigned a = (l & 32u) ? 0u : one, b = (l & 32u) ? one : 0u;
  s.S1 = (u32x4){a, b, 0u, 0u}; s.S2 = (u32x4){0u, 0u, a, b};
  asm volatile("s_nop 1" : "+v"(s.S1), "+v"(s.S2));
}
template <int FIRST>
__device__ __forceinline__ void reduce4(St& s, int j) {
  if constexpr (FIRST == 1)
    asm volatile("v_mfma_f32_16x16x32_bf16 %0, %2, %4, 0\n\tv_mfma_f32_16x16x32_bf16 %1, %3, %4, 0\n\t"
                 "v_mfma_f32_16x16x32_bf16 %0, %2, %5, %0\n\tv_mfma_f32_16x16x32_bf16 %1, %3, %5, %1\n\t"
                 "v_mfma_f32_16x16x32_bf16 %0, %2, %6, %0\n\tv_mfma_f32_16x16x32_bf16 %1, %3, %6, %1\n\t"
                 "v_mfma_f32_16x16x32_bf16 %0, %2, %7, %0\n\tv_mfma_f32_16x16x32_bf16 %1, %3, %7, %1"
                 : "=&v"(s.D1), "=&v"(s.D2) : "v"(s.S1), "v"(s.S2), "v"(s.tmp[j]), "v"(s.tmp[j + 1]), "v"(s.tmp[j + 2]), "v"(s.tmp[j + 3]));
  else if constexpr (FIRST == 2)
    asm volatile("v_mfma_f32_16x16x32_bf16 %0, %2, %4, 0\n\tv_mfma_f32_16x16x32_bf16 %1, %3, %4, 0\n\t"
                 "v_mfma_f32_16x16x32_bf16 %0, %2, %5, %0\n\tv_mfma_f32_16x16x32_bf16 %1, %3, %5, %1\n\t"
                 "v_mfma_f32_16x16x32_bf16 %0, %2, %6, %0\n\tv_mfma_f32_16x16x32_bf16 %1, %3, %6, %1\n\t"
                 "v_mfma_f32_16x16x32_bf16 %0, %2, %7, %0\n\tv_mfma_f32_16x16x32_bf16 %1, %3, %7, %1"
                 : "+v"(s.D1), "+v"(s.D2) : "v"(s.S1), "v"(s.S2), "v"(s.tmp[j]), "v"(s.tmp[j + 1]), "v"(s.tmp[j + 2]), "v"(s.tmp[j + 3]));
  else
    asm volatile("v_mfma_f32_16x16x32_bf16 %0, %2, %4, %0\n\tv_mfma_f32_16x16x32_bf16 %1, %3, %4, %1\n\t"
                 "v_mfma_f32_16x16x32_bf16 %0, %2, %5, %0\n\tv_mfma_f32_16x16x32_bf16 %1, %3, %5, %1\n\t"
                 "v_mfma_f32_16x16x32_bf16 %0, %2, %6, %0\n\tv_mfma_f32_16x16x32_bf16 %1, %3, %6, %1\n\t"
                 "v_mfma_f32_16x16x32_bf16 %0, %2, %7, %0\n\tv_mfma_f32_16x16x32_bf16 %1, %3, %7, %1"
                 : "+v"(s.D1), "+v"(s.D2) : "v"(s.S1), "v"(s.S2), "v"(s.tmp[j]), "v"(s.tmp[j + 1]), "v"(s.tmp[j + 2]), "v"(s.tmp[j + 3]));
}
template <bool NOP>
__device__ __forceinline__ u32x4 reduce_fin(St& s) {
  if constexpr (NOP) asm volatile("s_nop 7" : "+v"(s.D1), "+v"(s.D2)); else asm volatile("" : "+v"(s.D1), "+v"(s.D2));
  u32x4 r;
  r.x = pk2(s.D1[0], s.D1[1]) | s.poison; r.y = pk2(s.D1[2], s.D1[3]) | s.poison;
  r.z = pk2(s.D2[0], s.D2[1]) | s.poison; r.w = pk2(s.D2[2], s.D2[3]) | s.poison;
  asm volatile("" : "+v"(r));
  return r;
}
__device__ __forceinline__ unsigned spin_flags(const St& s, unsigned blk) {
  unsigned bad, v0, v1, st, t0, t1;
  asm volatile(
      "s_memrealtime vcc\n\ts_mov_b32 %0, 0\n\ts_waitcnt lgkmcnt(0)\n\ts_mov_b32 %4, vcc_lo\n"
      "1:\n\t"
      "buffer_load_dword %1, %6, %7, %8 offen sc0 sc1\n\t"
      "buffer_load_dword %2, off, %7, %9 sc0 sc1\n\t"
      "s_memrealtime vcc\n\t"
      "s_waitcnt vmcnt(0) lgkmcnt(0)\n\t"
      "s_sub_u32 %5, vcc_lo, %4\n\t"
      "v_cmp_gt_u32 vcc, %10, %1\n\t"
      "v_readfirstlane_b32 %3, %2\n\t"
      "s_cmp_eq_u64 vcc, 0\n\t"
      "s_cbranch_scc1 3f\n\t"
      "s_cmp_eq_u32 %3, %10\n\t"
      "s_cbranch_scc1 2f\n\t"
      "s_cmp_lt_u32 %5, %11\n\t"
      "s_cbranch_scc1 1b\n"
      "2:\n\t"
      "v_mov_b32 %1, %10\n\ts_mov_b64 exec, 1\n\tbuffer_store_dword %1, off, %7, %9 sc0 sc1\n\ts_mov_b64 exec, -1\n\t"
      "s_mov_b32 %0, 0x7fc07fc0\n\ts_waitcnt vmcnt(0)\n"
      "3:"
      : "=&s"(bad), "=&v"(v0), "=&v"(v1), "=&s"(st), "=&s"(t0), "=&s"(t1)
      : "v"(s.vlane16), "s"(s.srdWs), "s"(blk), "s"(WS_TMO), "s"(s.call), "s"((unsigned)DEADLINE)
      : "memory", "scc", "vcc");
  return bad;
}
#define SEND(R, lo, hi, off) do { const u32x4 _srd = {lo, hi, OUT_BYTES, 0x20000u}; ST128(R, s.vdst, _srd, off, "sc0 sc1"); } while (0)

#define SLOT_SEND(R, dst, ts, when) asm volatile("s_cmp_eq_u32 %0, %1\n\ts_cbranch_scc1 1f\n2:\n\t.subsection 1\n" \
    "1:\n\tglobal_store_dwordx4 %2, %3, %4 sc0 sc1\n\ts_nop 0\n\ts_branch 2b\n\t.subsection 0" :: "s"(ts), "i"(when), "v"(s.vdst), "v"(R), "s"(dst) : "memory", "scc")
#define PFLAG(blk) do { unsigned _c; asm volatile("v_mov_b32 %0, %1\n\ts_mov_b64 exec, 1\n\tbuffer_store_dword %0, off, %2, %3 sc1\n\ts_mov_b64 exec, -1" : "=&v"(_c) : "s"(s.call), "s"(s.srdWs), "s"((blk) + (unsigned)s.wslot * 16u) : "memory"); } while (0)
#define GUARD(q) asm volatile("s_nop 7" : "+v"(q[0][0]), "+v"(q[0][1]), "+v"(q[1][0]), "+v"(q[1][1]), "+v"(q[2][0]), "+v"(q[2][1]), "+v"(q[3][0]), "+v"(q[3][1]))

template <int I, int V>
__device__ __forceinline__ void ktile(St& s) {
  constexpr unsigned b = I & 1, nb = b ^ 1;
  constexpr bool BG = (V != V_PLAIN && V != V_FIRST);
  constexpr bool E15 = (I == 15 && (V == V_L || V == V_R)), E0 = (I == 0 && (V == V_L || V == V_R));
  constexpr int hv = (V == V_R) ? 1 : 0;
  constexpr bool INL = (E0 && hv == 0) || V == V_FIRST, INR = (E0 && hv == 1) || V == V_FIRST;
  constexpr bool S0 = BG && slot_at(S0_FIRST, S0_N, I), S1 = BG && slot_at(S1_FIRST, S1_N, I);
  constexpr int KT = E15 ? -1 : (BG && I != 15 && I != 14) ? I : -100;
  constexpr int NX = count_at(PSTP, KT) + count_at(PLDP, KT) - (PLDP[0] / 4 == KT ? 1 : 0) - (PLDP[4] / 4 == KT ? 1 : 0);
#define BGMEM(ph) do { _Pragma("unroll") for (int u = 0; u < 8; ++u) { if (PSTP[u] == KT * 4 + (ph)) PST(u); if (PLDP[u] == KT * 4 + (ph)) PLD(u); } } while (0)
  LDA(b, 0);
  STAGE_A(SA(nb, 1), s.a1, hA);
  BGMEM(0);
  if constexpr (V == V_FIRST) wait_v<11>();
  BAR; WAIT_A(); mma<INL>(s.acc[0][0], s.At, s.B0);
  if constexpr (E0) pack_quad(&s.tmp[4], s.acc[1][hv]);
  if constexpr (KT == T_TIE0 + 1) { make_sel(s); reduce4<1>(s, 0); }
  if constexpr (KT == T_TIE1 + 1 && T_TIE1 != T_TIE0) { if (s.keep != 0u) reduce4<0>(s, 4); unsigned k2 = s.keep; asm volatile("" : "+s"(k2)); if (k2 == 0u) reduce4<2>(s, 4); }
  ENDM;
  LDB(s.B1, b, 1);
  if constexpr (KT == T_POLL) asm volatile("buffer_load_dword %0, %1, %2, %3 offen sc1" : "=v"(s.pollv) : "v"(s.vlane16), "s"(s.srdWs), "s"(s.pflag) : "memory");
  STAGE_B(SB(b, 0), s.bl2, 0u);
  BGMEM(1);
  if constexpr (V == V_FIRST) wait_v<11>();
  BAR; WAIT_B(s.B1); mma<INR>(s.acc[0][1], s.At, s.B1);
  if constexpr (KT == T_TIE0 + 1) { s.R0 = reduce_fin<false>(s); s.d0 = s.pd; s.ts0 = (4 + (s.TS0 & 15)) | (int)s.keep; asm volatile("" : "+s"(s.d0), "+s"(s.ts0)); }
  if constexpr (KT == T_TIE1 + 1) { s.R1 = reduce_fin<false>(s); s.d1 = s.pd + (~s.keep & (8u * 2048u)); s.ts1 = 4 + ((s.TS0 >> 4) & 15); asm volatile("" : "+s"(s.d1), "+s"(s.ts1)); }
  ENDM;
  LDA(b, 1);
  STAGE_A(SA(b, 0), s.a2, 0u);
  BGMEM(2);
  if constexpr (S0) SLOT_SEND(s.R0, s.d0, s.ts0, I);
  if constexpr (S1) SLOT_SEND(s.R1, s.d1, s.ts1, (I < S1_FIRST ? I + 16 : I));
  wait_v<lag5(KT * 4 + 2)>();
  BAR; WAIT_A(); mma<INL>(s.acc[1][0], s.At, s.B0);
  if constexpr (E15) pack_quad(&s.tmp[0], s.acc[0][hv]);
  ENDM;
  LDB(s.B0, nb, 0);
  STAGE_B(SB(b, 1), s.br2, 0u);
  BGMEM(3);
  if constexpr (KT == T_TIE0) asm volatile("s_waitcnt vmcnt(%4)" : "+v"(s.tmp[0]), "+v"(s.tmp[1]), "+v"(s.tmp[2]), "+v"(s.tmp[3]) : "i"(6 + NX) : "memory");
  else if constexpr (KT == T_TIE1) asm volatile("s_waitcnt vmcnt(%4)" : "+v"(s.tmp[4]), "+v"(s.tmp[5]), "+v"(s.tmp[6]), "+v"(s.tmp[7]) : "i"(6 + NX) : "memory");
  else if constexpr (KT == T_POLL) asm volatile("s_waitcnt vmcnt(%1)" : "+v"(s.pollv) : "i"(6 + NX) : "memory");
  else wait_v<6 + NX>();
  if constexpr (KT == T_FLAG) PFLAG(s.pflag);
  if constexpr (I == 13) { s.sgBL = s.nxtBL; s.sgBR = s.nxtBR; }
  BAR; mma_step<INR>(s, s.acc[1][1], s.At, s.B1);
  WAIT_B(s.B0);
  ENDM;
#undef BGMEM
  if constexpr (KT == T_POLL) {
    if (!__all(s.pollv >= s.call || (s.vlane16 >> 7) == ((unsigned)s.wslot >> 3))) s.poison |= spin_flags(s, s.pflag);
    SCHED;
  }
}

__device__ __forceinline__ void final_pair(St& s, unsigned flagL, unsigned own8, unsigned adj8) {
  LDA(0u, 0); STAGE_A(SA(1u, 1), s.a1, hA);
  BAR; WAIT_A(); mma<false>(s.acc[0][0], s.At, s.B0); ENDM;
  LDA(0u, 1);
  BAR; WAIT_A(); mma<false>(s.acc[1][0], s.At, s.B0); ENDM;
  LDB(s.B1, 0u, 1); wait_v<2>();
  BAR; WAIT_B(s.B1); mma<false>(s.acc[1][1], s.At, s.B1); ENDM;
  LDB(s.B0, 1u, 0); LDA(1u, 0); wait_v<0>();
  BAR; WAIT_B(s.B0); WAIT_A(); mma<false>(s.acc[0][0], s.At, s.B0); ENDM;
  LDA(1u, 1); GUARD(s.acc[0][0]); pack_quad(&s.tmp[0], s.acc[0][0]); PST(0); PST(1); PST(2); PST(3);
  BAR; WAIT_A(); mma<false>(s.acc[1][0], s.At, s.B0); ENDM;
  LDB(s.B1, 1u, 1); GUARD(s.acc[1][0]); pack_quad(&s.tmp[4], s.acc[1][0]); PST(4); PST(5); PST(6); PST(7);
  BAR; WAIT_B(s.B1); mma<false>(s.acc[1][1], s.At, s.B1); ENDM;
  LDA(1u, 0);
  BAR; WAIT_A(); mma<false>(s.acc[0][1], s.At, s.B1); ENDM;
  LDB(s.B1, 0u, 1); LDA(0u, 0); wait_v<0>(); PFLAG(flagL);
  s.pw += REGION; s.own = own8; s.vls += adj8;
  pack_quad(&s.tmp[4], s.acc[1][1]); PST(4); PST(5); PST(6); PST(7);
  BAR; WAIT_B(s.B1); WAIT_A(); mma<false>(s.acc[0][1], s.At, s.B1); ENDM;
}

__global__ __launch_bounds__(512, 2) void gemm_a2a(Params p) {
  __shared__ __attribute__((aligned(1024))) unsigned char smem[8 * HTB + 32768];
  const int tid = threadIdx.x, wid = __builtin_amdgcn_readfirstlane(tid >> 6), lane = tid & 63;
  const int wr = wid >> 2, wc = wid & 3, fr = lane & 15, fq = lane >> 4;
  const unsigned blk = blockIdx.x, mem = blk & 7u, grp = (blk >> 3) & 31u;
  const unsigned pm = grp & 3u, cg = grp >> 2, sub = mem >> 2, mi = mem & 3u;
  const unsigned rank = (unsigned)p.rank & 7u;
  St s;
  char* wsp = p.ws[0];
  const unsigned long long pa = (unsigned long long)p.A, pw = (unsigned long long)p.W, pws = (unsigned long long)wsp;
  s.srdA = (u32x4){(unsigned)pa, (unsigned)(pa >> 32) & 0xffffu, A_BYTES, 0x20000u};
  s.srdB = (u32x4){(unsigned)pw, (unsigned)(pw >> 32) & 0xffffu, W_BYTES, 0x20000u};
  s.srdWs = (u32x4){(unsigned)pws, (unsigned)(pws >> 32) & 0xffffu, WS_USED, 0x20000u};
  unsigned info;
  if (cg < 7u) {
    const unsigned j = (cg + 1u) << 3;
    info = (j | 0u) | ((j | (1u + sub)) << 6) | ((j | (3u + sub)) << 12) | ((j | (5u + sub)) << 18) | (cg << 24);
  } else {
    const unsigned j0 = (0x6427u >> (4u * pm)) & 7u;
    const unsigned j1 = ((4u + 2u * pm + sub) % 7u) + 1u, j2 = ((2u + 2u * pm + sub) % 7u) + 1u, j3 = ((2u * pm + sub) % 7u) + 1u;
    info = ((j0 << 3) | 7u) | (((j1 << 3) | 7u) << 6) | (((j2 << 3) | 7u) << 12) | (((j3 << 3) | 7u) << 18) | (7u << 24);
  }
  auto ent = [&](unsigned k) -> unsigned { return (info >> (6u * k)) & 63u; };
  auto colB = [&](unsigned e) -> unsigned { return (((rank + (e >> 3)) & 7u) * 8u + (e & 7u)) * 256u; };
  auto offB = [&](unsigned k) -> unsigned { return mi * (32u * ktB) + colB(ent(k)); };
  const unsigned lds0 = (unsigned)(size_t)(LAS unsigned char*)smem;
  s.ldsw = lds0 + (unsigned)wid * 1024u;
  s.voffA = (unsigned)((8 * wid + (lane >> 3)) * (K_ * 2) + (((lane & 7) ^ (lane >> 3)) << 4));
  {
    const unsigned fB = (unsigned)((lane >> 4) | (((wid >> 1) & 1) << 2));
    s.voffB = (unsigned)((4 * wid + (lane >> 4)) * (N_ * 2)) + ((((unsigned)lane & 15u) ^ (fB << 1)) << 4);
  }
  s.sgA = pm * (256u * K_ * 2u) + mi * (32u * ktA); s.sgBL = offB(1); s.sgBR = offB(0);
  {
    const unsigned k0 = (sub * 16u + pm) * ktA;
    const unsigned a0 = s.sgA + k0, l0 = s.sgBL + (k0 << 13), q0 = s.sgBR + (k0 << 13);
    STAGE_B(SB(0, 0), l0, 0u); STAGE_A(SA(0, 0), a0, 0u); STAGE_B(SB(0, 1), q0, 0u); STAGE_A(SA(0, 1), a0, hA);
    STAGE_B(SB(1, 0), l0, ktB); STAGE_A(SA(1, 0), a0, ktA); STAGE_B(SB(1, 1), q0, ktB);
    s.a1 = a0 + ktA; s.a2 = a0 + 2u * ktA; s.bl2 = l0 + 2u * ktB; s.br2 = q0 + 2u * ktB; s.kA = k0 + 2u * ktA;
  }
  unsigned callv;
  asm volatile("buffer_load_dword %0, off, %1, %2 sc1" : "=v"(callv) : "s"(s.srdWs), "s"(WS_CALL + blk * 64u) : "memory");
  unsigned long long ob0 = (unsigned long long)p.out[ent(0) >> 3], ob1 = (unsigned long long)p.out[ent(1) >> 3], ob2 = (unsigned long long)p.out[ent(2) >> 3];
  {
    const int q = (lane & 15) >> 2, pp = lane & 3, g = lane >> 4;
    const int f = q | ((g & 1) << 2);
    s.aoff0 = lds0 + (unsigned)((wr * 64 + fr) * 128 + (((0 + fq) ^ (fr & 7)) << 4));
    s.aoff1 = lds0 + (unsigned)((wr * 64 + fr) * 128 + (((4 + fq) ^ (fr & 7)) << 4));
    s.boff0 = lds0 + 65536u + (unsigned)((8 * g + q) * 256 + (((2 * wc + 0) ^ f) << 5) + 8 * pp);
    s.boff1 = lds0 + 65536u + (unsigned)((8 * g + q) * 256 + (((2 * wc + 1) ^ f) << 5) + 8 * pp);
  }
  s.vlane16 = (unsigned)lane * 16u;
  {
    const unsigned r = (unsigned)lane >> 3, c = (unsigned)lane & 7u, cc = c & 3u;
    const unsigned sfq = ((cc & 1u) << 1) | (cc >> 1);
    s.vgat = (c >> 2) * 4096u + (sfq * 16u + r) * 16u;
    s.vdst = r * 2048u + c * 16u;
  }
  const unsigned uw = (unsigned)wid, rwr = uw >> 2, rwcp = uw & 1u, b1 = (uw >> 1) & 1u;
  const unsigned ai4 = mi >> 1, m4 = 2u * (mi & 1u) + b1, ai8 = mem >> 2, m8 = mem & 3u;
  unsigned rd4 = blk * SLOT + ((ai4 * 8u + rwr * 4u + 2u * rwcp) * 4u + m4) * 1024u;
  unsigned rd8 = blk * SLOT + ((ai8 * 8u + rwr * 4u + 2u * rwcp) * 4u + m8) * 1024u + b1 * 128u;
  const unsigned row0 = rank * 1024u + pm * 256u;
  unsigned ds4 = (row0 + ai4 * 128u + rwr * 64u + m4 * 16u) * 2048u + rwcp * 128u;
  unsigned ds8 = (row0 + ai8 * 128u + rwr * 64u + m8 * 16u + b1 * 8u) * 2048u + rwcp * 128u;
  unsigned pw0 = WS_PART + blk * SLOT + uw * 4096u;
  s.wslot = (int)(mem * 8u + uw);
  unsigned adj8 = (((2u * (mi & 1u)) ^ 2u) - m8) * 1024u;
  const unsigned lsp = lds0 + 8u * HTB;
  unsigned tsl;
  {
    const unsigned key = grp * 3u + mem + uw;
    constexpr unsigned F0 = S0_FIRST - 4, F1 = S1_FIRST - 4;
    constexpr unsigned W2 = (20 - S0_FIRST) / 2;
    constexpr unsigned G1 = (S0_FIRST + 4u > S1_FIRST ? S0_FIRST + 4u : S1_FIRST) - 4;
    constexpr unsigned G2 = (F0 + W2 > F1 ? F0 + W2 : F1);
    tsl = ((F1 + key % (16u - F1)) << 4) | (((F0 + key % W2) | ((G2 + key % (16u - G2)) << 4)) << 8) | (((F0 + key % 4u) | ((G1 + key % 4u) << 4)) << 16);
    static_assert(G1 + 3u + 4u <= 13u, "the last segment's pushes must leave before the final K-tile pair");
  }
  s.ts0 = -1; s.ts1 = -1; s.poison = 0u;
  s.d0 = s.d1 = 0ull;
  asm volatile("" : "+s"(s.d0), "+s"(s.d1), "+s"(s.ts0), "+s"(s.ts1));
  s.R0 = (u32x4){0u, 0u, 0u, 0u}; s.R1 = s.R0;
#pragma unroll
  for (int a = 0; a < 2; ++a)
#pragma unroll
    for (int bb = 0; bb < 2; ++bb)
#pragma unroll
      for (int m = 0; m < 4; ++m)
#pragma unroll
        for (int n = 0; n < 2; ++n) asm volatile("" : "=v"(s.acc[a][bb][m][n]));
  s.nxtBL = s.sgBL; s.nxtBR = offB(2);
  unsigned bl2 = offB(3), br3 = offB(4);
  ob0 += ds8 + (ent(0) & 7u) * 256u; ob1 += ds4 + (ent(1) & 7u) * 256u; ob2 += ds4 + (ent(2) & 7u) * 256u;
  unsigned pf0 = WS_PFLAG + grp * 1024u;
  asm volatile("" : "+s"(bl2), "+s"(br3), "+s"(ob0), "+s"(ob1), "+s"(ob2), "+s"(pf0));
  asm volatile("s_waitcnt lgkmcnt(0)" : "+s"(rd4), "+s"(rd8), "+s"(ds4), "+s"(ds8), "+s"(pw0), "+s"(adj8), "+s"(tsl), "+s"(s.wslot), "+s"(s.nxtBL), "+s"(s.nxtBR) :: "memory");
  if (wr == 1) BAR;
  wait_v<11>(); BAR;
  BAR;
  {
    LDB(s.B0, 0u, 0); WAIT_B(s.B0);
    ktile<0, V_FIRST>(s);
    asm volatile("" : "+v"(callv));
    s.call = __builtin_amdgcn_readfirstlane(callv) + 1u;
    ktile<1, V_PLAIN>(s);
    for (int t = 2; t < 12; t += 2) { ktile<0, V_PLAIN>(s); ktile<1, V_PLAIN>(s); }
    ktile<12, V_PLAIN>(s); ktile<13, V_PLAIN>(s);
  }
  const unsigned call = s.call;
  for (unsigned seg = 1; seg <= 3u; ++seg) {
    const unsigned k = seg - 1u;
    s.pw = pw0 + k * REGION;
    s.own = k == 0u ? 1u << mem : 3u << (2u * mi);
    s.vls = s.vlane16 + (lsp + uw * 4096u + (k == 0u ? adj8 : 0u));
    ktile<14, V_BG>(s);
    if (seg == 2u) {
      ktile<15, V_L>(s);
      ktile<0, V_L>(s);
    }
    unsigned seg2 = seg; asm volatile("" : "+s"(seg2));
    if (seg2 != 2u) {
      ktile<15, V_R>(s);
      ktile<0, V_R>(s);
    }
    s.keep = (unsigned)((int)(k - 1u) >> 31);
    s.prd0 = WS_PART + k * REGION + (k == 0u ? rd8 : rd4);
    s.prd1 = k == 0u ? s.prd0 ^ (4u * SLOT) : s.prd0 + 128u;
    s.lrd = lsp + (s.prd0 & 0x7fffu) + (k == 0u ? adj8 : 0u);
    s.pflag = pf0 + k * 32768u;
    s.TS0 = (int)(tsl >> (8u * k));
    s.pd = k == 0u ? ob0 : k == 1u ? ob1 : ob2;
    s.nxtBL = bl2;
    s.nxtBR = k == 0u ? s.sgBR : br3;
    asm volatile("" : "+s"(s.nxtBL), "+s"(s.nxtBR), "+s"(s.pd), "+s"(s.pflag), "+s"(s.prd0), "+s"(s.prd1), "+s"(s.lrd), "+s"(s.TS0), "+s"(s.keep));
    ktile<1, V_BG>(s); ktile<2, V_BG>(s); ktile<3, V_BG>(s); ktile<4, V_BG>(s); ktile<5, V_BG>(s); ktile<6, V_BG>(s); ktile<7, V_BG>(s);
    ktile<8, V_BG>(s); ktile<9, V_BG>(s); ktile<10, V_BG>(s); ktile<11, V_BG>(s); ktile<12, V_BG>(s); ktile<13, V_BG>(s);
  }
  s.pw = pw0 + 3u * REGION;
  const unsigned fblk = pf0 + 3u * 32768u;
  final_pair(s, fblk, 1u << mem, adj8);
  GUARD(s.acc[0][1]); pack_quad(&s.tmp[0], s.acc[0][1]); PST(0); PST(1); PST(2); PST(3);
  asm volatile("s_waitcnt vmcnt(0) lgkmcnt(0)" ::: "memory"); PFLAG(fblk + 32768u);
  const int lane2 = lane_id();
  const unsigned long long o3 = (unsigned long long)p.out[ent(3) >> 3], o4 = (unsigned long long)p.out[0];
  char* const w0 = p.ws[ent(0) >> 3]; char* const w1 = p.ws[ent(1) >> 3]; char* const w2 = p.ws[ent(2) >> 3]; char* const w3 = p.ws[ent(3) >> 3];
  const unsigned o3lo = (unsigned)o3, o3hi = (unsigned)(o3 >> 32) & 0xffffu, o4lo = (unsigned)o4, o4hi = (unsigned)(o4 >> 32) & 0xffffu;
  if (wr == 0) BAR;
  LAS unsigned* lflag = (LAS unsigned*)(smem + 8 * HTB + ((((2u * (mi & 1u)) ^ 2u) + 1u) * 1024u));
  if (wid == 0) {
    const unsigned bad = spin_flags(s, fblk);
    if (lane2 == 0) lflag[0] = bad;
    asm volatile("s_waitcnt lgkmcnt(0)" ::: "memory");
  }
  BAR;
  {
    s.prd0 = WS_PART + 3u * REGION + rd4; s.prd1 = s.prd0 + 128u; s.lrd = lsp + (rd4 & 0x7fffu);
    PLDG(1); PLDG(2); PLDG(3); PLDG(5); PLDG(6); PLDG(7); PLDL(0, 0); PLDL(4, 128);
    make_sel(s);
    if (wid == 0) {
      const unsigned bad = spin_flags(s, fblk + 32768u);
      if (lane2 == 0) lflag[1] = bad;
      asm volatile("s_waitcnt lgkmcnt(0)" ::: "memory");
    }
    s.poison |= __builtin_amdgcn_readfirstlane(*(volatile LAS unsigned*)lflag);
    asm volatile("s_waitcnt vmcnt(3) lgkmcnt(0)" : "+v"(s.tmp[0]), "+v"(s.tmp[1]), "+v"(s.tmp[2]), "+v"(s.tmp[3]) :: "memory");
    reduce4<1>(s, 0);
    const u32x4 o0 = reduce_fin<true>(s);
    asm volatile("s_waitcnt vmcnt(0)" : "+v"(s.tmp[4]), "+v"(s.tmp[5]), "+v"(s.tmp[6]), "+v"(s.tmp[7]) :: "memory");
    reduce4<1>(s, 4);
    const u32x4 o1 = reduce_fin<true>(s);
    BAR;
    s.prd0 = WS_PART + 4u * REGION + rd8; s.prd1 = s.prd0 ^ (4u * SLOT); s.lrd = lsp + (rd8 & 0x7fffu) + adj8;
    PLDG(1); PLDG(2); PLDG(3); PLDG(4); PLDG(5); PLDG(6); PLDG(7); PLDL(0, 0);
    const unsigned doff = ds4 + (ent(3) & 7u) * 256u;
    SEND(o0, o3lo, o3hi, doff);
    SEND(o1, o3lo, o3hi, doff + 8u * 2048u);
    s.poison |= __builtin_amdgcn_readfirstlane(*(volatile LAS unsigned*)(lflag + 1));
    asm volatile("s_waitcnt vmcnt(2) lgkmcnt(0)" : "+v"(s.tmp[0]), "+v"(s.tmp[1]), "+v"(s.tmp[2]), "+v"(s.tmp[3]), "+v"(s.tmp[4]), "+v"(s.tmp[5]), "+v"(s.tmp[6]), "+v"(s.tmp[7]) :: "memory");
    reduce4<1>(s, 0); reduce4<0>(s, 4);
    const u32x4 o2 = reduce_fin<true>(s);
    SEND(o2, o4lo, o4hi, ds8 + (ent(4) & 7u) * 256u);
  }
  wait_v<0>();
  BAR;
  if (wid == 0) {
    if (lane2 < 4) {
      const unsigned e = (info >> (6u * (unsigned)lane2)) & 63u;
      char* const wd = lane2 == 0 ? w0 : lane2 == 1 ? w1 : lane2 == 2 ? w2 : w3;
      unsigned* f = (unsigned*)(wd + WS_DONE + rank * 1024u) + ((e & 7u) * 4u + pm) * 8u;
      if (lane2 == 0) __hip_atomic_store(f + mem, call, __ATOMIC_RELAXED, __HIP_MEMORY_SCOPE_SYSTEM);
      else __hip_atomic_store((unsigned long long*)f + mi, ((unsigned long long)call << 32) | call, __ATOMIC_RELAXED, __HIP_MEMORY_SCOPE_SYSTEM);
    }
    if (lane2 == 0) __hip_atomic_store((unsigned*)(wsp + WS_CALL + blk * 64u), call, __ATOMIC_RELAXED, __HIP_MEMORY_SCOPE_AGENT);
    if (blk == 0) {
      const unsigned long long* f = (const unsigned long long*)(wsp + WS_DONE) + lane2;
      unsigned* tmo = (unsigned*)(wsp + WS_TMO);
      const unsigned long long t0 = __builtin_amdgcn_s_memrealtime();
      for (unsigned n = 1;; ++n) {
        bool ok = true;
#pragma unroll
        for (unsigned j = 1; j < 8; ++j) {
          const unsigned long long v0 = __hip_atomic_load(f + ((rank + j) & 7u) * 128u, __ATOMIC_RELAXED, __HIP_MEMORY_SCOPE_SYSTEM);
          const unsigned long long v1 = __hip_atomic_load(f + ((rank + j) & 7u) * 128u + 64u, __ATOMIC_RELAXED, __HIP_MEMORY_SCOPE_SYSTEM);
          ok = ok && ((unsigned)v0 >= call) && ((unsigned)(v0 >> 32) >= call) && ((unsigned)v1 >= call) && ((unsigned)(v1 >> 32) >= call);
        }
        if (__all(ok)) break;
        if ((n & 15u) == 0u &&
            (__builtin_amdgcn_readfirstlane(__hip_atomic_load(tmo, __ATOMIC_RELAXED, __HIP_MEMORY_SCOPE_AGENT)) == call ||
             __builtin_amdgcn_s_memrealtime() - t0 > DEADLINE)) {
          if (lane2 == 0) {
            __hip_atomic_store(tmo, call, __ATOMIC_RELAXED, __HIP_MEMORY_SCOPE_AGENT);
            __hip_atomic_store((unsigned*)(o4 + (size_t)rank * 1024u * 2048u), 0x7fc07fc0u, __ATOMIC_RELAXED, __HIP_MEMORY_SCOPE_SYSTEM);
          }
          break;
        }
      }
      __builtin_amdgcn_fence(__ATOMIC_ACQUIRE, "");
    }
  }
}

}

extern "C" void kernel_launch(int rank, int nranks, ncclComm_t comm,
                              hipStream_t stream,
                              void* const* d_in, const int* in_sizes, int n_in,
                              void* d_out, int out_size,
                              const int* in_dtypes, int out_dtype,
                              void* const* d_out_all,
                              void* const* d_ws_all, size_t ws_bytes) {
  using namespace ga2a;
  (void)comm; (void)in_sizes; (void)n_in; (void)d_out; (void)out_size; (void)in_dtypes; (void)out_dtype;
  if (nranks != NR || ws_bytes < (size_t)WS_USED) { fprintf(stderr, "unsupported configuration\n"); return; }
  Params p;
  p.A = (const u16*)d_in[0];
  p.W = (const u16*)d_in[1];
  for (int j = 0; j < NR; ++j) { p.out[j] = (char*)d_out_all[(rank + j) & 7]; p.ws[j] = (char*)d_ws_all[(rank + j) & 7]; }
  p.rank = rank;
  hipLaunchKernelGGL(gemm_a2a, dim3(256), dim3(512), 0, stream, p);
  HIP_CHECK(hipGetLastError());
}
```
